```python
import numpy as np

kernel_name = "gpu_ag_gemm_m4096_k4096_n16384_f32_silu"


ABSMAX_RTOL = 0.001

COMPUTE_DTYPE = "float32"
EPILOGUE = "silu"
IN_QUANT = None
OUT_QUANT = None
IN_DTYPES = {"x": "float32", "w_mat": "float32"}
WORLD_SIZE = 8

def setup_inputs():

    x = np.random.randn(4096, 4096).astype(np.float32)
    w_mat = (np.random.randn(4096, 16384) / np.sqrt(4096)).astype(np.float32)
    return {"x": x, "w_mat": w_mat}

def _epilogue(y):
    return (y / (1.0 + np.exp(-np.clip(y, -60.0, 60.0)))).astype(np.float32)

def reference(x, w_mat):
    y = (x @ w_mat).astype(np.float32)
    return _epilogue(y)
```

```cpp
#include <hip/hip_runtime.h>
#include <rccl/rccl.h>
#include <cstdio>

#define HIP_CHECK(expr) do { hipError_t _e = (expr); if (_e != hipSuccess) { \
    fprintf(stderr, "HIP error %s at %s:%d\n", hipGetErrorString(_e), __FILE__, __LINE__); \
    return; } } while (0)

#define LAS __attribute__((address_space(3)))
typedef _Float16 f16x8 __attribute__((ext_vector_type(8)));
typedef _Float16 f16x2 __attribute__((ext_vector_type(2)));
typedef float f32x4 __attribute__((ext_vector_type(4)));
typedef float f32x2 __attribute__((ext_vector_type(2)));
typedef unsigned u32x4 __attribute__((ext_vector_type(4)));
typedef unsigned u32x2 __attribute__((ext_vector_type(2)));

namespace agk {

constexpr int NR = 8;
constexpr int KD = 4096;
constexpr int NL = 2048;
constexpr int NH = 128;
constexpr int NSLOT = 6;
constexpr int SLOT_B = 24576;
constexpr int CTRL = NSLOT * SLOT_B;
constexpr int LDS_B = CTRL + 256;
constexpr int NWAVE = 12;
#define COMM_P 4
#define COMM_D 2
#define POLL_SLEEP 4

constexpr size_t OFF_CNT = 0;
constexpr size_t OFF_TMO = 32768;
constexpr size_t OFF_XFLAG = 65536;
constexpr size_t OFF_WFLAG = 131072;
constexpr size_t OFF_DBG = 262144;
constexpr size_t OFF_X16 = 1ull << 20;
constexpr size_t X16_B = 32ull << 20;
constexpr size_t OFF_W16 = OFF_X16 + 2 * X16_B;
constexpr size_t WS_NEED = OFF_W16 + (16ull << 20);
constexpr unsigned long long DEADLINE = 50000000ull;

struct Params {
    const float* x;
    const float* w;
    float* out;
    char* myws;
    char* dst[NR];
    int rank;
};

template <int V> struct IC { static constexpr int value = V; };
template <int N, int I = 0, class F> __device__ __forceinline__ void static_for(F&& f) {
    if constexpr (I < N) { f(IC<I>{}); static_for<N, I + 1>(f); }
}

__device__ __forceinline__ unsigned long long rt() { return __builtin_amdgcn_s_memrealtime(); }

__device__ __forceinline__ unsigned pk16(float a, float b) {
    f32x2 v = {a, b};
    f16x2 h = __builtin_convertvector(v, f16x2);
    return __builtin_bit_cast(unsigned, h);
}

__device__ __forceinline__ void st16_sys(const char* sbase, unsigned voff, u32x4 v) {
    asm volatile("global_store_dwordx4 %0, %1, %2 sc0 sc1\n\ts_nop 1" :: "v"(voff), "v"(v), "s"(sbase) : "memory");
}
__device__ __forceinline__ void st16_sys_p(const char* p, u32x4 v) {
    asm volatile("global_store_dwordx4 %0, %1, off sc0 sc1\n\ts_nop 1" :: "v"(p), "v"(v) : "memory");
}
__device__ __forceinline__ void st4_sys_p(const char* p, unsigned v) {
    asm volatile("global_store_dword %0, %1, off sc0 sc1" :: "v"(p), "v"(v) : "memory");
}

#define VMW(n) asm volatile("s_waitcnt vmcnt(" #n ")" ::: "memory")

__device__ __forceinline__ void wait_vm_dyn(int n) {
    switch (n) {
        case 0: VMW(0); break;   case 1: VMW(1); break;
        case 8: VMW(8); break;   case 9: VMW(9); break;
        case 16: VMW(16); break; case 17: VMW(17); break;
        case 24: VMW(24); break; case 25: VMW(25); break;
        case 32: VMW(32); break; case 33: VMW(33); break;
        case 40: VMW(40); break; case 41: VMW(41); break;
        default: VMW(0); break;
    }
}

__device__ __forceinline__ void glds8(const char* sbase, unsigned voff, unsigned voff2, unsigned ldst) {
    unsigned keep;
    asm volatile(
        "s_mov_b32 %0, m0\n\t"
        "s_mov_b32 m0, %4\n\t"
        "s_nop 0\n\t"
        "global_load_lds_dwordx4 %1, %3 sc0 sc1\n\t"
        "global_load_lds_dwordx4 %1, %3 offset:1024 sc0 sc1\n\t"
        "global_load_lds_dwordx4 %1, %3 offset:2048 sc0 sc1\n\t"
        "global_load_lds_dwordx4 %1, %3 offset:3072 sc0 sc1\n\t"
        "s_add_u32 m0, %4, 4096\n\t"
        "s_nop 0\n\t"
        "global_load_lds_dwordx4 %2, %3 sc0 sc1\n\t"
        "global_load_lds_dwordx4 %2, %3 offset:1024 sc0 sc1\n\t"
        "global_load_lds_dwordx4 %2, %3 offset:2048 sc0 sc1\n\t"
        "global_load_lds_dwordx4 %2, %3 offset:3072 sc0 sc1\n\t"
        "s_mov_b32 m0, %0"
        : "=&s"(keep) : "v"(voff), "v"(voff2), "s"(sbase), "s"(ldst) : "memory", "scc");
}

__device__ __forceinline__ void lds_wr32(unsigned addr, unsigned v) {
    asm volatile("ds_write_b32 %0, %1" :: "v"(addr), "v"(v) : "memory");
}
__device__ __forceinline__ u32x4 lds_rd128_wait(unsigned addr) {
    u32x4 v;
    asm volatile("ds_read_b128 %0, %1\n\ts_waitcnt lgkmcnt(0)" : "=&v"(v) : "v"(addr) : "memory");
    return v;
}
__device__ __forceinline__ unsigned rfl(unsigned v) { return __builtin_amdgcn_readfirstlane(v); }
__device__ __forceinline__ unsigned umin(unsigned a, unsigned b) { return a < b ? a : b; }

__device__ __forceinline__ float silu(float y) {
    float t = __builtin_amdgcn_fmed3f(y, -60.0f, 60.0f);
    float e = __builtin_amdgcn_exp2f(t * -1.44269504088896340736f);
    return y * __builtin_amdgcn_rcpf(1.0f + e);
}

#define STAMP(i) do { } while (0)

__global__ void __launch_bounds__(NWAVE * 64, 3) ag_gemm_silu(Params p) {
    extern __shared__ __attribute__((aligned(16))) char lds_raw[];
    LAS char* lds = (LAS char*)lds_raw;
    const unsigned ldsb = (unsigned)(size_t)lds;
    const int tid = threadIdx.x;
    const int wave = rfl(tid >> 6), lane = tid & 63;
    const int b = blockIdx.x;
    const int xcd = b & 7, idx = b >> 3;
    const int ti = (xcd >> 1) * 4 + (idx >> 3);
    const int tj = (xcd & 1) * 8 + (idx & 7);
    char* myws = p.myws;
    STAMP(0);

    unsigned* cntp = (unsigned*)(myws + OFF_CNT) + b * 16 + wave;
    const unsigned call = rfl(__hip_atomic_load(cntp, __ATOMIC_RELAXED, __HIP_MEMORY_SCOPE_AGENT)) + 1u;
    if (lane == 0) __hip_atomic_store(cntp, call, __ATOMIC_RELAXED, __HIP_MEMORY_SCOPE_AGENT);
    unsigned* tmo = (unsigned*)(myws + OFF_TMO);
    const unsigned fbase = call * 256u;
    const size_t x16off = OFF_X16 + (size_t)(call & 1u) * X16_B;

    if (tid < 64) *(LAS unsigned*)(lds + CTRL + tid * 4) = 0u;
    asm volatile("s_waitcnt lgkmcnt(0)" ::: "memory");
    __builtin_amdgcn_s_barrier();
    asm volatile("" ::: "memory");

    const int g = lane >> 4, q = lane & 15;

    if (wave < 8) {
        {
            const int wi = ((ti & 3) == 0) ? 12 + (ti >> 2) : (ti >> 2) * 3 + (ti & 3) - 1;
            const int h = wi * 8 + wave;
            const float* wsrc = p.w + (size_t)(h * 32 + 4 * g) * NL + tj * 128 + 4 * q;
            f32x4 v[2][8];
#pragma unroll
            for (int cg = 0; cg < 2; ++cg)
#pragma unroll
                for (int i = 0; i < 8; ++i)
                    v[cg][i] = __builtin_nontemporal_load((const f32x4*)(wsrc + (size_t)((i & 3) + 16 * (i >> 2)) * NL + cg * 64));
            char* dst = myws + OFF_W16 + (size_t)(tj * NH + h) * 8192 + lane * 16;
#pragma unroll
            for (int cg = 0; cg < 2; ++cg)
#pragma unroll
                for (int f = 0; f < 4; ++f) {
                    u32x4 o;
                    o.x = pk16(v[cg][0][f], v[cg][1][f]); o.y = pk16(v[cg][2][f], v[cg][3][f]);
                    o.z = pk16(v[cg][4][f], v[cg][5][f]); o.w = pk16(v[cg][6][f], v[cg][7][f]);
                    st16_sys_p(dst + (cg * 4 + f) * 1024, o);
                }
            VMW(0);
            if (lane == 0) st4_sys_p(myws + OFF_WFLAG + (size_t)(tj * NH + h) * 4, call);
        }
        STAMP(1);
        const int wr = wave >> 1, wc = wave & 1;
        f32x4 acc[4][4];
#pragma unroll
        for (int m = 0; m < 4; ++m)
#pragma unroll
            for (int n = 0; n < 4; ++n) acc[m][n] = (f32x4){0.f, 0.f, 0.f, 0.f};
        f16x8 a0[4], b0[4], a1[4], b1[4];
        const unsigned aoff = wr * 4096 + lane * 16, boff = 16384 + wc * 4096 + lane * 16;
        const unsigned fullad = ldsb + CTRL, freead = ldsb + CTRL + 64 + wave * 4;
        unsigned known = 0;
        unsigned sb = 0;

        auto refresh = [&]() {
            u32x4 f = lds_rd128_wait(fullad);
            known = rfl(umin(umin(f.x, f.y), f.z));
        };
        auto wait_full = [&](unsigned need) {
            if (known >= need) return;
            refresh();
            if (known >= need) return;
            const unsigned long long t0 = rt();
            unsigned sp = 0;
            for (;;) {
                refresh();
                if (known >= need) break;
                if ((++sp & 1023u) == 0u && rt() - t0 > 2 * DEADLINE) {
                    lds_wr32(ldsb + CTRL + 12, 1u);
                    known = NH;
                    break;
                }
            }
        };
#define RD_FRAGS(A, B) do { \
            _Pragma("unroll") for (int n = 0; n < 4; ++n) B[n] = *(const LAS f16x8*)(lds + sb + boff + n * 1024); \
            _Pragma("unroll") for (int m = 0; m < 4; ++m) A[m] = *(const LAS f16x8*)(lds + sb + aoff + m * 1024); \
            sb = (sb == (NSLOT - 1) * SLOT_B) ? 0u : sb + SLOT_B; } while (0)
#define MMA(A, B) do { __builtin_amdgcn_s_setprio(1); \
            _Pragma("unroll") for (int m = 0; m < 4; ++m) _Pragma("unroll") for (int n = 0; n < 4; ++n) \
                acc[m][n] = __builtin_amdgcn_mfma_f32_16x16x32_f16(A[m], B[n], acc[m][n], 0, 0, 0); \
            __builtin_amdgcn_s_setprio(0); } while (0)
#define STEP(CA, CB, NA, NB, hn) do {   \
            const unsigned need_ = (unsigned)(hn) + 1u; \
            const bool more_ = (hn) < NH; \
            __builtin_amdgcn_s_waitcnt(0xC07F);   \
            if (more_ && known < need_) refresh(); \
            const bool early_ = more_ && known >= need_; \
            if (early_) { RD_FRAGS(NA, NB); lds_wr32(freead, need_); } \
            MMA(CA, CB); \
            if (more_ && !early_) { wait_full(need_); RD_FRAGS(NA, NB); lds_wr32(freead, need_); } \
            } while (0)

        wait_full(1);
        STAMP(2);
        RD_FRAGS(a0, b0); lds_wr32(freead, 1);
        for (int h = 0; h < NH; h += 2) {
            STEP(a0, b0, a1, b1, h + 1);
            STEP(a1, b1, a0, b0, h + 2);
        }
        STAMP(3);
        u32x4 cw = lds_rd128_wait(fullad);
        const bool bad = rfl(cw.w) != 0u ||
                         rfl(__hip_atomic_load(tmo, __ATOMIC_RELAXED, __HIP_MEMORY_SCOPE_AGENT)) == call;
        float* obase = p.out + (size_t)(ti * 256 + wr * 64 + 4 * g) * NL + tj * 128 + wc * 64 + 4 * q;
#pragma unroll
        for (int m = 0; m < 4; ++m)
#pragma unroll
            for (int r = 0; r < 4; ++r) {
                f32x4 o;
                o.x = silu(acc[m][0][r]); o.y = silu(acc[m][1][r]);
                o.z = silu(acc[m][2][r]); o.w = silu(acc[m][3][r]);
                if (bad) { const float qn = __builtin_nanf(""); o = (f32x4){qn, qn, qn, qn}; }
                st16_sys_p((const char*)(obase + (size_t)(m * 16 + r) * NL), __builtin_bit_cast(u32x4, o));
            }
        STAMP(4);
    } else if (wave < 11) {
        const int L = wave - 8;
        const unsigned soff = (L < 2) ? (unsigned)x16off + (unsigned)ti * (NH * 16384) + L * 8192
                                      : (unsigned)OFF_W16 + (unsigned)tj * (NH * 8192);
        const char* src = myws + rfl(soff);
        const unsigned step = (L < 2) ? 16384u : 8192u;
        const unsigned foff = (L < 2) ? (unsigned)OFF_XFLAG + (unsigned)(ti * 16 + L * 8 + (lane & 7)) * 128u
                                      : (unsigned)OFF_WFLAG + (unsigned)(tj * NH + lane * 2) * 4u;
        const char* flagp = myws + foff;
        const unsigned ldst0 = ldsb + L * 8192;
        const unsigned fullad = ldsb + CTRL + L * 4, freead = ldsb + CTRL + 64;
        unsigned voff = lane * 16;
        unsigned h = 0, pub = 0, ready = 0, freed = 0, slot = 0;
        bool dead = false;
        unsigned long long tw = 0; unsigned sp = 0;
        while (h < NH) {
            if (h < ready && h < freed + NSLOT) {
                glds8(src, voff, voff + 4096, ldst0 + slot);
                voff += step; ++h;
                slot = (slot == (NSLOT - 1) * SLOT_B) ? 0u : slot + SLOT_B;
                sp = 0;
                continue;
            }
            if (h >= ready) {
                u32x2 fv;
                asm volatile("global_load_dwordx2 %0, %1, off sc0 sc1" : "=v"(fv) : "v"(flagp) : "memory");
                while (pub < h) {
                    wait_vm_dyn(8 * (int)(h - pub - 1) + 1);
                    ++pub; lds_wr32(fullad, pub);
                }
                asm volatile("s_waitcnt vmcnt(0)" : "+v"(fv) :: "memory");
                if (L < 2) {
                    unsigned d = fv.x - fbase;
                    d = (d <= (unsigned)NH) ? d : ((int)d > 0 ? (unsigned)NH : 0u);
                    d = umin(d, (unsigned)__builtin_amdgcn_update_dpp(0, (int)d, 0xB1, 0xf, 0xf, false));
                    d = umin(d, (unsigned)__builtin_amdgcn_update_dpp(0, (int)d, 0x4E, 0xf, 0xf, false));
                    d = umin(d, (unsigned)__builtin_amdgcn_update_dpp(0, (int)d, 0x141, 0xf, 0xf, false));
                    ready = rfl(d);
                } else {
                    const bool ok = (fv.x == call) && (fv.y == call);
                    const unsigned long long m = __ballot(ok);
                    ready = (m == ~0ull) ? (unsigned)NH : 2u * (unsigned)__builtin_ctzll(~m);
                }
                if (h >= ready) {
                    if (sp == 0) tw = rt();
                    ++sp;
                    if ((sp & 63u) == 0u) {
                        const unsigned tv = rfl(__hip_atomic_load(tmo, __ATOMIC_RELAXED, __HIP_MEMORY_SCOPE_AGENT));
                        if (tv == call || rt() - tw > DEADLINE) dead = true;
                    }
                    if (dead) {
                        if (lane == 0) __hip_atomic_store(tmo, call, __ATOMIC_RELAXED, __HIP_MEMORY_SCOPE_AGENT);
                        lds_wr32(ldsb + CTRL + 12, 1u);
                        VMW(0);
                        ready = NH;
                    } else {
                        __builtin_amdgcn_s_sleep(POLL_SLEEP);
                    }
                }
            } else {
                if (pub < h) {
                    wait_vm_dyn(8 * (int)(h - pub - 1));
                    ++pub; lds_wr32(fullad, pub);
                }
                u32x4 f0 = lds_rd128_wait(freead), f1 = lds_rd128_wait(freead + 16);
                freed = rfl(umin(umin(umin(f0.x, f0.y), umin(f0.z, f0.w)), umin(umin(f1.x, f1.y), umin(f1.z, f1.w))));
                if (pub == h && h >= freed + NSLOT) {
                    if (sp == 0) tw = rt();
                    ++sp;
                    if ((sp & 1023u) == 0u && rt() - tw > 3 * DEADLINE) { lds_wr32(ldsb + CTRL + 12, 1u); freed = NH; }
                    __builtin_amdgcn_s_sleep(1);
                }
            }
        }
        while (pub < h) {
            wait_vm_dyn(8 * (int)(h - pub - 1));
            ++pub; lds_wr32(fullad, pub);
        }
        STAMP(1);
    } else if (b < 32) {
        const int rb = b;
        const float* xs = p.x + (size_t)(rb * 16 + q) * KD + 4 * g;
        const size_t doff = x16off + (size_t)(p.rank * 2 + (rb >> 4)) * (NH * 16384) + (size_t)(rb & 15) * 1024;
        const char* dst[NR];
#pragma unroll
        for (int j = 0; j < NR; ++j) dst[j] = p.dst[j] + doff;
        const char* flagp = p.dst[0];
#pragma unroll
        for (int j = 1; j < NR; ++j) flagp = ((lane & 7) == j) ? p.dst[j] : flagp;
        flagp += OFF_XFLAG + (size_t)(p.rank * 32 + rb) * 128;
        unsigned voff = lane * 16;
        constexpr int P = COMM_P;
        constexpr int D = COMM_D;
        static_assert(D >= 1 && D <= P && NH % P == 0 && 11 * (D - 1) + 1 <= 63 && 11 * P <= 63 + 11, "comm window");
        f32x4 xa[P][2];
#define XLD(slot_, hh) do { const float* _s = xs + (hh) * 32; \
            asm volatile("global_load_dwordx4 %0, %2, off nt\n\tglobal_load_dwordx4 %1, %2, off offset:64 nt" \
                         : "=&v"(xa[slot_][0]), "=&v"(xa[slot_][1]) : "v"(_s) : "memory"); } while (0)
#define XWAIT(n_, slot_) asm volatile("s_waitcnt vmcnt(%2)" : "+v"(xa[slot_][0]), "+v"(xa[slot_][1]) : "i"(n_) : "memory")
#pragma unroll
        for (int s = 0; s < P; ++s) XLD(s, s);
        auto body = [&](auto sc, auto wc_, int h) {
            constexpr int s = decltype(sc)::value;
            XWAIT(decltype(wc_)::value, s);
            u32x4 o;
            o.x = pk16(xa[s][0].x, xa[s][0].y); o.y = pk16(xa[s][0].z, xa[s][0].w);
            o.z = pk16(xa[s][1].x, xa[s][1].y); o.w = pk16(xa[s][1].z, xa[s][1].w);
            asm volatile("" : "+v"(o));
            const int hn = (h + P < NH) ? h + P : NH - 1;
            XLD(s, hn);
#pragma unroll
            for (int j = 0; j < NR; ++j) st16_sys(dst[j], voff, o);
            voff += 16384;
            if (lane < 8) st4_sys_p(flagp, fbase + (unsigned)(h >= D ? h - D + 1 : 0));
        };
        constexpr int WS_ = 11 * (D - 1) + 1;
        static_for<P>([&](auto sc) {
            constexpr int s = decltype(sc)::value;
            constexpr int nl = 2 * (P - 1 - s) + 11 * s;
            constexpr int w = (s >= D && WS_ < nl) ? WS_ : nl;
            body(sc, IC<(w > 63 ? 63 : w)>{}, s);
        });
        for (int h0 = P; h0 < NH; h0 += P) {
            static_for<P>([&](auto sc) { body(sc, IC<WS_>{}, h0 + decltype(sc)::value); });
        }
        static_for<D>([&](auto sc) {
            constexpr int i = decltype(sc)::value;
            constexpr int w = 11 * (D - 1 - i) + 1 + i;
            asm volatile("s_waitcnt vmcnt(%0)" :: "i"(w) : "memory");
            if (lane < 8) st4_sys_p(flagp, fbase + (unsigned)(NH - D + i + 1));
        });
        STAMP(1);
    }
}

}

extern "C" void kernel_launch(int rank, int nranks, ncclComm_t comm,
                              hipStream_t stream,
                              void* const* d_in, const int* in_sizes, int n_in,
                              void* d_out, int out_size,
                              const int* in_dtypes, int out_dtype,
                              void* const* d_out_all,
                              void* const* d_ws_all, size_t ws_bytes) {
    using namespace agk;
    if (nranks != NR || (unsigned)rank >= (unsigned)NR || ws_bytes < WS_NEED || n_in < 2) {
        fprintf(stderr, "ag_gemm_silu: unexpected configuration\n");
        return;
    }
    static bool attr_done = false;
    if (!attr_done) {
        HIP_CHECK(hipFuncSetAttribute((const void*)ag_gemm_silu, hipFuncAttributeMaxDynamicSharedMemorySize, LDS_B));
        attr_done = true;
    }
    Params p;
    p.x = (const float*)d_in[0];
    p.w = (const float*)d_in[1];
    p.out = (float*)d_out;
    p.myws = (char*)d_ws_all[rank];
    for (int i = 0; i < NR; ++i) p.dst[i] = (char*)d_ws_all[(rank + 1 + i) & 7];
    p.rank = rank;
    ag_gemm_silu<<<dim3(256), dim3(NWAVE * 64), LDS_B, stream>>>(p);
    HIP_CHECK(hipGetLastError());
}
```

```cpp
#include <hip/hip_runtime.h>
#include <rccl/rccl.h>
#include <cstdio>

#define HIP_CHECK(expr) do { hipError_t _e = (expr); if (_e != hipSuccess) { \
    fprintf(stderr, "HIP error %s at %s:%d\n", hipGetErrorString(_e), __FILE__, __LINE__); \
    return; } } while (0)

#define LAS __attribute__((address_space(3)))
typedef _Float16 f16x8 __attribute__((ext_vector_type(8)));
typedef _Float16 f16x2 __attribute__((ext_vector_type(2)));
typedef float f32x4 __attribute__((ext_vector_type(4)));
typedef float f32x2 __attribute__((ext_vector_type(2)));
typedef unsigned u32x4 __attribute__((ext_vector_type(4)));
typedef unsigned u32x2 __attribute__((ext_vector_type(2)));

namespace agk {

constexpr int NR = 8;
constexpr int KD = 4096;
constexpr int NL = 2048;
constexpr int NH = 128;
constexpr int NSLOT = 6;
constexpr int WB = 9216;
constexpr int SLOT_B = 16384 + WB;
constexpr int CTRL = NSLOT * SLOT_B;
constexpr int LDS_B = CTRL + 256;
constexpr int NWAVE = 12;
constexpr int NCWG = 16;
#define COMM_P 3
#define COMM_R 1
#define COMM_WGS 16
#define COMM_WPG 2
#define COMM_SIDES 1
#define COMM_SWAP 0
#define POLL_SLEEP 2
#define POLL_LAG 2
#define POLL_NX 8
#define W_STAG 0
#define W_WSTAG 4
#define W_DELAY0 0
#define POLL_W 2
#define POLL_MODE 0
#define POLL_ND 2
constexpr int CWPG = COMM_WPG;
constexpr int NCW = COMM_WGS * COMM_WPG;
constexpr int IPH = 32 / COMM_R;
constexpr int NITEM = NH * IPH;

constexpr int C_FULL = 0;
constexpr int C_RDYA = 12;
constexpr int C_RDYB = 16;
constexpr int C_BAD = 20;
constexpr int C_WCNT = 24;
constexpr int C_FREE = 64;

constexpr size_t OFF_CNT = 0;
constexpr size_t OFF_TMO = 32768;
constexpr size_t OFF_ICTR = 65536;
constexpr size_t OFF_WFLAG = 131072;
constexpr size_t OFF_DBG = 262144;
constexpr size_t OFF_X16 = 1ull << 20;
constexpr size_t X16_B = 32ull << 20;
constexpr size_t OFF_W16 = OFF_X16 + 2 * X16_B;
constexpr size_t WS_NEED = OFF_W16 + (size_t)15 * NH * WB;
constexpr unsigned long long DEADLINE = 50000000ull;

struct Params {
    const float* x;
    const float* w;
    float* out;
    char* myws;
    char* dst[NR];
    int rank;
};
#define DBGRET(bit) do { } while (0)

template <int V> struct IC { static constexpr int value = V; };
template <int N, int I = 0, class F> __device__ __forceinline__ void static_for(F&& f) {
    if constexpr (I < N) { f(IC<I>{}); static_for<N, I + 1>(f); }
}

__device__ __forceinline__ unsigned long long rt() {
    unsigned long long t;
    asm volatile("s_memrealtime %0\n\ts_waitcnt lgkmcnt(0)" : "=s"(t) :: "memory");
    return t;
}

__device__ __forceinline__ unsigned pk16(float a, float b) {
    f32x2 v = {a, b};
    f16x2 h = __builtin_convertvector(v, f16x2);
    return __builtin_bit_cast(unsigned, h);
}

__device__ __forceinline__ void st16_sys(const char* sbase, unsigned voff, u32x4 v) {
    asm volatile("global_store_dwordx4 %0, %1, %2 sc0 sc1\n\ts_nop 1" :: "v"(voff), "v"(v), "s"(sbase) : "memory");
}
__device__ __forceinline__ void st16_sys_o(const char* sbase, unsigned voff, u32x4 v) {
    asm volatile("global_store_dwordx4 %0, %1, %2 offset:1024 sc0 sc1\n\ts_nop 1" :: "v"(voff), "v"(v), "s"(sbase) : "memory");
}
__device__ __forceinline__ void st16_sys_p(const char* p, u32x4 v) {
    asm volatile("global_store_dwordx4 %0, %1, off sc0 sc1\n\ts_nop 1" :: "v"(p), "v"(v) : "memory");
}
__device__ __forceinline__ void st4_sys_p(const char* p, unsigned v) {
    asm volatile("global_store_dword %0, %1, off sc0 sc1" :: "v"(p), "v"(v) : "memory");
}

#define VMW(n) asm volatile("s_waitcnt vmcnt(" #n ")" ::: "memory")

__device__ __forceinline__ unsigned vm_outstanding() {
    unsigned v;
    asm volatile("s_getreg_b32 %0, hwreg(HW_REG_IB_STS)" : "=s"(v) :: "memory");
    return (v & 15u) | ((v >> 18) & 48u);
}

__device__ __forceinline__ void glds8(const char* sbase, unsigned voff, unsigned voff2, unsigned ldst) {
    unsigned keep;
    asm volatile(
        "s_mov_b32 %0, m0\n\t"
        "s_mov_b32 m0, %4\n\t"
        "s_nop 0\n\t"
        "global_load_lds_dwordx4 %1, %3 sc0 sc1\n\t"
        "global_load_lds_dwordx4 %1, %3 offset:1024 sc0 sc1\n\t"
        "global_load_lds_dwordx4 %1, %3 offset:2048 sc0 sc1\n\t"
        "global_load_lds_dwordx4 %1, %3 offset:3072 sc0 sc1\n\t"
        "s_add_u32 m0, %4, 4096\n\t"
        "s_nop 0\n\t"
        "global_load_lds_dwordx4 %2, %3 sc0 sc1\n\t"
        "global_load_lds_dwordx4 %2, %3 offset:1024 sc0 sc1\n\t"
        "global_load_lds_dwordx4 %2, %3 offset:2048 sc0 sc1\n\t"
        "global_load_lds_dwordx4 %2, %3 offset:3072 sc0 sc1\n\t"
        "s_mov_b32 m0, %0"
        : "=&s"(keep) : "v"(voff), "v"(voff2), "s"(sbase), "s"(ldst) : "memory", "scc");
}
__device__ __forceinline__ void glds1(const char* sbase, unsigned voff, unsigned ldst) {
    unsigned keep;
    asm volatile(
        "s_mov_b32 %0, m0\n\t"
        "s_mov_b32 m0, %3\n\t"
        "s_nop 0\n\t"
        "global_load_lds_dwordx4 %1, %2 sc0 sc1\n\t"
        "s_mov_b32 m0, %0"
        : "=&s"(keep) : "v"(voff), "s"(sbase), "s"(ldst) : "memory");
}

__device__ __forceinline__ void lds_wr32(unsigned addr, unsigned v) {
    asm volatile("s_mov_b64 exec, 1\n\tds_write_b32 %0, %1\n\ts_mov_b64 exec, -1" :: "v"(addr), "v"(v) : "memory");
}
__device__ __forceinline__ u32x4 lds_rd128_wait(unsigned addr) {
    u32x4 v;
    asm volatile("ds_read_b128 %0, %1\n\ts_waitcnt lgkmcnt(0)" : "=&v"(v) : "v"(addr) : "memory");
    return v;
}
__device__ __forceinline__ unsigned lds_rd32_wait(unsigned addr) {
    unsigned v;
    asm volatile("ds_read_b32 %0, %1\n\ts_waitcnt lgkmcnt(0)" : "=&v"(v) : "v"(addr) : "memory");
    return v;
}
__device__ __forceinline__ unsigned rfl(unsigned v) { return __builtin_amdgcn_readfirstlane(v); }
__device__ __forceinline__ unsigned umin(unsigned a, unsigned b) { return a < b ? a : b; }
template <int CTL> __device__ __forceinline__ unsigned dppmin(unsigned d) {
    return umin(d, (unsigned)__builtin_amdgcn_update_dpp((int)d, (int)d, CTL, 0xf, 0xf, false));
}

__device__ __forceinline__ float silu(float y) {
    float t = __builtin_amdgcn_fmed3f(y, -60.0f, 60.0f);
    float e = __builtin_amdgcn_exp2f(t * -1.44269504088896340736f);
    return y * __builtin_amdgcn_rcpf(1.0f + e);
}

#define STAMP(i) do { } while (0)

#define SB0() __builtin_amdgcn_sched_barrier(0)
#define DSR(dst, addr, off) asm volatile("ds_read_b128 %0, %1 offset:%2" : "=v"(dst) : "v"(addr), "i"(off) : "memory")

template <int MOFF>
__device__ __forceinline__ void fix_frag(f16x8& X, unsigned long long badmask, const char* sbase, unsigned hh,
                                         unsigned lane16, unsigned tag, unsigned badad) {
    unsigned t, vo;
    asm volatile(
        "s_cmp_eq_u64 %[bm], 0\n\t"
        "s_cbranch_scc1 3f\n\t"
        "s_lshl_b32 s92, %[hh], 14\n\t"
        "s_memrealtime s[90:91]\n\t"
        "v_add_u32 %[vo], s92, %[l16]\n\t"
        "s_waitcnt lgkmcnt(0)\n"
        "1:\n\t"
        "global_load_dword %[t], %[vo], %[sb] offset:%[mo] sc0 sc1\n\t"
        "s_waitcnt vmcnt(0)\n\t"
        "v_and_b32 %[t], 1, %[t]\n\t"
        "v_cmp_ne_u32 vcc, %[tag], %[t]\n\t"
        "s_nop 4\n\t"
        "s_cmp_eq_u64 vcc, 0\n\t"
        "s_cbranch_scc1 2f\n\t"
        "s_sleep 1\n\t"
        "s_memrealtime s[92:93]\n\t"
        "s_waitcnt lgkmcnt(0)\n\t"
        "s_sub_u32 s92, s92, s90\n\t"
        "s_cmp_lt_u32 s92, %[dl]\n\t"
        "s_cbranch_scc1 1b\n\t"
        "v_mov_b32 %[t], %[ba]\n\t"
        "ds_write_b32 %[t], %[t]\n\t"
        "s_waitcnt lgkmcnt(0)\n"
        "2:\n\t"
        "global_load_dwordx4 %[x], %[vo], %[sb] offset:%[mo] sc0 sc1\n\t"
        "s_waitcnt vmcnt(0)\n"
        "3:"
        : [x] "+v"(X), [t] "=&v"(t), [vo] "=&v"(vo)
        : [bm] "s"(badmask), [hh] "s"(hh), [l16] "v"(lane16), [sb] "s"(sbase), [tag] "s"(tag), [ba] "s"(badad),
          [dl] "i"((unsigned)DEADLINE), [mo] "i"(MOFF)
        : "memory", "vcc", "scc", "s90", "s91", "s92", "s93");
}

template <int NB>
__device__ __forceinline__ void consume(const Params& p, const unsigned ldsb, const int wave, const int lane,
                                        const int ti, const int tj, const int col0, const unsigned call) {
    const int wr = wave & 3, wc = wave >> 2;
    const int g = lane >> 4, q = lane & 15;
    f32x4 acc[4][NB];
#pragma unroll
    for (int m = 0; m < 4; ++m)
#pragma unroll
        for (int n = 0; n < NB; ++n) acc[m][n] = (f32x4){0.f, 0.f, 0.f, 0.f};
    f16x8 A0, A1, A2, A3, Bx[NB], By[NB];
    u32x4 fw;
    const unsigned va0 = ldsb + wr * 4096 + lane * 16, vb0 = ldsb + 16384 + wc * 4096 + lane * 16;
    const unsigned fullad = ldsb + CTRL + C_FULL, freead = ldsb + CTRL + C_FREE + wave * 4;
    const unsigned badad = ldsb + CTRL + C_BAD;
    const unsigned lane16 = lane * 16;
    const unsigned tag = ((call + 1u) >> 1) & 1u;
    const char* xsrc = p.myws + rfl((unsigned)OFF_X16 + (call & 1u) * (unsigned)X16_B + (unsigned)ti * (NH * 16384) + wr * 4096);
    unsigned sb = 0;
#define CNTFIX(m_) do { } while (0)

    auto spin_full = [&](unsigned need) {
        unsigned long long t0 = 0;
        unsigned sp = 0;
        for (;;) {
            u32x4 f = lds_rd128_wait(fullad);
            if (rfl(umin(umin(f.x, f.y), f.z)) >= need) break;
            if (sp == 0) t0 = rt();
            if ((++sp & 1023u) == 0u && rt() - t0 > 3 * DEADLINE) { lds_wr32(badad, 1u); break; }
            __builtin_amdgcn_s_sleep(1);
        }
    };
#define MM(m, A, B, n0, n1) do { _Pragma("unroll") for (int n = n0; n < n1; ++n) \
        acc[m][n] = __builtin_amdgcn_mfma_f32_16x16x32_f16(A, B[n], acc[m][n], 0, 0, 0); } while (0)
#define CHK(A) __ballot((__builtin_bit_cast(u32x4, A).x & 1u) != tag)
#define FIX(A, bm, hh, m) do { CNTFIX(bm); fix_frag<(m) * 1024>(A, bm, xsrc, (unsigned)(hh), lane16, tag, badad); } while (0)
#define LDB(B) do { DSR(B[0], vb, 0); DSR(B[1], vb, 1024); DSR(B[2], vb, 2048); DSR(B[3], vb, 3072); \
        if constexpr (NB == 5) DSR(B[NB - 1], vb, 8192); \
        asm volatile("ds_read_b128 %0, %1" : "=v"(fw) : "v"(fullad) : "memory"); } while (0)
#define WTOP(B) do { if constexpr (NB == 5) \
        asm volatile("s_waitcnt lgkmcnt(2)" : "+v"(B[0]), "+v"(B[1]), "+v"(B[2]), "+v"(B[3]), "+v"(B[NB - 1]), "+v"(fw), "+v"(A0) :: "memory"); \
        else asm volatile("s_waitcnt lgkmcnt(2)" : "+v"(B[0]), "+v"(B[1]), "+v"(B[2]), "+v"(B[3]), "+v"(fw), "+v"(A0) :: "memory"); } while (0)
#define WROW(A) asm volatile("s_waitcnt lgkmcnt(%1)" : "+v"(A) : "i"(NB + 4) : "memory")
#define WALL() asm volatile("s_waitcnt lgkmcnt(0)" : "+v"(A1), "+v"(A2), "+v"(A3) :: "memory")
#define PLAIN(C, hc) do { \
        WALL(); \
        const unsigned long long m1_ = CHK(A1), m2_ = CHK(A2), m3_ = CHK(A3); \
        MM(0, A0, C, 0, NB); SB0(); \
        FIX(A1, m1_, hc, 1); MM(1, A1, C, 0, NB); SB0(); \
        FIX(A2, m2_, hc, 2); MM(2, A2, C, 0, NB); SB0(); \
        FIX(A3, m3_, hc, 3); MM(3, A3, C, 0, NB); SB0(); } while (0)
#define FROW(m, A, AN, C, hc) do { \
        MM(m, A, C, 0, 2); SB0(); \
        WROW(AN); const unsigned long long mm_ = CHK(AN); SB0(); \
        MM(m, A, C, 2, NB); SB0(); \
        FIX(AN, mm_, hc, m + 1); } while (0)
#define STEP(C, N, hn) do { \
        const unsigned need_ = (unsigned)(hn) + 1u; \
        const bool fast_ = rfl(umin(umin(fw.x, fw.y), fw.z)) >= need_; \
        if (!fast_) { PLAIN(C, (hn) - 1); spin_full(need_); } \
        const unsigned va = va0 + sb, vb = vb0 + sb; \
        sb = (sb == (NSLOT - 1) * SLOT_B) ? 0u : sb + SLOT_B; \
        LDB(N); \
        SB0(); \
        if (fast_) FROW(0, A0, A1, C, (hn) - 1); \
        DSR(A0, va, 0); \
        if (fast_) FROW(1, A1, A2, C, (hn) - 1); \
        DSR(A1, va, 1024); \
        if (fast_) FROW(2, A2, A3, C, (hn) - 1); \
        DSR(A2, va, 2048); \
        if (fast_) { MM(3, A3, C, 0, 2); SB0(); } \
        WTOP(N); \
        const unsigned long long m0_ = CHK(A0); \
        SB0(); \
        if (fast_) { MM(3, A3, C, 2, NB); SB0(); } \
        DSR(A3, va, 3072); \
        lds_wr32(freead, need_); \
        FIX(A0, m0_, hn, 0); \
        } while (0)

    spin_full(1);
    STAMP(2);
    {
        const unsigned va = va0, vb = vb0;
        sb = SLOT_B;
        LDB(Bx);
        DSR(A0, va, 0); DSR(A1, va, 1024); DSR(A2, va, 2048);
        WTOP(Bx);
        const unsigned long long m0_ = CHK(A0);
        DSR(A3, va, 3072);
        lds_wr32(freead, 1u);
        FIX(A0, m0_, 0, 0);
    }
    __builtin_amdgcn_s_setprio(1);
    for (int h = 0; h < NH - 2; h += 2) {
        STEP(Bx, By, h + 1);
        STEP(By, Bx, h + 2);
    }
    STEP(Bx, By, NH - 1);
    PLAIN(By, NH - 1);
    __builtin_amdgcn_s_setprio(0);
    STAMP(3);
    const unsigned* tmo = (const unsigned*)(p.myws + OFF_TMO);
    const bool bad = rfl(lds_rd32_wait(badad)) != 0u ||
                     rfl(__hip_atomic_load(tmo, __ATOMIC_RELAXED, __HIP_MEMORY_SCOPE_AGENT)) == call;
    float* orow = p.out + (size_t)(ti * 256 + wr * 64 + 4 * g) * NL + col0;
    float* obase = orow + wc * 64 + 4 * q;
    const float qn = __builtin_nanf("");
#pragma unroll
    for (int m = 0; m < 4; ++m)
#pragma unroll
        for (int r = 0; r < 4; ++r) {
            f32x4 o;
            o.x = silu(acc[m][0][r]); o.y = silu(acc[m][1][r]);
            o.z = silu(acc[m][2][r]); o.w = silu(acc[m][3][r]);
            if (bad) o = (f32x4){qn, qn, qn, qn};
            st16_sys_p((const char*)(obase + (size_t)(m * 16 + r) * NL), __builtin_bit_cast(u32x4, o));
            if constexpr (NB == 5) {
                float e = silu(acc[m][NB - 1][r]);
                if (bad) e = qn;
                st4_sys_p((const char*)(orow + (size_t)(m * 16 + r) * NL + 128 + q), __builtin_bit_cast(unsigned, e));
            }
        }
    STAMP(4);
}

__global__ void __launch_bounds__(NWAVE * 64, 3) ag_gemm_silu(Params p) {
    extern __shared__ __attribute__((aligned(16))) char lds_raw[];
    LAS char* lds = (LAS char*)lds_raw;
    const unsigned ldsb = (unsigned)(size_t)lds;
    const int tid = threadIdx.x;
    const int wave = rfl(tid >> 6), lane = tid & 63;
    const int b = blockIdx.x;
    char* myws = p.myws;
    const int g = lane >> 4, q = lane & 15;
    unsigned* cntp = (unsigned*)(myws + OFF_CNT) + b * 16 + wave;
    STAMP(0);

    if (b < NCWG) {
        if (b >= COMM_WGS || wave >= CWPG) return;
        constexpr int P = COMM_P;
        constexpr int R = COMM_R;
        constexpr int ND = NR / COMM_SIDES;
        constexpr int NW = NCW / COMM_SIDES;
        constexpr int OPS = 1 + (2 + ND) * R;
        static_assert(ND * R + (P - 1) * OPS <= 63 && 2 * P * NW <= NITEM, "comm window");
        const int side = COMM_SIDES == 2 ? (b >> 2) & 1 : 0;
        const int cw = COMM_SIDES == 2 ? wave * (COMM_WGS / 2) + (b >> 3) * 4 + (b & 3) : wave * COMM_WGS + b;
        const float* xs = p.x + (size_t)q * KD + 4 * g;
        f32x4 xa[P][R][2];
        unsigned av[P];
        unsigned cur[P];
#define XLD(s_, it_) do { const unsigned i_ = (it_) < (unsigned)NITEM ? (it_) : (unsigned)NITEM - 1u; \
            const float* _s = xs + (size_t)((i_ % IPH) * (R * 16 * KD) + (i_ / IPH) * 32); \
            _Pragma("unroll") for (int r_ = 0; r_ < R; ++r_) \
            asm volatile("global_load_dwordx4 %0, %2, off nt\n\tglobal_load_dwordx4 %1, %2, off offset:64 nt" \
                         : "=&v"(xa[s_][r_][0]), "=&v"(xa[s_][r_][1]) : "v"(_s + r_ * 16 * KD) : "memory"); } while (0)
        unsigned cv;
        asm volatile("global_load_dword %0, %1, off sc1" : "=v"(cv) : "v"(cntp) : "memory");
#pragma unroll
        for (int s = 0; s < P; ++s) { cur[s] = s * NW + cw; XLD(s, cur[s]); }
        asm volatile("s_waitcnt vmcnt(%1)" : "+v"(cv) : "i"(2 * P * R) : "memory");
        const unsigned call = rfl(cv) + 1u;
        const unsigned tag = ((call + 1u) >> 1) & 1u;
        const size_t doff = OFF_X16 + (size_t)(call & 1u) * X16_B + (size_t)(p.rank * 2) * (NH * 16384);
        const char* dst[ND];
#pragma unroll
        for (int j = 0; j < ND; ++j) dst[j] = (side ? p.dst[NR - ND + j] : p.dst[j]) + doff;
        const char* ictr = myws + OFF_ICTR + (call & 1u) * 256u + side * 128u;
        const unsigned vzero = 0u, vone = 1u;
        auto body = [&](auto sc, auto first) -> bool {
            constexpr int s = decltype(sc)::value;
            constexpr bool F = decltype(first)::value;
            constexpr int W = F ? 2 * R * (P - 1 - s) + s * OPS : ND * R + (P - 1) * OPS;
            if constexpr (F) av[s] = 0u;
            if constexpr (R == 1)
                asm volatile("s_waitcnt vmcnt(%3)" : "+v"(xa[s][0][0]), "+v"(xa[s][0][1]), "+v"(av[s]) : "i"(W) : "memory");
            else
                asm volatile("s_waitcnt vmcnt(%5)" : "+v"(xa[s][0][0]), "+v"(xa[s][0][1]), "+v"(xa[s][R - 1][0]), "+v"(xa[s][R - 1][1]), "+v"(av[s])
                             : "i"(W) : "memory");
            const unsigned it = cur[s];
            if (it >= (unsigned)NITEM) return false;
            const unsigned nxt = F ? (unsigned)((P + s) * NW + cw) : (unsigned)(2 * P * NW) + rfl(av[s]);
            u32x4 o[R];
#pragma unroll
            for (int r = 0; r < R; ++r) {
                const unsigned rn = pk16(xa[s][r][0].x, xa[s][r][0].y);
                unsigned t = __builtin_bit_cast(unsigned, __builtin_amdgcn_cvt_pkrtz(xa[s][r][0].x, 0.0f));
                t += (t ^ tag) & 1u;
                o[r].x = (rn & 0xffff0000u) | (t & 0xffffu);
                o[r].y = pk16(xa[s][r][0].z, xa[s][r][0].w);
                o[r].z = pk16(xa[s][r][1].x, xa[s][r][1].y); o[r].w = pk16(xa[s][r][1].z, xa[s][r][1].w);
                asm volatile("" : "+v"(o[r]));
            }
            asm volatile("s_mov_b64 exec, 1\n\tglobal_atomic_add %0, %1, %2, %3 sc0 sc1\n\ts_mov_b64 exec, -1"
                         : "=&v"(av[s]) : "v"(vzero), "v"(vone), "s"(ictr) : "memory");
            XLD(s, nxt);
            cur[s] = nxt;
            const unsigned rb = (it % IPH) * R;
            const unsigned voff = (rb >> 4) * (NH * 16384u) + (it / IPH) * 16384u + (rb & 15u) * 1024u + lane * 16u;
#pragma unroll
            for (int j = 0; j < ND; ++j) st16_sys(dst[j], voff, o[0]);
            if constexpr (R == 2) {
#pragma unroll
                for (int j = 0; j < ND; ++j) st16_sys_o(dst[j], voff, o[R - 1]);
            }
            return true;
        };
        static_for<P>([&](auto sc) { body(sc, IC<1>{}); });
        for (bool go = true; go;) {
            static_for<P>([&](auto sc) { if (go) go = body(sc, IC<0>{}); });
        }
#pragma unroll
        for (int s = 0; s < P; ++s)
#pragma unroll
            for (int r = 0; r < R; ++r)
                asm volatile("s_waitcnt vmcnt(0)" : "+v"(xa[s][r][0]), "+v"(xa[s][r][1]), "+v"(av[s]) :: "memory");
        if (lane == 0) {
            __hip_atomic_store(cntp, call, __ATOMIC_RELAXED, __HIP_MEMORY_SCOPE_AGENT);
            if (cw == 0) st4_sys_p(myws + OFF_ICTR + ((call + 1u) & 1u) * 256u + side * 128u, 0u);
        }
        STAMP(1);
        return;
    }

    const int xcd = b & 7;
    const int t = (xcd & 1) * 30 + (b >> 3) - 2;
    const int tj = t >> 2;
    const int ti = (xcd >> 1) * 4 + (t & 3);
    const bool wide = (tj & 1) || tj == 14;
    const int col0 = ((tj * 17) >> 1) * 16;

    const unsigned call = rfl(__hip_atomic_load(cntp, __ATOMIC_RELAXED, __HIP_MEMORY_SCOPE_AGENT)) + 1u;
    if (lane == 0) __hip_atomic_store(cntp, call, __ATOMIC_RELAXED, __HIP_MEMORY_SCOPE_AGENT);
    unsigned* tmo = (unsigned*)(myws + OFF_TMO);

    if (tid < 64) *(LAS unsigned*)(lds + CTRL + tid * 4) = 0u;
    asm volatile("s_waitcnt lgkmcnt(0)" ::: "memory");
    __builtin_amdgcn_s_barrier();
    asm volatile("" ::: "memory");

    if (wave < 8) {
        {
            const int h = ti * 8 + wave;
            for (int i = 0; i < W_DELAY0 + ti * W_STAG + wave * W_WSTAG; ++i) __builtin_amdgcn_s_sleep(8);
            const float* wrow = p.w + (size_t)(h * 32 + 4 * g) * NL + col0;
            const float* wsrc = wrow + 4 * q;
            f32x4 v[2][8];
            float e[8];
#pragma unroll
            for (int cg = 0; cg < 2; ++cg)
#pragma unroll
                for (int i = 0; i < 8; ++i)
                    v[cg][i] = __builtin_nontemporal_load((const f32x4*)(wsrc + (size_t)((i & 3) + 16 * (i >> 2)) * NL + cg * 64));
            if (wide) {
#pragma unroll
                for (int i = 0; i < 8; ++i)
                    e[i] = __builtin_nontemporal_load(wrow + (size_t)((i & 3) + 16 * (i >> 2)) * NL + 128 + q);
            }
            char* dst = myws + OFF_W16 + (size_t)(tj * NH + h) * WB + lane * 16;
#pragma unroll
            for (int cg = 0; cg < 2; ++cg)
#pragma unroll
                for (int f = 0; f < 4; ++f) {
                    u32x4 o;
                    o.x = pk16(v[cg][0][f], v[cg][1][f]); o.y = pk16(v[cg][2][f], v[cg][3][f]);
                    o.z = pk16(v[cg][4][f], v[cg][5][f]); o.w = pk16(v[cg][6][f], v[cg][7][f]);
                    st16_sys_p(dst + (cg * 4 + f) * 1024, o);
                }
            if (wide) {
                u32x4 o;
                o.x = pk16(e[0], e[1]); o.y = pk16(e[2], e[3]); o.z = pk16(e[4], e[5]); o.w = pk16(e[6], e[7]);
                st16_sys_p(dst + 8192, o);
            }
            VMW(0);
            if (lane == 0) {
                unsigned old;
                asm volatile("ds_add_rtn_u32 %0, %1, %2\n\ts_waitcnt lgkmcnt(0)"
                             : "=&v"(old) : "v"(ldsb + CTRL + C_WCNT), "v"(1u) : "memory");
                if (old == 7u) st4_sys_p(myws + OFF_WFLAG + (size_t)(tj * 16 + ti) * 128, call);
            }
        }
        STAMP(1);
        DBGRET(1);
        if (wide && wave < 4) consume<5>(p, ldsb, wave, lane, ti, tj, col0, call);
        else consume<4>(p, ldsb, wave, lane, ti, tj, col0, call);
    } else if (wave < 11) {
        DBGRET(1);
        const int L = wave - 8;
        const bool nine = (L == 2) && wide;
        const unsigned soff = (L < 2) ? (unsigned)OFF_X16 + (call & 1u) * (unsigned)X16_B + (unsigned)ti * (NH * 16384) + L * 8192
                                      : (unsigned)OFF_W16 + (unsigned)tj * (NH * WB);
        const char* src = myws + rfl(soff);
        const unsigned step = (L < 2) ? 16384u : (unsigned)WB;
        const unsigned ldst0 = ldsb + L * 8192;
        const unsigned fullad = ldsb + CTRL + C_FULL + L * 4;
        const unsigned ctlad = ldsb + CTRL + ((lane < 8) ? C_FREE + lane * 4 : (L < 2 ? C_RDYA : C_RDYB));
        unsigned voff = lane * 16;
        unsigned h = 0, pub = 0, slot = 0;
        unsigned long long tw = 0; unsigned sp = 0;
        bool dead = false;
        VMW(0);
        while (pub < NH) {
            const unsigned cv = lds_rd32_wait(ctlad);
            const unsigned ready = __builtin_amdgcn_readlane(cv, 8);
            const unsigned freed = rfl(dppmin<0x141>(dppmin<0x4E>(dppmin<0xB1>(cv))));
            const unsigned lim = dead ? (unsigned)NH : umin(ready, freed + NSLOT);
            bool act = false;
            if (h < lim) {
                glds8(src, voff, voff + 4096, ldst0 + slot);
                if (nine) glds1(src, voff + 8192, ldst0 + slot + 8192);
                voff += step; ++h;
                slot = (slot == (NSLOT - 1) * SLOT_B) ? 0u : slot + SLOT_B;
                act = true;
                if (h < lim) {
                    glds8(src, voff, voff + 4096, ldst0 + slot);
                    if (nine) glds1(src, voff + 8192, ldst0 + slot + 8192);
                    voff += step; ++h;
                    slot = (slot == (NSLOT - 1) * SLOT_B) ? 0u : slot + SLOT_B;
                }
            }
            const unsigned vm = vm_outstanding();
            const unsigned landed = h - (nine ? ((vm + 8u) * 57u) >> 9 : (vm + 7u) >> 3);
            if (landed > pub) { pub = landed; lds_wr32(fullad, pub); act = true; }
            if (act) { sp = 0; continue; }
            if (sp == 0) tw = rt();
            if ((++sp & 1023u) == 0u && rt() - tw > 2 * DEADLINE) { lds_wr32(ldsb + CTRL + C_BAD, 1u); dead = true; }
            if (h == pub) __builtin_amdgcn_s_sleep(1);
        }
        STAMP(1);
    } else {
        const unsigned tag = ((call + 1u) >> 1) & 1u;
        const char* wflagp = myws + OFF_WFLAG + (unsigned)(tj * 16 + (lane & 15)) * 128u;
        const char* xsrc = myws + rfl((unsigned)OFF_X16 + (call & 1u) * (unsigned)X16_B + (unsigned)ti * (NH * 16384));
        unsigned ra = 0;
        unsigned pp = 0;
        unsigned rbw = 0, pw = POLL_W;
        bool wok = false;
        unsigned long long tw = 0; unsigned sp = 0;
        while (ra < NH || !wok) {
            unsigned fv = tag ^ 1u, wv = 0;
            const unsigned hx = ra + (lane >> 4) - 1u;
            const bool probe = lane < pw && pp + lane < (unsigned)NH;
            const bool piece = lane >= 16 && lane < 48 && hx >= (unsigned)(NH - POLL_NX) && hx < (unsigned)NH &&
                               (hx < pp || hx >= (unsigned)(NH - POLL_ND));
            const unsigned off = probe ? (pp + lane) * 16384u + 15u * 1024u + 1008u : hx * 16384u + (lane & 15) * 1024u + 1008u;
            if (!wok && lane < 16)
                asm volatile("global_load_dword %0, %1, off sc0 sc1" : "=v"(wv) : "v"(wflagp) : "memory");
            if (probe || piece)
                asm volatile("global_load_dword %0, %1, %2 sc0 sc1" : "=v"(fv) : "v"(off), "s"(xsrc) : "memory");
            asm volatile("s_waitcnt vmcnt(0)" : "+v"(fv), "+v"(wv) :: "memory");
            bool act = false;
            if (!wok) {
                const unsigned long long m = ~__ballot(wv == call);
                const unsigned nb = (unsigned)__builtin_ctzll(m | 0x10000ull) * 8u;
                if (nb > rbw) { rbw = nb; lds_wr32(ldsb + CTRL + C_RDYB, nb); act = true; }
                wok = nb == (unsigned)NH;
            }
            if (ra < NH) {
                const unsigned long long m = ~__ballot((fv & 1u) == tag);
                const unsigned np = (unsigned)__builtin_ctzll(m | 0x10000ull);
                pw = (np == pw) ? umin(2u * pw, 16u) : (unsigned)POLL_W;
                const unsigned nx = (unsigned)__builtin_ctzll((m >> 16) | (1ull << 32)) >> 4;
                pp += np;
                unsigned rn = ra + nx;
                const unsigned rb = umin(pp > (unsigned)POLL_LAG ? pp - POLL_LAG : 0u, (unsigned)(NH - POLL_NX));
                rn = rn > rb ? rn : rb;
                if (rn > ra) { ra = rn; lds_wr32(ldsb + CTRL + C_RDYA, ra); act = true; }
                if (np) act = true;
            }
            if (act) { sp = 0; continue; }
            if (sp == 0) tw = rt();
            if ((++sp & 63u) == 0u) {
                const unsigned tv = rfl(__hip_atomic_load(tmo, __ATOMIC_RELAXED, __HIP_MEMORY_SCOPE_AGENT));
                if (tv == call || rt() - tw > DEADLINE) {
                    if (lane == 0) __hip_atomic_store(tmo, call, __ATOMIC_RELAXED, __HIP_MEMORY_SCOPE_AGENT);
                    lds_wr32(ldsb + CTRL + C_BAD, 1u);
                    lds_wr32(ldsb + CTRL + C_RDYA, NH);
                    lds_wr32(ldsb + CTRL + C_RDYB, NH);
                    break;
                }
            }
            __builtin_amdgcn_s_sleep(POLL_SLEEP);
        }
        STAMP(1);
    }
}

}

extern "C" void kernel_launch(int rank, int nranks, ncclComm_t comm,
                              hipStream_t stream,
                              void* const* d_in, const int* in_sizes, int n_in,
                              void* d_out, int out_size,
                              const int* in_dtypes, int out_dtype,
                              void* const* d_out_all,
                              void* const* d_ws_all, size_t ws_bytes) {
    using namespace agk;
    if (nranks != NR || (unsigned)rank >= (unsigned)NR || ws_bytes < WS_NEED || n_in < 2) {
        fprintf(stderr, "ag_gemm_silu: unexpected configuration\n");
        return;
    }
    static bool attr_done = false;
    if (!attr_done) {
        HIP_CHECK(hipFuncSetAttribute((const void*)ag_gemm_silu, hipFuncAttributeMaxDynamicSharedMemorySize, LDS_B));
        attr_done = true;
    }
    Params p;
    p.x = (const float*)d_in[0];
    p.w = (const float*)d_in[1];
    p.out = (float*)d_out;
    p.myws = (char*)d_ws_all[rank];
    if (COMM_SIDES == 2) {
        static const int grp[2][4] = {{0, 2, 3, 6}, {1, 4, 5, 7}};
        const int gi = (rank == 1 || rank == 4 || rank == 5 || rank == 7) ? 1 : 0;
        int pos = 0;
        for (int i = 0; i < 4; ++i) if (grp[gi][i] == rank) pos = i;
        for (int i = 0; i < 4; ++i) {
            p.dst[(COMM_SWAP ? 4 : 0) + i] = (char*)d_ws_all[grp[gi][(pos + 1 + i) & 3]];
            p.dst[(COMM_SWAP ? 0 : 4) + i] = (char*)d_ws_all[grp[gi ^ 1][(pos + i) & 3]];
        }
    } else {
        for (int i = 0; i < NR; ++i) p.dst[i] = (char*)d_ws_all[(rank + 1 + i) & 7];
    }
    p.rank = rank;
    ag_gemm_silu<<<dim3(256), dim3(NWAVE * 64), LDS_B, stream>>>(p);
    HIP_CHECK(hipGetLastError());
}
```

```cpp
#include <hip/hip_runtime.h>
#include <rccl/rccl.h>
#include <cstdio>

#define HIP_CHECK(expr) do { hipError_t _e = (expr); if (_e != hipSuccess) { \
    fprintf(stderr, "HIP error %s at %s:%d\n", hipGetErrorString(_e), __FILE__, __LINE__); \
    return; } } while (0)

#define LAS __attribute__((address_space(3)))
typedef _Float16 f16x8 __attribute__((ext_vector_type(8)));
typedef _Float16 f16x2 __attribute__((ext_vector_type(2)));
typedef float f32x4 __attribute__((ext_vector_type(4)));
typedef float f32x2 __attribute__((ext_vector_type(2)));
typedef unsigned u32x4 __attribute__((ext_vector_type(4)));
typedef unsigned u32x2 __attribute__((ext_vector_type(2)));

namespace agk {

constexpr int NR = 8;
constexpr int KD = 4096;
constexpr int NL = 2048;
constexpr int NH = 128;
constexpr int NSLOT = 6;
constexpr int WB = 9216;
constexpr int SLOT_B = 16384 + WB;
constexpr int CTRL = NSLOT * SLOT_B;
constexpr int LDS_B = CTRL + 256;
constexpr int NWAVE = 12;
constexpr int NCWG = 16;
#define COMM_P 3
#define COMM_R 1
#define COMM_WGS 16
#define COMM_WPG 2
#define COMM_SIDES 1
#define COMM_SWAP 0
#define POLL_SLEEP 2
#define POLL_LAG 2
#define POLL_NX 8
#define W_JC 1
#define W_LEAD 20
#define W_TICK 55
#define POLL_W 2
#define POLL_ND 2
constexpr int CWPG = COMM_WPG;
constexpr int NCW = COMM_WGS * COMM_WPG;
constexpr int IPH = 32 / COMM_R;
constexpr int NITEM = NH * IPH;

constexpr int C_FULL = 0;
constexpr int C_RDYA = 12;
constexpr int C_RDYB = 16;
constexpr int C_BAD = 20;
constexpr int C_WCNT = 24;
constexpr int C_XEXP = 28;
constexpr int C_FREE = 64;

constexpr size_t OFF_CNT = 0;
constexpr size_t OFF_TMO = 32768;
constexpr size_t OFF_ICTR = 65536;
constexpr size_t OFF_XEXP = 98304;
constexpr size_t OFF_WFLAG = 131072;
constexpr size_t OFF_DBG = 262144;
constexpr size_t OFF_X16 = 1ull << 20;
constexpr size_t X16_B = 32ull << 20;
constexpr size_t OFF_W16 = OFF_X16 + 2 * X16_B;
constexpr size_t WS_NEED = OFF_W16 + (size_t)15 * NH * WB;
constexpr unsigned long long DEADLINE = 50000000ull;

struct Params {
    const float* x;
    const float* w;
    float* out;
    char* myws;
    char* dst[NR];
    int rank;
};
#define DBGRET(bit) do { } while (0)

template <int V> struct IC { static constexpr int value = V; };
template <int N, int I = 0, class F> __device__ __forceinline__ void static_for(F&& f) {
    if constexpr (I < N) { f(IC<I>{}); static_for<N, I + 1>(f); }
}

__device__ __forceinline__ unsigned long long rt() {
    unsigned long long t;
    asm volatile("s_memrealtime %0\n\ts_waitcnt lgkmcnt(0)" : "=s"(t) :: "memory");
    return t;
}

__device__ __forceinline__ unsigned pk16(float a, float b) {
    f32x2 v = {a, b};
    f16x2 h = __builtin_convertvector(v, f16x2);
    return __builtin_bit_cast(unsigned, h);
}

#define POL_COMM "sc0 sc1"
#define POL_OUT "sc0 sc1"
#define POL_DMA "sc0 sc1"
__device__ __forceinline__ void st16_sys(const char* sbase, unsigned voff, u32x4 v) {
    asm volatile("global_store_dwordx4 %0, %1, %2 " POL_COMM "\n\ts_nop 1" :: "v"(voff), "v"(v), "s"(sbase) : "memory");
}
__device__ __forceinline__ void st16_out(const char* p, u32x4 v) {
    asm volatile("global_store_dwordx4 %0, %1, off " POL_OUT "\n\ts_nop 1" :: "v"(p), "v"(v) : "memory");
}
__device__ __forceinline__ void st4_out(const char* p, unsigned v) {
    asm volatile("global_store_dword %0, %1, off " POL_OUT :: "v"(p), "v"(v) : "memory");
}
__device__ __forceinline__ void st16_sys_o(const char* sbase, unsigned voff, u32x4 v) {
    asm volatile("global_store_dwordx4 %0, %1, %2 offset:1024 sc0 sc1\n\ts_nop 1" :: "v"(voff), "v"(v), "s"(sbase) : "memory");
}
__device__ __forceinline__ void st16_sys_p(const char* p, u32x4 v) {
    asm volatile("global_store_dwordx4 %0, %1, off sc0 sc1\n\ts_nop 1" :: "v"(p), "v"(v) : "memory");
}
__device__ __forceinline__ void st4_sys_p(const char* p, unsigned v) {
    asm volatile("global_store_dword %0, %1, off sc0 sc1" :: "v"(p), "v"(v) : "memory");
}

#define VMW(n) asm volatile("s_waitcnt vmcnt(" #n ")" ::: "memory")

__device__ __forceinline__ unsigned vm_outstanding() {
    unsigned v;
    asm volatile("s_getreg_b32 %0, hwreg(HW_REG_IB_STS)" : "=s"(v) :: "memory");
    return (v & 15u) | ((v >> 18) & 48u);
}

__device__ __forceinline__ void glds8(const char* sbase, unsigned voff, unsigned voff2, unsigned ldst) {
    unsigned keep;
    asm volatile(
        "s_mov_b32 %0, m0\n\t"
        "s_mov_b32 m0, %4\n\t"
        "s_nop 0\n\t"
        "global_load_lds_dwordx4 %1, %3 " POL_DMA "\n\t"
        "global_load_lds_dwordx4 %1, %3 offset:1024 " POL_DMA "\n\t"
        "global_load_lds_dwordx4 %1, %3 offset:2048 " POL_DMA "\n\t"
        "global_load_lds_dwordx4 %1, %3 offset:3072 " POL_DMA "\n\t"
        "s_add_u32 m0, %4, 4096\n\t"
        "s_nop 0\n\t"
        "global_load_lds_dwordx4 %2, %3 " POL_DMA "\n\t"
        "global_load_lds_dwordx4 %2, %3 offset:1024 " POL_DMA "\n\t"
        "global_load_lds_dwordx4 %2, %3 offset:2048 " POL_DMA "\n\t"
        "global_load_lds_dwordx4 %2, %3 offset:3072 " POL_DMA "\n\t"
        "s_mov_b32 m0, %0"
        : "=&s"(keep) : "v"(voff), "v"(voff2), "s"(sbase), "s"(ldst) : "memory", "scc");
}
__device__ __forceinline__ void glds1(const char* sbase, unsigned voff, unsigned ldst) {
    unsigned keep;
    asm volatile(
        "s_mov_b32 %0, m0\n\t"
        "s_mov_b32 m0, %3\n\t"
        "s_nop 0\n\t"
        "global_load_lds_dwordx4 %1, %2 " POL_DMA "\n\t"
        "s_mov_b32 m0, %0"
        : "=&s"(keep) : "v"(voff), "s"(sbase), "s"(ldst) : "memory");
}

__device__ __forceinline__ void lds_wr32(unsigned addr, unsigned v) {
    asm volatile("s_mov_b64 exec, 1\n\tds_write_b32 %0, %1\n\ts_mov_b64 exec, -1" :: "v"(addr), "v"(v) : "memory");
}
__device__ __forceinline__ u32x4 lds_rd128_wait(unsigned addr) {
    u32x4 v;
    asm volatile("ds_read_b128 %0, %1\n\ts_waitcnt lgkmcnt(0)" : "=&v"(v) : "v"(addr) : "memory");
    return v;
}
__device__ __forceinline__ unsigned lds_rd32_wait(unsigned addr) {
    unsigned v;
    asm volatile("ds_read_b32 %0, %1\n\ts_waitcnt lgkmcnt(0)" : "=&v"(v) : "v"(addr) : "memory");
    return v;
}
__device__ __forceinline__ unsigned rfl(unsigned v) { return __builtin_amdgcn_readfirstlane(v); }
__device__ __forceinline__ unsigned umin(unsigned a, unsigned b) { return a < b ? a : b; }
template <int CTL> __device__ __forceinline__ unsigned dppmin(unsigned d) {
    return umin(d, (unsigned)__builtin_amdgcn_update_dpp((int)d, (int)d, CTL, 0xf, 0xf, false));
}

template <int CTL> __device__ __forceinline__ unsigned dppmax(unsigned d) {
    const unsigned o = (unsigned)__builtin_amdgcn_update_dpp((int)d, (int)d, CTL, 0xf, 0xf, false);
    return d > o ? d : o;
}
__device__ __forceinline__ int scale_exp(f32x4 v) {
    const u32x4 u = __builtin_bit_cast(u32x4, v) & 0x7fffffffu;
    unsigned m = u.x > u.y ? u.x : u.y, n = u.z > u.w ? u.z : u.w;
    m = m > n ? m : n;
    m = dppmax<0x140>(dppmax<0x141>(dppmax<0x4E>(dppmax<0xB1>(m))));
    const unsigned a = __builtin_amdgcn_readlane(m, 0), b = __builtin_amdgcn_readlane(m, 16);
    const unsigned c = __builtin_amdgcn_readlane(m, 32), d = __builtin_amdgcn_readlane(m, 48);
    const unsigned ab = a > b ? a : b, cd = c > d ? c : d;
    const int e = (int)((ab > cd ? ab : cd) >> 23);
    if (e == 0 || e == 255) return 0;
    const int sx = 126 - e;
    return sx < -60 ? -60 : (sx > 60 ? 60 : sx);
}
__device__ __forceinline__ float exp2i(int e) { return __builtin_bit_cast(float, (unsigned)(127 + e) << 23); }
__device__ __forceinline__ const f32x4* x_sample(const float* x, int lane) {
    return (const f32x4*)(x + (size_t)(lane * 8 + 3) * KD + ((lane * 260) & (KD - 1)));
}
__device__ __forceinline__ const f32x4* w_sample(const float* w, int lane) {
    return (const f32x4*)(w + (size_t)(lane * 64 + 17) * NL + ((lane * 132) & (NL - 1)));
}
__device__ __forceinline__ void fp16_saturate() {
    asm volatile("s_setreg_imm32_b32 hwreg(HW_REG_MODE, 23, 1), 1" ::: "memory");
}

__device__ __forceinline__ float silu(float y) {
    float t = __builtin_amdgcn_fmed3f(y, -60.0f, 60.0f);
    float e = __builtin_amdgcn_exp2f(t * -1.44269504088896340736f);
    return y * __builtin_amdgcn_rcpf(1.0f + e);
}

#define STAMP(i) do { } while (0)

#define SB0() __builtin_amdgcn_sched_barrier(0)
#define DSR(dst, addr, off) asm volatile("ds_read_b128 %0, %1 offset:%2" : "=v"(dst) : "v"(addr), "i"(off) : "memory")

template <int MOFF>
__device__ __forceinline__ void fix_frag(f16x8& X, unsigned long long badmask, const char* sbase, unsigned hh,
                                         unsigned lane16, unsigned tag, unsigned badad) {
    unsigned t, vo;
    asm volatile(
        "s_cmp_eq_u64 %[bm], 0\n\t"
        "s_cbranch_scc1 3f\n\t"
        "s_lshl_b32 s92, %[hh], 14\n\t"
        "s_memrealtime s[90:91]\n\t"
        "v_add_u32 %[vo], s92, %[l16]\n\t"
        "s_waitcnt lgkmcnt(0)\n"
        "1:\n\t"
        "global_load_dword %[t], %[vo], %[sb] offset:%[mo] sc0 sc1\n\t"
        "s_waitcnt vmcnt(0)\n\t"
        "v_and_b32 %[t], 1, %[t]\n\t"
        "v_cmp_ne_u32 vcc, %[tag], %[t]\n\t"
        "s_nop 4\n\t"
        "s_cmp_eq_u64 vcc, 0\n\t"
        "s_cbranch_scc1 2f\n\t"
        "s_sleep 1\n\t"
        "s_memrealtime s[92:93]\n\t"
        "s_waitcnt lgkmcnt(0)\n\t"
        "s_sub_u32 s92, s92, s90\n\t"
        "s_cmp_lt_u32 s92, %[dl]\n\t"
        "s_cbranch_scc1 1b\n\t"
        "v_mov_b32 %[t], %[ba]\n\t"
        "ds_write_b32 %[t], %[t]\n\t"
        "s_waitcnt lgkmcnt(0)\n"
        "2:\n\t"
        "global_load_dwordx4 %[x], %[vo], %[sb] offset:%[mo] sc0 sc1\n\t"
        "s_waitcnt vmcnt(0)\n"
        "3:"
        : [x] "+v"(X), [t] "=&v"(t), [vo] "=&v"(vo)
        : [bm] "s"(badmask), [hh] "s"(hh), [l16] "v"(lane16), [sb] "s"(sbase), [tag] "s"(tag), [ba] "s"(badad),
          [dl] "i"((unsigned)DEADLINE), [mo] "i"(MOFF)
        : "memory", "vcc", "scc", "s90", "s91", "s92", "s93");
}

__device__ __forceinline__ void conv_w64(const Params& p, int tj, int col0, int h, int cg, int lane, const f32x4* smp, int& se) {
    const int g = lane >> 4, q = lane & 15;
    const float* wsrc = p.w + (size_t)(h * 32 + 4 * g) * NL + col0 + cg * 64 + 4 * q;
    f32x4 v[8];
#pragma unroll
    for (int i = 0; i < 8; ++i)
        v[i] = __builtin_nontemporal_load((const f32x4*)(wsrc + (size_t)((i & 3) + 16 * (i >> 2)) * NL));
    se = scale_exp(*smp);
    const float sc = exp2i(se);
#pragma unroll
    for (int i = 0; i < 8; ++i) v[i] *= sc;
    char* dst = p.myws + OFF_W16 + (size_t)(tj * NH + h) * WB + cg * 4096 + lane * 16;
#pragma unroll
    for (int f = 0; f < 4; ++f) {
        u32x4 o;
        o.x = pk16(v[0][f], v[1][f]); o.y = pk16(v[2][f], v[3][f]);
        o.z = pk16(v[4][f], v[5][f]); o.w = pk16(v[6][f], v[7][f]);
        st16_sys_p(dst + f * 1024, o);
    }
}
__device__ __forceinline__ void conv_w16(const Params& p, int tj, int col0, int h, int lane, const f32x4* smp, int& se) {
    const int g = lane >> 4, q = lane & 15;
    const float* wsrc = p.w + (size_t)(h * 32 + 4 * g) * NL + col0 + 128 + q;
    float e[8];
#pragma unroll
    for (int i = 0; i < 8; ++i) e[i] = __builtin_nontemporal_load(wsrc + (size_t)((i & 3) + 16 * (i >> 2)) * NL);
    se = scale_exp(*smp);
    const float sc = exp2i(se);
#pragma unroll
    for (int i = 0; i < 8; ++i) e[i] *= sc;
    u32x4 o;
    o.x = pk16(e[0], e[1]); o.y = pk16(e[2], e[3]); o.z = pk16(e[4], e[5]); o.w = pk16(e[6], e[7]);
    st16_sys_p(p.myws + OFF_W16 + (size_t)(tj * NH + h) * WB + 8192 + lane * 16, o);
}
__device__ __forceinline__ void conv_wblock(const Params& p, int tj, int col0, bool wide, int h, int lane, float sc) {
    const int g = lane >> 4, q = lane & 15;
    const float* wrow = p.w + (size_t)(h * 32 + 4 * g) * NL + col0;
    f32x4 v[2][8];
    float e[8];
#pragma unroll
    for (int cg = 0; cg < 2; ++cg)
#pragma unroll
        for (int i = 0; i < 8; ++i)
            v[cg][i] = __builtin_nontemporal_load((const f32x4*)(wrow + 4 * q + (size_t)((i & 3) + 16 * (i >> 2)) * NL + cg * 64));
    if (wide) {
#pragma unroll
        for (int i = 0; i < 8; ++i)
            e[i] = __builtin_nontemporal_load(wrow + (size_t)((i & 3) + 16 * (i >> 2)) * NL + 128 + q);
    }
    char* dst = p.myws + OFF_W16 + (size_t)(tj * NH + h) * WB + lane * 16;
#pragma unroll
    for (int cg = 0; cg < 2; ++cg)
#pragma unroll
        for (int f = 0; f < 4; ++f) {
            u32x4 o;
            o.x = pk16(v[cg][0][f] * sc, v[cg][1][f] * sc); o.y = pk16(v[cg][2][f] * sc, v[cg][3][f] * sc);
            o.z = pk16(v[cg][4][f] * sc, v[cg][5][f] * sc); o.w = pk16(v[cg][6][f] * sc, v[cg][7][f] * sc);
            st16_sys_p(dst + (cg * 4 + f) * 1024, o);
        }
    if (wide) {
        u32x4 o;
        o.x = pk16(e[0] * sc, e[1] * sc); o.y = pk16(e[2] * sc, e[3] * sc);
        o.z = pk16(e[4] * sc, e[5] * sc); o.w = pk16(e[6] * sc, e[7] * sc);
        st16_sys_p(dst + 8192, o);
    }
}

template <int NB>
__device__ __forceinline__ void consume(const Params& p, const unsigned ldsb, const int wave, const int lane,
                                        const int ti, const int col0, const unsigned call, const int wexp) {
    const int wr = wave & 3, wc = wave >> 2;
    const int g = lane >> 4, q = lane & 15;
    f32x4 acc[4][NB];
#pragma unroll
    for (int m = 0; m < 4; ++m)
#pragma unroll
        for (int n = 0; n < NB; ++n) acc[m][n] = (f32x4){0.f, 0.f, 0.f, 0.f};
    f16x8 A0, A1, A2, A3, Bx[NB], By[NB];
    u32x4 fw;
    const unsigned va0 = ldsb + wr * 4096 + lane * 16, vb0 = ldsb + 16384 + wc * 4096 + lane * 16;
    const unsigned fullad = ldsb + CTRL + C_FULL, freead = ldsb + CTRL + C_FREE + wave * 4;
    const unsigned badad = ldsb + CTRL + C_BAD;
    const unsigned lane16 = lane * 16;
    const unsigned tag = ((call + 1u) >> 1) & 1u;
    const char* xsrc = p.myws + rfl((unsigned)OFF_X16 + (call & 1u) * (unsigned)X16_B + (unsigned)ti * (NH * 16384) + wr * 4096);
    unsigned sb = 0;
#define CNTFIX(m_) do { } while (0)

    auto spin_full = [&](unsigned need) {
        unsigned long long t0 = 0;
        unsigned sp = 0;
        for (;;) {
            u32x4 f = lds_rd128_wait(fullad);
            if (rfl(umin(umin(f.x, f.y), f.z)) >= need) break;
            if (sp == 0) t0 = rt();
            if ((++sp & 1023u) == 0u && rt() - t0 > 3 * DEADLINE) { lds_wr32(badad, 1u); break; }
            __builtin_amdgcn_s_sleep(1);
        }
    };
#define MM(m, A, B, n0, n1) do { _Pragma("unroll") for (int n = n0; n < n1; ++n) \
        acc[m][n] = __builtin_amdgcn_mfma_f32_16x16x32_f16(A, B[n], acc[m][n], 0, 0, 0); } while (0)
#define CHK(A) __ballot((__builtin_bit_cast(u32x4, A).x & 1u) != tag)
#define FIX(A, bm, hh, m) do { CNTFIX(bm); fix_frag<(m) * 1024>(A, bm, xsrc, (unsigned)(hh), lane16, tag, badad); } while (0)
#define LDB(B) do { DSR(B[0], vb, 0); DSR(B[1], vb, 1024); DSR(B[2], vb, 2048); DSR(B[3], vb, 3072); \
        if constexpr (NB == 5) DSR(B[NB - 1], vb, 8192); \
        asm volatile("ds_read_b128 %0, %1" : "=v"(fw) : "v"(fullad) : "memory"); } while (0)
#define WTOP(B) do { if constexpr (NB == 5) \
        asm volatile("s_waitcnt lgkmcnt(2)" : "+v"(B[0]), "+v"(B[1]), "+v"(B[2]), "+v"(B[3]), "+v"(B[NB - 1]), "+v"(fw), "+v"(A0) :: "memory"); \
        else asm volatile("s_waitcnt lgkmcnt(2)" : "+v"(B[0]), "+v"(B[1]), "+v"(B[2]), "+v"(B[3]), "+v"(fw), "+v"(A0) :: "memory"); } while (0)
#define WROW(A) asm volatile("s_waitcnt lgkmcnt(%1)" : "+v"(A) : "i"(NB + 4) : "memory")
#define WALL() asm volatile("s_waitcnt lgkmcnt(0)" : "+v"(A1), "+v"(A2), "+v"(A3) :: "memory")
#define PLAIN(C, hc) do { \
        WALL(); \
        const unsigned long long m1_ = CHK(A1), m2_ = CHK(A2), m3_ = CHK(A3); \
        MM(0, A0, C, 0, NB); SB0(); \
        FIX(A1, m1_, hc, 1); MM(1, A1, C, 0, NB); SB0(); \
        FIX(A2, m2_, hc, 2); MM(2, A2, C, 0, NB); SB0(); \
        FIX(A3, m3_, hc, 3); MM(3, A3, C, 0, NB); SB0(); } while (0)
#define FROW(m, A, AN, C, hc) do { \
        MM(m, A, C, 0, 2); SB0(); \
        WROW(AN); const unsigned long long mm_ = CHK(AN); SB0(); \
        MM(m, A, C, 2, NB); SB0(); \
        FIX(AN, mm_, hc, m + 1); } while (0)
#define STEP(C, N, hn) do { \
        const unsigned need_ = (unsigned)(hn) + 1u; \
        const bool fast_ = rfl(umin(umin(fw.x, fw.y), fw.z)) >= need_; \
        if (!fast_) { PLAIN(C, (hn) - 1); spin_full(need_); } \
        const unsigned va = va0 + sb, vb = vb0 + sb; \
        sb = (sb == (NSLOT - 1) * SLOT_B) ? 0u : sb + SLOT_B; \
        LDB(N); \
        SB0(); \
        if (fast_) FROW(0, A0, A1, C, (hn) - 1); \
        DSR(A0, va, 0); \
        if (fast_) FROW(1, A1, A2, C, (hn) - 1); \
        DSR(A1, va, 1024); \
        if (fast_) FROW(2, A2, A3, C, (hn) - 1); \
        DSR(A2, va, 2048); \
        if (fast_) { MM(3, A3, C, 0, 2); SB0(); } \
        WTOP(N); \
        const unsigned long long m0_ = CHK(A0); \
        SB0(); \
        if (fast_) { MM(3, A3, C, 2, NB); SB0(); } \
        DSR(A3, va, 3072); \
        lds_wr32(freead, need_); \
        FIX(A0, m0_, hn, 0); \
        } while (0)

    spin_full(1);
    STAMP(2);
    {
        const unsigned va = va0, vb = vb0;
        sb = SLOT_B;
        LDB(Bx);
        DSR(A0, va, 0); DSR(A1, va, 1024); DSR(A2, va, 2048);
        WTOP(Bx);
        const unsigned long long m0_ = CHK(A0);
        DSR(A3, va, 3072);
        lds_wr32(freead, 1u);
        FIX(A0, m0_, 0, 0);
    }
    __builtin_amdgcn_s_setprio(1);
    for (int h = 0; h < NH - 2; h += 2) {
        STEP(Bx, By, h + 1);
        STEP(By, Bx, h + 2);
    }
    STEP(Bx, By, NH - 1);
    PLAIN(By, NH - 1);
    __builtin_amdgcn_s_setprio(0);
    STAMP(3);
    const unsigned* tmo = (const unsigned*)(p.myws + OFF_TMO);
    const bool bad = rfl(lds_rd32_wait(badad)) != 0u ||
                     rfl(__hip_atomic_load(tmo, __ATOMIC_RELAXED, __HIP_MEMORY_SCOPE_AGENT)) == call;
    const float osc = exp2i(-wexp - ((int)rfl(lds_rd32_wait(ldsb + CTRL + C_XEXP)) - 64));
    float* orow = p.out + (size_t)(ti * 256 + wr * 64 + 4 * g) * NL + col0;
    float* obase = orow + wc * 64 + 4 * q;
    const float qn = __builtin_nanf("");
#pragma unroll
    for (int m = 0; m < 4; ++m)
#pragma unroll
        for (int r = 0; r < 4; ++r) {
            f32x4 o;
            o.x = silu(acc[m][0][r] * osc); o.y = silu(acc[m][1][r] * osc);
            o.z = silu(acc[m][2][r] * osc); o.w = silu(acc[m][3][r] * osc);
            if (bad) o = (f32x4){qn, qn, qn, qn};
            st16_out((const char*)(obase + (size_t)(m * 16 + r) * NL), __builtin_bit_cast(u32x4, o));
            if constexpr (NB == 5) {
                float e = silu(acc[m][NB - 1][r] * osc);
                if (bad) e = qn;
                st4_out((const char*)(orow + (size_t)(m * 16 + r) * NL + 128 + q), __builtin_bit_cast(unsigned, e));
            }
        }
    STAMP(4);
}

__global__ void __launch_bounds__(NWAVE * 64, 3) ag_gemm_silu(Params p) {
    extern __shared__ __attribute__((aligned(16))) char lds_raw[];
    LAS char* lds = (LAS char*)lds_raw;
    const unsigned ldsb = (unsigned)(size_t)lds;
    const int tid = threadIdx.x;
    const int wave = rfl(tid >> 6), lane = tid & 63;
    const int b = blockIdx.x;
    char* myws = p.myws;
    const int g = lane >> 4, q = lane & 15;
    unsigned* cntp = (unsigned*)(myws + OFF_CNT) + b * 16 + wave;
    STAMP(0);

    if (b < NCWG) {
        if (b == 0 && wave == CWPG) {
            const f32x4 sv = *x_sample(p.x, lane);
            const unsigned call = rfl(__hip_atomic_load(cntp, __ATOMIC_RELAXED, __HIP_MEMORY_SCOPE_AGENT)) + 1u;
            const int xe = scale_exp(sv);
            const char* d = p.dst[0];
#pragma unroll
            for (int j = 1; j < NR; ++j) d = (lane == j) ? p.dst[j] : d;
            if (lane < NR) st4_sys_p(d + OFF_XEXP + (call & 1u) * 1024u + p.rank * 128, call * 256u + (unsigned)(xe + 64));
            if (lane == 0) __hip_atomic_store(cntp, call, __ATOMIC_RELAXED, __HIP_MEMORY_SCOPE_AGENT);
            return;
        }
        if (b >= COMM_WGS || wave >= CWPG) return;
        fp16_saturate();
        constexpr int P = COMM_P;
        constexpr int R = COMM_R;
        constexpr int ND = NR / COMM_SIDES;
        constexpr int NW = NCW / COMM_SIDES;
        constexpr int OPS = 1 + (2 + ND) * R;
        static_assert(ND * R + (P - 1) * OPS <= 63 && 2 * P * NW <= NITEM, "comm window");
        const int side = COMM_SIDES == 2 ? (b >> 2) & 1 : 0;
        const int cw = COMM_SIDES == 2 ? wave * (COMM_WGS / 2) + (b >> 3) * 4 + (b & 3) : wave * COMM_WGS + b;
        const float* xs = p.x + (size_t)q * KD + 4 * g;
        f32x4 xa[P][R][2];
        unsigned av[P];
        unsigned cur[P];
#define XLD(s_, it_) do { const unsigned i_ = (it_) < (unsigned)NITEM ? (it_) : (unsigned)NITEM - 1u; \
            const float* _s = xs + (size_t)((i_ % IPH) * (R * 16 * KD) + (i_ / IPH) * 32); \
            _Pragma("unroll") for (int r_ = 0; r_ < R; ++r_) \
            asm volatile("global_load_dwordx4 %0, %2, off nt\n\tglobal_load_dwordx4 %1, %2, off offset:64 nt" \
                         : "=&v"(xa[s_][r_][0]), "=&v"(xa[s_][r_][1]) : "v"(_s + r_ * 16 * KD) : "memory"); } while (0)
        unsigned cv;
        f32x4 sv;
        asm volatile("global_load_dwordx4 %0, %1, off" : "=v"(sv) : "v"(x_sample(p.x, lane)) : "memory");
        asm volatile("global_load_dword %0, %1, off sc1" : "=v"(cv) : "v"(cntp) : "memory");
#pragma unroll
        for (int s = 0; s < P; ++s) { cur[s] = s * NW + cw; XLD(s, cur[s]); }
        asm volatile("s_waitcnt vmcnt(%2)" : "+v"(cv), "+v"(sv) : "i"(2 * P * R) : "memory");
        const float xsc = exp2i(scale_exp(sv));
        const unsigned call = rfl(cv) + 1u;
        const unsigned tag = ((call + 1u) >> 1) & 1u;
        const size_t doff = OFF_X16 + (size_t)(call & 1u) * X16_B + (size_t)(p.rank * 2) * (NH * 16384);
        const char* dst[ND];
#pragma unroll
        for (int j = 0; j < ND; ++j) dst[j] = (side ? p.dst[NR - ND + j] : p.dst[j]) + doff;
        const char* ictr = myws + OFF_ICTR + (call & 1u) * 256u + side * 128u;
        const unsigned vzero = 0u, vone = 1u;
        auto body = [&](auto sc, auto first) -> bool {
            constexpr int s = decltype(sc)::value;
            constexpr bool F = decltype(first)::value;
            constexpr int W = F ? 2 * R * (P - 1 - s) + s * OPS : ND * R + (P - 1) * OPS;
            if constexpr (F) av[s] = 0u;
            if constexpr (R == 1)
                asm volatile("s_waitcnt vmcnt(%3)" : "+v"(xa[s][0][0]), "+v"(xa[s][0][1]), "+v"(av[s]) : "i"(W) : "memory");
            else
                asm volatile("s_waitcnt vmcnt(%5)" : "+v"(xa[s][0][0]), "+v"(xa[s][0][1]), "+v"(xa[s][R - 1][0]), "+v"(xa[s][R - 1][1]), "+v"(av[s])
                             : "i"(W) : "memory");
            const unsigned it = cur[s];
            if (it >= (unsigned)NITEM) return false;
            const unsigned nxt = F ? (unsigned)((P + s) * NW + cw) : (unsigned)(2 * P * NW) + rfl(av[s]);
            u32x4 o[R];
#pragma unroll
            for (int r = 0; r < R; ++r) {
                const f32x4 u0 = xa[s][r][0] * xsc, u1 = xa[s][r][1] * xsc;
                const unsigned rn = pk16(u0.x, u0.y);
                unsigned t = __builtin_bit_cast(unsigned, __builtin_amdgcn_cvt_pkrtz(u0.x, 0.0f));
                t += (t ^ tag) & 1u;
                o[r].x = (rn & 0xffff0000u) | (t & 0xffffu);
                o[r].y = pk16(u0.z, u0.w);
                o[r].z = pk16(u1.x, u1.y); o[r].w = pk16(u1.z, u1.w);
                asm volatile("" : "+v"(o[r]));
            }
            asm volatile("s_mov_b64 exec, 1\n\tglobal_atomic_add %0, %1, %2, %3 sc0 sc1\n\ts_mov_b64 exec, -1"
                         : "=&v"(av[s]) : "v"(vzero), "v"(vone), "s"(ictr) : "memory");
            XLD(s, nxt);
            cur[s] = nxt;
            const unsigned rb = (it % IPH) * R;
            const unsigned voff = (rb >> 4) * (NH * 16384u) + (it / IPH) * 16384u + (rb & 15u) * 1024u + lane * 16u;
#pragma unroll
            for (int j = 0; j < ND; ++j) st16_sys(dst[j], voff, o[0]);
            if constexpr (R == 2) {
#pragma unroll
                for (int j = 0; j < ND; ++j) st16_sys_o(dst[j], voff, o[R - 1]);
            }
            return true;
        };
        static_for<P>([&](auto sc) { body(sc, IC<1>{}); });
        for (bool go = true; go;) {
            static_for<P>([&](auto sc) { if (go) go = body(sc, IC<0>{}); });
        }
#pragma unroll
        for (int s = 0; s < P; ++s)
#pragma unroll
            for (int r = 0; r < R; ++r)
                asm volatile("s_waitcnt vmcnt(0)" : "+v"(xa[s][r][0]), "+v"(xa[s][r][1]), "+v"(av[s]) :: "memory");
        if (lane == 0) {
            __hip_atomic_store(cntp, call, __ATOMIC_RELAXED, __HIP_MEMORY_SCOPE_AGENT);
            if (cw == 0) st4_sys_p(myws + OFF_ICTR + ((call + 1u) & 1u) * 256u + side * 128u, 0u);
        }
        STAMP(1);
        return;
    }

    const int xcd = b & 7;
    const int t = (xcd & 1) * 30 + (b >> 3) - 2;
    const int tj = t >> 2;
    const int ti = (xcd >> 1) * 4 + (t & 3);
    const bool wide = (tj & 1) || tj == 14;
    const int col0 = ((tj * 17) >> 1) * 16;

    const unsigned call = rfl(__hip_atomic_load(cntp, __ATOMIC_RELAXED, __HIP_MEMORY_SCOPE_AGENT)) + 1u;
    if (lane == 0) __hip_atomic_store(cntp, call, __ATOMIC_RELAXED, __HIP_MEMORY_SCOPE_AGENT);
    unsigned* tmo = (unsigned*)(myws + OFF_TMO);

    if (tid < 64) *(LAS unsigned*)(lds + CTRL + tid * 4) = 0u;
    asm volatile("s_waitcnt lgkmcnt(0)" ::: "memory");
    __builtin_amdgcn_s_barrier();
    asm volatile("" ::: "memory");

    if (wave < 8) {
        static_assert(W_JC <= 2, "phase 0");
        fp16_saturate();
        int wexp;
        {
            const int j = wave / 3, part = wave - 3 * j;
            const f32x4* smp = w_sample(p.w, lane);
            if (j < W_JC && part < 2) conv_w64(p, tj, col0, 16 * j + ti, part, lane, smp, wexp);
            else if (j < W_JC && wide) conv_w16(p, tj, col0, 16 * j + ti, lane, smp, wexp);
            else wexp = scale_exp(*smp);
        }
        VMW(0);
        if (lane == 0) asm volatile("ds_add_u32 %0, %1" :: "v"(ldsb + CTRL + C_WCNT), "v"(1u) : "memory");
        STAMP(1);
        DBGRET(1);
        if (wide && wave < 4) consume<5>(p, ldsb, wave, lane, ti, col0, call, wexp);
        else consume<4>(p, ldsb, wave, lane, ti, col0, call, wexp);
    } else if (wave < 11) {
        DBGRET(1);
        const int L = wave - 8;
        const bool nine = (L == 2) && wide;
        const unsigned soff = (L < 2) ? (unsigned)OFF_X16 + (call & 1u) * (unsigned)X16_B + (unsigned)ti * (NH * 16384) + L * 8192
                                      : (unsigned)OFF_W16 + (unsigned)tj * (NH * WB);
        const char* src = myws + rfl(soff);
        const unsigned step = (L < 2) ? 16384u : (unsigned)WB;
        const unsigned ldst0 = ldsb + L * 8192;
        const unsigned fullad = ldsb + CTRL + C_FULL + L * 4;
        const unsigned ctlad = ldsb + CTRL + ((lane < 8) ? C_FREE + lane * 4 : (L < 2 ? C_RDYA : C_RDYB));
        unsigned voff = lane * 16;
        unsigned h = 0, pub = 0, slot = 0;
        unsigned long long tw = 0; unsigned sp = 0;
        bool dead = false;
        VMW(0);
        while (pub < NH) {
            const unsigned cv = lds_rd32_wait(ctlad);
            const unsigned ready = __builtin_amdgcn_readlane(cv, 8);
            const unsigned freed = rfl(dppmin<0x141>(dppmin<0x4E>(dppmin<0xB1>(cv))));
            const unsigned lim = dead ? (unsigned)NH : umin(ready, freed + NSLOT);
            bool act = false;
            if (h < lim) {
                glds8(src, voff, voff + 4096, ldst0 + slot);
                if (nine) glds1(src, voff + 8192, ldst0 + slot + 8192);
                voff += step; ++h;
                slot = (slot == (NSLOT - 1) * SLOT_B) ? 0u : slot + SLOT_B;
                act = true;
                if (h < lim) {
                    glds8(src, voff, voff + 4096, ldst0 + slot);
                    if (nine) glds1(src, voff + 8192, ldst0 + slot + 8192);
                    voff += step; ++h;
                    slot = (slot == (NSLOT - 1) * SLOT_B) ? 0u : slot + SLOT_B;
                }
            }
            const unsigned vm = vm_outstanding();
            const unsigned landed = h - (nine ? ((vm + 8u) * 57u) >> 9 : (vm + 7u) >> 3);
            if (landed > pub) { pub = landed; lds_wr32(fullad, pub); act = true; }
            if (act) { sp = 0; continue; }
            if (sp == 0) tw = rt();
            if ((++sp & 1023u) == 0u && rt() - tw > 2 * DEADLINE) { lds_wr32(ldsb + CTRL + C_BAD, 1u); dead = true; }
            if (h == pub) __builtin_amdgcn_s_sleep(1);
        }
        STAMP(1);
    } else {
        const unsigned tag = ((call + 1u) >> 1) & 1u;
        const char* wflagp = myws + OFF_WFLAG + (unsigned)(tj * 16 + (lane & 15)) * 128u;
        const char* xsrc = myws + rfl((unsigned)OFF_X16 + (call & 1u) * (unsigned)X16_B + (unsigned)ti * (NH * 16384));
        unsigned ra = 0;
        unsigned pp = 0;
        unsigned rbw = 0, pw = POLL_W;
        bool wok = false;
        const unsigned wbase = call * 256u;
        int jn = W_JC;
        int jpub = 0;
        bool p0 = false;
        const unsigned tstart = (unsigned)rt();
        fp16_saturate();
        const float wsc = exp2i(scale_exp(*w_sample(p.w, lane)));
        const char* xexpp = myws + OFF_XEXP + (call & 1u) * 1024u + (ti >> 1) * 128;
        bool xok = false;
        unsigned long long tw = 0; unsigned sp = 0;
        while (ra < NH || !wok) {
            unsigned fv = tag ^ 1u, wv = 0, xv = 0;
            const unsigned hx = ra + (lane >> 4) - 1u;
            const bool probe = lane < pw && pp + lane < (unsigned)NH;
            const bool piece = lane >= 16 && lane < 48 && hx >= (unsigned)(NH - POLL_NX) && hx < (unsigned)NH &&
                               (hx < pp || hx >= (unsigned)(NH - POLL_ND));
            const unsigned off = probe ? (pp + lane) * 16384u + 15u * 1024u + 1008u : hx * 16384u + (lane & 15) * 1024u + 1008u;
            if (!wok && lane < 16)
                asm volatile("global_load_dword %0, %1, off sc0 sc1" : "=v"(wv) : "v"(wflagp) : "memory");
            if (!xok && lane == 0)
                asm volatile("global_load_dword %0, %1, off sc0 sc1" : "=v"(xv) : "v"(xexpp) : "memory");
            if (probe || piece)
                asm volatile("global_load_dword %0, %1, %2 sc0 sc1" : "=v"(fv) : "v"(off), "s"(xsrc) : "memory");
            asm volatile("s_waitcnt vmcnt(0)" : "+v"(fv), "+v"(wv), "+v"(xv) :: "memory");
            bool act = false;
            if (!xok) {
                const unsigned xe = rfl(xv) - wbase;
                if (xe < 256u) { xok = true; lds_wr32(ldsb + CTRL + C_XEXP, xe); }
            }
            if (jpub < 8) {
                if (!p0) p0 = rfl(lds_rd32_wait(ldsb + CTRL + C_WCNT)) == 8u;
                if (p0 && jn > jpub) {
                    jpub = jn;
                    if (lane == 0) st4_sys_p(myws + OFF_WFLAG + (size_t)(tj * 16 + ti) * 128, wbase + (unsigned)jpub);
                }
            }
            if (!wok) {
                unsigned c = wv - wbase;
                c = (c <= 8u ? c : 0u) * 16u + (lane & 15);
                const unsigned nb = umin(rfl(dppmin<0x140>(dppmin<0x141>(dppmin<0x4E>(dppmin<0xB1>(c))))), (unsigned)NH);
                if (nb > rbw) { rbw = nb; lds_wr32(ldsb + CTRL + C_RDYB, nb); act = true; }
                wok = nb == (unsigned)NH;
            }
            if (ra < NH) {
                const unsigned long long m = ~__ballot((fv & 1u) == tag);
                const unsigned np = (unsigned)__builtin_ctzll(m | 0x10000ull);
                pw = (np == pw) ? umin(2u * pw, 16u) : (unsigned)POLL_W;
                const unsigned nx = (unsigned)__builtin_ctzll((m >> 16) | (1ull << 32)) >> 4;
                pp += np;
                unsigned rn = ra + nx;
                const unsigned rb = umin(pp > (unsigned)POLL_LAG ? pp - POLL_LAG : 0u, (unsigned)(NH - POLL_NX));
                rn = rn > rb ? rn : rb;
                if (!xok) rn = umin(rn, (unsigned)(NH - 1));
                if (rn > ra) { ra = rn; lds_wr32(ldsb + CTRL + C_RDYA, ra); act = true; }
                if (np) act = true;
            }
            if (jn < 8) {
                const int hw = 16 * jn + ti;
                if ((int)ra + W_LEAD >= hw || (int)((unsigned)rt() - tstart) >= (hw - W_LEAD) * W_TICK) {
                    conv_wblock(p, tj, col0, wide, hw, lane, wsc);
                    ++jn; act = true;
                }
            }
            if (act) { sp = 0; continue; }
            if (sp == 0) tw = rt();
            if ((++sp & 63u) == 0u) {
                const unsigned tv = rfl(__hip_atomic_load(tmo, __ATOMIC_RELAXED, __HIP_MEMORY_SCOPE_AGENT));
                if (tv == call || rt() - tw > DEADLINE) {
                    if (lane == 0) __hip_atomic_store(tmo, call, __ATOMIC_RELAXED, __HIP_MEMORY_SCOPE_AGENT);
                    lds_wr32(ldsb + CTRL + C_BAD, 1u);
                    lds_wr32(ldsb + CTRL + C_RDYA, NH);
                    lds_wr32(ldsb + CTRL + C_RDYB, NH);
                    break;
                }
            }
            __builtin_amdgcn_s_sleep(POLL_SLEEP);
        }
        STAMP(1);
    }
}

}

extern "C" void kernel_launch(int rank, int nranks, ncclComm_t comm,
                              hipStream_t stream,
                              void* const* d_in, const int* in_sizes, int n_in,
                              void* d_out, int out_size,
                              const int* in_dtypes, int out_dtype,
                              void* const* d_out_all,
                              void* const* d_ws_all, size_t ws_bytes) {
    using namespace agk;
    if (nranks != NR || (unsigned)rank >= (unsigned)NR || ws_bytes < WS_NEED || n_in < 2) {
        fprintf(stderr, "ag_gemm_silu: unexpected configuration\n");
        return;
    }
    static bool attr_done = false;
    if (!attr_done) {
        HIP_CHECK(hipFuncSetAttribute((const void*)ag_gemm_silu, hipFuncAttributeMaxDynamicSharedMemorySize, LDS_B));
        attr_done = true;
    }
    Params p;
    p.x = (const float*)d_in[0];
    p.w = (const float*)d_in[1];
    p.out = (float*)d_out;
    p.myws = (char*)d_ws_all[rank];
    if (COMM_SIDES == 2) {
        static const int grp[2][4] = {{0, 2, 3, 6}, {1, 4, 5, 7}};
        const int gi = (rank == 1 || rank == 4 || rank == 5 || rank == 7) ? 1 : 0;
        int pos = 0;
        for (int i = 0; i < 4; ++i) if (grp[gi][i] == rank) pos = i;
        for (int i = 0; i < 4; ++i) {
            p.dst[(COMM_SWAP ? 4 : 0) + i] = (char*)d_ws_all[grp[gi][(pos + 1 + i) & 3]];
            p.dst[(COMM_SWAP ? 0 : 4) + i] = (char*)d_ws_all[grp[gi ^ 1][(pos + i) & 3]];
        }
    } else {
        for (int i = 0; i < NR; ++i) p.dst[i] = (char*)d_ws_all[(rank + 1 + i) & 7];
    }
    p.rank = rank;
    ag_gemm_silu<<<dim3(256), dim3(NWAVE * 64), LDS_B, stream>>>(p);
    HIP_CHECK(hipGetLastError());
}
```

```cpp
#include <hip/hip_runtime.h>
#include <rccl/rccl.h>
#include <cstdio>

#define HIP_CHECK(expr) do { hipError_t _e = (expr); if (_e != hipSuccess) { \
    fprintf(stderr, "HIP error %s at %s:%d\n", hipGetErrorString(_e), __FILE__, __LINE__); \
    return; } } while (0)

#define LAS __attribute__((address_space(3)))
typedef _Float16 f16x8 __attribute__((ext_vector_type(8)));
typedef _Float16 f16x2 __attribute__((ext_vector_type(2)));
typedef float f32x4 __attribute__((ext_vector_type(4)));
typedef float f32x2 __attribute__((ext_vector_type(2)));
typedef unsigned u32x4 __attribute__((ext_vector_type(4)));
typedef unsigned u32x2 __attribute__((ext_vector_type(2)));

namespace agk {

constexpr int NR = 8;
constexpr int KD = 4096;
constexpr int NL = 2048;
constexpr int NH = 128;
constexpr int NSLOT = 6;
constexpr int WB = 9216;
constexpr int SLOT_B = 16384 + WB;
constexpr int CTRL = NSLOT * SLOT_B;
constexpr int LDS_B = CTRL + 256;
constexpr int NWAVE = 12;
constexpr int NCWG = 16;
#define COMM_P 3
#define COMM_R 1
#define COMM_WGS 16
#define COMM_WPG 2
#define COMM_SIDES 1
#define COMM_SWAP 0
#define POLL_SLEEP 2
#define POLL_LAG 2
#define POLL_NX 8
#define W_JC 1
#define W_LEAD 20
#define W_TICK 55
#define POLL_W 2
#define POLL_ND 2
constexpr int CWPG = COMM_WPG;
constexpr int NCW = COMM_WGS * COMM_WPG;
constexpr int IPH = 32 / COMM_R;
constexpr int NITEM = NH * IPH;

constexpr int C_FULL = 0;
constexpr int C_RDYA = 12;
constexpr int C_RDYB = 16;
constexpr int C_BAD = 20;
constexpr int C_WCNT = 24;
constexpr int C_XEXP = 28;
constexpr int C_FREE = 64;

constexpr size_t OFF_CNT = 0;
constexpr size_t OFF_TMO = 32768;
constexpr size_t OFF_ICTR = 65536;
constexpr size_t OFF_XEXP = 98304;
constexpr size_t OFF_WFLAG = 131072;
constexpr size_t OFF_DBG = 262144;
constexpr size_t OFF_X16 = 1ull << 20;
constexpr size_t X16_B = 32ull << 20;
constexpr size_t OFF_W16 = OFF_X16 + 2 * X16_B;
constexpr size_t WS_NEED = OFF_W16 + (size_t)15 * NH * WB;
constexpr unsigned long long DEADLINE = 50000000ull;

struct Params {
    const float* x;
    const float* w;
    float* out;
    char* myws;
    char* dst[NR];
    int rank;
};
#define DBGRET(bit) do { } while (0)

template <int V> struct IC { static constexpr int value = V; };
template <int N, int I = 0, class F> __device__ __forceinline__ void static_for(F&& f) {
    if constexpr (I < N) { f(IC<I>{}); static_for<N, I + 1>(f); }
}

__device__ __forceinline__ unsigned long long rt() {
    unsigned long long t;
    asm volatile("s_memrealtime %0\n\ts_waitcnt lgkmcnt(0)" : "=s"(t) :: "memory");
    return t;
}

__device__ __forceinline__ unsigned pk16(float a, float b) {
    f32x2 v = {a, b};
    f16x2 h = __builtin_convertvector(v, f16x2);
    return __builtin_bit_cast(unsigned, h);
}

__device__ __forceinline__ u32x4 cvt_tag(f32x4 u0, f32x4 u1, unsigned tag) {
    const unsigned rn = pk16(u0.x, u0.y);
    unsigned t = __builtin_bit_cast(unsigned, __builtin_amdgcn_cvt_pkrtz(u0.x, 0.0f));
    t += (t ^ tag) & 1u;
    u32x4 o;
    o.x = (rn & 0xffff0000u) | (t & 0xffffu);
    o.y = pk16(u0.z, u0.w);
    o.z = pk16(u1.x, u1.y); o.w = pk16(u1.z, u1.w);
    return o;
}

#define POL_COMM "sc0 sc1"
#define POL_OUT "sc0 sc1"
#define POL_DMA "sc0 sc1"
__device__ __forceinline__ void st16_sys(const char* sbase, unsigned voff, u32x4 v) {
    asm volatile("global_store_dwordx4 %0, %1, %2 " POL_COMM "\n\ts_nop 1" :: "v"(voff), "v"(v), "s"(sbase) : "memory");
}
__device__ __forceinline__ void st16_out(const char* p, u32x4 v) {
    asm volatile("global_store_dwordx4 %0, %1, off " POL_OUT "\n\ts_nop 1" :: "v"(p), "v"(v) : "memory");
}
__device__ __forceinline__ void st4_out(const char* p, unsigned v) {
    asm volatile("global_store_dword %0, %1, off " POL_OUT :: "v"(p), "v"(v) : "memory");
}
__device__ __forceinline__ void st16_sys_o(const char* sbase, unsigned voff, u32x4 v) {
    asm volatile("global_store_dwordx4 %0, %1, %2 offset:1024 sc0 sc1\n\ts_nop 1" :: "v"(voff), "v"(v), "s"(sbase) : "memory");
}
__device__ __forceinline__ void st16_sys_p(const char* p, u32x4 v) {
    asm volatile("global_store_dwordx4 %0, %1, off sc0 sc1\n\ts_nop 1" :: "v"(p), "v"(v) : "memory");
}
__device__ __forceinline__ void st4_sys_p(const char* p, unsigned v) {
    asm volatile("global_store_dword %0, %1, off sc0 sc1" :: "v"(p), "v"(v) : "memory");
}

#define VMW(n) asm volatile("s_waitcnt vmcnt(" #n ")" ::: "memory")

__device__ __forceinline__ unsigned vm_outstanding() {
    unsigned v;
    asm volatile("s_getreg_b32 %0, hwreg(HW_REG_IB_STS)" : "=s"(v) :: "memory");
    return (v & 15u) | ((v >> 18) & 48u);
}

__device__ __forceinline__ void glds8(const char* sbase, unsigned voff, unsigned voff2, unsigned ldst) {
    unsigned keep;
    asm volatile(
        "s_mov_b32 %0, m0\n\t"
        "s_mov_b32 m0, %4\n\t"
        "s_nop 0\n\t"
        "global_load_lds_dwordx4 %1, %3 " POL_DMA "\n\t"
        "global_load_lds_dwordx4 %1, %3 offset:1024 " POL_DMA "\n\t"
        "global_load_lds_dwordx4 %1, %3 offset:2048 " POL_DMA "\n\t"
        "global_load_lds_dwordx4 %1, %3 offset:3072 " POL_DMA "\n\t"
        "s_add_u32 m0, %4, 4096\n\t"
        "s_nop 0\n\t"
        "global_load_lds_dwordx4 %2, %3 " POL_DMA "\n\t"
        "global_load_lds_dwordx4 %2, %3 offset:1024 " POL_DMA "\n\t"
        "global_load_lds_dwordx4 %2, %3 offset:2048 " POL_DMA "\n\t"
        "global_load_lds_dwordx4 %2, %3 offset:3072 " POL_DMA "\n\t"
        "s_mov_b32 m0, %0"
        : "=&s"(keep) : "v"(voff), "v"(voff2), "s"(sbase), "s"(ldst) : "memory", "scc");
}
__device__ __forceinline__ void glds1(const char* sbase, unsigned voff, unsigned ldst) {
    unsigned keep;
    asm volatile(
        "s_mov_b32 %0, m0\n\t"
        "s_mov_b32 m0, %3\n\t"
        "s_nop 0\n\t"
        "global_load_lds_dwordx4 %1, %2 " POL_DMA "\n\t"
        "s_mov_b32 m0, %0"
        : "=&s"(keep) : "v"(voff), "s"(sbase), "s"(ldst) : "memory");
}

__device__ __forceinline__ void lds_wr32(unsigned addr, unsigned v) {
    asm volatile("s_mov_b64 exec, 1\n\tds_write_b32 %0, %1\n\ts_mov_b64 exec, -1" :: "v"(addr), "v"(v) : "memory");
}
__device__ __forceinline__ u32x4 lds_rd128_wait(unsigned addr) {
    u32x4 v;
    asm volatile("ds_read_b128 %0, %1\n\ts_waitcnt lgkmcnt(0)" : "=&v"(v) : "v"(addr) : "memory");
    return v;
}
__device__ __forceinline__ unsigned lds_rd32_wait(unsigned addr) {
    unsigned v;
    asm volatile("ds_read_b32 %0, %1\n\ts_waitcnt lgkmcnt(0)" : "=&v"(v) : "v"(addr) : "memory");
    return v;
}
__device__ __forceinline__ unsigned rfl(unsigned v) { return __builtin_amdgcn_readfirstlane(v); }
__device__ __forceinline__ unsigned umin(unsigned a, unsigned b) { return a < b ? a : b; }
template <int CTL> __device__ __forceinline__ unsigned dppmin(unsigned d) {
    return umin(d, (unsigned)__builtin_amdgcn_update_dpp((int)d, (int)d, CTL, 0xf, 0xf, false));
}

template <int CTL> __device__ __forceinline__ unsigned dppmax(unsigned d) {
    const unsigned o = (unsigned)__builtin_amdgcn_update_dpp((int)d, (int)d, CTL, 0xf, 0xf, false);
    return d > o ? d : o;
}
__device__ __forceinline__ int scale_exp(f32x4 v) {
    const u32x4 u = __builtin_bit_cast(u32x4, v) & 0x7fffffffu;
    unsigned m = u.x > u.y ? u.x : u.y, n = u.z > u.w ? u.z : u.w;
    m = m > n ? m : n;
    m = dppmax<0x140>(dppmax<0x141>(dppmax<0x4E>(dppmax<0xB1>(m))));
    const unsigned a = __builtin_amdgcn_readlane(m, 0), b = __builtin_amdgcn_readlane(m, 16);
    const unsigned c = __builtin_amdgcn_readlane(m, 32), d = __builtin_amdgcn_readlane(m, 48);
    const unsigned ab = a > b ? a : b, cd = c > d ? c : d;
    const int e = (int)((ab > cd ? ab : cd) >> 23);
    if (e == 0 || e == 255) return 0;
    const int sx = 126 - e;
    return sx < -60 ? -60 : (sx > 60 ? 60 : sx);
}
__device__ __forceinline__ float exp2i(int e) { return __builtin_bit_cast(float, (unsigned)(127 + e) << 23); }
__device__ __forceinline__ const f32x4* x_sample(const float* x, int lane) {
    return (const f32x4*)(x + (size_t)(lane * 8 + 3) * KD + ((lane * 260) & (KD - 1)));
}
__device__ __forceinline__ const f32x4* w_sample(const float* w, int lane) {
    return (const f32x4*)(w + (size_t)(lane * 64 + 17) * NL + ((lane * 132) & (NL - 1)));
}
__device__ __forceinline__ void fp16_saturate() {
    asm volatile("s_setreg_imm32_b32 hwreg(HW_REG_MODE, 23, 1), 1" ::: "memory");
}

__device__ __forceinline__ float silu(float y) {
    float t = __builtin_amdgcn_fmed3f(y, -60.0f, 60.0f);
    float e = __builtin_amdgcn_exp2f(t * -1.44269504088896340736f);
    return y * __builtin_amdgcn_rcpf(1.0f + e);
}

#define STAMP(i) do { } while (0)

#define SB0() __builtin_amdgcn_sched_barrier(0)
#define DSR(dst, addr, off) asm volatile("ds_read_b128 %0, %1 offset:%2" : "=v"(dst) : "v"(addr), "i"(off) : "memory")

template <int MOFF>
__device__ __forceinline__ void fix_frag(f16x8& X, unsigned long long badmask, const char* sbase, unsigned hh,
                                         unsigned lane16, unsigned tag, unsigned badad) {
    unsigned t, vo;
    asm volatile(
        "s_cmp_eq_u64 %[bm], 0\n\t"
        "s_cbranch_scc1 3f\n\t"
        "s_lshl_b32 s92, %[hh], 14\n\t"
        "s_memrealtime s[90:91]\n\t"
        "v_add_u32 %[vo], s92, %[l16]\n\t"
        "s_waitcnt lgkmcnt(0)\n"
        "1:\n\t"
        "global_load_dword %[t], %[vo], %[sb] offset:%[mo] sc0 sc1\n\t"
        "s_waitcnt vmcnt(0)\n\t"
        "v_and_b32 %[t], 1, %[t]\n\t"
        "v_cmp_ne_u32 vcc, %[tag], %[t]\n\t"
        "s_nop 4\n\t"
        "s_cmp_eq_u64 vcc, 0\n\t"
        "s_cbranch_scc1 2f\n\t"
        "s_sleep 1\n\t"
        "s_memrealtime s[92:93]\n\t"
        "s_waitcnt lgkmcnt(0)\n\t"
        "s_sub_u32 s92, s92, s90\n\t"
        "s_cmp_lt_u32 s92, %[dl]\n\t"
        "s_cbranch_scc1 1b\n\t"
        "v_mov_b32 %[t], %[ba]\n\t"
        "ds_write_b32 %[t], %[t]\n\t"
        "s_waitcnt lgkmcnt(0)\n"
        "2:\n\t"
        "global_load_dwordx4 %[x], %[vo], %[sb] offset:%[mo] sc0 sc1\n\t"
        "s_waitcnt vmcnt(0)\n"
        "3:"
        : [x] "+v"(X), [t] "=&v"(t), [vo] "=&v"(vo)
        : [bm] "s"(badmask), [hh] "s"(hh), [l16] "v"(lane16), [sb] "s"(sbase), [tag] "s"(tag), [ba] "s"(badad),
          [dl] "i"((unsigned)DEADLINE), [mo] "i"(MOFF)
        : "memory", "vcc", "scc", "s90", "s91", "s92", "s93");
}

__device__ __forceinline__ void conv_w64(const Params& p, int tj, int col0, int h, int cg, int lane, const f32x4* smp, int& se) {
    const int g = lane >> 4, q = lane & 15;
    const float* wsrc = p.w + (size_t)(h * 32 + 4 * g) * NL + col0 + cg * 64 + 4 * q;
    f32x4 v[8];
#pragma unroll
    for (int i = 0; i < 8; ++i)
        v[i] = __builtin_nontemporal_load((const f32x4*)(wsrc + (size_t)((i & 3) + 16 * (i >> 2)) * NL));
    se = scale_exp(*smp);
    const float sc = exp2i(se);
#pragma unroll
    for (int i = 0; i < 8; ++i) v[i] *= sc;
    char* dst = p.myws + OFF_W16 + (size_t)(tj * NH + h) * WB + cg * 4096 + lane * 16;
#pragma unroll
    for (int f = 0; f < 4; ++f) {
        u32x4 o;
        o.x = pk16(v[0][f], v[1][f]); o.y = pk16(v[2][f], v[3][f]);
        o.z = pk16(v[4][f], v[5][f]); o.w = pk16(v[6][f], v[7][f]);
        st16_sys_p(dst + f * 1024, o);
    }
}
__device__ __forceinline__ void conv_w16(const Params& p, int tj, int col0, int h, int lane, const f32x4* smp, int& se) {
    const int g = lane >> 4, q = lane & 15;
    const float* wsrc = p.w + (size_t)(h * 32 + 4 * g) * NL + col0 + 128 + q;
    float e[8];
#pragma unroll
    for (int i = 0; i < 8; ++i) e[i] = __builtin_nontemporal_load(wsrc + (size_t)((i & 3) + 16 * (i >> 2)) * NL);
    se = scale_exp(*smp);
    const float sc = exp2i(se);
#pragma unroll
    for (int i = 0; i < 8; ++i) e[i] *= sc;
    u32x4 o;
    o.x = pk16(e[0], e[1]); o.y = pk16(e[2], e[3]); o.z = pk16(e[4], e[5]); o.w = pk16(e[6], e[7]);
    st16_sys_p(p.myws + OFF_W16 + (size_t)(tj * NH + h) * WB + 8192 + lane * 16, o);
}
__device__ __forceinline__ void conv_wblock(const Params& p, int tj, int col0, bool wide, int h, int lane, float sc) {
    const int g = lane >> 4, q = lane & 15;
    const float* wrow = p.w + (size_t)(h * 32 + 4 * g) * NL + col0;
    f32x4 v[2][8];
    float e[8];
#pragma unroll
    for (int cg = 0; cg < 2; ++cg)
#pragma unroll
        for (int i = 0; i < 8; ++i)
            v[cg][i] = __builtin_nontemporal_load((const f32x4*)(wrow + 4 * q + (size_t)((i & 3) + 16 * (i >> 2)) * NL + cg * 64));
    if (wide) {
#pragma unroll
        for (int i = 0; i < 8; ++i)
            e[i] = __builtin_nontemporal_load(wrow + (size_t)((i & 3) + 16 * (i >> 2)) * NL + 128 + q);
    }
    char* dst = p.myws + OFF_W16 + (size_t)(tj * NH + h) * WB + lane * 16;
#pragma unroll
    for (int cg = 0; cg < 2; ++cg)
#pragma unroll
        for (int f = 0; f < 4; ++f) {
            u32x4 o;
            o.x = pk16(v[cg][0][f] * sc, v[cg][1][f] * sc); o.y = pk16(v[cg][2][f] * sc, v[cg][3][f] * sc);
            o.z = pk16(v[cg][4][f] * sc, v[cg][5][f] * sc); o.w = pk16(v[cg][6][f] * sc, v[cg][7][f] * sc);
            st16_sys_p(dst + (cg * 4 + f) * 1024, o);
        }
    if (wide) {
        u32x4 o;
        o.x = pk16(e[0] * sc, e[1] * sc); o.y = pk16(e[2] * sc, e[3] * sc);
        o.z = pk16(e[4] * sc, e[5] * sc); o.w = pk16(e[6] * sc, e[7] * sc);
        st16_sys_p(dst + 8192, o);
    }
}

template <int NB>
__device__ __forceinline__ void consume(const Params& p, const unsigned ldsb, const int wave, const int lane,
                                        const int ti, const int col0, const unsigned call, const int wexp) {
    const int wr = wave & 3, wc = wave >> 2;
    const int g = lane >> 4, q = lane & 15;
    f32x4 acc[4][NB];
#pragma unroll
    for (int m = 0; m < 4; ++m)
#pragma unroll
        for (int n = 0; n < NB; ++n) acc[m][n] = (f32x4){0.f, 0.f, 0.f, 0.f};
    f16x8 A0, A1, A2, A3, Bx[NB], By[NB];
    u32x4 fw;
    const unsigned va0 = ldsb + wr * 4096 + lane * 16, vb0 = ldsb + 16384 + wc * 4096 + lane * 16;
    const unsigned fullad = ldsb + CTRL + C_FULL, freead = ldsb + CTRL + C_FREE + wave * 4;
    const unsigned badad = ldsb + CTRL + C_BAD;
    const unsigned lane16 = lane * 16;
    const unsigned tag = ((call + 1u) >> 1) & 1u;
    const char* xsrc = p.myws + rfl((unsigned)OFF_X16 + (call & 1u) * (unsigned)X16_B + (unsigned)ti * (NH * 16384) + wr * 4096);
    unsigned sb = 0;
#define CNTFIX(m_) do { } while (0)

    auto spin_full = [&](unsigned need) {
        unsigned long long t0 = 0;
        unsigned sp = 0;
        for (;;) {
            u32x4 f = lds_rd128_wait(fullad);
            if (rfl(umin(umin(f.x, f.y), f.z)) >= need) break;
            if (sp == 0) t0 = rt();
            if ((++sp & 1023u) == 0u && rt() - t0 > 3 * DEADLINE) { lds_wr32(badad, 1u); break; }
            __builtin_amdgcn_s_sleep(1);
        }
    };
#define MM(m, A, B, n0, n1) do { _Pragma("unroll") for (int n = n0; n < n1; ++n) \
        acc[m][n] = __builtin_amdgcn_mfma_f32_16x16x32_f16(A, B[n], acc[m][n], 0, 0, 0); } while (0)
#define CHK(A) __ballot((__builtin_bit_cast(u32x4, A).x & 1u) != tag)
#define FIX(A, bm, hh, m) do { CNTFIX(bm); fix_frag<(m) * 1024>(A, bm, xsrc, (unsigned)(hh), lane16, tag, badad); } while (0)
#define LDB(B) do { DSR(B[0], vb, 0); DSR(B[1], vb, 1024); DSR(B[2], vb, 2048); DSR(B[3], vb, 3072); \
        if constexpr (NB == 5) DSR(B[NB - 1], vb, 8192); \
        asm volatile("ds_read_b128 %0, %1" : "=v"(fw) : "v"(fullad) : "memory"); } while (0)
#define WTOP(B) do { if constexpr (NB == 5) \
        asm volatile("s_waitcnt lgkmcnt(2)" : "+v"(B[0]), "+v"(B[1]), "+v"(B[2]), "+v"(B[3]), "+v"(B[NB - 1]), "+v"(fw), "+v"(A0) :: "memory"); \
        else asm volatile("s_waitcnt lgkmcnt(2)" : "+v"(B[0]), "+v"(B[1]), "+v"(B[2]), "+v"(B[3]), "+v"(fw), "+v"(A0) :: "memory"); } while (0)
#define WROW(A) asm volatile("s_waitcnt lgkmcnt(%1)" : "+v"(A) : "i"(NB + 4) : "memory")
#define WALL() asm volatile("s_waitcnt lgkmcnt(0)" : "+v"(A1), "+v"(A2), "+v"(A3) :: "memory")
#define PLAIN(C, hc) do { \
        WALL(); \
        const unsigned long long m1_ = CHK(A1), m2_ = CHK(A2), m3_ = CHK(A3); \
        MM(0, A0, C, 0, NB); SB0(); \
        FIX(A1, m1_, hc, 1); MM(1, A1, C, 0, NB); SB0(); \
        FIX(A2, m2_, hc, 2); MM(2, A2, C, 0, NB); SB0(); \
        FIX(A3, m3_, hc, 3); MM(3, A3, C, 0, NB); SB0(); } while (0)
#define FROW(m, A, AN, C, hc) do { \
        MM(m, A, C, 0, 2); SB0(); \
        WROW(AN); const unsigned long long mm_ = CHK(AN); SB0(); \
        MM(m, A, C, 2, NB); SB0(); \
        FIX(AN, mm_, hc, m + 1); } while (0)
#define STEP(C, N, hn) do { \
        const unsigned need_ = (unsigned)(hn) + 1u; \
        const bool fast_ = rfl(umin(umin(fw.x, fw.y), fw.z)) >= need_; \
        if (!fast_) { PLAIN(C, (hn) - 1); spin_full(need_); } \
        const unsigned va = va0 + sb, vb = vb0 + sb; \
        sb = (sb == (NSLOT - 1) * SLOT_B) ? 0u : sb + SLOT_B; \
        LDB(N); \
        SB0(); \
        if (fast_) FROW(0, A0, A1, C, (hn) - 1); \
        DSR(A0, va, 0); \
        if (fast_) FROW(1, A1, A2, C, (hn) - 1); \
        DSR(A1, va, 1024); \
        if (fast_) FROW(2, A2, A3, C, (hn) - 1); \
        DSR(A2, va, 2048); \
        if (fast_) { MM(3, A3, C, 0, 2); SB0(); } \
        WTOP(N); \
        const unsigned long long m0_ = CHK(A0); \
        SB0(); \
        if (fast_) { MM(3, A3, C, 2, NB); SB0(); } \
        DSR(A3, va, 3072); \
        lds_wr32(freead, need_); \
        FIX(A0, m0_, hn, 0); \
        } while (0)

    spin_full(1);
    STAMP(2);
    {
        const unsigned va = va0, vb = vb0;
        sb = SLOT_B;
        LDB(Bx);
        DSR(A0, va, 0); DSR(A1, va, 1024); DSR(A2, va, 2048);
        WTOP(Bx);
        const unsigned long long m0_ = CHK(A0);
        DSR(A3, va, 3072);
        lds_wr32(freead, 1u);
        FIX(A0, m0_, 0, 0);
    }
    __builtin_amdgcn_s_setprio(1);
    for (int h = 0; h < NH - 2; h += 2) {
        STEP(Bx, By, h + 1);
        STEP(By, Bx, h + 2);
    }
    STEP(Bx, By, NH - 1);
    PLAIN(By, NH - 1);
    __builtin_amdgcn_s_setprio(0);
    STAMP(3);
    const unsigned* tmo = (const unsigned*)(p.myws + OFF_TMO);
    const bool bad = rfl(lds_rd32_wait(badad)) != 0u ||
                     rfl(__hip_atomic_load(tmo, __ATOMIC_RELAXED, __HIP_MEMORY_SCOPE_AGENT)) == call;
    const float osc = exp2i(-wexp - ((int)rfl(lds_rd32_wait(ldsb + CTRL + C_XEXP)) - 64));
    float* orow = p.out + (size_t)(ti * 256 + wr * 64 + 4 * g) * NL + col0;
    float* obase = orow + wc * 64 + 4 * q;
    const float qn = __builtin_nanf("");
#pragma unroll
    for (int m = 0; m < 4; ++m)
#pragma unroll
        for (int r = 0; r < 4; ++r) {
            f32x4 o;
            o.x = silu(acc[m][0][r] * osc); o.y = silu(acc[m][1][r] * osc);
            o.z = silu(acc[m][2][r] * osc); o.w = silu(acc[m][3][r] * osc);
            if (bad) o = (f32x4){qn, qn, qn, qn};
            st16_out((const char*)(obase + (size_t)(m * 16 + r) * NL), __builtin_bit_cast(u32x4, o));
            if constexpr (NB == 5) {
                float e = silu(acc[m][NB - 1][r] * osc);
                if (bad) e = qn;
                st4_out((const char*)(orow + (size_t)(m * 16 + r) * NL + 128 + q), __builtin_bit_cast(unsigned, e));
            }
        }
    STAMP(4);
}

__global__ void __launch_bounds__(NWAVE * 64, 3) ag_gemm_silu(Params p) {
    extern __shared__ __attribute__((aligned(16))) char lds_raw[];
    LAS char* lds = (LAS char*)lds_raw;
    const unsigned ldsb = (unsigned)(size_t)lds;
    const int tid = threadIdx.x;
    const int wave = rfl(tid >> 6), lane = tid & 63;
    const int b = blockIdx.x;
    char* myws = p.myws;
    const int g = lane >> 4, q = lane & 15;
    unsigned* cntp = (unsigned*)(myws + OFF_CNT) + b * 16 + wave;
    STAMP(0);

    if (b < NCWG) {
        if (b == 0 && wave == CWPG) {
            const f32x4 sv = *x_sample(p.x, lane);
            const unsigned call = rfl(__hip_atomic_load(cntp, __ATOMIC_RELAXED, __HIP_MEMORY_SCOPE_AGENT)) + 1u;
            const int xe = scale_exp(sv);
            const char* d = p.dst[0];
#pragma unroll
            for (int j = 1; j < NR; ++j) d = (lane == j) ? p.dst[j] : d;
            if (lane < NR) st4_sys_p(d + OFF_XEXP + (call & 1u) * 1024u + p.rank * 128, call * 256u + (unsigned)(xe + 64));
            if (lane == 0) __hip_atomic_store(cntp, call, __ATOMIC_RELAXED, __HIP_MEMORY_SCOPE_AGENT);
            return;
        }
        if (b >= COMM_WGS || wave >= CWPG) return;
        fp16_saturate();
        constexpr int P = COMM_P;
        constexpr int R = COMM_R;
        constexpr int ND = NR - 1;
        static_assert(COMM_SIDES == 1, "sides");
        constexpr int NW = NCW / COMM_SIDES;
        constexpr int OPS = 1 + (2 + ND) * R;
        static_assert(ND * R + (P - 1) * OPS <= 63 && 2 * P * NW <= NITEM, "comm window");
        const int side = COMM_SIDES == 2 ? (b >> 2) & 1 : 0;
        const int cw = COMM_SIDES == 2 ? wave * (COMM_WGS / 2) + (b >> 3) * 4 + (b & 3) : wave * COMM_WGS + b;
        const float* xs = p.x + (size_t)q * KD + 4 * g;
        f32x4 xa[P][R][2];
        unsigned av[P];
        unsigned cur[P];
#define XLD(s_, it_) do { const unsigned i_ = (it_) < (unsigned)NITEM ? (it_) : (unsigned)NITEM - 1u; \
            const float* _s = xs + (size_t)((i_ % IPH) * (R * 16 * KD) + (i_ / IPH) * 32); \
            _Pragma("unroll") for (int r_ = 0; r_ < R; ++r_) \
            asm volatile("global_load_dwordx4 %0, %2, off nt\n\tglobal_load_dwordx4 %1, %2, off offset:64 nt" \
                         : "=&v"(xa[s_][r_][0]), "=&v"(xa[s_][r_][1]) : "v"(_s + r_ * 16 * KD) : "memory"); } while (0)
        unsigned cv;
        f32x4 sv;
        asm volatile("global_load_dwordx4 %0, %1, off" : "=v"(sv) : "v"(x_sample(p.x, lane)) : "memory");
        asm volatile("global_load_dword %0, %1, off sc1" : "=v"(cv) : "v"(cntp) : "memory");
#pragma unroll
        for (int s = 0; s < P; ++s) { cur[s] = s * NW + cw; XLD(s, cur[s]); }
        asm volatile("s_waitcnt vmcnt(%2)" : "+v"(cv), "+v"(sv) : "i"(2 * P * R) : "memory");
        const float xsc = exp2i(scale_exp(sv));
        const unsigned call = rfl(cv) + 1u;
        const unsigned tag = ((call + 1u) >> 1) & 1u;
        const size_t doff = OFF_X16 + (size_t)(call & 1u) * X16_B + (size_t)(p.rank * 2) * (NH * 16384);
        const char* dst[ND];
#pragma unroll
        for (int j = 0; j < ND; ++j) dst[j] = (side ? p.dst[NR - ND + j] : p.dst[j]) + doff;
        const char* ictr = myws + OFF_ICTR + (call & 1u) * 256u + side * 128u;
        const unsigned vzero = 0u, vone = 1u;
        auto body = [&](auto sc, auto first) -> bool {
            constexpr int s = decltype(sc)::value;
            constexpr bool F = decltype(first)::value;
            constexpr int W = F ? 2 * R * (P - 1 - s) + s * OPS : ND * R + (P - 1) * OPS;
            if constexpr (F) av[s] = 0u;
            if constexpr (R == 1)
                asm volatile("s_waitcnt vmcnt(%3)" : "+v"(xa[s][0][0]), "+v"(xa[s][0][1]), "+v"(av[s]) : "i"(W) : "memory");
            else
                asm volatile("s_waitcnt vmcnt(%5)" : "+v"(xa[s][0][0]), "+v"(xa[s][0][1]), "+v"(xa[s][R - 1][0]), "+v"(xa[s][R - 1][1]), "+v"(av[s])
                             : "i"(W) : "memory");
            const unsigned it = cur[s];
            if (it >= (unsigned)NITEM) return false;
            const unsigned nxt = F ? (unsigned)((P + s) * NW + cw) : (unsigned)(2 * P * NW) + rfl(av[s]);
            u32x4 o[R];
#pragma unroll
            for (int r = 0; r < R; ++r) {
                o[r] = cvt_tag(xa[s][r][0] * xsc, xa[s][r][1] * xsc, tag);
                asm volatile("" : "+v"(o[r]));
            }
            asm volatile("s_mov_b64 exec, 1\n\tglobal_atomic_add %0, %1, %2, %3 sc0 sc1\n\ts_mov_b64 exec, -1"
                         : "=&v"(av[s]) : "v"(vzero), "v"(vone), "s"(ictr) : "memory");
            XLD(s, nxt);
            cur[s] = nxt;
            const unsigned rb = (it % IPH) * R;
            const unsigned voff = (rb >> 4) * (NH * 16384u) + (it / IPH) * 16384u + (rb & 15u) * 1024u + lane * 16u;
#pragma unroll
            for (int j = 0; j < ND; ++j) st16_sys(dst[j], voff, o[0]);
            if constexpr (R == 2) {
#pragma unroll
                for (int j = 0; j < ND; ++j) st16_sys_o(dst[j], voff, o[R - 1]);
            }
            return true;
        };
        static_for<P>([&](auto sc) { body(sc, IC<1>{}); });
        for (bool go = true; go;) {
            static_for<P>([&](auto sc) { if (go) go = body(sc, IC<0>{}); });
        }
#pragma unroll
        for (int s = 0; s < P; ++s)
#pragma unroll
            for (int r = 0; r < R; ++r)
                asm volatile("s_waitcnt vmcnt(0)" : "+v"(xa[s][r][0]), "+v"(xa[s][r][1]), "+v"(av[s]) :: "memory");
        if (lane == 0) {
            __hip_atomic_store(cntp, call, __ATOMIC_RELAXED, __HIP_MEMORY_SCOPE_AGENT);
            if (cw == 0) st4_sys_p(myws + OFF_ICTR + ((call + 1u) & 1u) * 256u + side * 128u, 0u);
        }
        STAMP(1);
        return;
    }

    const int xcd = b & 7;
    const int t = (xcd & 1) * 30 + (b >> 3) - 2;
    const int tj = t >> 2;
    const int ti = (xcd >> 1) * 4 + (t & 3);
    const bool wide = (tj & 1) || tj == 14;
    const int col0 = ((tj * 17) >> 1) * 16;

    const unsigned call = rfl(__hip_atomic_load(cntp, __ATOMIC_RELAXED, __HIP_MEMORY_SCOPE_AGENT)) + 1u;
    if (lane == 0) __hip_atomic_store(cntp, call, __ATOMIC_RELAXED, __HIP_MEMORY_SCOPE_AGENT);
    unsigned* tmo = (unsigned*)(myws + OFF_TMO);

    if (tid < 64) *(LAS unsigned*)(lds + CTRL + tid * 4) = 0u;
    asm volatile("s_waitcnt lgkmcnt(0)" ::: "memory");
    __builtin_amdgcn_s_barrier();
    asm volatile("" ::: "memory");

    if (wave < 8) {
        static_assert(W_JC == 1, "phase 0");
        fp16_saturate();
        int wexp;
        const f32x4* smp = w_sample(p.w, lane);
        if (wave < 3) {
            if (wave < 2) conv_w64(p, tj, col0, ti, wave, lane, smp, wexp);
            else if (wide) conv_w16(p, tj, col0, ti, lane, smp, wexp);
            else wexp = scale_exp(*smp);
            VMW(0);
            if (lane == 0) asm volatile("ds_add_u32 %0, %1" :: "v"(ldsb + CTRL + C_WCNT), "v"(1u) : "memory");
        } else {
            const unsigned u = (wave - 3) * (256 - NCWG) + (b - NCWG);
            if (u < 1024u) {
                const float* xs = p.x + (size_t)((u & 31u) * 16u + q) * KD + (u >> 5) * 128u + 4 * g;
                f32x4 v[8];
#pragma unroll
                for (int i = 0; i < 8; ++i) v[i] = *(const f32x4*)(xs + i * 16);
                const float xsc = exp2i(scale_exp(*x_sample(p.x, lane)));
                const unsigned tag = ((call + 1u) >> 1) & 1u;
                char* dst = myws + OFF_X16 + (size_t)(call & 1u) * X16_B + (size_t)(p.rank * 2 + ((u >> 4) & 1u)) * (NH * 16384) +
                            (u >> 5) * 65536u + (u & 15u) * 1024u + lane * 16;
#pragma unroll
                for (int i = 0; i < 4; ++i) st16_sys_p(dst + i * 16384, cvt_tag(v[2 * i] * xsc, v[2 * i + 1] * xsc, tag));
            }
            wexp = scale_exp(*smp);
        }
        STAMP(1);
        DBGRET(1);
        if (wide && wave < 4) consume<5>(p, ldsb, wave, lane, ti, col0, call, wexp);
        else consume<4>(p, ldsb, wave, lane, ti, col0, call, wexp);
    } else if (wave < 11) {
        DBGRET(1);
        const int L = wave - 8;
        const bool nine = (L == 2) && wide;
        const unsigned soff = (L < 2) ? (unsigned)OFF_X16 + (call & 1u) * (unsigned)X16_B + (unsigned)ti * (NH * 16384) + L * 8192
                                      : (unsigned)OFF_W16 + (unsigned)tj * (NH * WB);
        const char* src = myws + rfl(soff);
        const unsigned step = (L < 2) ? 16384u : (unsigned)WB;
        const unsigned ldst0 = ldsb + L * 8192;
        const unsigned fullad = ldsb + CTRL + C_FULL + L * 4;
        const unsigned ctlad = ldsb + CTRL + ((lane < 8) ? C_FREE + lane * 4 : (L < 2 ? C_RDYA : C_RDYB));
        unsigned voff = lane * 16;
        unsigned h = 0, pub = 0, slot = 0;
        unsigned long long tw = 0; unsigned sp = 0;
        bool dead = false;
        VMW(0);
        while (pub < NH) {
            const unsigned cv = lds_rd32_wait(ctlad);
            const unsigned ready = __builtin_amdgcn_readlane(cv, 8);
            const unsigned freed = rfl(dppmin<0x141>(dppmin<0x4E>(dppmin<0xB1>(cv))));
            const unsigned lim = dead ? (unsigned)NH : umin(ready, freed + NSLOT);
            bool act = false;
            if (h < lim) {
                glds8(src, voff, voff + 4096, ldst0 + slot);
                if (nine) glds1(src, voff + 8192, ldst0 + slot + 8192);
                voff += step; ++h;
                slot = (slot == (NSLOT - 1) * SLOT_B) ? 0u : slot + SLOT_B;
                act = true;
                if (h < lim) {
                    glds8(src, voff, voff + 4096, ldst0 + slot);
                    if (nine) glds1(src, voff + 8192, ldst0 + slot + 8192);
                    voff += step; ++h;
                    slot = (slot == (NSLOT - 1) * SLOT_B) ? 0u : slot + SLOT_B;
                }
            }
            const unsigned vm = vm_outstanding();
            const unsigned landed = h - (nine ? ((vm + 8u) * 57u) >> 9 : (vm + 7u) >> 3);
            if (landed > pub) { pub = landed; lds_wr32(fullad, pub); act = true; }
            if (act) { sp = 0; continue; }
            if (sp == 0) tw = rt();
            if ((++sp & 1023u) == 0u && rt() - tw > 2 * DEADLINE) { lds_wr32(ldsb + CTRL + C_BAD, 1u); dead = true; }
            if (h == pub) __builtin_amdgcn_s_sleep(1);
        }
        STAMP(1);
    } else {
        const unsigned tag = ((call + 1u) >> 1) & 1u;
        const char* wflagp = myws + OFF_WFLAG + (unsigned)(tj * 16 + (lane & 15)) * 128u;
        const char* xsrc = myws + rfl((unsigned)OFF_X16 + (call & 1u) * (unsigned)X16_B + (unsigned)ti * (NH * 16384));
        unsigned ra = 0;
        unsigned pp = 0;
        unsigned rbw = 0, pw = POLL_W;
        bool wok = false;
        const unsigned wbase = call * 256u;
        int jn = W_JC;
        int jpub = 0;
        bool p0 = false;
        const unsigned tstart = (unsigned)rt();
        fp16_saturate();
        const float wsc = exp2i(scale_exp(*w_sample(p.w, lane)));
        const char* xexpp = myws + OFF_XEXP + (call & 1u) * 1024u + (ti >> 1) * 128;
        bool xok = false;
        unsigned long long tw = 0; unsigned sp = 0;
        while (ra < NH || !wok) {
            unsigned fv = tag ^ 1u, wv = 0, xv = 0;
            const unsigned hx = ra + (lane >> 4) - 1u;
            const bool probe = lane < pw && pp + lane < (unsigned)NH;
            const bool piece = lane >= 16 && lane < 48 && hx >= (unsigned)(NH - POLL_NX) && hx < (unsigned)NH &&
                               (hx < pp || hx >= (unsigned)(NH - POLL_ND));
            const unsigned off = probe ? (pp + lane) * 16384u + 15u * 1024u + 1008u : hx * 16384u + (lane & 15) * 1024u + 1008u;
            if (!wok && lane < 16)
                asm volatile("global_load_dword %0, %1, off sc0 sc1" : "=v"(wv) : "v"(wflagp) : "memory");
            if (!xok && lane == 0)
                asm volatile("global_load_dword %0, %1, off sc0 sc1" : "=v"(xv) : "v"(xexpp) : "memory");
            if (probe || piece)
                asm volatile("global_load_dword %0, %1, %2 sc0 sc1" : "=v"(fv) : "v"(off), "s"(xsrc) : "memory");
            asm volatile("s_waitcnt vmcnt(0)" : "+v"(fv), "+v"(wv), "+v"(xv) :: "memory");
            bool act = false;
            if (!xok) {
                const unsigned xe = rfl(xv) - wbase;
                if (xe < 256u) { xok = true; lds_wr32(ldsb + CTRL + C_XEXP, xe); }
            }
            if (jpub < 8) {
                if (!p0) p0 = rfl(lds_rd32_wait(ldsb + CTRL + C_WCNT)) == 3u;
                if (p0 && jn > jpub) {
                    jpub = jn;
                    if (lane == 0) st4_sys_p(myws + OFF_WFLAG + (size_t)(tj * 16 + ti) * 128, wbase + (unsigned)jpub);
                }
            }
            if (!wok) {
                unsigned c = wv - wbase;
                c = (c <= 8u ? c : 0u) * 16u + (lane & 15);
                const unsigned nb = umin(rfl(dppmin<0x140>(dppmin<0x141>(dppmin<0x4E>(dppmin<0xB1>(c))))), (unsigned)NH);
                if (nb > rbw) { rbw = nb; lds_wr32(ldsb + CTRL + C_RDYB, nb); act = true; }
                wok = nb == (unsigned)NH;
            }
            if (ra < NH) {
                const unsigned long long m = ~__ballot((fv & 1u) == tag);
                const unsigned np = (unsigned)__builtin_ctzll(m | 0x10000ull);
                pw = (np == pw) ? umin(2u * pw, 16u) : (unsigned)POLL_W;
                const unsigned nx = (unsigned)__builtin_ctzll((m >> 16) | (1ull << 32)) >> 4;
                pp += np;
                unsigned rn = ra + nx;
                const unsigned rb = umin(pp > (unsigned)POLL_LAG ? pp - POLL_LAG : 0u, (unsigned)(NH - POLL_NX));
                rn = rn > rb ? rn : rb;
                if (!xok) rn = umin(rn, (unsigned)(NH - 1));
                if (rn > ra) { ra = rn; lds_wr32(ldsb + CTRL + C_RDYA, ra); act = true; }
                if (np) act = true;
            }
            if (jn < 8) {
                const int hw = 16 * jn + ti;
                if ((int)ra + W_LEAD >= hw || (int)((unsigned)rt() - tstart) >= (hw - W_LEAD) * W_TICK) {
                    conv_wblock(p, tj, col0, wide, hw, lane, wsc);
                    ++jn; act = true;
                }
            }
            if (act) { sp = 0; continue; }
            if (sp == 0) tw = rt();
            if ((++sp & 63u) == 0u) {
                const unsigned tv = rfl(__hip_atomic_load(tmo, __ATOMIC_RELAXED, __HIP_MEMORY_SCOPE_AGENT));
                if (tv == call || rt() - tw > DEADLINE) {
                    if (lane == 0) __hip_atomic_store(tmo, call, __ATOMIC_RELAXED, __HIP_MEMORY_SCOPE_AGENT);
                    lds_wr32(ldsb + CTRL + C_BAD, 1u);
                    lds_wr32(ldsb + CTRL + C_RDYA, NH);
                    lds_wr32(ldsb + CTRL + C_RDYB, NH);
                    break;
                }
            }
            __builtin_amdgcn_s_sleep(POLL_SLEEP);
        }
        STAMP(1);
    }
}

}

extern "C" void kernel_launch(int rank, int nranks, ncclComm_t comm,
                              hipStream_t stream,
                              void* const* d_in, const int* in_sizes, int n_in,
                              void* d_out, int out_size,
                              const int* in_dtypes, int out_dtype,
                              void* const* d_out_all,
                              void* const* d_ws_all, size_t ws_bytes) {
    using namespace agk;
    if (nranks != NR || (unsigned)rank >= (unsigned)NR || ws_bytes < WS_NEED || n_in < 2) {
        fprintf(stderr, "ag_gemm_silu: unexpected configuration\n");
        return;
    }
    static bool attr_done = false;
    if (!attr_done) {
        HIP_CHECK(hipFuncSetAttribute((const void*)ag_gemm_silu, hipFuncAttributeMaxDynamicSharedMemorySize, LDS_B));
        attr_done = true;
    }
    Params p;
    p.x = (const float*)d_in[0];
    p.w = (const float*)d_in[1];
    p.out = (float*)d_out;
    p.myws = (char*)d_ws_all[rank];
    if (COMM_SIDES == 2) {
        static const int grp[2][4] = {{0, 2, 3, 6}, {1, 4, 5, 7}};
        const int gi = (rank == 1 || rank == 4 || rank == 5 || rank == 7) ? 1 : 0;
        int pos = 0;
        for (int i = 0; i < 4; ++i) if (grp[gi][i] == rank) pos = i;
        for (int i = 0; i < 4; ++i) {
            p.dst[(COMM_SWAP ? 4 : 0) + i] = (char*)d_ws_all[grp[gi][(pos + 1 + i) & 3]];
            p.dst[(COMM_SWAP ? 0 : 4) + i] = (char*)d_ws_all[grp[gi ^ 1][(pos + i) & 3]];
        }
    } else {
        for (int i = 0; i < NR; ++i) p.dst[i] = (char*)d_ws_all[(rank + 1 + i) & 7];
    }
    p.rank = rank;
    ag_gemm_silu<<<dim3(256), dim3(NWAVE * 64), LDS_B, stream>>>(p);
    HIP_CHECK(hipGetLastError());
}
```

```cpp
#include <hip/hip_runtime.h>
#include <rccl/rccl.h>
#include <cstdio>

#define HIP_CHECK(expr) do { hipError_t _e = (expr); if (_e != hipSuccess) { \
    fprintf(stderr, "HIP error %s at %s:%d\n", hipGetErrorString(_e), __FILE__, __LINE__); \
    return; } } while (0)

#define LAS __attribute__((address_space(3)))
typedef _Float16 f16x8 __attribute__((ext_vector_type(8)));
typedef _Float16 f16x2 __attribute__((ext_vector_type(2)));
typedef float f32x4 __attribute__((ext_vector_type(4)));
typedef float f32x2 __attribute__((ext_vector_type(2)));
typedef unsigned u32x4 __attribute__((ext_vector_type(4)));
typedef unsigned u32x2 __attribute__((ext_vector_type(2)));

namespace agk {

constexpr int NR = 8;
constexpr int KD = 4096;
constexpr int NL = 2048;
constexpr int NH = 128;
constexpr int NSLOT = 6;
constexpr int WB = 9216;
constexpr int SLOT_B = 16384 + WB;
constexpr int CTRL = NSLOT * SLOT_B;
constexpr int LDS_B = CTRL + 256;
constexpr int NWAVE = 12;
constexpr int NCWG = 16;
#define COMM_P 3
#define COMM_NF 3
#define COMM_WGS 16
#define COMM_WPG 2
#define PRIO_C 1
#define PRIO_L 0
#define PRIO_P 0
#define POLL_SLEEP 2
#define POLL_LAG 2
#define POLL_NX 8
#define W_JC 1
#define W_LEAD 20
#define W_TICK 55
#define POLL_W 2
#define POLL_ND 2
constexpr int CWPG = COMM_WPG;
constexpr int NCW = COMM_WGS * COMM_WPG;
constexpr int NITEM = NH * 32;

constexpr int C_FULL = 0;
constexpr int C_RDYA = 12;
constexpr int C_RDYB = 16;
constexpr int C_BAD = 20;
constexpr int C_WCNT = 24;
constexpr int C_XEXP = 28;
constexpr int C_FREE = 64;

constexpr size_t OFF_CNT = 0;
constexpr size_t OFF_TMO = 32768;
constexpr size_t OFF_ICTR = 65536;
constexpr size_t OFF_XEXP = 98304;
constexpr size_t OFF_WFLAG = 131072;
constexpr size_t OFF_DBG = 262144;
constexpr size_t OFF_X16 = 1ull << 20;
constexpr size_t X16_B = 32ull << 20;
constexpr size_t OFF_W16 = OFF_X16 + 2 * X16_B;
constexpr size_t WS_NEED = OFF_W16 + (size_t)15 * NH * WB;
constexpr unsigned long long DEADLINE = 50000000ull;

struct Params {
    const float* x;
    const float* w;
    float* out;
    char* myws;
    char* dst[NR];
    int rank;
};
#define DBGRET(bit) do { } while (0)

template <int V> struct IC { static constexpr int value = V; };
template <int N, int I = 0, class F> __device__ __forceinline__ void static_for(F&& f) {
    if constexpr (I < N) { f(IC<I>{}); static_for<N, I + 1>(f); }
}

__device__ __forceinline__ unsigned long long rt() {
    unsigned long long t;
    asm volatile("s_memrealtime %0\n\ts_waitcnt lgkmcnt(0)" : "=s"(t) :: "memory");
    return t;
}

__device__ __forceinline__ unsigned pk16(float a, float b) {
    f32x2 v = {a, b};
    f16x2 h = __builtin_convertvector(v, f16x2);
    return __builtin_bit_cast(unsigned, h);
}

__device__ __forceinline__ u32x4 cvt_tag(f32x4 u0, f32x4 u1, unsigned tag) {
    const unsigned rn = pk16(u0.x, u0.y);
    unsigned t = __builtin_bit_cast(unsigned, __builtin_amdgcn_cvt_pkrtz(u0.x, 0.0f));
    t += (t ^ tag) & 1u;
    u32x4 o;
    o.x = (rn & 0xffff0000u) | (t & 0xffffu);
    o.y = pk16(u0.z, u0.w);
    o.z = pk16(u1.x, u1.y); o.w = pk16(u1.z, u1.w);
    return o;
}

#define POL_COMM "sc0 sc1"
#define POL_OUT "sc0 sc1"
#define POL_DMA "sc0 sc1"
__device__ __forceinline__ void st16_sys(const char* sbase, unsigned voff, u32x4 v) {
    asm volatile("global_store_dwordx4 %0, %1, %2 " POL_COMM "\n\ts_nop 1" :: "v"(voff), "v"(v), "s"(sbase) : "memory");
}
__device__ __forceinline__ void st16_out(const char* p, u32x4 v) {
    asm volatile("global_store_dwordx4 %0, %1, off " POL_OUT "\n\ts_nop 1" :: "v"(p), "v"(v) : "memory");
}
__device__ __forceinline__ void st4_out(const char* p, unsigned v) {
    asm volatile("global_store_dword %0, %1, off " POL_OUT :: "v"(p), "v"(v) : "memory");
}
__device__ __forceinline__ void st16_sys_p(const char* p, u32x4 v) {
    asm volatile("global_store_dwordx4 %0, %1, off sc0 sc1\n\ts_nop 1" :: "v"(p), "v"(v) : "memory");
}
__device__ __forceinline__ void st4_sys_p(const char* p, unsigned v) {
    asm volatile("global_store_dword %0, %1, off sc0 sc1" :: "v"(p), "v"(v) : "memory");
}

#define VMW(n) asm volatile("s_waitcnt vmcnt(" #n ")" ::: "memory")

__device__ __forceinline__ unsigned vm_outstanding() {
    unsigned v;
    asm volatile("s_getreg_b32 %0, hwreg(HW_REG_IB_STS)" : "=s"(v) :: "memory");
    return (v & 15u) | ((v >> 18) & 48u);
}

__device__ __forceinline__ void glds8(const char* sbase, unsigned voff, unsigned voff2, unsigned ldst) {
    unsigned keep;
    asm volatile(
        "s_mov_b32 %0, m0\n\t"
        "s_mov_b32 m0, %4\n\t"
        "s_nop 0\n\t"
        "global_load_lds_dwordx4 %1, %3 " POL_DMA "\n\t"
        "global_load_lds_dwordx4 %1, %3 offset:1024 " POL_DMA "\n\t"
        "global_load_lds_dwordx4 %1, %3 offset:2048 " POL_DMA "\n\t"
        "global_load_lds_dwordx4 %1, %3 offset:3072 " POL_DMA "\n\t"
        "s_add_u32 m0, %4, 4096\n\t"
        "s_nop 0\n\t"
        "global_load_lds_dwordx4 %2, %3 " POL_DMA "\n\t"
        "global_load_lds_dwordx4 %2, %3 offset:1024 " POL_DMA "\n\t"
        "global_load_lds_dwordx4 %2, %3 offset:2048 " POL_DMA "\n\t"
        "global_load_lds_dwordx4 %2, %3 offset:3072 " POL_DMA "\n\t"
        "s_mov_b32 m0, %0"
        : "=&s"(keep) : "v"(voff), "v"(voff2), "s"(sbase), "s"(ldst) : "memory", "scc");
}
__device__ __forceinline__ void glds1(const char* sbase, unsigned voff, unsigned ldst) {
    unsigned keep;
    asm volatile(
        "s_mov_b32 %0, m0\n\t"
        "s_mov_b32 m0, %3\n\t"
        "s_nop 0\n\t"
        "global_load_lds_dwordx4 %1, %2 " POL_DMA "\n\t"
        "s_mov_b32 m0, %0"
        : "=&s"(keep) : "v"(voff), "s"(sbase), "s"(ldst) : "memory");
}

__device__ __forceinline__ void lds_wr32(unsigned addr, unsigned v) {
    asm volatile("s_mov_b64 exec, 1\n\tds_write_b32 %0, %1\n\ts_mov_b64 exec, -1" :: "v"(addr), "v"(v) : "memory");
}
__device__ __forceinline__ u32x4 lds_rd128_wait(unsigned addr) {
    u32x4 v;
    asm volatile("ds_read_b128 %0, %1\n\ts_waitcnt lgkmcnt(0)" : "=&v"(v) : "v"(addr) : "memory");
    return v;
}
__device__ __forceinline__ unsigned lds_rd32_wait(unsigned addr) {
    unsigned v;
    asm volatile("ds_read_b32 %0, %1\n\ts_waitcnt lgkmcnt(0)" : "=&v"(v) : "v"(addr) : "memory");
    return v;
}
__device__ __forceinline__ unsigned rfl(unsigned v) { return __builtin_amdgcn_readfirstlane(v); }
__device__ __forceinline__ unsigned umin(unsigned a, unsigned b) { return a < b ? a : b; }
template <int CTL> __device__ __forceinline__ unsigned dppmin(unsigned d) {
    return umin(d, (unsigned)__builtin_amdgcn_update_dpp((int)d, (int)d, CTL, 0xf, 0xf, false));
}

template <int CTL> __device__ __forceinline__ unsigned dppmax(unsigned d) {
    const unsigned o = (unsigned)__builtin_amdgcn_update_dpp((int)d, (int)d, CTL, 0xf, 0xf, false);
    return d > o ? d : o;
}
__device__ __forceinline__ int scale_exp(f32x4 v) {
    const u32x4 u = __builtin_bit_cast(u32x4, v) & 0x7fffffffu;
    unsigned m = u.x > u.y ? u.x : u.y, n = u.z > u.w ? u.z : u.w;
    m = m > n ? m : n;
    m = dppmax<0x140>(dppmax<0x141>(dppmax<0x4E>(dppmax<0xB1>(m))));
    const unsigned a = __builtin_amdgcn_readlane(m, 0), b = __builtin_amdgcn_readlane(m, 16);
    const unsigned c = __builtin_amdgcn_readlane(m, 32), d = __builtin_amdgcn_readlane(m, 48);
    const unsigned ab = a > b ? a : b, cd = c > d ? c : d;
    const int e = (int)((ab > cd ? ab : cd) >> 23);
    if (e == 0 || e == 255) return 0;
    const int sx = 126 - e;
    return sx < -60 ? -60 : (sx > 60 ? 60 : sx);
}
__device__ __forceinline__ float exp2i(int e) { return __builtin_bit_cast(float, (unsigned)(127 + e) << 23); }
__device__ __forceinline__ const f32x4* x_sample(const float* x, int lane) {
    return (const f32x4*)(x + (size_t)(lane * 8 + 3) * KD + ((lane * 260) & (KD - 1)));
}
__device__ __forceinline__ const f32x4* w_sample(const float* w, int lane) {
    return (const f32x4*)(w + (size_t)(lane * 64 + 17) * NL + ((lane * 132) & (NL - 1)));
}
__device__ __forceinline__ void fp16_saturate() {
    asm volatile("s_setreg_imm32_b32 hwreg(HW_REG_MODE, 23, 1), 1" ::: "memory");
}

#define STAMP(i) do { } while (0)

#define SB0() __builtin_amdgcn_sched_barrier(0)

struct KArgs { unsigned va0, vb0, vfull, vfree, l16, tag, bad; const char* xsrc; };
#define MF(x) x
#define TG(x) x
#define NTG(x)
__device__ __forceinline__ void kloop(f32x4 (&acc)[4][4], const KArgs& k, unsigned& nfix) {
    asm volatile(
        "v_mov_b32 v162, 1\n\t"
        "s_mov_b32 s88, 0\n\t"
        "s_mov_b32 s89, 1\n\t"
        "s_branch LOADX_%=\n\t"
        ".p2align 6\n\t"
        "TOPX_%=:\n\t"
        "s_waitcnt lgkmcnt(4)\n\t"
        MF("v_mfma_f32_16x16x32_f16 %[c00], v[104:107], v[136:139], %[c00]\n\t")
        "v_add_u32 v157, s88, %[va0]\n\t"
        "v_add_u32 v158, s88, %[vb0]\n\t"
        MF("v_mfma_f32_16x16x32_f16 %[c10], v[108:111], v[136:139], %[c10]\n\t")
        "ds_read_b128 v[120:123], v157\n\t"
        "ds_read_b128 v[124:127], v157 offset:1024\n\t"
        MF("v_mfma_f32_16x16x32_f16 %[c20], v[112:115], v[136:139], %[c20]\n\t")
        "ds_read_b128 v[128:131], v157 offset:2048\n\t"
        "ds_read_b128 v[132:135], v157 offset:3072\n\t"
        MF("v_mfma_f32_16x16x32_f16 %[c30], v[116:119], v[136:139], %[c30]\n\t")
        "ds_read_b32 v156, %[vfull]\n\t"
        "ds_read_b128 v[136:139], v158\n\t"
        "s_waitcnt lgkmcnt(9)\n\t"
        MF("v_mfma_f32_16x16x32_f16 %[c01], v[104:107], v[140:143], %[c01]\n\t")
        "s_add_u32 s88, s88, %[slotb]\n\t"
        "s_cmp_eq_u32 s88, %[ringb]\n\t"
        "s_cselect_b32 s88, 0, s88\n\t"
        "s_add_u32 s89, s89, 1\n\t"
        MF("v_mfma_f32_16x16x32_f16 %[c11], v[108:111], v[140:143], %[c11]\n\t")
        MF("v_mfma_f32_16x16x32_f16 %[c21], v[112:115], v[140:143], %[c21]\n\t")
        MF("v_mfma_f32_16x16x32_f16 %[c31], v[116:119], v[140:143], %[c31]\n\t")
        "ds_read_b128 v[140:143], v158 offset:1024\n\t"
        "s_waitcnt lgkmcnt(9)\n\t"
        MF("v_mfma_f32_16x16x32_f16 %[c02], v[104:107], v[144:147], %[c02]\n\t")
        MF("v_mfma_f32_16x16x32_f16 %[c12], v[108:111], v[144:147], %[c12]\n\t")
        MF("v_mfma_f32_16x16x32_f16 %[c22], v[112:115], v[144:147], %[c22]\n\t")
        MF("v_mfma_f32_16x16x32_f16 %[c32], v[116:119], v[144:147], %[c32]\n\t")
        "ds_read_b128 v[144:147], v158 offset:2048\n\t"
        "s_waitcnt lgkmcnt(3)\n\t"
        MF("v_mfma_f32_16x16x32_f16 %[c03], v[104:107], v[148:151], %[c03]\n\t")
        "v_bitop3_b32 v159, v120, v124, %[tag] bitop3:0x7e\n\t"
        "v_bitop3_b32 v160, v128, v132, %[tag] bitop3:0x7e\n\t"
        MF("v_mfma_f32_16x16x32_f16 %[c13], v[108:111], v[148:151], %[c13]\n\t")
        "v_bitop3_b32 v159, v159, 1, v160 bitop3:0xc8\n\t"
        "v_cmp_gt_u32_e64 s[86:87], s89, v156\n\t"
        MF("v_mfma_f32_16x16x32_f16 %[c23], v[112:115], v[148:151], %[c23]\n\t")
        TG("v_cmp_ne_u32_e64 s[84:85], 0, v159\n\t")
        NTG("s_mov_b64 s[84:85], 0\n\t")
        MF("v_mfma_f32_16x16x32_f16 %[c33], v[116:119], v[148:151], %[c33]\n\t")
        "ds_read_b128 v[148:151], v158 offset:3072\n\t"
        "s_mov_b64 exec, 1\n\t"
        "ds_add_u32 %[vfree], v162\n\t"
        "s_mov_b64 exec, -1\n\t"
        "s_or_b64 s[90:91], s[84:85], s[86:87]\n\t"
        "s_cbranch_scc1 SLOWY_%=\n\t"
        "TOPY_%=:\n\t"
        "s_waitcnt lgkmcnt(4)\n\t"
        MF("v_mfma_f32_16x16x32_f16 %[c00], v[120:123], v[136:139], %[c00]\n\t")
        "v_add_u32 v157, s88, %[va0]\n\t"
        "v_add_u32 v158, s88, %[vb0]\n\t"
        MF("v_mfma_f32_16x16x32_f16 %[c10], v[124:127], v[136:139], %[c10]\n\t")
        "ds_read_b128 v[104:107], v157\n\t"
        "ds_read_b128 v[108:111], v157 offset:1024\n\t"
        MF("v_mfma_f32_16x16x32_f16 %[c20], v[128:131], v[136:139], %[c20]\n\t")
        "ds_read_b128 v[112:115], v157 offset:2048\n\t"
        "ds_read_b128 v[116:119], v157 offset:3072\n\t"
        MF("v_mfma_f32_16x16x32_f16 %[c30], v[132:135], v[136:139], %[c30]\n\t")
        "ds_read_b32 v156, %[vfull]\n\t"
        "ds_read_b128 v[136:139], v158\n\t"
        "s_waitcnt lgkmcnt(9)\n\t"
        MF("v_mfma_f32_16x16x32_f16 %[c01], v[120:123], v[140:143], %[c01]\n\t")
        "s_add_u32 s88, s88, %[slotb]\n\t"
        "s_cmp_eq_u32 s88, %[ringb]\n\t"
        "s_cselect_b32 s88, 0, s88\n\t"
        "s_add_u32 s89, s89, 1\n\t"
        MF("v_mfma_f32_16x16x32_f16 %[c11], v[124:127], v[140:143], %[c11]\n\t")
        MF("v_mfma_f32_16x16x32_f16 %[c21], v[128:131], v[140:143], %[c21]\n\t")
        MF("v_mfma_f32_16x16x32_f16 %[c31], v[132:135], v[140:143], %[c31]\n\t")
        "ds_read_b128 v[140:143], v158 offset:1024\n\t"
        "s_waitcnt lgkmcnt(9)\n\t"
        MF("v_mfma_f32_16x16x32_f16 %[c02], v[120:123], v[144:147], %[c02]\n\t")
        MF("v_mfma_f32_16x16x32_f16 %[c12], v[124:127], v[144:147], %[c12]\n\t")
        MF("v_mfma_f32_16x16x32_f16 %[c22], v[128:131], v[144:147], %[c22]\n\t")
        MF("v_mfma_f32_16x16x32_f16 %[c32], v[132:135], v[144:147], %[c32]\n\t")
        "ds_read_b128 v[144:147], v158 offset:2048\n\t"
        "s_waitcnt lgkmcnt(3)\n\t"
        MF("v_mfma_f32_16x16x32_f16 %[c03], v[120:123], v[148:151], %[c03]\n\t")
        "v_bitop3_b32 v159, v104, v108, %[tag] bitop3:0x7e\n\t"
        "v_bitop3_b32 v160, v112, v116, %[tag] bitop3:0x7e\n\t"
        MF("v_mfma_f32_16x16x32_f16 %[c13], v[124:127], v[148:151], %[c13]\n\t")
        "v_bitop3_b32 v159, v159, 1, v160 bitop3:0xc8\n\t"
        "v_cmp_gt_u32_e64 s[86:87], s89, v156\n\t"
        MF("v_mfma_f32_16x16x32_f16 %[c23], v[128:131], v[148:151], %[c23]\n\t")
        TG("v_cmp_ne_u32_e64 s[84:85], 0, v159\n\t")
        NTG("s_mov_b64 s[84:85], 0\n\t")
        MF("v_mfma_f32_16x16x32_f16 %[c33], v[132:135], v[148:151], %[c33]\n\t")
        "ds_read_b128 v[148:151], v158 offset:3072\n\t"
        "s_mov_b64 exec, 1\n\t"
        "ds_add_u32 %[vfree], v162\n\t"
        "s_mov_b64 exec, -1\n\t"
        "s_or_b64 s[90:91], s[84:85], s[86:87]\n\t"
        "s_cbranch_scc1 SLOWX_%=\n\t"
        "s_waitcnt lgkmcnt(4)\n\t"
        MF("v_mfma_f32_16x16x32_f16 %[c00], v[104:107], v[136:139], %[c00]\n\t")
        "v_add_u32 v157, s88, %[va0]\n\t"
        "v_add_u32 v158, s88, %[vb0]\n\t"
        MF("v_mfma_f32_16x16x32_f16 %[c10], v[108:111], v[136:139], %[c10]\n\t")
        "ds_read_b128 v[120:123], v157\n\t"
        "ds_read_b128 v[124:127], v157 offset:1024\n\t"
        MF("v_mfma_f32_16x16x32_f16 %[c20], v[112:115], v[136:139], %[c20]\n\t")
        "ds_read_b128 v[128:131], v157 offset:2048\n\t"
        "ds_read_b128 v[132:135], v157 offset:3072\n\t"
        MF("v_mfma_f32_16x16x32_f16 %[c30], v[116:119], v[136:139], %[c30]\n\t")
        "ds_read_b32 v156, %[vfull]\n\t"
        "ds_read_b128 v[136:139], v158\n\t"
        "s_waitcnt lgkmcnt(9)\n\t"
        MF("v_mfma_f32_16x16x32_f16 %[c01], v[104:107], v[140:143], %[c01]\n\t")
        "s_add_u32 s88, s88, %[slotb]\n\t"
        "s_cmp_eq_u32 s88, %[ringb]\n\t"
        "s_cselect_b32 s88, 0, s88\n\t"
        "s_add_u32 s89, s89, 1\n\t"
        MF("v_mfma_f32_16x16x32_f16 %[c11], v[108:111], v[140:143], %[c11]\n\t")
        MF("v_mfma_f32_16x16x32_f16 %[c21], v[112:115], v[140:143], %[c21]\n\t")
        MF("v_mfma_f32_16x16x32_f16 %[c31], v[116:119], v[140:143], %[c31]\n\t")
        "ds_read_b128 v[140:143], v158 offset:1024\n\t"
        "s_waitcnt lgkmcnt(9)\n\t"
        MF("v_mfma_f32_16x16x32_f16 %[c02], v[104:107], v[144:147], %[c02]\n\t")
        MF("v_mfma_f32_16x16x32_f16 %[c12], v[108:111], v[144:147], %[c12]\n\t")
        MF("v_mfma_f32_16x16x32_f16 %[c22], v[112:115], v[144:147], %[c22]\n\t")
        MF("v_mfma_f32_16x16x32_f16 %[c32], v[116:119], v[144:147], %[c32]\n\t")
        "ds_read_b128 v[144:147], v158 offset:2048\n\t"
        "s_waitcnt lgkmcnt(3)\n\t"
        MF("v_mfma_f32_16x16x32_f16 %[c03], v[104:107], v[148:151], %[c03]\n\t")
        "v_bitop3_b32 v159, v120, v124, %[tag] bitop3:0x7e\n\t"
        "v_bitop3_b32 v160, v128, v132, %[tag] bitop3:0x7e\n\t"
        MF("v_mfma_f32_16x16x32_f16 %[c13], v[108:111], v[148:151], %[c13]\n\t")
        "v_bitop3_b32 v159, v159, 1, v160 bitop3:0xc8\n\t"
        "v_cmp_gt_u32_e64 s[86:87], s89, v156\n\t"
        MF("v_mfma_f32_16x16x32_f16 %[c23], v[112:115], v[148:151], %[c23]\n\t")
        TG("v_cmp_ne_u32_e64 s[84:85], 0, v159\n\t")
        NTG("s_mov_b64 s[84:85], 0\n\t")
        MF("v_mfma_f32_16x16x32_f16 %[c33], v[116:119], v[148:151], %[c33]\n\t")
        "ds_read_b128 v[148:151], v158 offset:3072\n\t"
        "s_mov_b64 exec, 1\n\t"
        "ds_add_u32 %[vfree], v162\n\t"
        "s_mov_b64 exec, -1\n\t"
        "s_or_b64 s[90:91], s[84:85], s[86:87]\n\t"
        "s_cbranch_scc1 SLOWY_%=\n\t"
        "s_waitcnt lgkmcnt(4)\n\t"
        MF("v_mfma_f32_16x16x32_f16 %[c00], v[120:123], v[136:139], %[c00]\n\t")
        "v_add_u32 v157, s88, %[va0]\n\t"
        "v_add_u32 v158, s88, %[vb0]\n\t"
        MF("v_mfma_f32_16x16x32_f16 %[c10], v[124:127], v[136:139], %[c10]\n\t")
        "ds_read_b128 v[104:107], v157\n\t"
        "ds_read_b128 v[108:111], v157 offset:1024\n\t"
        MF("v_mfma_f32_16x16x32_f16 %[c20], v[128:131], v[136:139], %[c20]\n\t")
        "ds_read_b128 v[112:115], v157 offset:2048\n\t"
        "ds_read_b128 v[116:119], v157 offset:3072\n\t"
        MF("v_mfma_f32_16x16x32_f16 %[c30], v[132:135], v[136:139], %[c30]\n\t")
        "ds_read_b32 v156, %[vfull]\n\t"
        "ds_read_b128 v[136:139], v158\n\t"
        "s_waitcnt lgkmcnt(9)\n\t"
        MF("v_mfma_f32_16x16x32_f16 %[c01], v[120:123], v[140:143], %[c01]\n\t")
        "s_add_u32 s88, s88, %[slotb]\n\t"
        "s_cmp_eq_u32 s88, %[ringb]\n\t"
        "s_cselect_b32 s88, 0, s88\n\t"
        "s_add_u32 s89, s89, 1\n\t"
        MF("v_mfma_f32_16x16x32_f16 %[c11], v[124:127], v[140:143], %[c11]\n\t")
        MF("v_mfma_f32_16x16x32_f16 %[c21], v[128:131], v[140:143], %[c21]\n\t")
        MF("v_mfma_f32_16x16x32_f16 %[c31], v[132:135], v[140:143], %[c31]\n\t")
        "ds_read_b128 v[140:143], v158 offset:1024\n\t"
        "s_waitcnt lgkmcnt(9)\n\t"
        MF("v_mfma_f32_16x16x32_f16 %[c02], v[120:123], v[144:147], %[c02]\n\t")
        MF("v_mfma_f32_16x16x32_f16 %[c12], v[124:127], v[144:147], %[c12]\n\t")
        MF("v_mfma_f32_16x16x32_f16 %[c22], v[128:131], v[144:147], %[c22]\n\t")
        MF("v_mfma_f32_16x16x32_f16 %[c32], v[132:135], v[144:147], %[c32]\n\t")
        "ds_read_b128 v[144:147], v158 offset:2048\n\t"
        "s_waitcnt lgkmcnt(3)\n\t"
        MF("v_mfma_f32_16x16x32_f16 %[c03], v[120:123], v[148:151], %[c03]\n\t")
        "v_bitop3_b32 v159, v104, v108, %[tag] bitop3:0x7e\n\t"
        "v_bitop3_b32 v160, v112, v116, %[tag] bitop3:0x7e\n\t"
        MF("v_mfma_f32_16x16x32_f16 %[c13], v[124:127], v[148:151], %[c13]\n\t")
        "v_bitop3_b32 v159, v159, 1, v160 bitop3:0xc8\n\t"
        "v_cmp_gt_u32_e64 s[86:87], s89, v156\n\t"
        MF("v_mfma_f32_16x16x32_f16 %[c23], v[128:131], v[148:151], %[c23]\n\t")
        TG("v_cmp_ne_u32_e64 s[84:85], 0, v159\n\t")
        NTG("s_mov_b64 s[84:85], 0\n\t")
        MF("v_mfma_f32_16x16x32_f16 %[c33], v[132:135], v[148:151], %[c33]\n\t")
        "ds_read_b128 v[148:151], v158 offset:3072\n\t"
        "s_mov_b64 exec, 1\n\t"
        "ds_add_u32 %[vfree], v162\n\t"
        "s_mov_b64 exec, -1\n\t"
        "s_or_b64 s[90:91], s[84:85], s[86:87]\n\t"
        "s_cbranch_scc0 TOPX_%=\n\t"
        "SLOWX_%=:\n\t"
        "s_cmp_eq_u64 s[84:85], 0\n\t"
        "s_cbranch_scc1 NOFIXX_%=\n\t"
        "s_add_u32 %[nfix], %[nfix], 1\n\t"
        "v_mov_b32 v159, %[bad]\n\t"
        "ds_read_b32 v160, v159\n\t"
        "s_sub_u32 s92, s89, 2\n\t"
        "s_lshl_b32 s92, s92, 14\n\t"
        "v_add_u32 v161, s92, %[l16]\n\t"
        "s_memrealtime s[90:91]\n\t"
        "s_waitcnt lgkmcnt(0)\n\t"
        "v_cmp_ne_u32_e64 s[92:93], 0, v160\n\t"
        "s_cmp_lg_u64 s[92:93], 0\n\t"
        "s_cbranch_scc1 NOFIXX_%=\n\t"
        "FIXLX_%=:\n\t"
        "global_load_dwordx4 v[104:107], v161, %[xsrc] offset:0 sc0 sc1\n\t"
        "global_load_dwordx4 v[108:111], v161, %[xsrc] offset:1024 sc0 sc1\n\t"
        "global_load_dwordx4 v[112:115], v161, %[xsrc] offset:2048 sc0 sc1\n\t"
        "global_load_dwordx4 v[116:119], v161, %[xsrc] offset:3072 sc0 sc1\n\t"
        "s_waitcnt vmcnt(0)\n\t"
        "v_bitop3_b32 v159, v104, v108, %[tag] bitop3:0x7e\n\t"
        "v_bitop3_b32 v160, v112, v116, %[tag] bitop3:0x7e\n\t"
        "v_bitop3_b32 v159, v159, 1, v160 bitop3:0xc8\n\t"
        "v_cmp_ne_u32_e64 s[92:93], 0, v159\n\t"
        "s_cmp_eq_u64 s[92:93], 0\n\t"
        "s_cbranch_scc1 NOFIXX_%=\n\t"
        "s_sleep 1\n\t"
        "s_memrealtime s[92:93]\n\t"
        "s_waitcnt lgkmcnt(0)\n\t"
        "s_sub_u32 s92, s92, s90\n\t"
        "s_cmp_lt_u32 s92, %[dl]\n\t"
        "s_cbranch_scc1 FIXLX_%=\n\t"
        "v_mov_b32 v159, %[bad]\n\t"
        "ds_write_b32 v159, v159\n\t"
        "NOFIXX_%=:\n\t"
        "s_cmp_gt_u32 s89, %[nh]\n\t"
        "s_cbranch_scc1 PLAINX_%=\n\t"
        "ds_read_b32 v156, %[vfull]\n\t"
        "s_waitcnt lgkmcnt(0)\n\t"
        "v_cmp_gt_u32_e64 s[86:87], s89, v156\n\t"
        "s_cmp_eq_u64 s[86:87], 0\n\t"
        "s_cbranch_scc1 TOPX_%=\n\t"
        "PLAINX_%=:\n\t"
        "s_waitcnt lgkmcnt(0)\n\t"
        MF("v_mfma_f32_16x16x32_f16 %[c00], v[104:107], v[136:139], %[c00]\n\t")
        MF("v_mfma_f32_16x16x32_f16 %[c10], v[108:111], v[136:139], %[c10]\n\t")
        MF("v_mfma_f32_16x16x32_f16 %[c20], v[112:115], v[136:139], %[c20]\n\t")
        MF("v_mfma_f32_16x16x32_f16 %[c30], v[116:119], v[136:139], %[c30]\n\t")
        MF("v_mfma_f32_16x16x32_f16 %[c01], v[104:107], v[140:143], %[c01]\n\t")
        MF("v_mfma_f32_16x16x32_f16 %[c11], v[108:111], v[140:143], %[c11]\n\t")
        MF("v_mfma_f32_16x16x32_f16 %[c21], v[112:115], v[140:143], %[c21]\n\t")
        MF("v_mfma_f32_16x16x32_f16 %[c31], v[116:119], v[140:143], %[c31]\n\t")
        MF("v_mfma_f32_16x16x32_f16 %[c02], v[104:107], v[144:147], %[c02]\n\t")
        MF("v_mfma_f32_16x16x32_f16 %[c12], v[108:111], v[144:147], %[c12]\n\t")
        MF("v_mfma_f32_16x16x32_f16 %[c22], v[112:115], v[144:147], %[c22]\n\t")
        MF("v_mfma_f32_16x16x32_f16 %[c32], v[116:119], v[144:147], %[c32]\n\t")
        MF("v_mfma_f32_16x16x32_f16 %[c03], v[104:107], v[148:151], %[c03]\n\t")
        MF("v_mfma_f32_16x16x32_f16 %[c13], v[108:111], v[148:151], %[c13]\n\t")
        MF("v_mfma_f32_16x16x32_f16 %[c23], v[112:115], v[148:151], %[c23]\n\t")
        MF("v_mfma_f32_16x16x32_f16 %[c33], v[116:119], v[148:151], %[c33]\n\t")
        "s_cmp_gt_u32 s89, %[nh]\n\t"
        "s_cbranch_scc1 DONE_%=\n\t"
        "LOADX_%=:\n\t"
        "s_mov_b32 s94, 0\n\t"
        "v_add_u32 v157, s88, %[va0]\n\t"
        "v_add_u32 v158, s88, %[vb0]\n\t"
        "SPINX_%=:\n\t"
        "ds_read_b32 v156, %[vfull]\n\t"
        "s_waitcnt lgkmcnt(0)\n\t"
        "v_cmp_gt_u32_e64 s[86:87], s89, v156\n\t"
        "s_cmp_eq_u64 s[86:87], 0\n\t"
        "s_cbranch_scc1 GOX_%=\n\t"
        "s_sleep 1\n\t"
        "s_add_u32 s94, s94, 1\n\t"
        "s_and_b32 s92, s94, 0x3ff\n\t"
        "s_cbranch_scc1 SPINX_%=\n\t"
        "s_memrealtime s[92:93]\n\t"
        "s_waitcnt lgkmcnt(0)\n\t"
        "s_cmp_eq_u32 s94, 0x400\n\t"
        "s_cselect_b32 s90, s92, s90\n\t"
        "s_sub_u32 s92, s92, s90\n\t"
        "s_cmp_lt_u32 s92, %[dl3]\n\t"
        "s_cbranch_scc1 SPINX_%=\n\t"
        "v_mov_b32 v159, %[bad]\n\t"
        "ds_write_b32 v159, v159\n\t"
        "GOX_%=:\n\t"
        "ds_read_b128 v[104:107], v157 offset:0\n\t"
        "ds_read_b128 v[108:111], v157 offset:1024\n\t"
        "ds_read_b128 v[112:115], v157 offset:2048\n\t"
        "ds_read_b128 v[116:119], v157 offset:3072\n\t"
        "ds_read_b32 v156, %[vfull]\n\t"
        "ds_read_b128 v[136:139], v158 offset:0\n\t"
        "ds_read_b128 v[140:143], v158 offset:1024\n\t"
        "ds_read_b128 v[144:147], v158 offset:2048\n\t"
        "ds_read_b128 v[148:151], v158 offset:3072\n\t"
        "s_mov_b64 exec, 1\n\t"
        "ds_add_u32 %[vfree], v162\n\t"
        "s_mov_b64 exec, -1\n\t"
        "s_add_u32 s88, s88, %[slotb]\n\t"
        "s_cmp_eq_u32 s88, %[ringb]\n\t"
        "s_cselect_b32 s88, 0, s88\n\t"
        "s_add_u32 s89, s89, 1\n\t"
        "s_waitcnt lgkmcnt(5)\n\t"
        "v_bitop3_b32 v159, v104, v108, %[tag] bitop3:0x7e\n\t"
        "v_bitop3_b32 v160, v112, v116, %[tag] bitop3:0x7e\n\t"
        "v_bitop3_b32 v159, v159, 1, v160 bitop3:0xc8\n\t"
        "v_cmp_gt_u32_e64 s[86:87], s89, v156\n\t"
        TG("v_cmp_ne_u32_e64 s[84:85], 0, v159\n\t")
        NTG("s_mov_b64 s[84:85], 0\n\t")
        "s_or_b64 s[90:91], s[84:85], s[86:87]\n\t"
        "s_cbranch_scc0 TOPX_%=\n\t"
        "s_branch SLOWX_%=\n\t"
        "SLOWY_%=:\n\t"
        "s_cmp_eq_u64 s[84:85], 0\n\t"
        "s_cbranch_scc1 NOFIXY_%=\n\t"
        "s_add_u32 %[nfix], %[nfix], 1\n\t"
        "v_mov_b32 v159, %[bad]\n\t"
        "ds_read_b32 v160, v159\n\t"
        "s_sub_u32 s92, s89, 2\n\t"
        "s_lshl_b32 s92, s92, 14\n\t"
        "v_add_u32 v161, s92, %[l16]\n\t"
        "s_memrealtime s[90:91]\n\t"
        "s_waitcnt lgkmcnt(0)\n\t"
        "v_cmp_ne_u32_e64 s[92:93], 0, v160\n\t"
        "s_cmp_lg_u64 s[92:93], 0\n\t"
        "s_cbranch_scc1 NOFIXY_%=\n\t"
        "FIXLY_%=:\n\t"
        "global_load_dwordx4 v[120:123], v161, %[xsrc] offset:0 sc0 sc1\n\t"
        "global_load_dwordx4 v[124:127], v161, %[xsrc] offset:1024 sc0 sc1\n\t"
        "global_load_dwordx4 v[128:131], v161, %[xsrc] offset:2048 sc0 sc1\n\t"
        "global_load_dwordx4 v[132:135], v161, %[xsrc] offset:3072 sc0 sc1\n\t"
        "s_waitcnt vmcnt(0)\n\t"
        "v_bitop3_b32 v159, v120, v124, %[tag] bitop3:0x7e\n\t"
        "v_bitop3_b32 v160, v128, v132, %[tag] bitop3:0x7e\n\t"
        "v_bitop3_b32 v159, v159, 1, v160 bitop3:0xc8\n\t"
        "v_cmp_ne_u32_e64 s[92:93], 0, v159\n\t"
        "s_cmp_eq_u64 s[92:93], 0\n\t"
        "s_cbranch_scc1 NOFIXY_%=\n\t"
        "s_sleep 1\n\t"
        "s_memrealtime s[92:93]\n\t"
        "s_waitcnt lgkmcnt(0)\n\t"
        "s_sub_u32 s92, s92, s90\n\t"
        "s_cmp_lt_u32 s92, %[dl]\n\t"
        "s_cbranch_scc1 FIXLY_%=\n\t"
        "v_mov_b32 v159, %[bad]\n\t"
        "ds_write_b32 v159, v159\n\t"
        "NOFIXY_%=:\n\t"
        "s_cmp_gt_u32 s89, %[nh]\n\t"
        "s_cbranch_scc1 PLAINY_%=\n\t"
        "ds_read_b32 v156, %[vfull]\n\t"
        "s_waitcnt lgkmcnt(0)\n\t"
        "v_cmp_gt_u32_e64 s[86:87], s89, v156\n\t"
        "s_cmp_eq_u64 s[86:87], 0\n\t"
        "s_cbranch_scc1 TOPY_%=\n\t"
        "PLAINY_%=:\n\t"
        "s_waitcnt lgkmcnt(0)\n\t"
        MF("v_mfma_f32_16x16x32_f16 %[c00], v[120:123], v[136:139], %[c00]\n\t")
        MF("v_mfma_f32_16x16x32_f16 %[c10], v[124:127], v[136:139], %[c10]\n\t")
        MF("v_mfma_f32_16x16x32_f16 %[c20], v[128:131], v[136:139], %[c20]\n\t")
        MF("v_mfma_f32_16x16x32_f16 %[c30], v[132:135], v[136:139], %[c30]\n\t")
        MF("v_mfma_f32_16x16x32_f16 %[c01], v[120:123], v[140:143], %[c01]\n\t")
        MF("v_mfma_f32_16x16x32_f16 %[c11], v[124:127], v[140:143], %[c11]\n\t")
        MF("v_mfma_f32_16x16x32_f16 %[c21], v[128:131], v[140:143], %[c21]\n\t")
        MF("v_mfma_f32_16x16x32_f16 %[c31], v[132:135], v[140:143], %[c31]\n\t")
        MF("v_mfma_f32_16x16x32_f16 %[c02], v[120:123], v[144:147], %[c02]\n\t")
        MF("v_mfma_f32_16x16x32_f16 %[c12], v[124:127], v[144:147], %[c12]\n\t")
        MF("v_mfma_f32_16x16x32_f16 %[c22], v[128:131], v[144:147], %[c22]\n\t")
        MF("v_mfma_f32_16x16x32_f16 %[c32], v[132:135], v[144:147], %[c32]\n\t")
        MF("v_mfma_f32_16x16x32_f16 %[c03], v[120:123], v[148:151], %[c03]\n\t")
        MF("v_mfma_f32_16x16x32_f16 %[c13], v[124:127], v[148:151], %[c13]\n\t")
        MF("v_mfma_f32_16x16x32_f16 %[c23], v[128:131], v[148:151], %[c23]\n\t")
        MF("v_mfma_f32_16x16x32_f16 %[c33], v[132:135], v[148:151], %[c33]\n\t")
        "s_cmp_gt_u32 s89, %[nh]\n\t"
        "s_cbranch_scc1 DONE_%=\n\t"
        "LOADY_%=:\n\t"
        "s_mov_b32 s94, 0\n\t"
        "v_add_u32 v157, s88, %[va0]\n\t"
        "v_add_u32 v158, s88, %[vb0]\n\t"
        "SPINY_%=:\n\t"
        "ds_read_b32 v156, %[vfull]\n\t"
        "s_waitcnt lgkmcnt(0)\n\t"
        "v_cmp_gt_u32_e64 s[86:87], s89, v156\n\t"
        "s_cmp_eq_u64 s[86:87], 0\n\t"
        "s_cbranch_scc1 GOY_%=\n\t"
        "s_sleep 1\n\t"
        "s_add_u32 s94, s94, 1\n\t"
        "s_and_b32 s92, s94, 0x3ff\n\t"
        "s_cbranch_scc1 SPINY_%=\n\t"
        "s_memrealtime s[92:93]\n\t"
        "s_waitcnt lgkmcnt(0)\n\t"
        "s_cmp_eq_u32 s94, 0x400\n\t"
        "s_cselect_b32 s90, s92, s90\n\t"
        "s_sub_u32 s92, s92, s90\n\t"
        "s_cmp_lt_u32 s92, %[dl3]\n\t"
        "s_cbranch_scc1 SPINY_%=\n\t"
        "v_mov_b32 v159, %[bad]\n\t"
        "ds_write_b32 v159, v159\n\t"
        "GOY_%=:\n\t"
        "ds_read_b128 v[120:123], v157 offset:0\n\t"
        "ds_read_b128 v[124:127], v157 offset:1024\n\t"
        "ds_read_b128 v[128:131], v157 offset:2048\n\t"
        "ds_read_b128 v[132:135], v157 offset:3072\n\t"
        "ds_read_b32 v156, %[vfull]\n\t"
        "ds_read_b128 v[136:139], v158 offset:0\n\t"
        "ds_read_b128 v[140:143], v158 offset:1024\n\t"
        "ds_read_b128 v[144:147], v158 offset:2048\n\t"
        "ds_read_b128 v[148:151], v158 offset:3072\n\t"
        "s_mov_b64 exec, 1\n\t"
        "ds_add_u32 %[vfree], v162\n\t"
        "s_mov_b64 exec, -1\n\t"
        "s_add_u32 s88, s88, %[slotb]\n\t"
        "s_cmp_eq_u32 s88, %[ringb]\n\t"
        "s_cselect_b32 s88, 0, s88\n\t"
        "s_add_u32 s89, s89, 1\n\t"
        "s_waitcnt lgkmcnt(5)\n\t"
        "v_bitop3_b32 v159, v120, v124, %[tag] bitop3:0x7e\n\t"
        "v_bitop3_b32 v160, v128, v132, %[tag] bitop3:0x7e\n\t"
        "v_bitop3_b32 v159, v159, 1, v160 bitop3:0xc8\n\t"
        "v_cmp_gt_u32_e64 s[86:87], s89, v156\n\t"
        TG("v_cmp_ne_u32_e64 s[84:85], 0, v159\n\t")
        NTG("s_mov_b64 s[84:85], 0\n\t")
        "s_or_b64 s[90:91], s[84:85], s[86:87]\n\t"
        "s_cbranch_scc0 TOPY_%=\n\t"
        "s_branch SLOWY_%=\n\t"
        "DONE_%=:\n\t"
        "s_nop 7\n\t"
        "s_nop 1\n\t"
        : [c00] "+v"(acc[0][0]), [c01] "+v"(acc[0][1]), [c02] "+v"(acc[0][2]), [c03] "+v"(acc[0][3]), [c10] "+v"(acc[1][0]), [c11] "+v"(acc[1][1]), [c12] "+v"(acc[1][2]), [c13] "+v"(acc[1][3]), [c20] "+v"(acc[2][0]), [c21] "+v"(acc[2][1]), [c22] "+v"(acc[2][2]), [c23] "+v"(acc[2][3]), [c30] "+v"(acc[3][0]), [c31] "+v"(acc[3][1]), [c32] "+v"(acc[3][2]), [c33] "+v"(acc[3][3]), [nfix] "+s"(nfix)
        : [va0] "v"(k.va0), [vb0] "v"(k.vb0), [vfull] "v"(k.vfull), [vfree] "v"(k.vfree), [l16] "v"(k.l16),
          [tag] "s"(k.tag), [bad] "s"(k.bad), [xsrc] "s"(k.xsrc), [slotb] "i"(SLOT_B), [ringb] "i"(NSLOT * SLOT_B), [nh] "i"(NH),
          [dl] "i"((unsigned)DEADLINE), [dl3] "i"(3u * (unsigned)DEADLINE)
        : "memory", "scc", "v104", "v105", "v106", "v107", "v108", "v109", "v110", "v111", "v112", "v113", "v114", "v115", "v116", "v117", "v118", "v119", "v120", "v121", "v122", "v123", "v124", "v125", "v126", "v127", "v128", "v129", "v130", "v131", "v132", "v133", "v134", "v135", "v136", "v137", "v138", "v139", "v140", "v141", "v142", "v143", "v144", "v145", "v146", "v147", "v148", "v149", "v150", "v151", "v152", "v153", "v154", "v155", "v156", "v157", "v158", "v159", "v160", "v161", "v162", "s84", "s85", "s86", "s87", "s88", "s89", "s90", "s91", "s92", "s93", "s94");
}
__device__ __forceinline__ void kloop(f32x4 (&acc)[4][5], const KArgs& k, unsigned& nfix) {
    asm volatile(
        "v_mov_b32 v162, 1\n\t"
        "s_mov_b32 s88, 0\n\t"
        "s_mov_b32 s89, 1\n\t"
        "s_branch LOADX_%=\n\t"
        ".p2align 6\n\t"
        "TOPX_%=:\n\t"
        "s_waitcnt lgkmcnt(5)\n\t"
        MF("v_mfma_f32_16x16x32_f16 %[c00], v[104:107], v[136:139], %[c00]\n\t")
        "v_add_u32 v157, s88, %[va0]\n\t"
        "v_add_u32 v158, s88, %[vb0]\n\t"
        MF("v_mfma_f32_16x16x32_f16 %[c10], v[108:111], v[136:139], %[c10]\n\t")
        "ds_read_b128 v[120:123], v157\n\t"
        "ds_read_b128 v[124:127], v157 offset:1024\n\t"
        MF("v_mfma_f32_16x16x32_f16 %[c20], v[112:115], v[136:139], %[c20]\n\t")
        "ds_read_b128 v[128:131], v157 offset:2048\n\t"
        "ds_read_b128 v[132:135], v157 offset:3072\n\t"
        MF("v_mfma_f32_16x16x32_f16 %[c30], v[116:119], v[136:139], %[c30]\n\t")
        "ds_read_b32 v156, %[vfull]\n\t"
        "ds_read_b128 v[136:139], v158\n\t"
        "s_waitcnt lgkmcnt(10)\n\t"
        MF("v_mfma_f32_16x16x32_f16 %[c01], v[104:107], v[140:143], %[c01]\n\t")
        "s_add_u32 s88, s88, %[slotb]\n\t"
        "s_cmp_eq_u32 s88, %[ringb]\n\t"
        "s_cselect_b32 s88, 0, s88\n\t"
        "s_add_u32 s89, s89, 1\n\t"
        MF("v_mfma_f32_16x16x32_f16 %[c11], v[108:111], v[140:143], %[c11]\n\t")
        MF("v_mfma_f32_16x16x32_f16 %[c21], v[112:115], v[140:143], %[c21]\n\t")
        MF("v_mfma_f32_16x16x32_f16 %[c31], v[116:119], v[140:143], %[c31]\n\t")
        "ds_read_b128 v[140:143], v158 offset:1024\n\t"
        "s_waitcnt lgkmcnt(10)\n\t"
        MF("v_mfma_f32_16x16x32_f16 %[c02], v[104:107], v[144:147], %[c02]\n\t")
        MF("v_mfma_f32_16x16x32_f16 %[c12], v[108:111], v[144:147], %[c12]\n\t")
        MF("v_mfma_f32_16x16x32_f16 %[c22], v[112:115], v[144:147], %[c22]\n\t")
        MF("v_mfma_f32_16x16x32_f16 %[c32], v[116:119], v[144:147], %[c32]\n\t")
        "ds_read_b128 v[144:147], v158 offset:2048\n\t"
        "s_waitcnt lgkmcnt(10)\n\t"
        MF("v_mfma_f32_16x16x32_f16 %[c03], v[104:107], v[148:151], %[c03]\n\t")
        MF("v_mfma_f32_16x16x32_f16 %[c13], v[108:111], v[148:151], %[c13]\n\t")
        MF("v_mfma_f32_16x16x32_f16 %[c23], v[112:115], v[148:151], %[c23]\n\t")
        MF("v_mfma_f32_16x16x32_f16 %[c33], v[116:119], v[148:151], %[c33]\n\t")
        "ds_read_b128 v[148:151], v158 offset:3072\n\t"
        "s_waitcnt lgkmcnt(4)\n\t"
        MF("v_mfma_f32_16x16x32_f16 %[c04], v[104:107], v[152:155], %[c04]\n\t")
        "v_bitop3_b32 v159, v120, v124, %[tag] bitop3:0x7e\n\t"
        "v_bitop3_b32 v160, v128, v132, %[tag] bitop3:0x7e\n\t"
        MF("v_mfma_f32_16x16x32_f16 %[c14], v[108:111], v[152:155], %[c14]\n\t")
        "v_bitop3_b32 v159, v159, 1, v160 bitop3:0xc8\n\t"
        "v_cmp_gt_u32_e64 s[86:87], s89, v156\n\t"
        MF("v_mfma_f32_16x16x32_f16 %[c24], v[112:115], v[152:155], %[c24]\n\t")
        TG("v_cmp_ne_u32_e64 s[84:85], 0, v159\n\t")
        NTG("s_mov_b64 s[84:85], 0\n\t")
        MF("v_mfma_f32_16x16x32_f16 %[c34], v[116:119], v[152:155], %[c34]\n\t")
        "ds_read_b128 v[152:155], v158 offset:8192\n\t"
        "s_mov_b64 exec, 1\n\t"
        "ds_add_u32 %[vfree], v162\n\t"
        "s_mov_b64 exec, -1\n\t"
        "s_or_b64 s[90:91], s[84:85], s[86:87]\n\t"
        "s_cbranch_scc1 SLOWY_%=\n\t"
        "TOPY_%=:\n\t"
        "s_waitcnt lgkmcnt(5)\n\t"
        MF("v_mfma_f32_16x16x32_f16 %[c00], v[120:123], v[136:139], %[c00]\n\t")
        "v_add_u32 v157, s88, %[va0]\n\t"
        "v_add_u32 v158, s88, %[vb0]\n\t"
        MF("v_mfma_f32_16x16x32_f16 %[c10], v[124:127], v[136:139], %[c10]\n\t")
        "ds_read_b128 v[104:107], v157\n\t"
        "ds_read_b128 v[108:111], v157 offset:1024\n\t"
        MF("v_mfma_f32_16x16x32_f16 %[c20], v[128:131], v[136:139], %[c20]\n\t")
        "ds_read_b128 v[112:115], v157 offset:2048\n\t"
        "ds_read_b128 v[116:119], v157 offset:3072\n\t"
        MF("v_mfma_f32_16x16x32_f16 %[c30], v[132:135], v[136:139], %[c30]\n\t")
        "ds_read_b32 v156, %[vfull]\n\t"
        "ds_read_b128 v[136:139], v158\n\t"
        "s_waitcnt lgkmcnt(10)\n\t"
        MF("v_mfma_f32_16x16x32_f16 %[c01], v[120:123], v[140:143], %[c01]\n\t")
        "s_add_u32 s88, s88, %[slotb]\n\t"
        "s_cmp_eq_u32 s88, %[ringb]\n\t"
        "s_cselect_b32 s88, 0, s88\n\t"
        "s_add_u32 s89, s89, 1\n\t"
        MF("v_mfma_f32_16x16x32_f16 %[c11], v[124:127], v[140:143], %[c11]\n\t")
        MF("v_mfma_f32_16x16x32_f16 %[c21], v[128:131], v[140:143], %[c21]\n\t")
        MF("v_mfma_f32_16x16x32_f16 %[c31], v[132:135], v[140:143], %[c31]\n\t")
        "ds_read_b128 v[140:143], v158 offset:1024\n\t"
        "s_waitcnt lgkmcnt(10)\n\t"
        MF("v_mfma_f32_16x16x32_f16 %[c02], v[120:123], v[144:147], %[c02]\n\t")
        MF("v_mfma_f32_16x16x32_f16 %[c12], v[124:127], v[144:147], %[c12]\n\t")
        MF("v_mfma_f32_16x16x32_f16 %[c22], v[128:131], v[144:147], %[c22]\n\t")
        MF("v_mfma_f32_16x16x32_f16 %[c32], v[132:135], v[144:147], %[c32]\n\t")
        "ds_read_b128 v[144:147], v158 offset:2048\n\t"
        "s_waitcnt lgkmcnt(10)\n\t"
        MF("v_mfma_f32_16x16x32_f16 %[c03], v[120:123], v[148:151], %[c03]\n\t")
        MF("v_mfma_f32_16x16x32_f16 %[c13], v[124:127], v[148:151], %[c13]\n\t")
        MF("v_mfma_f32_16x16x32_f16 %[c23], v[128:131], v[148:151], %[c23]\n\t")
        MF("v_mfma_f32_16x16x32_f16 %[c33], v[132:135], v[148:151], %[c33]\n\t")
        "ds_read_b128 v[148:151], v158 offset:3072\n\t"
        "s_waitcnt lgkmcnt(4)\n\t"
        MF("v_mfma_f32_16x16x32_f16 %[c04], v[120:123], v[152:155], %[c04]\n\t")
        "v_bitop3_b32 v159, v104, v108, %[tag] bitop3:0x7e\n\t"
        "v_bitop3_b32 v160, v112, v116, %[tag] bitop3:0x7e\n\t"
        MF("v_mfma_f32_16x16x32_f16 %[c14], v[124:127], v[152:155], %[c14]\n\t")
        "v_bitop3_b32 v159, v159, 1, v160 bitop3:0xc8\n\t"
        "v_cmp_gt_u32_e64 s[86:87], s89, v156\n\t"
        MF("v_mfma_f32_16x16x32_f16 %[c24], v[128:131], v[152:155], %[c24]\n\t")
        TG("v_cmp_ne_u32_e64 s[84:85], 0, v159\n\t")
        NTG("s_mov_b64 s[84:85], 0\n\t")
        MF("v_mfma_f32_16x16x32_f16 %[c34], v[132:135], v[152:155], %[c34]\n\t")
        "ds_read_b128 v[152:155], v158 offset:8192\n\t"
        "s_mov_b64 exec, 1\n\t"
        "ds_add_u32 %[vfree], v162\n\t"
        "s_mov_b64 exec, -1\n\t"
        "s_or_b64 s[90:91], s[84:85], s[86:87]\n\t"
        "s_cbranch_scc1 SLOWX_%=\n\t"
        "s_waitcnt lgkmcnt(5)\n\t"
        MF("v_mfma_f32_16x16x32_f16 %[c00], v[104:107], v[136:139], %[c00]\n\t")
        "v_add_u32 v157, s88, %[va0]\n\t"
        "v_add_u32 v158, s88, %[vb0]\n\t"
        MF("v_mfma_f32_16x16x32_f16 %[c10], v[108:111], v[136:139], %[c10]\n\t")
        "ds_read_b128 v[120:123], v157\n\t"
        "ds_read_b128 v[124:127], v157 offset:1024\n\t"
        MF("v_mfma_f32_16x16x32_f16 %[c20], v[112:115], v[136:139], %[c20]\n\t")
        "ds_read_b128 v[128:131], v157 offset:2048\n\t"
        "ds_read_b128 v[132:135], v157 offset:3072\n\t"
        MF("v_mfma_f32_16x16x32_f16 %[c30], v[116:119], v[136:139], %[c30]\n\t")
        "ds_read_b32 v156, %[vfull]\n\t"
        "ds_read_b128 v[136:139], v158\n\t"
        "s_waitcnt lgkmcnt(10)\n\t"
        MF("v_mfma_f32_16x16x32_f16 %[c01], v[104:107], v[140:143], %[c01]\n\t")
        "s_add_u32 s88, s88, %[slotb]\n\t"
        "s_cmp_eq_u32 s88, %[ringb]\n\t"
        "s_cselect_b32 s88, 0, s88\n\t"
        "s_add_u32 s89, s89, 1\n\t"
        MF("v_mfma_f32_16x16x32_f16 %[c11], v[108:111], v[140:143], %[c11]\n\t")
        MF("v_mfma_f32_16x16x32_f16 %[c21], v[112:115], v[140:143], %[c21]\n\t")
        MF("v_mfma_f32_16x16x32_f16 %[c31], v[116:119], v[140:143], %[c31]\n\t")
        "ds_read_b128 v[140:143], v158 offset:1024\n\t"
        "s_waitcnt lgkmcnt(10)\n\t"
        MF("v_mfma_f32_16x16x32_f16 %[c02], v[104:107], v[144:147], %[c02]\n\t")
        MF("v_mfma_f32_16x16x32_f16 %[c12], v[108:111], v[144:147], %[c12]\n\t")
        MF("v_mfma_f32_16x16x32_f16 %[c22], v[112:115], v[144:147], %[c22]\n\t")
        MF("v_mfma_f32_16x16x32_f16 %[c32], v[116:119], v[144:147], %[c32]\n\t")
        "ds_read_b128 v[144:147], v158 offset:2048\n\t"
        "s_waitcnt lgkmcnt(10)\n\t"
        MF("v_mfma_f32_16x16x32_f16 %[c03], v[104:107], v[148:151], %[c03]\n\t")
        MF("v_mfma_f32_16x16x32_f16 %[c13], v[108:111], v[148:151], %[c13]\n\t")
        MF("v_mfma_f32_16x16x32_f16 %[c23], v[112:115], v[148:151], %[c23]\n\t")
        MF("v_mfma_f32_16x16x32_f16 %[c33], v[116:119], v[148:151], %[c33]\n\t")
        "ds_read_b128 v[148:151], v158 offset:3072\n\t"
        "s_waitcnt lgkmcnt(4)\n\t"
        MF("v_mfma_f32_16x16x32_f16 %[c04], v[104:107], v[152:155], %[c04]\n\t")
        "v_bitop3_b32 v159, v120, v124, %[tag] bitop3:0x7e\n\t"
        "v_bitop3_b32 v160, v128, v132, %[tag] bitop3:0x7e\n\t"
        MF("v_mfma_f32_16x16x32_f16 %[c14], v[108:111], v[152:155], %[c14]\n\t")
        "v_bitop3_b32 v159, v159, 1, v160 bitop3:0xc8\n\t"
        "v_cmp_gt_u32_e64 s[86:87], s89, v156\n\t"
        MF("v_mfma_f32_16x16x32_f16 %[c24], v[112:115], v[152:155], %[c24]\n\t")
        TG("v_cmp_ne_u32_e64 s[84:85], 0, v159\n\t")
        NTG("s_mov_b64 s[84:85], 0\n\t")
        MF("v_mfma_f32_16x16x32_f16 %[c34], v[116:119], v[152:155], %[c34]\n\t")
        "ds_read_b128 v[152:155], v158 offset:8192\n\t"
        "s_mov_b64 exec, 1\n\t"
        "ds_add_u32 %[vfree], v162\n\t"
        "s_mov_b64 exec, -1\n\t"
        "s_or_b64 s[90:91], s[84:85], s[86:87]\n\t"
        "s_cbranch_scc1 SLOWY_%=\n\t"
        "s_waitcnt lgkmcnt(5)\n\t"
        MF("v_mfma_f32_16x16x32_f16 %[c00], v[120:123], v[136:139], %[c00]\n\t")
        "v_add_u32 v157, s88, %[va0]\n\t"
        "v_add_u32 v158, s88, %[vb0]\n\t"
        MF("v_mfma_f32_16x16x32_f16 %[c10], v[124:127], v[136:139], %[c10]\n\t")
        "ds_read_b128 v[104:107], v157\n\t"
        "ds_read_b128 v[108:111], v157 offset:1024\n\t"
        MF("v_mfma_f32_16x16x32_f16 %[c20], v[128:131], v[136:139], %[c20]\n\t")
        "ds_read_b128 v[112:115], v157 offset:2048\n\t"
        "ds_read_b128 v[116:119], v157 offset:3072\n\t"
        MF("v_mfma_f32_16x16x32_f16 %[c30], v[132:135], v[136:139], %[c30]\n\t")
        "ds_read_b32 v156, %[vfull]\n\t"
        "ds_read_b128 v[136:139], v158\n\t"
        "s_waitcnt lgkmcnt(10)\n\t"
        MF("v_mfma_f32_16x16x32_f16 %[c01], v[120:123], v[140:143], %[c01]\n\t")
        "s_add_u32 s88, s88, %[slotb]\n\t"
        "s_cmp_eq_u32 s88, %[ringb]\n\t"
        "s_cselect_b32 s88, 0, s88\n\t"
        "s_add_u32 s89, s89, 1\n\t"
        MF("v_mfma_f32_16x16x32_f16 %[c11], v[124:127], v[140:143], %[c11]\n\t")
        MF("v_mfma_f32_16x16x32_f16 %[c21], v[128:131], v[140:143], %[c21]\n\t")
        MF("v_mfma_f32_16x16x32_f16 %[c31], v[132:135], v[140:143], %[c31]\n\t")
        "ds_read_b128 v[140:143], v158 offset:1024\n\t"
        "s_waitcnt lgkmcnt(10)\n\t"
        MF("v_mfma_f32_16x16x32_f16 %[c02], v[120:123], v[144:147], %[c02]\n\t")
        MF("v_mfma_f32_16x16x32_f16 %[c12], v[124:127], v[144:147], %[c12]\n\t")
        MF("v_mfma_f32_16x16x32_f16 %[c22], v[128:131], v[144:147], %[c22]\n\t")
        MF("v_mfma_f32_16x16x32_f16 %[c32], v[132:135], v[144:147], %[c32]\n\t")
        "ds_read_b128 v[144:147], v158 offset:2048\n\t"
        "s_waitcnt lgkmcnt(10)\n\t"
        MF("v_mfma_f32_16x16x32_f16 %[c03], v[120:123], v[148:151], %[c03]\n\t")
        MF("v_mfma_f32_16x16x32_f16 %[c13], v[124:127], v[148:151], %[c13]\n\t")
        MF("v_mfma_f32_16x16x32_f16 %[c23], v[128:131], v[148:151], %[c23]\n\t")
        MF("v_mfma_f32_16x16x32_f16 %[c33], v[132:135], v[148:151], %[c33]\n\t")
        "ds_read_b128 v[148:151], v158 offset:3072\n\t"
        "s_waitcnt lgkmcnt(4)\n\t"
        MF("v_mfma_f32_16x16x32_f16 %[c04], v[120:123], v[152:155], %[c04]\n\t")
        "v_bitop3_b32 v159, v104, v108, %[tag] bitop3:0x7e\n\t"
        "v_bitop3_b32 v160, v112, v116, %[tag] bitop3:0x7e\n\t"
        MF("v_mfma_f32_16x16x32_f16 %[c14], v[124:127], v[152:155], %[c14]\n\t")
        "v_bitop3_b32 v159, v159, 1, v160 bitop3:0xc8\n\t"
        "v_cmp_gt_u32_e64 s[86:87], s89, v156\n\t"
        MF("v_mfma_f32_16x16x32_f16 %[c24], v[128:131], v[152:155], %[c24]\n\t")
        TG("v_cmp_ne_u32_e64 s[84:85], 0, v159\n\t")
        NTG("s_mov_b64 s[84:85], 0\n\t")
        MF("v_mfma_f32_16x16x32_f16 %[c34], v[132:135], v[152:155], %[c34]\n\t")
        "ds_read_b128 v[152:155], v158 offset:8192\n\t"
        "s_mov_b64 exec, 1\n\t"
        "ds_add_u32 %[vfree], v162\n\t"
        "s_mov_b64 exec, -1\n\t"
        "s_or_b64 s[90:91], s[84:85], s[86:87]\n\t"
        "s_cbranch_scc0 TOPX_%=\n\t"
        "SLOWX_%=:\n\t"
        "s_cmp_eq_u64 s[84:85], 0\n\t"
        "s_cbranch_scc1 NOFIXX_%=\n\t"
        "s_add_u32 %[nfix], %[nfix], 1\n\t"
        "v_mov_b32 v159, %[bad]\n\t"
        "ds_read_b32 v160, v159\n\t"
        "s_sub_u32 s92, s89, 2\n\t"
        "s_lshl_b32 s92, s92, 14\n\t"
        "v_add_u32 v161, s92, %[l16]\n\t"
        "s_memrealtime s[90:91]\n\t"
        "s_waitcnt lgkmcnt(0)\n\t"
        "v_cmp_ne_u32_e64 s[92:93], 0, v160\n\t"
        "s_cmp_lg_u64 s[92:93], 0\n\t"
        "s_cbranch_scc1 NOFIXX_%=\n\t"
        "FIXLX_%=:\n\t"
        "global_load_dwordx4 v[104:107], v161, %[xsrc] offset:0 sc0 sc1\n\t"
        "global_load_dwordx4 v[108:111], v161, %[xsrc] offset:1024 sc0 sc1\n\t"
        "global_load_dwordx4 v[112:115], v161, %[xsrc] offset:2048 sc0 sc1\n\t"
        "global_load_dwordx4 v[116:119], v161, %[xsrc] offset:3072 sc0 sc1\n\t"
        "s_waitcnt vmcnt(0)\n\t"
        "v_bitop3_b32 v159, v104, v108, %[tag] bitop3:0x7e\n\t"
        "v_bitop3_b32 v160, v112, v116, %[tag] bitop3:0x7e\n\t"
        "v_bitop3_b32 v159, v159, 1, v160 bitop3:0xc8\n\t"
        "v_cmp_ne_u32_e64 s[92:93], 0, v159\n\t"
        "s_cmp_eq_u64 s[92:93], 0\n\t"
        "s_cbranch_scc1 NOFIXX_%=\n\t"
        "s_sleep 1\n\t"
        "s_memrealtime s[92:93]\n\t"
        "s_waitcnt lgkmcnt(0)\n\t"
        "s_sub_u32 s92, s92, s90\n\t"
        "s_cmp_lt_u32 s92, %[dl]\n\t"
        "s_cbranch_scc1 FIXLX_%=\n\t"
        "v_mov_b32 v159, %[bad]\n\t"
        "ds_write_b32 v159, v159\n\t"
        "NOFIXX_%=:\n\t"
        "s_cmp_gt_u32 s89, %[nh]\n\t"
        "s_cbranch_scc1 PLAINX_%=\n\t"
        "ds_read_b32 v156, %[vfull]\n\t"
        "s_waitcnt lgkmcnt(0)\n\t"
        "v_cmp_gt_u32_e64 s[86:87], s89, v156\n\t"
        "s_cmp_eq_u64 s[86:87], 0\n\t"
        "s_cbranch_scc1 TOPX_%=\n\t"
        "PLAINX_%=:\n\t"
        "s_waitcnt lgkmcnt(0)\n\t"
        MF("v_mfma_f32_16x16x32_f16 %[c00], v[104:107], v[136:139], %[c00]\n\t")
        MF("v_mfma_f32_16x16x32_f16 %[c10], v[108:111], v[136:139], %[c10]\n\t")
        MF("v_mfma_f32_16x16x32_f16 %[c20], v[112:115], v[136:139], %[c20]\n\t")
        MF("v_mfma_f32_16x16x32_f16 %[c30], v[116:119], v[136:139], %[c30]\n\t")
        MF("v_mfma_f32_16x16x32_f16 %[c01], v[104:107], v[140:143], %[c01]\n\t")
        MF("v_mfma_f32_16x16x32_f16 %[c11], v[108:111], v[140:143], %[c11]\n\t")
        MF("v_mfma_f32_16x16x32_f16 %[c21], v[112:115], v[140:143], %[c21]\n\t")
        MF("v_mfma_f32_16x16x32_f16 %[c31], v[116:119], v[140:143], %[c31]\n\t")
        MF("v_mfma_f32_16x16x32_f16 %[c02], v[104:107], v[144:147], %[c02]\n\t")
        MF("v_mfma_f32_16x16x32_f16 %[c12], v[108:111], v[144:147], %[c12]\n\t")
        MF("v_mfma_f32_16x16x32_f16 %[c22], v[112:115], v[144:147], %[c22]\n\t")
        MF("v_mfma_f32_16x16x32_f16 %[c32], v[116:119], v[144:147], %[c32]\n\t")
        MF("v_mfma_f32_16x16x32_f16 %[c03], v[104:107], v[148:151], %[c03]\n\t")
        MF("v_mfma_f32_16x16x32_f16 %[c13], v[108:111], v[148:151], %[c13]\n\t")
        MF("v_mfma_f32_16x16x32_f16 %[c23], v[112:115], v[148:151], %[c23]\n\t")
        MF("v_mfma_f32_16x16x32_f16 %[c33], v[116:119], v[148:151], %[c33]\n\t")
        MF("v_mfma_f32_16x16x32_f16 %[c04], v[104:107], v[152:155], %[c04]\n\t")
        MF("v_mfma_f32_16x16x32_f16 %[c14], v[108:111], v[152:155], %[c14]\n\t")
        MF("v_mfma_f32_16x16x32_f16 %[c24], v[112:115], v[152:155], %[c24]\n\t")
        MF("v_mfma_f32_16x16x32_f16 %[c34], v[116:119], v[152:155], %[c34]\n\t")
        "s_cmp_gt_u32 s89, %[nh]\n\t"
        "s_cbranch_scc1 DONE_%=\n\t"
        "LOADX_%=:\n\t"
        "s_mov_b32 s94, 0\n\t"
        "v_add_u32 v157, s88, %[va0]\n\t"
        "v_add_u32 v158, s88, %[vb0]\n\t"
        "SPINX_%=:\n\t"
        "ds_read_b32 v156, %[vfull]\n\t"
        "s_waitcnt lgkmcnt(0)\n\t"
        "v_cmp_gt_u32_e64 s[86:87], s89, v156\n\t"
        "s_cmp_eq_u64 s[86:87], 0\n\t"
        "s_cbranch_scc1 GOX_%=\n\t"
        "s_sleep 1\n\t"
        "s_add_u32 s94, s94, 1\n\t"
        "s_and_b32 s92, s94, 0x3ff\n\t"
        "s_cbranch_scc1 SPINX_%=\n\t"
        "s_memrealtime s[92:93]\n\t"
        "s_waitcnt lgkmcnt(0)\n\t"
        "s_cmp_eq_u32 s94, 0x400\n\t"
        "s_cselect_b32 s90, s92, s90\n\t"
        "s_sub_u32 s92, s92, s90\n\t"
        "s_cmp_lt_u32 s92, %[dl3]\n\t"
        "s_cbranch_scc1 SPINX_%=\n\t"
        "v_mov_b32 v159, %[bad]\n\t"
        "ds_write_b32 v159, v159\n\t"
        "GOX_%=:\n\t"
        "ds_read_b128 v[104:107], v157 offset:0\n\t"
        "ds_read_b128 v[108:111], v157 offset:1024\n\t"
        "ds_read_b128 v[112:115], v157 offset:2048\n\t"
        "ds_read_b128 v[116:119], v157 offset:3072\n\t"
        "ds_read_b32 v156, %[vfull]\n\t"
        "ds_read_b128 v[136:139], v158 offset:0\n\t"
        "ds_read_b128 v[140:143], v158 offset:1024\n\t"
        "ds_read_b128 v[144:147], v158 offset:2048\n\t"
        "ds_read_b128 v[148:151], v158 offset:3072\n\t"
        "ds_read_b128 v[152:155], v158 offset:8192\n\t"
        "s_mov_b64 exec, 1\n\t"
        "ds_add_u32 %[vfree], v162\n\t"
        "s_mov_b64 exec, -1\n\t"
        "s_add_u32 s88, s88, %[slotb]\n\t"
        "s_cmp_eq_u32 s88, %[ringb]\n\t"
        "s_cselect_b32 s88, 0, s88\n\t"
        "s_add_u32 s89, s89, 1\n\t"
        "s_waitcnt lgkmcnt(6)\n\t"
        "v_bitop3_b32 v159, v104, v108, %[tag] bitop3:0x7e\n\t"
        "v_bitop3_b32 v160, v112, v116, %[tag] bitop3:0x7e\n\t"
        "v_bitop3_b32 v159, v159, 1, v160 bitop3:0xc8\n\t"
        "v_cmp_gt_u32_e64 s[86:87], s89, v156\n\t"
        TG("v_cmp_ne_u32_e64 s[84:85], 0, v159\n\t")
        NTG("s_mov_b64 s[84:85], 0\n\t")
        "s_or_b64 s[90:91], s[84:85], s[86:87]\n\t"
        "s_cbranch_scc0 TOPX_%=\n\t"
        "s_branch SLOWX_%=\n\t"
        "SLOWY_%=:\n\t"
        "s_cmp_eq_u64 s[84:85], 0\n\t"
        "s_cbranch_scc1 NOFIXY_%=\n\t"
        "s_add_u32 %[nfix], %[nfix], 1\n\t"
        "v_mov_b32 v159, %[bad]\n\t"
        "ds_read_b32 v160, v159\n\t"
        "s_sub_u32 s92, s89, 2\n\t"
        "s_lshl_b32 s92, s92, 14\n\t"
        "v_add_u32 v161, s92, %[l16]\n\t"
        "s_memrealtime s[90:91]\n\t"
        "s_waitcnt lgkmcnt(0)\n\t"
        "v_cmp_ne_u32_e64 s[92:93], 0, v160\n\t"
        "s_cmp_lg_u64 s[92:93], 0\n\t"
        "s_cbranch_scc1 NOFIXY_%=\n\t"
        "FIXLY_%=:\n\t"
        "global_load_dwordx4 v[120:123], v161, %[xsrc] offset:0 sc0 sc1\n\t"
        "global_load_dwordx4 v[124:127], v161, %[xsrc] offset:1024 sc0 sc1\n\t"
        "global_load_dwordx4 v[128:131], v161, %[xsrc] offset:2048 sc0 sc1\n\t"
        "global_load_dwordx4 v[132:135], v161, %[xsrc] offset:3072 sc0 sc1\n\t"
        "s_waitcnt vmcnt(0)\n\t"
        "v_bitop3_b32 v159, v120, v124, %[tag] bitop3:0x7e\n\t"
        "v_bitop3_b32 v160, v128, v132, %[tag] bitop3:0x7e\n\t"
        "v_bitop3_b32 v159, v159, 1, v160 bitop3:0xc8\n\t"
        "v_cmp_ne_u32_e64 s[92:93], 0, v159\n\t"
        "s_cmp_eq_u64 s[92:93], 0\n\t"
        "s_cbranch_scc1 NOFIXY_%=\n\t"
        "s_sleep 1\n\t"
        "s_memrealtime s[92:93]\n\t"
        "s_waitcnt lgkmcnt(0)\n\t"
        "s_sub_u32 s92, s92, s90\n\t"
        "s_cmp_lt_u32 s92, %[dl]\n\t"
        "s_cbranch_scc1 FIXLY_%=\n\t"
        "v_mov_b32 v159, %[bad]\n\t"
        "ds_write_b32 v159, v159\n\t"
        "NOFIXY_%=:\n\t"
        "s_cmp_gt_u32 s89, %[nh]\n\t"
        "s_cbranch_scc1 PLAINY_%=\n\t"
        "ds_read_b32 v156, %[vfull]\n\t"
        "s_waitcnt lgkmcnt(0)\n\t"
        "v_cmp_gt_u32_e64 s[86:87], s89, v156\n\t"
        "s_cmp_eq_u64 s[86:87], 0\n\t"
        "s_cbranch_scc1 TOPY_%=\n\t"
        "PLAINY_%=:\n\t"
        "s_waitcnt lgkmcnt(0)\n\t"
        MF("v_mfma_f32_16x16x32_f16 %[c00], v[120:123], v[136:139], %[c00]\n\t")
        MF("v_mfma_f32_16x16x32_f16 %[c10], v[124:127], v[136:139], %[c10]\n\t")
        MF("v_mfma_f32_16x16x32_f16 %[c20], v[128:131], v[136:139], %[c20]\n\t")
        MF("v_mfma_f32_16x16x32_f16 %[c30], v[132:135], v[136:139], %[c30]\n\t")
        MF("v_mfma_f32_16x16x32_f16 %[c01], v[120:123], v[140:143], %[c01]\n\t")
        MF("v_mfma_f32_16x16x32_f16 %[c11], v[124:127], v[140:143], %[c11]\n\t")
        MF("v_mfma_f32_16x16x32_f16 %[c21], v[128:131], v[140:143], %[c21]\n\t")
        MF("v_mfma_f32_16x16x32_f16 %[c31], v[132:135], v[140:143], %[c31]\n\t")
        MF("v_mfma_f32_16x16x32_f16 %[c02], v[120:123], v[144:147], %[c02]\n\t")
        MF("v_mfma_f32_16x16x32_f16 %[c12], v[124:127], v[144:147], %[c12]\n\t")
        MF("v_mfma_f32_16x16x32_f16 %[c22], v[128:131], v[144:147], %[c22]\n\t")
        MF("v_mfma_f32_16x16x32_f16 %[c32], v[132:135], v[144:147], %[c32]\n\t")
        MF("v_mfma_f32_16x16x32_f16 %[c03], v[120:123], v[148:151], %[c03]\n\t")
        MF("v_mfma_f32_16x16x32_f16 %[c13], v[124:127], v[148:151], %[c13]\n\t")
        MF("v_mfma_f32_16x16x32_f16 %[c23], v[128:131], v[148:151], %[c23]\n\t")
        MF("v_mfma_f32_16x16x32_f16 %[c33], v[132:135], v[148:151], %[c33]\n\t")
        MF("v_mfma_f32_16x16x32_f16 %[c04], v[120:123], v[152:155], %[c04]\n\t")
        MF("v_mfma_f32_16x16x32_f16 %[c14], v[124:127], v[152:155], %[c14]\n\t")
        MF("v_mfma_f32_16x16x32_f16 %[c24], v[128:131], v[152:155], %[c24]\n\t")
        MF("v_mfma_f32_16x16x32_f16 %[c34], v[132:135], v[152:155], %[c34]\n\t")
        "s_cmp_gt_u32 s89, %[nh]\n\t"
        "s_cbranch_scc1 DONE_%=\n\t"
        "LOADY_%=:\n\t"
        "s_mov_b32 s94, 0\n\t"
        "v_add_u32 v157, s88, %[va0]\n\t"
        "v_add_u32 v158, s88, %[vb0]\n\t"
        "SPINY_%=:\n\t"
        "ds_read_b32 v156, %[vfull]\n\t"
        "s_waitcnt lgkmcnt(0)\n\t"
        "v_cmp_gt_u32_e64 s[86:87], s89, v156\n\t"
        "s_cmp_eq_u64 s[86:87], 0\n\t"
        "s_cbranch_scc1 GOY_%=\n\t"
        "s_sleep 1\n\t"
        "s_add_u32 s94, s94, 1\n\t"
        "s_and_b32 s92, s94, 0x3ff\n\t"
        "s_cbranch_scc1 SPINY_%=\n\t"
        "s_memrealtime s[92:93]\n\t"
        "s_waitcnt lgkmcnt(0)\n\t"
        "s_cmp_eq_u32 s94, 0x400\n\t"
        "s_cselect_b32 s90, s92, s90\n\t"
        "s_sub_u32 s92, s92, s90\n\t"
        "s_cmp_lt_u32 s92, %[dl3]\n\t"
        "s_cbranch_scc1 SPINY_%=\n\t"
        "v_mov_b32 v159, %[bad]\n\t"
        "ds_write_b32 v159, v159\n\t"
        "GOY_%=:\n\t"
        "ds_read_b128 v[120:123], v157 offset:0\n\t"
        "ds_read_b128 v[124:127], v157 offset:1024\n\t"
        "ds_read_b128 v[128:131], v157 offset:2048\n\t"
        "ds_read_b128 v[132:135], v157 offset:3072\n\t"
        "ds_read_b32 v156, %[vfull]\n\t"
        "ds_read_b128 v[136:139], v158 offset:0\n\t"
        "ds_read_b128 v[140:143], v158 offset:1024\n\t"
        "ds_read_b128 v[144:147], v158 offset:2048\n\t"
        "ds_read_b128 v[148:151], v158 offset:3072\n\t"
        "ds_read_b128 v[152:155], v158 offset:8192\n\t"
        "s_mov_b64 exec, 1\n\t"
        "ds_add_u32 %[vfree], v162\n\t"
        "s_mov_b64 exec, -1\n\t"
        "s_add_u32 s88, s88, %[slotb]\n\t"
        "s_cmp_eq_u32 s88, %[ringb]\n\t"
        "s_cselect_b32 s88, 0, s88\n\t"
        "s_add_u32 s89, s89, 1\n\t"
        "s_waitcnt lgkmcnt(6)\n\t"
        "v_bitop3_b32 v159, v120, v124, %[tag] bitop3:0x7e\n\t"
        "v_bitop3_b32 v160, v128, v132, %[tag] bitop3:0x7e\n\t"
        "v_bitop3_b32 v159, v159, 1, v160 bitop3:0xc8\n\t"
        "v_cmp_gt_u32_e64 s[86:87], s89, v156\n\t"
        TG("v_cmp_ne_u32_e64 s[84:85], 0, v159\n\t")
        NTG("s_mov_b64 s[84:85], 0\n\t")
        "s_or_b64 s[90:91], s[84:85], s[86:87]\n\t"
        "s_cbranch_scc0 TOPY_%=\n\t"
        "s_branch SLOWY_%=\n\t"
        "DONE_%=:\n\t"
        "s_nop 7\n\t"
        "s_nop 1\n\t"
        : [c00] "+v"(acc[0][0]), [c01] "+v"(acc[0][1]), [c02] "+v"(acc[0][2]), [c03] "+v"(acc[0][3]), [c04] "+v"(acc[0][4]), [c10] "+v"(acc[1][0]), [c11] "+v"(acc[1][1]), [c12] "+v"(acc[1][2]), [c13] "+v"(acc[1][3]), [c14] "+v"(acc[1][4]), [c20] "+v"(acc[2][0]), [c21] "+v"(acc[2][1]), [c22] "+v"(acc[2][2]), [c23] "+v"(acc[2][3]), [c24] "+v"(acc[2][4]), [c30] "+v"(acc[3][0]), [c31] "+v"(acc[3][1]), [c32] "+v"(acc[3][2]), [c33] "+v"(acc[3][3]), [c34] "+v"(acc[3][4]), [nfix] "+s"(nfix)
        : [va0] "v"(k.va0), [vb0] "v"(k.vb0), [vfull] "v"(k.vfull), [vfree] "v"(k.vfree), [l16] "v"(k.l16),
          [tag] "s"(k.tag), [bad] "s"(k.bad), [xsrc] "s"(k.xsrc), [slotb] "i"(SLOT_B), [ringb] "i"(NSLOT * SLOT_B), [nh] "i"(NH),
          [dl] "i"((unsigned)DEADLINE), [dl3] "i"(3u * (unsigned)DEADLINE)
        : "memory", "scc", "v104", "v105", "v106", "v107", "v108", "v109", "v110", "v111", "v112", "v113", "v114", "v115", "v116", "v117", "v118", "v119", "v120", "v121", "v122", "v123", "v124", "v125", "v126", "v127", "v128", "v129", "v130", "v131", "v132", "v133", "v134", "v135", "v136", "v137", "v138", "v139", "v140", "v141", "v142", "v143", "v144", "v145", "v146", "v147", "v148", "v149", "v150", "v151", "v152", "v153", "v154", "v155", "v156", "v157", "v158", "v159", "v160", "v161", "v162", "s84", "s85", "s86", "s87", "s88", "s89", "s90", "s91", "s92", "s93", "s94");
}

__device__ __forceinline__ void conv_w64(const Params& p, int tj, int col0, int h, int cg, int lane, const f32x4* smp, int& se) {
    const int g = lane >> 4, q = lane & 15;
    const float* wsrc = p.w + (size_t)(h * 32 + 4 * g) * NL + col0 + cg * 64 + 4 * q;
    f32x4 v[8];
#pragma unroll
    for (int i = 0; i < 8; ++i)
        v[i] = __builtin_nontemporal_load((const f32x4*)(wsrc + (size_t)((i & 3) + 16 * (i >> 2)) * NL));
    se = scale_exp(*smp);
    const float sc = exp2i(se);
#pragma unroll
    for (int i = 0; i < 8; ++i) v[i] *= sc;
    char* dst = p.myws + OFF_W16 + (size_t)(tj * NH + h) * WB + cg * 4096 + lane * 16;
#pragma unroll
    for (int f = 0; f < 4; ++f) {
        u32x4 o;
        o.x = pk16(v[0][f], v[1][f]); o.y = pk16(v[2][f], v[3][f]);
        o.z = pk16(v[4][f], v[5][f]); o.w = pk16(v[6][f], v[7][f]);
        st16_sys_p(dst + f * 1024, o);
    }
}
__device__ __forceinline__ void conv_w16(const Params& p, int tj, int col0, int h, int lane, const f32x4* smp, int& se) {
    const int g = lane >> 4, q = lane & 15;
    const float* wsrc = p.w + (size_t)(h * 32 + 4 * g) * NL + col0 + 128 + q;
    float e[8];
#pragma unroll
    for (int i = 0; i < 8; ++i) e[i] = __builtin_nontemporal_load(wsrc + (size_t)((i & 3) + 16 * (i >> 2)) * NL);
    se = scale_exp(*smp);
    const float sc = exp2i(se);
#pragma unroll
    for (int i = 0; i < 8; ++i) e[i] *= sc;
    u32x4 o;
    o.x = pk16(e[0], e[1]); o.y = pk16(e[2], e[3]); o.z = pk16(e[4], e[5]); o.w = pk16(e[6], e[7]);
    st16_sys_p(p.myws + OFF_W16 + (size_t)(tj * NH + h) * WB + 8192 + lane * 16, o);
}
__device__ __forceinline__ void conv_wblock(const Params& p, int tj, int col0, bool wide, int h, int lane, float sc) {
    const int g = lane >> 4, q = lane & 15;
    const float* wrow = p.w + (size_t)(h * 32 + 4 * g) * NL + col0;
    f32x4 v[2][8];
    float e[8];
#pragma unroll
    for (int cg = 0; cg < 2; ++cg)
#pragma unroll
        for (int i = 0; i < 8; ++i)
            v[cg][i] = __builtin_nontemporal_load((const f32x4*)(wrow + 4 * q + (size_t)((i & 3) + 16 * (i >> 2)) * NL + cg * 64));
    if (wide) {
#pragma unroll
        for (int i = 0; i < 8; ++i)
            e[i] = __builtin_nontemporal_load(wrow + (size_t)((i & 3) + 16 * (i >> 2)) * NL + 128 + q);
    }
    char* dst = p.myws + OFF_W16 + (size_t)(tj * NH + h) * WB + lane * 16;
#pragma unroll
    for (int cg = 0; cg < 2; ++cg)
#pragma unroll
        for (int f = 0; f < 4; ++f) {
            u32x4 o;
            o.x = pk16(v[cg][0][f] * sc, v[cg][1][f] * sc); o.y = pk16(v[cg][2][f] * sc, v[cg][3][f] * sc);
            o.z = pk16(v[cg][4][f] * sc, v[cg][5][f] * sc); o.w = pk16(v[cg][6][f] * sc, v[cg][7][f] * sc);
            st16_sys_p(dst + (cg * 4 + f) * 1024, o);
        }
    if (wide) {
        u32x4 o;
        o.x = pk16(e[0] * sc, e[1] * sc); o.y = pk16(e[2] * sc, e[3] * sc);
        o.z = pk16(e[4] * sc, e[5] * sc); o.w = pk16(e[6] * sc, e[7] * sc);
        st16_sys_p(dst + 8192, o);
    }
}

template <int NB>
__device__ __forceinline__ void consume(const Params& p, const unsigned ldsb, const int wave, const int lane,
                                        const int ti, const int col0, const unsigned call, const int wexp) {
    const int wr = wave & 3, wc = wave >> 2;
    const int g = lane >> 4, q = lane & 15;
    f32x4 acc[4][NB];
#pragma unroll
    for (int m = 0; m < 4; ++m)
#pragma unroll
        for (int n = 0; n < NB; ++n) acc[m][n] = (f32x4){0.f, 0.f, 0.f, 0.f};
    KArgs k;
    k.va0 = ldsb + wr * 4096 + lane * 16;
    k.vb0 = ldsb + 16384 + wc * 4096 + lane * 16;
    k.vfull = ldsb + CTRL + C_FULL + (lane < 2 ? lane : 2) * 4;
    k.vfree = ldsb + CTRL + C_FREE + wave * 4;
    k.l16 = lane * 16;
    k.tag = ((call + 1u) >> 1) & 1u;
    k.bad = ldsb + CTRL + C_BAD;
    k.xsrc = p.myws + rfl((unsigned)OFF_X16 + (call & 1u) * (unsigned)X16_B + (unsigned)ti * (NH * 16384) + wr * 4096);
    float* obase = p.out + (size_t)(ti * 256 + wr * 64 + 4 * g) * NL + col0 + wc * 64 + 4 * q;
    asm volatile("" : "+v"(obase));
    unsigned nfix = 0;
    __builtin_amdgcn_s_setprio(PRIO_C);
    kloop(acc, k, nfix);
    __builtin_amdgcn_s_setprio(0);
    STAMP(3);
    const u32x4 cw = lds_rd128_wait(ldsb + CTRL + C_RDYB);
    const int oe = wexp + (int)rfl(cw.w) - 64;
    const float c1 = exp2i(-oe) * -1.44269504088896340736f;
    const float io = rfl(cw.y) != 0u ? __builtin_nanf("") : exp2i(oe);
    const f32x2 c1v = {c1, c1}, iov = {io, io};
    float* oext = obase + 128 - 3 * q;
#pragma unroll
    for (int m = 0; m < 4; ++m)
#pragma unroll
        for (int rp = 0; rp < 2; ++rp) {
            f32x4 o[2];
            float ex[2];
            f32x2 e[NB];
#pragma unroll
            for (int n = 0; n < NB; ++n) e[n] = (f32x2){acc[m][n][2 * rp], acc[m][n][2 * rp + 1]} * c1v;
#pragma unroll
            for (int n = 0; n < NB; ++n) {
                e[n].x = __builtin_amdgcn_fmed3f(e[n].x, -86.5617024533378f, 86.5617024533378f);
                e[n].y = __builtin_amdgcn_fmed3f(e[n].y, -86.5617024533378f, 86.5617024533378f);
            }
            SB0();
#pragma unroll
            for (int n = 0; n < NB; ++n) { e[n].x = __builtin_amdgcn_exp2f(e[n].x); e[n].y = __builtin_amdgcn_exp2f(e[n].y); }
            SB0();
#pragma unroll
            for (int n = 0; n < NB; ++n) e[n] = __builtin_elementwise_fma(e[n], iov, iov);
            SB0();
#pragma unroll
            for (int n = 0; n < NB; ++n) { e[n].x = __builtin_amdgcn_rcpf(e[n].x); e[n].y = __builtin_amdgcn_rcpf(e[n].y); }
            SB0();
#pragma unroll
            for (int n = 0; n < NB; ++n) {
                float v0 = acc[m][n][2 * rp] * e[n].x, v1 = acc[m][n][2 * rp + 1] * e[n].y;
                asm("" : "+v"(v0)); asm("" : "+v"(v1));
                if (n < 4) { o[0][n] = v0; o[1][n] = v1; } else { ex[0] = v0; ex[1] = v1; }
            }
#pragma unroll
            for (int r = 0; r < 2; ++r) {
                const size_t ro = (size_t)(m * 16 + 2 * rp + r) * NL;
                st16_out((const char*)(obase + ro), __builtin_bit_cast(u32x4, o[r]));
                if constexpr (NB == 5) st4_out((const char*)(oext + ro), __builtin_bit_cast(unsigned, ex[r]));
            }
        }
    STAMP(4);
}

__global__ void __launch_bounds__(NWAVE * 64, 3) ag_gemm_silu(Params p) {
    extern __shared__ __attribute__((aligned(16))) char lds_raw[];
    LAS char* lds = (LAS char*)lds_raw;
    const unsigned ldsb = (unsigned)(size_t)lds;
    const int tid = threadIdx.x;
    const int wave = rfl(tid >> 6), lane = tid & 63;
    const int b = blockIdx.x;
    char* myws = p.myws;
    const int g = lane >> 4, q = lane & 15;
    unsigned* cntp = (unsigned*)(myws + OFF_CNT) + b * 16 + wave;
    STAMP(0);

    if (b < NCWG) {
        if (b == 0 && wave == CWPG) {
            const f32x4 sv = *x_sample(p.x, lane);
            const unsigned call = rfl(__hip_atomic_load(cntp, __ATOMIC_RELAXED, __HIP_MEMORY_SCOPE_AGENT)) + 1u;
            const int xe = scale_exp(sv);
            const char* d = p.dst[0];
#pragma unroll
            for (int j = 1; j < NR; ++j) d = (lane == j) ? p.dst[j] : d;
            if (lane < NR) st4_sys_p(d + OFF_XEXP + (call & 1u) * 1024u + p.rank * 128, call * 256u + (unsigned)(xe + 64));
            if (lane == 0) __hip_atomic_store(cntp, call, __ATOMIC_RELAXED, __HIP_MEMORY_SCOPE_AGENT);
            return;
        }
        if (b >= COMM_WGS || wave >= CWPG) return;
        fp16_saturate();
        constexpr int P = COMM_P, NF = COMM_NF;
        constexpr int ND = NR - 1;
        constexpr int NW = NCW;
        static_assert(ND + (P - 1) * (3 + ND) <= 63 && NF >= 2 && NF * P * NW <= NITEM, "comm window");
        const int cw = wave * COMM_WGS + b;
        const float* xs = p.x + (size_t)q * KD + 4 * g;
        f32x4 xa[P][2];
        unsigned av[P];
        unsigned cur[P];
#define XLD(d0_, d1_, i_) asm volatile("global_load_dwordx4 %0, %2, off nt\n\tglobal_load_dwordx4 %1, %2, off offset:64 nt" \
                         : "=&v"(d0_), "=&v"(d1_) : "v"(xs + (size_t)(((i_) & 31u) * (16 * KD) + ((i_) >> 5) * 32)) : "memory")
#define VOFF(i_) ((((i_) >> 4) & 1u) * (NH * 16384u) + ((i_) >> 5) * 16384u + ((i_) & 15u) * 1024u + lane * 16u)
        unsigned cv;
        f32x4 sv;
        asm volatile("global_load_dwordx4 %0, %1, off" : "=v"(sv) : "v"(x_sample(p.x, lane)) : "memory");
        asm volatile("global_load_dword %0, %1, off sc1" : "=v"(cv) : "v"(cntp) : "memory");
#pragma unroll
        for (int s = 0; s < P; ++s) { cur[s] = s * NW + cw; av[s] = 0u; XLD(xa[s][0], xa[s][1], cur[s]); }
        asm volatile("s_waitcnt vmcnt(%2)" : "+v"(cv), "+v"(sv) : "i"(2 * P) : "memory");
        const float xsc = exp2i(scale_exp(sv));
        const unsigned call = rfl(cv) + 1u;
        const unsigned tag = ((call + 1u) >> 1) & 1u;
        const size_t doff = OFF_X16 + (size_t)(call & 1u) * X16_B + (size_t)(p.rank * 2) * (NH * 16384);
        const char* dst[ND];
#pragma unroll
        for (int j = 0; j < ND; ++j) dst[j] = p.dst[j] + doff;
        const char* own = myws + doff;
        const char* ictr = myws + OFF_ICTR + (call & 1u) * 256u;
        const unsigned vzero = 0u, vone = 1u;
        auto body = [&](auto sc, auto rc) -> bool {
            constexpr int s = decltype(sc)::value, r = decltype(rc)::value;
            constexpr int OPR = ND + (r == NF - 2 ? 3 : 2), OPP = ND + (r - 1 == NF - 2 ? 3 : 2);
            constexpr int W = r == 0 ? 2 * (P - 1 - s) + s * OPR : ND + (P - 1 - s) * OPP + s * OPR;
            if constexpr (r < NF)
                asm volatile("s_waitcnt vmcnt(%3)" : "+v"(xa[s][0]), "+v"(xa[s][1]), "+v"(av[s]) : "i"(W) : "memory");
            else
                asm volatile("s_waitcnt vmcnt(%2)" : "+v"(xa[s][0]), "+v"(av[s]) : "i"(W) : "memory");
            const unsigned it = cur[s];
            if (it >= (unsigned)NITEM) return false;
            const unsigned nxt = r < NF - 1 ? (unsigned)(((r + 1) * P + s) * NW + cw) : (unsigned)(NF * P * NW) + rfl(av[s]);
            u32x4 o;
            if constexpr (r < NF) {
                o = cvt_tag(xa[s][0] * xsc, xa[s][1] * xsc, tag);
            } else {
                asm volatile("v_mov_b32 %0, %4\n\tv_mov_b32 %1, %5\n\tv_mov_b32 %2, %6\n\tv_mov_b32 %3, %7"
                             : "=&v"(o.x), "=&v"(o.y), "=&v"(o.z), "=&v"(o.w)
                             : "v"(xa[s][0].x), "v"(xa[s][0].y), "v"(xa[s][0].z), "v"(xa[s][0].w));
                if (__ballot((o.x & 1u) != tag) != 0ull) {
                    f32x4 y0, y1;
                    XLD(y0, y1, it);
                    asm volatile("s_waitcnt vmcnt(0)" : "+v"(y0), "+v"(y1) :: "memory");
                    o = cvt_tag(y0 * xsc, y1 * xsc, tag);
                }
            }
            asm volatile("" : "+v"(o));
            if constexpr (r >= NF - 2)
                asm volatile("s_mov_b64 exec, 1\n\tglobal_atomic_add %0, %1, %2, %3 sc0 sc1\n\ts_mov_b64 exec, -1"
                             : "=&v"(av[s]) : "v"(vzero), "v"(vone), "s"(ictr) : "memory");
            if constexpr (r < NF - 1) {
                XLD(xa[s][0], xa[s][1], nxt);
            } else {
                const unsigned in = nxt < (unsigned)NITEM ? nxt : (unsigned)NITEM - 1u;
                asm volatile("global_load_dwordx4 %0, %1, %2 sc0 sc1" : "+v"(xa[s][0]) : "v"(VOFF(in)), "s"(own) : "memory");
            }
            cur[s] = nxt;
            const unsigned voff = VOFF(it);
#pragma unroll
            for (int j = 0; j < ND; ++j) st16_sys(dst[j], voff, o);
            return true;
        };
        static_for<NF>([&](auto rc) { static_for<P>([&](auto sc) { body(sc, rc); }); });
        for (bool go = true; go;) {
            static_for<P>([&](auto sc) { if (go) go = body(sc, IC<NF>{}); });
        }
#pragma unroll
        for (int s = 0; s < P; ++s)
            asm volatile("s_waitcnt vmcnt(0)" : "+v"(xa[s][0]), "+v"(av[s]) :: "memory");
        if (lane == 0) {
            __hip_atomic_store(cntp, call, __ATOMIC_RELAXED, __HIP_MEMORY_SCOPE_AGENT);
            if (cw == 0) st4_sys_p(myws + OFF_ICTR + ((call + 1u) & 1u) * 256u, 0u);
        }
        STAMP(1);
        return;
    }

    const int xcd = b & 7;
    const int t = (xcd & 1) * 30 + (b >> 3) - 2;
    const int tj = t >> 2;
    const int ti = (xcd >> 1) * 4 + (t & 3);
    const bool wide = !(tj & 1);
    const int col0 = ((tj * 17 + 1) >> 1) * 16;

    const unsigned call = rfl(__hip_atomic_load(cntp, __ATOMIC_RELAXED, __HIP_MEMORY_SCOPE_AGENT)) + 1u;
    if (lane == 0) __hip_atomic_store(cntp, call, __ATOMIC_RELAXED, __HIP_MEMORY_SCOPE_AGENT);
    unsigned* tmo = (unsigned*)(myws + OFF_TMO);

    if (tid < 64) *(LAS unsigned*)(lds + CTRL + tid * 4) = tid >= C_FREE / 4 ? (unsigned)NSLOT : 0u;
    asm volatile("s_waitcnt lgkmcnt(0)" ::: "memory");
    __builtin_amdgcn_s_barrier();
    asm volatile("" ::: "memory");

    if (wave < 8) {
        static_assert(W_JC == 1, "phase 0");
        fp16_saturate();
        int wexp;
        const f32x4* smp = w_sample(p.w, lane);
        if (wave < 3) {
            if (wave < 2) conv_w64(p, tj, col0, ti, wave, lane, smp, wexp);
            else if (wide) conv_w16(p, tj, col0, ti, lane, smp, wexp);
            else wexp = scale_exp(*smp);
            VMW(0);
            if (lane == 0) asm volatile("ds_add_u32 %0, %1" :: "v"(ldsb + CTRL + C_WCNT), "v"(1u) : "memory");
        } else {
            const unsigned u = (wave - 3) * (256 - NCWG) + (b - NCWG);
            if (u < 1024u) {
                const float* xs = p.x + (size_t)((u & 31u) * 16u + q) * KD + (u >> 5) * 128u + 4 * g;
                f32x4 v[8];
#pragma unroll
                for (int i = 0; i < 8; ++i) v[i] = *(const f32x4*)(xs + i * 16);
                const float xsc = exp2i(scale_exp(*x_sample(p.x, lane)));
                const unsigned tag = ((call + 1u) >> 1) & 1u;
                char* dst = myws + OFF_X16 + (size_t)(call & 1u) * X16_B + (size_t)(p.rank * 2 + ((u >> 4) & 1u)) * (NH * 16384) +
                            (u >> 5) * 65536u + (u & 15u) * 1024u + lane * 16;
#pragma unroll
                for (int i = 0; i < 4; ++i) st16_sys_p(dst + i * 16384, cvt_tag(v[2 * i] * xsc, v[2 * i + 1] * xsc, tag));
            }
            wexp = scale_exp(*smp);
        }
        STAMP(1);
        DBGRET(1);
        if (wide && wave < 4) consume<5>(p, ldsb, wave, lane, ti, col0, call, wexp);
        else consume<4>(p, ldsb, wave, lane, ti, col0, call, wexp);
    } else if (wave < 11) {
        DBGRET(1);
        __builtin_amdgcn_s_setprio(PRIO_L);
        const int L = wave - 8;
        const bool nine = (L == 2) && wide;
        const unsigned soff = (L < 2) ? (unsigned)OFF_X16 + (call & 1u) * (unsigned)X16_B + (unsigned)ti * (NH * 16384) + L * 8192
                                      : (unsigned)OFF_W16 + (unsigned)tj * (NH * WB);
        const char* src = myws + rfl(soff);
        const unsigned step = (L < 2) ? 16384u : (unsigned)WB;
        const unsigned ldst0 = ldsb + L * 8192;
        const unsigned fullad = ldsb + CTRL + C_FULL + L * 4;
        const unsigned ctlad = ldsb + CTRL + ((lane < 8) ? C_FREE + lane * 4 : (L < 2 ? C_RDYA : C_RDYB));
        const unsigned voff = lane * 16;
        unsigned h = 0, pub = 0, slot = 0;
        unsigned long long tw = 0; unsigned sp = 0;
        bool dead = false;
        auto issue = [&]() {
            glds8(src, voff, voff + 4096, ldst0 + slot);
            if (nine) glds1(src, voff + 8192, ldst0 + slot + 8192);
            src += step; ++h;
            slot = (slot == (NSLOT - 1) * SLOT_B) ? 0u : slot + SLOT_B;
        };
        VMW(0);
        while (pub < NH) {
            unsigned cv = lds_rd32_wait(ctlad);
            if (dead) cv = NH;
            bool act = false;
            if (__ballot(cv <= h) == 0ull) {
                issue();
                act = true;
                if (__ballot(cv <= h) == 0ull) issue();
            }
            const unsigned vm = vm_outstanding();
            const unsigned landed = h - (nine ? ((vm + 8u) * 57u) >> 9 : (vm + 7u) >> 3);
            if (landed > pub) { pub = landed; lds_wr32(fullad, pub); act = true; }
            if (act) { sp = 0; continue; }
            if (sp == 0) tw = rt();
            if ((++sp & 1023u) == 0u && rt() - tw > 2 * DEADLINE) { lds_wr32(ldsb + CTRL + C_BAD, 1u); dead = true; }
            if (h == pub) __builtin_amdgcn_s_sleep(1);
        }
        STAMP(1);
    } else {
        __builtin_amdgcn_s_setprio(PRIO_P);
        const unsigned tag = ((call + 1u) >> 1) & 1u;
        const char* wflagp = myws + OFF_WFLAG + (unsigned)(tj * 16 + (lane & 15)) * 128u;
        const char* xsrc = myws + rfl((unsigned)OFF_X16 + (call & 1u) * (unsigned)X16_B + (unsigned)ti * (NH * 16384));
        unsigned ra = 0;
        unsigned pp = 0;
        unsigned rbw = 0, pw = POLL_W;
        bool wok = false;
        const unsigned wbase = call * 256u;
        int jn = W_JC;
        int jpub = 0;
        bool p0 = false;
        const unsigned tstart = (unsigned)rt();
        fp16_saturate();
        const float wsc = exp2i(scale_exp(*w_sample(p.w, lane)));
        const char* xexpp = myws + OFF_XEXP + (call & 1u) * 1024u + (ti >> 1) * 128;
        bool xok = false;
        unsigned long long tw = 0; unsigned sp = 0;
        while (ra < NH || !wok) {
            unsigned fv = tag ^ 1u, wv = 0, xv = 0;
            const unsigned hx = ra + (lane >> 4) - 1u;
            const bool probe = lane < pw && pp + lane < (unsigned)NH;
            const bool piece = lane >= 16 && lane < 48 && hx >= (unsigned)(NH - POLL_NX) && hx < (unsigned)NH &&
                               (hx < pp || hx >= (unsigned)(NH - POLL_ND));
            const unsigned off = probe ? (pp + lane) * 16384u + 15u * 1024u + 1008u : hx * 16384u + (lane & 15) * 1024u + 1008u;
            if (!wok && lane < 16)
                asm volatile("global_load_dword %0, %1, off sc0 sc1" : "=v"(wv) : "v"(wflagp) : "memory");
            if (!xok && lane == 0)
                asm volatile("global_load_dword %0, %1, off sc0 sc1" : "=v"(xv) : "v"(xexpp) : "memory");
            if (probe || piece)
                asm volatile("global_load_dword %0, %1, %2 sc0 sc1" : "=v"(fv) : "v"(off), "s"(xsrc) : "memory");
            asm volatile("s_waitcnt vmcnt(0)" : "+v"(fv), "+v"(wv), "+v"(xv) :: "memory");
            bool act = false;
            if (!xok) {
                const unsigned xe = rfl(xv) - wbase;
                if (xe < 256u) { xok = true; lds_wr32(ldsb + CTRL + C_XEXP, xe); }
            }
            if (jpub < 8) {
                if (!p0) p0 = rfl(lds_rd32_wait(ldsb + CTRL + C_WCNT)) == 3u;
                if (p0 && jn > jpub) {
                    jpub = jn;
                    if (lane == 0) st4_sys_p(myws + OFF_WFLAG + (size_t)(tj * 16 + ti) * 128, wbase + (unsigned)jpub);
                }
            }
            if (!wok) {
                unsigned c = wv - wbase;
                c = (c <= 8u ? c : 0u) * 16u + (lane & 15);
                const unsigned nb = umin(rfl(dppmin<0x140>(dppmin<0x141>(dppmin<0x4E>(dppmin<0xB1>(c))))), (unsigned)NH);
                if (nb > rbw) { rbw = nb; lds_wr32(ldsb + CTRL + C_RDYB, nb); act = true; }
                wok = nb == (unsigned)NH;
            }
            if (ra < NH) {
                const unsigned long long m = ~__ballot((fv & 1u) == tag);
                const unsigned np = (unsigned)__builtin_ctzll(m | 0x10000ull);
                pw = (np == pw) ? umin(2u * pw, 16u) : (unsigned)POLL_W;
                const unsigned nx = (unsigned)__builtin_ctzll((m >> 16) | (1ull << 32)) >> 4;
                pp += np;
                unsigned rn = ra + nx;
                const unsigned rb = umin(pp > (unsigned)POLL_LAG ? pp - POLL_LAG : 0u, (unsigned)(NH - POLL_NX));
                rn = rn > rb ? rn : rb;
                if (!xok) rn = umin(rn, (unsigned)(NH - 1));
                if (rn > ra) { ra = rn; lds_wr32(ldsb + CTRL + C_RDYA, ra); act = true; }
                if (np) act = true;
            }
            if (jn < 8) {
                const int hw = 16 * jn + ti;
                if ((int)ra + W_LEAD >= hw || (int)((unsigned)rt() - tstart) >= (hw - W_LEAD) * W_TICK) {
                    conv_wblock(p, tj, col0, wide, hw, lane, wsc);
                    ++jn; act = true;
                }
            }
            if (act) { sp = 0; continue; }
            if (sp == 0) tw = rt();
            if ((++sp & 63u) == 0u) {
                const unsigned tv = rfl(__hip_atomic_load(tmo, __ATOMIC_RELAXED, __HIP_MEMORY_SCOPE_AGENT));
                if (tv == call || rt() - tw > DEADLINE) {
                    if (lane == 0) __hip_atomic_store(tmo, call, __ATOMIC_RELAXED, __HIP_MEMORY_SCOPE_AGENT);
                    lds_wr32(ldsb + CTRL + C_BAD, 1u);
                    lds_wr32(ldsb + CTRL + C_RDYA, NH);
                    lds_wr32(ldsb + CTRL + C_RDYB, NH);
                    break;
                }
            }
            __builtin_amdgcn_s_sleep(POLL_SLEEP);
        }
        STAMP(1);
    }
}

}

extern "C" void kernel_launch(int rank, int nranks, ncclComm_t comm,
                              hipStream_t stream,
                              void* const* d_in, const int* in_sizes, int n_in,
                              void* d_out, int out_size,
                              const int* in_dtypes, int out_dtype,
                              void* const* d_out_all,
                              void* const* d_ws_all, size_t ws_bytes) {
    using namespace agk;
    if (nranks != NR || (unsigned)rank >= (unsigned)NR || ws_bytes < WS_NEED || n_in < 2) {
        fprintf(stderr, "ag_gemm_silu: unexpected configuration\n");
        return;
    }
    static bool attr_done = false;
    if (!attr_done) {
        HIP_CHECK(hipFuncSetAttribute((const void*)ag_gemm_silu, hipFuncAttributeMaxDynamicSharedMemorySize, LDS_B));
        attr_done = true;
    }
    Params p;
    p.x = (const float*)d_in[0];
    p.w = (const float*)d_in[1];
    p.out = (float*)d_out;
    p.myws = (char*)d_ws_all[rank];
    for (int i = 0; i < NR; ++i) p.dst[i] = (char*)d_ws_all[(rank + 1 + i) & 7];
    p.rank = rank;
    ag_gemm_silu<<<dim3(256), dim3(NWAVE * 64), LDS_B, stream>>>(p);
    HIP_CHECK(hipGetLastError());
}
```

```cpp
#include <hip/hip_runtime.h>
#include <rccl/rccl.h>
#include <cstdio>
#include <cstddef>

#define HIP_CHECK(expr) do { hipError_t _e = (expr); if (_e != hipSuccess) { \
    fprintf(stderr, "HIP error %s at %s:%d\n", hipGetErrorString(_e), __FILE__, __LINE__); \
    return; } } while (0)

#define LAS __attribute__((address_space(3)))
typedef _Float16 f16x8 __attribute__((ext_vector_type(8)));
typedef _Float16 f16x2 __attribute__((ext_vector_type(2)));
typedef float f32x4 __attribute__((ext_vector_type(4)));
typedef float f32x2 __attribute__((ext_vector_type(2)));
typedef unsigned u32x4 __attribute__((ext_vector_type(4)));
typedef unsigned u32x2 __attribute__((ext_vector_type(2)));

namespace agk {

constexpr int NR = 8;
constexpr int KD = 4096;
constexpr int NL = 2048;
constexpr int NH = 128;
constexpr int NSLOT = 16;
#define TAIL_N 16
constexpr int NT = TAIL_N;
constexpr int NM = NH - NT;
#define KDA 3
constexpr int KD_A = KDA;
constexpr unsigned NS = 1000;
constexpr int WB = 9216;
constexpr int CTRL = NSLOT * WB;
constexpr int LDS_B = CTRL + 512;
constexpr int NWAVE = 12;
constexpr int NCWG = 16;
#define COMM_P 3
#define COMM_NF 3
#define COMM_DIR 0
#define DRY_N 3
#define CODE_PF 64
#define SW_N 0
#define SW_I 1
#define SW_W 9
#define COMM_WGS 16
#define COMM_WPG 2
#define PRIO_C 2
#define PRIO_S 0
#define PRIO_E 0
#define PRIO_E1 0
#define PRIO_E3 0
#define PRIO_L 1
#define PRIO_P 1
#define LD_SLEEP 1
#define POLL_SLEEP 16
#define POLL_SLEEP2 16
#define W_JC 1
#define W_LEAD 40
#define W_NUM 16
#define W_TICK 55
#define W_TMIN 500
#define W_TSTEP 100
#define W0_DLY 100
#define W0_CAP 16
#define W0_STEP 20
#define X0_DLY 100
#define X_DSTEP 20
#define X_DLY 200
#define X_DK 8
#define LSTEP 3
#define POLL_W0 2
#define XC_MAP 0xAAABA
#define POLL_W 2
constexpr int CWPG = COMM_WPG;
constexpr int NCW = COMM_WGS * COMM_WPG;
#define PULL_F 17
#define PF_N (128 - PULL_F)
#define PF_W 5
#define PF_W2 PF_W
#define PF_S2 32
#define PF_W3 PF_W2
#define PF_S3 112
#define OWN_X 0
#define PF_WV 4
#define PF_DYN 0
#define PF_K 4
#define LD_WIN 40
#define X_NA 2
#define EARLY_N 0
#define EARLY_PF 2
#define EARLY_W 7
#define PF_IL 1
#define X_DK2 32
#define X_DLY2 0
#define LD_ORD 2
#define PF_PROBE 0
#define T_GO 0
#define T_POLL 500
#define PF_SLEEP 4
#define XP_LATE 0
constexpr int PF = PULL_F;
constexpr int NA = X_NA * (256 - NCWG) / 32 * 4;
static_assert(NA >= PF + EARLY_PF + 3 && NA <= NM, "early part");
static_assert(PF <= NM && PF + PF_N <= NH, "pushed part");
constexpr int NITEM = PF * 32;

constexpr int C_FULL = 256;
constexpr int C_ALL = 40;
constexpr int C_RDYB = 16;
constexpr int C_BAD = 20;
constexpr int C_WCNT = 24;
constexpr int C_XCNT = 12;
constexpr int C_XEXP = 32;
constexpr int C_RDYA = 28;
constexpr int C_TOK = 44;
constexpr int C_FREE = 64;

constexpr size_t OFF_CNT = 0;
constexpr size_t OFF_TMO = 32768;
constexpr size_t OFF_ICTR = 65536;
constexpr size_t OFF_XCNT = 81920;
constexpr size_t OFF_XEXP = 98304;
constexpr size_t OFF_PLAN = OFF_XEXP + 12288;
constexpr size_t OFF_WFLAG = 131072;
constexpr size_t OFF_DBG = 262144;
#define TILE_GAP 4096
constexpr unsigned OFF_RX = 2u << 20;
constexpr unsigned RT_B = PF * 16384 + TILE_GAP;
static_assert(TILE_GAP + 4 * RT_B <= (2u << 20), "receive area");
constexpr size_t OFF_X16 = OFF_RX + (16u << 20) + TILE_GAP;
constexpr unsigned TILE_B = NH * 16384 + TILE_GAP;
constexpr size_t X16_B = 2ull * TILE_B;
constexpr size_t OFF_W16 = OFF_X16 + 2 * X16_B;
constexpr size_t WS_NEED = OFF_W16 + (size_t)15 * NH * WB;
constexpr unsigned long long DEADLINE = 50000000ull;

struct Params {
    const float* x;
    const float* w;
    float* out;
    char* myws;
    char* dst[NR];
    int rank;
    unsigned map;
    int dbg, pad;
};
#define DBGRET(bit) do { } while (0)

#define MAPX 8
inline unsigned xcd_map(int rank) {
    unsigned m = 0;
    const unsigned grp = 0xb2u;
    int n0 = 0, n1 = 4;
    for (int j = 0; j < 8; ++j) {
        const int r = (rank + j) & 7;
        const bool same = ((grp >> r) & 1u) == ((grp >> rank) & 1u);
        int x = (same != (MAPX == 9)) ? n0++ : n1++;
        x = x == 0 ? OWN_X : (x == OWN_X ? 0 : x);
        m |= (unsigned)r << (4 * x);
    }
    return m;
}
__host__ __device__ __forceinline__ int map_tj(int b) { return ((b >> 3) - 2) >> 1; }
__host__ __device__ __forceinline__ int map_src(int b, unsigned map) { return (map >> (4 * (b & 7))) & 7; }
__device__ __forceinline__ int my_src(unsigned map) { return map_src(__builtin_amdgcn_s_getreg(20 | (3 << 11)), map); }
inline int map_ti(int b, int rank) { return map_src(b, xcd_map(rank)) * 2 + ((b >> 3) & 1); }

template <int V> struct IC { static constexpr int value = V; };
template <int N, int I = 0, class F> __device__ __forceinline__ void static_for(F&& f) {
    if constexpr (I < N) { f(IC<I>{}); static_for<N, I + 1>(f); }
}

__device__ __forceinline__ unsigned long long rt() {
    unsigned long long t;
    asm volatile("s_memrealtime %0\n\ts_waitcnt lgkmcnt(0)" : "=s"(t) :: "memory");
    return t;
}

__device__ __forceinline__ const char* rank_ws(const Params& p, int r) {
    const int i = (r - p.rank - 1) & 7;
    const char* d = p.dst[0];
#pragma unroll
    for (int j = 1; j < NR; ++j) d = (i == j) ? p.dst[j] : d;
    const unsigned long long a = (unsigned long long)d;
    return (const char*)(((unsigned long long)(unsigned)__builtin_amdgcn_readfirstlane((unsigned)(a >> 32)) << 32) |
                         (unsigned)__builtin_amdgcn_readfirstlane((unsigned)a));
}

__device__ __forceinline__ unsigned pk16(float a, float b) {
    f32x2 v = {a, b};
    f16x2 h = __builtin_convertvector(v, f16x2);
    return __builtin_bit_cast(unsigned, h);
}

__device__ __forceinline__ unsigned pk16s(float a, float b, float sc) {
    a *= sc; b *= sc;
    asm("" : "+v"(a)); asm("" : "+v"(b));
    return pk16(a, b);
}

__device__ __forceinline__ unsigned piece_pos(unsigned h, unsigned rb) {
    const unsigned j = h - NM, st = (rb & 1u) * 2u + (rb >> 3);
    return h < (unsigned)NM ? h * 16u + rb : (unsigned)(NM * 16) + (st * (NT / 2) + (j >> 1)) * 8u + (rb & 6u) + (j & 1u);
}

__device__ __forceinline__ u32x4 cvt_tag(f32x4 u0, f32x4 u1, unsigned tag) {
    const unsigned rn = pk16(u0.x, u0.y);
    unsigned t = __builtin_bit_cast(unsigned, __builtin_amdgcn_cvt_pkrtz(u0.x, 0.0f));
    t += (t ^ tag) & 1u;
    u32x4 o;
    o.x = (rn & 0xffff0000u) | (t & 0xffffu);
    o.y = pk16(u0.z, u0.w);
    o.z = pk16(u1.x, u1.y); o.w = pk16(u1.z, u1.w);
    return o;
}

#define POL_COMM "sc0 sc1"
#define POL_OUT "nt"
#define POL_DMA "sc0 sc1"
#define POL_B2 "sc0 sc1 nt"
#define POL_CV "sc0 sc1"
#define XC_NT 0
#define POL_FULL ""
#define POL_FAR ""
#define POL_B "sc0 sc1"
#define TAIL_DRY 0
#define LD_WINT 9
constexpr int DRY_W = 8;
#define XS_MODE 4
#define CW_MAP 0
#define PRIO_M 0
#define MOV32 0
__device__ __forceinline__ void st16_sys(const char* sbase, unsigned voff, u32x4 v) {
    asm volatile("global_store_dwordx4 %0, %1, %2 " POL_COMM "\n\ts_nop 1" :: "v"(voff), "v"(v), "s"(sbase) : "memory");
}
template <int O> __device__ __forceinline__ void st16_out(const float* sbase, unsigned voff, u32x4 v) {
    asm volatile("global_store_dwordx4 %0, %1, %2 offset:%3 " POL_OUT "\n\ts_nop 1" :: "v"(voff), "v"(v), "s"(sbase), "i"(O) : "memory");
}
__device__ __forceinline__ void st16_sys_p(const char* p, u32x4 v) {
    asm volatile("global_store_dwordx4 %0, %1, off " POL_CV "\n\ts_nop 1" :: "v"(p), "v"(v) : "memory");
}
__device__ __forceinline__ void st4_sys_p(const char* p, unsigned v) {
    asm volatile("global_store_dword %0, %1, off sc0 sc1" :: "v"(p), "v"(v) : "memory");
}

#define VMW(n) asm volatile("s_waitcnt vmcnt(" #n ")" ::: "memory")

struct StepTab {
    unsigned v[NH + 1];
    constexpr StepTab() : v() { for (int i = 0; i <= NH; ++i) v[i] = (unsigned)i; }
};
__device__ const StepTab k_step = StepTab();

template <int NP, int P2>
__device__ __forceinline__ void glds_run(const char* src, unsigned step, unsigned d0, unsigned voff, unsigned ctrl, unsigned ctl) {
    unsigned cv = 0, vh = voff < 128u ? (unsigned)(LSTEP - NSLOT) : 0u, vnum = 4;
#define GL1(v_, o_) ".if %[p2]\n\tglobal_load_lds_dwordx4 " v_ ", s[84:85] offset:" o_ " " POL_B2 "\n\t.else\n\t" \
                    "global_load_lds_dwordx4 " v_ ", s[84:85] offset:" o_ " " POL_B "\n\t.endif\n\t"
#define GL4(v_) GL1(v_, "0") GL1(v_, "1024") GL1(v_, "2048") GL1(v_, "3072")
    asm volatile(
        "s_mov_b32 s88, m0\n\t"
        "s_mov_b64 s[84:85], %[src0]\n\t"
        "s_mov_b32 s92, %[d0]\n\t"
        "s_mov_b32 s93, %[n0]\n\t"
        "s_mov_b32 s89, 0\n\t"
        "s_mov_b32 s90, 0\n\t"
        "LOOP_%=:\n\t"
        "v_cmp_le_i32 vcc, %[cv], %[vh]\n\t"
        "s_cbranch_vccnz WAIT_%=\n\t"
        "ds_read_b32 %[cv], %[ctl]\n\t"
        "s_mov_b32 m0, s92\n\t"
        "s_waitcnt vmcnt(%[win])\n\t"
        GL4("%[v0]")
        "s_add_u32 m0, s92, 4096\n\t"
        "s_nop 0\n\t"
        GL4("%[v1]")
        ".if %[np] > 8\n\t"
        "s_add_u32 m0, s92, 8192\n\t"
        "s_nop 0\n\t"
        GL1("%[v2]", "0")
        ".endif\n\t"
        "s_add_u32 m0, %[ctrl], %[cful]\n\t"
        "s_add_u32 s92, s92, %[slotb]\n\t"
        "global_load_lds_dword %[vnum], %[tab] " POL_FULL "\n\t"
        "s_cmp_eq_u32 s92, %[ctrl]\n\t"
        "s_cselect_b32 s92, %[d0], s92\n\t"
        "v_add_u32 %[vh], 1, %[vh]\n\t"
        "s_add_u32 s84, s84, %[step]\n\t"
        "s_addc_u32 s85, s85, 0\n\t"
        "v_add_u32 %[vnum], 4, %[vnum]\n\t"
        "s_mov_b32 s89, 0\n\t"
        "s_add_u32 s93, s93, -1\n\t"
        "s_waitcnt lgkmcnt(0)\n\t"
        "s_cbranch_scc1 LOOP_%=\n\t"
        "s_branch DONE_%=\n\t"
        "WAIT_%=:\n\t"
        "s_cmp_eq_u32 s89, 0\n\t"
        "s_cbranch_scc1 W1_%=\n\t"
        "s_sleep %[slp]\n\t"
        "W1_%=:\n\t"
        "ds_read_b32 %[cv], %[ctl]\n\t"
        "s_add_u32 s89, s89, 1\n\t"
        "s_and_b32 s91, s89, 0x3ff\n\t"
        "s_waitcnt lgkmcnt(0)\n\t"
        "s_cbranch_scc1 LOOP_%=\n\t"
        "s_memrealtime s[86:87]\n\t"
        "s_waitcnt lgkmcnt(0)\n\t"
        "s_cmp_eq_u32 s89, 0x400\n\t"
        "s_cselect_b32 s90, s86, s90\n\t"
        "s_sub_u32 s91, s86, s90\n\t"
        "s_cmp_lt_u32 s91, %[dl]\n\t"
        "s_cbranch_scc1 LOOP_%=\n\t"
        "s_add_u32 s91, %[ctrl], %[cbad]\n\t"
        "v_mov_b32 %[ctl], s91\n\t"
        "ds_write_b32 %[ctl], %[ctl]\n\t"
        "s_add_u32 s91, %[ctrl], %[cdead]\n\t"
        "v_mov_b32 %[ctl], s91\n\t"
        "s_branch W1_%=\n\t"
        "DONE_%=:\n\t"
        "s_mov_b32 m0, s88"
        : [cv] "+v"(cv), [vh] "+v"(vh), [vnum] "+v"(vnum), [ctl] "+v"(ctl)
        : [v0] "v"(voff), [v1] "v"(voff + 4096u), [v2] "v"(voff + 8192u),
          [src0] "s"(src), [n0] "i"(NH - 1), [tab] "s"(&k_step.v[0]),
          [d0] "s"(d0), [step] "s"(step), [ctrl] "s"(ctrl),
          [np] "i"(NP), [p2] "i"(P2), [win] "i"(LD_WIN), [slotb] "i"(WB), [slp] "i"(LD_SLEEP), [dl] "i"(2u * (unsigned)DEADLINE),
          [cbad] "i"(C_BAD), [cdead] "i"(C_ALL), [cful] "i"(C_FULL)
        : "memory", "scc", "vcc", "s84", "s85", "s86", "s87", "s88", "s89", "s90", "s91", "s92", "s93");
#undef GL4
#undef GL1
}

__device__ __forceinline__ void lds_wr32(unsigned addr, unsigned v) {
    asm volatile("s_mov_b64 exec, 1\n\tds_write_b32 %0, %1\n\ts_mov_b64 exec, -1" :: "v"(addr), "v"(v) : "memory");
}
__device__ __forceinline__ void lds_wr64(unsigned addr, unsigned v0, unsigned v1) {
    const u32x2 v = {v0, v1};
    asm volatile("s_mov_b64 exec, 1\n\tds_write_b64 %0, %1\n\ts_mov_b64 exec, -1" :: "v"(addr), "v"(v) : "memory");
}
__device__ __forceinline__ u32x4 lds_rd128_wait(unsigned addr) {
    u32x4 v;
    asm volatile("ds_read_b128 %0, %1\n\ts_waitcnt lgkmcnt(0)" : "=&v"(v) : "v"(addr) : "memory");
    return v;
}
__device__ __forceinline__ unsigned lds_rd32_wait(unsigned addr) {
    unsigned v;
    asm volatile("ds_read_b32 %0, %1\n\ts_waitcnt lgkmcnt(0)" : "=&v"(v) : "v"(addr) : "memory");
    return v;
}
__device__ __forceinline__ unsigned rfl(unsigned v) { return __builtin_amdgcn_readfirstlane(v); }
__device__ __forceinline__ unsigned umin(unsigned a, unsigned b) { return a < b ? a : b; }
template <int CTL> __device__ __forceinline__ unsigned dppmin(unsigned d) {
    return umin(d, (unsigned)__builtin_amdgcn_update_dpp((int)d, (int)d, CTL, 0xf, 0xf, false));
}

template <int CTL> __device__ __forceinline__ unsigned dppmax(unsigned d) {
    const unsigned o = (unsigned)__builtin_amdgcn_update_dpp((int)d, (int)d, CTL, 0xf, 0xf, false);
    return d > o ? d : o;
}
__device__ __forceinline__ int scale_exp(f32x4 v) {
    unsigned m; int e;
#define DM(c_) "s_nop 1\n\tv_max_u32_dpp %0, %0, %0 " c_ " bank_mask:0xf\n\t"
    asm volatile("v_max3_f32 %0, |%2|, |%3|, |%4|\n\tv_max_f32 %0, %0, |%5|\n\t"
                 DM("quad_perm:[1,0,3,2] row_mask:0xf") DM("quad_perm:[2,3,0,1] row_mask:0xf") DM("row_half_mirror row_mask:0xf")
                 DM("row_mirror row_mask:0xf") DM("row_bcast:15 row_mask:0xa") DM("row_bcast:31 row_mask:0xc")
                 "s_nop 1\n\tv_readlane_b32 %1, %0, 63\n\ts_bfe_u32 %1, %1, 0x80017"
                 : "=&v"(m), "=s"(e) : "v"(v.x), "v"(v.y), "v"(v.z), "v"(v.w) : "scc");
#undef DM
    const int sx = e < 70 ? 56 : (e > 186 ? -60 : 126 - e);
    return (e == 0 || e == 255) ? 0 : sx;
}
__device__ __forceinline__ float exp2i(int e) { return __builtin_bit_cast(float, (unsigned)(127 + e) << 23); }
__device__ __forceinline__ unsigned x_soff(unsigned lane) {
    return (lane >> 2) * (32u * KD * 4u + 1024u) + (lane & 3u) * 16u;
}
__device__ __forceinline__ const f32x4* x_sample(const float* x, int lane) { return (const f32x4*)((const char*)x + x_soff(lane)); }
__device__ __forceinline__ const f32x4* w_sample(const float* w, int lane) {
    return (const f32x4*)(w + (size_t)(lane * 64 + 17) * NL + ((lane * 132) & (NL - 1)));
}
__device__ __forceinline__ void fp16_saturate() {
    asm volatile("s_setreg_imm32_b32 hwreg(HW_REG_MODE, 23, 1), 1" ::: "memory");
}

#define STAMP(i) do { } while (0)

#define SB0() __builtin_amdgcn_sched_barrier(0)

struct KArgs { unsigned vctl, l16, tag, bad, stage, xc; const char *xloc, *xfar, *xtail, *xt0, *xtl2; const float *outp, *out2; };
#define KSTR2(x) #x
#define KSTR(x) KSTR2(x)
#define KDL "50000000"
#define KDL3 "150000000"
#define KPRC KSTR(PRIO_C)
#define KPRM KSTR(PRIO_M)
#define KPRE KSTR(PRIO_E)
#define KTOK "24"
#define KXEXP "12"
static_assert(C_XEXP - C_BAD == 12 && NL == 2048, "KXEXP");
static_assert(C_TOK - C_BAD == 24, "KTOK");
static_assert(DEADLINE == 50000000ull, "KDL");
#define MF(x) x
#define TG(x) x
#define NTG(x)
#define TRM(x)
#define TRO
#define AGK_CNT 0
#define SC1(x)
#define SC2(x)
#define SC3(x)
static_assert(PF == 17 && NM == 112 && NSLOT == 16 && WB == 9216 && KD_A == 3, "generated for these");
__device__ __forceinline__ void kall8(KArgs& k, unsigned& nfix, unsigned& tr) {
    asm volatile(
        "v_mov_b32 v167, 1\n\t"
        "v_mov_b32 v8, 0\n\t"
        "v_mov_b32 v9, 0\n\t"
        "v_mov_b32 v10, 0\n\t"
        "v_mov_b32 v11, 0\n\t"
        "v_mov_b32 v12, 0\n\t"
        "v_mov_b32 v13, 0\n\t"
        "v_mov_b32 v14, 0\n\t"
        "v_mov_b32 v15, 0\n\t"
        "v_mov_b32 v16, 0\n\t"
        "v_mov_b32 v17, 0\n\t"
        "v_mov_b32 v18, 0\n\t"
        "v_mov_b32 v19, 0\n\t"
        "v_mov_b32 v20, 0\n\t"
        "v_mov_b32 v21, 0\n\t"
        "v_mov_b32 v22, 0\n\t"
        "v_mov_b32 v23, 0\n\t"
        "v_mov_b32 v24, 0\n\t"
        "v_mov_b32 v25, 0\n\t"
        "v_mov_b32 v26, 0\n\t"
        "v_mov_b32 v27, 0\n\t"
        "v_mov_b32 v28, 0\n\t"
        "v_mov_b32 v29, 0\n\t"
        "v_mov_b32 v30, 0\n\t"
        "v_mov_b32 v31, 0\n\t"
        "v_mov_b32 v32, 0\n\t"
        "v_mov_b32 v33, 0\n\t"
        "v_mov_b32 v34, 0\n\t"
        "v_mov_b32 v35, 0\n\t"
        "v_mov_b32 v36, 0\n\t"
        "v_mov_b32 v37, 0\n\t"
        "v_mov_b32 v38, 0\n\t"
        "v_mov_b32 v39, 0\n\t"
        "v_mov_b32 v44, 0\n\t"
        "v_mov_b32 v45, 0\n\t"
        "v_mov_b32 v46, 0\n\t"
        "v_mov_b32 v47, 0\n\t"
        "v_mov_b32 v48, 0\n\t"
        "v_mov_b32 v49, 0\n\t"
        "v_mov_b32 v50, 0\n\t"
        "v_mov_b32 v51, 0\n\t"
        "v_mov_b32 v52, 0\n\t"
        "v_mov_b32 v53, 0\n\t"
        "v_mov_b32 v54, 0\n\t"
        "v_mov_b32 v55, 0\n\t"
        "v_mov_b32 v56, 0\n\t"
        "v_mov_b32 v57, 0\n\t"
        "v_mov_b32 v58, 0\n\t"
        "v_mov_b32 v59, 0\n\t"
        "v_mov_b32 v60, 0\n\t"
        "v_mov_b32 v61, 0\n\t"
        "v_mov_b32 v62, 0\n\t"
        "v_mov_b32 v63, 0\n\t"
        "v_mov_b32 v64, 0\n\t"
        "v_mov_b32 v65, 0\n\t"
        "v_mov_b32 v66, 0\n\t"
        "v_mov_b32 v67, 0\n\t"
        "v_mov_b32 v68, 0\n\t"
        "v_mov_b32 v69, 0\n\t"
        "v_mov_b32 v70, 0\n\t"
        "v_mov_b32 v71, 0\n\t"
        "v_mov_b32 v72, 0\n\t"
        "v_mov_b32 v73, 0\n\t"
        "v_mov_b32 v74, 0\n\t"
        "v_mov_b32 v75, 0\n\t"
        "s_cmp_eq_u32 %[stage], 0\n\t"
        "s_cselect_b32 s98, 0x2000, 0\n\t"
        "s_mov_b32 s89, 1\n\t"
        "s_mov_b64 s[96:97], %[xloc]\n\t"
        "s_movk_i32 s95, 0x4000\n\t"
        "s_add_u32 s88, 0, 9216\n\t"
        "v_add_u32 v165, 0, %[l16]\n\t"
        "s_mov_b32 s94, 0\n\t"
        "SPINI0_%=:\n\t"
        "ds_read_b32 v164, %[vctl]\n\t"
        "s_waitcnt lgkmcnt(0)\n\t"
        "v_cmp_gt_u32_e64 s[86:87], s89, v164\n\t"
        "s_and_b64 s[86:87], s[86:87], 2\n\t"
        "s_cbranch_scc0 GOI0_%=\n\t"
        SC3("s_add_u32 %[nfix], %[nfix], 1\n\t")
        "s_sleep 1\n\t"
        "s_add_u32 s94, s94, 1\n\t"
        "s_and_b32 s92, s94, 0x3ff\n\t"
        "s_cbranch_scc1 SPINI0_%=\n\t"
        "s_memrealtime s[92:93]\n\t"
        "s_waitcnt lgkmcnt(0)\n\t"
        "s_cmp_eq_u32 s94, 0x400\n\t"
        "s_cselect_b32 s90, s92, s90\n\t"
        "s_sub_u32 s92, s92, s90\n\t"
        "s_cmp_lt_u32 s92, " KDL3 "\n\t"
        "s_cbranch_scc1 SPINI0_%=\n\t"
        "v_mov_b32 v166, %[bad]\n\t"
        "ds_write_b32 v166, v166\n\t"
        "s_branch GOI0_%=\n\t"
        "GOI0_%=:\n\t"
        "global_load_dwordx4 v[80:83], %[l16], s[96:97] offset:0 " POL_DMA "\n\t"
        "global_load_dwordx4 v[84:87], %[l16], s[96:97] offset:1024 " POL_DMA "\n\t"
        "s_add_u32 s96, s96, s95\n\t"
        "s_addc_u32 s97, s97, 0\n\t"
        "global_load_dwordx4 v[88:91], %[l16], s[96:97] offset:0 " POL_DMA "\n\t"
        "global_load_dwordx4 v[92:95], %[l16], s[96:97] offset:1024 " POL_DMA "\n\t"
        "s_add_u32 s96, s96, s95\n\t"
        "s_addc_u32 s97, s97, 0\n\t"
        "s_mov_b32 s89, 2\n\t"
        "s_mov_b32 s94, 0\n\t"
        "SPINI1_%=:\n\t"
        "ds_read_b32 v164, %[vctl]\n\t"
        "s_waitcnt lgkmcnt(0)\n\t"
        "v_cmp_gt_u32_e64 s[86:87], s89, v164\n\t"
        "s_cmp_eq_u64 s[86:87], 0\n\t"
        "s_cbranch_scc1 GOI1_%=\n\t"
        SC3("s_add_u32 %[nfix], %[nfix], 1\n\t")
        "s_sleep 1\n\t"
        "s_add_u32 s94, s94, 1\n\t"
        "s_and_b32 s92, s94, 0x3ff\n\t"
        "s_cbranch_scc1 SPINI1_%=\n\t"
        "s_memrealtime s[92:93]\n\t"
        "s_waitcnt lgkmcnt(0)\n\t"
        "s_cmp_eq_u32 s94, 0x400\n\t"
        "s_cselect_b32 s90, s92, s90\n\t"
        "s_sub_u32 s92, s92, s90\n\t"
        "s_cmp_lt_u32 s92, " KDL3 "\n\t"
        "s_cbranch_scc1 SPINI1_%=\n\t"
        "v_mov_b32 v166, %[bad]\n\t"
        "ds_write_b32 v166, v166\n\t"
        "s_branch GOI1_%=\n\t"
        "GOI1_%=:\n\t"
        "ds_read_b128 v[128:131], v165 offset:0\n\t"
        "ds_read_b128 v[132:135], v165 offset:1024\n\t"
        "ds_read_b128 v[136:139], v165 offset:2048\n\t"
        "ds_read_b128 v[140:143], v165 offset:3072\n\t"
        "ds_read_b128 v[144:147], v165 offset:4096\n\t"
        "ds_read_b128 v[148:151], v165 offset:5120\n\t"
        "ds_read_b128 v[152:155], v165 offset:6144\n\t"
        "ds_read_b128 v[156:159], v165 offset:7168\n\t"
        "s_mov_b64 exec, 1\n\t"
        "ds_add_u32 %[vctl], v167\n\t"
        "s_mov_b64 exec, -1\n\t"
        "s_waitcnt vmcnt(2)\n\t"
        "v_bitop3_b32 v166, v80, v84, %[tag] bitop3:0x7e\n\t"
        "v_and_b32 v166, 1, v166\n\t"
        TG("v_cmp_ne_u32_e64 s[84:85], 0, v166\n\t")
        NTG("s_mov_b64 s[84:85], 0\n\t")
        "s_waitcnt lgkmcnt(0)\n\t"
        "s_cmp_eq_u64 s[84:85], 0\n\t"
        "s_cbranch_scc0 SLOWP0_%=\n\t"
        "PREP0_%=:\n\t"
        "s_waitcnt lgkmcnt(9)\n\t"
        MF("v_mfma_f32_16x16x32_f16 v[8:11], v[80:83], v[128:131], v[8:11]\n\t")
        "v_add_u32 v165, s88, %[l16]\n\t"
        "global_load_dwordx4 v[96:99], %[l16], s[96:97] offset:0 " POL_DMA "\n\t"
        MF("v_mfma_f32_16x16x32_f16 v[44:47], v[84:87], v[128:131], v[44:47]\n\t")
        "global_load_dwordx4 v[100:103], %[l16], s[96:97] offset:1024 " POL_DMA "\n\t"
        "ds_read_b128 v[128:131], v165 offset:0\n\t"
        "s_waitcnt lgkmcnt(9)\n\t"
        MF("v_mfma_f32_16x16x32_f16 v[12:15], v[80:83], v[132:135], v[12:15]\n\t")
        "s_add_u32 s88, s88, 9216\n\t"
        "s_cmp_eq_u32 s88, 147456\n\t"
        "s_cselect_b32 s88, 0, s88\n\t"
        MF("v_mfma_f32_16x16x32_f16 v[48:51], v[84:87], v[132:135], v[48:51]\n\t")
        "ds_read_b128 v[132:135], v165 offset:1024\n\t"
        "s_waitcnt lgkmcnt(9)\n\t"
        MF("v_mfma_f32_16x16x32_f16 v[16:19], v[80:83], v[136:139], v[16:19]\n\t")
        "s_add_u32 s96, s96, s95\n\t"
        "s_addc_u32 s97, s97, 0\n\t"
        MF("v_mfma_f32_16x16x32_f16 v[52:55], v[84:87], v[136:139], v[52:55]\n\t")
        "ds_read_b128 v[136:139], v165 offset:2048\n\t"
        "ds_read_b32 v164, %[vctl]\n\t"
        "s_waitcnt lgkmcnt(9)\n\t"
        MF("v_mfma_f32_16x16x32_f16 v[20:23], v[80:83], v[140:143], v[20:23]\n\t")
        MF("v_mfma_f32_16x16x32_f16 v[56:59], v[84:87], v[140:143], v[56:59]\n\t")
        "ds_read_b128 v[140:143], v165 offset:3072\n\t"
        "s_waitcnt lgkmcnt(9)\n\t"
        MF("v_mfma_f32_16x16x32_f16 v[24:27], v[80:83], v[144:147], v[24:27]\n\t")
        "s_add_u32 s89, s89, 1\n\t"
        MF("v_mfma_f32_16x16x32_f16 v[60:63], v[84:87], v[144:147], v[60:63]\n\t")
        "ds_read_b128 v[144:147], v165 offset:4096\n\t"
        "s_waitcnt lgkmcnt(9)\n\t"
        MF("v_mfma_f32_16x16x32_f16 v[28:31], v[80:83], v[148:151], v[28:31]\n\t")
        MF("v_mfma_f32_16x16x32_f16 v[64:67], v[84:87], v[148:151], v[64:67]\n\t")
        "ds_read_b128 v[148:151], v165 offset:5120\n\t"
        "s_waitcnt lgkmcnt(9)\n\t"
        "s_waitcnt vmcnt(2)\n\t"
        MF("v_mfma_f32_16x16x32_f16 v[32:35], v[80:83], v[152:155], v[32:35]\n\t")
        "v_bitop3_b32 v166, v88, v92, %[tag] bitop3:0x7e\n\t"
        MF("v_mfma_f32_16x16x32_f16 v[68:71], v[84:87], v[152:155], v[68:71]\n\t")
        "v_and_b32 v166, 1, v166\n\t"
        "ds_read_b128 v[152:155], v165 offset:6144\n\t"
        "s_waitcnt lgkmcnt(4)\n\t"
        MF("v_mfma_f32_16x16x32_f16 v[36:39], v[80:83], v[156:159], v[36:39]\n\t")
        "v_cmp_gt_u32_e64 s[86:87], s89, v164\n\t"
        TG("v_cmp_ne_u32_e64 s[84:85], 0, v166\n\t")
        NTG("s_mov_b64 s[84:85], 0\n\t")
        MF("v_mfma_f32_16x16x32_f16 v[72:75], v[84:87], v[156:159], v[72:75]\n\t")
        "ds_read_b128 v[156:159], v165 offset:7168\n\t"
        "s_mov_b64 exec, 1\n\t"
        "ds_add_u32 %[vctl], v167\n\t"
        "s_mov_b64 exec, -1\n\t"
        "s_or_b64 s[90:91], s[84:85], s[86:87]\n\t"
        "s_cbranch_scc1 SLOWP1_%=\n\t"
        "PREP1_%=:\n\t"
        "s_waitcnt lgkmcnt(9)\n\t"
        MF("v_mfma_f32_16x16x32_f16 v[8:11], v[88:91], v[128:131], v[8:11]\n\t")
        "v_add_u32 v165, s88, %[l16]\n\t"
        "global_load_dwordx4 v[104:107], %[l16], s[96:97] offset:0 " POL_DMA "\n\t"
        MF("v_mfma_f32_16x16x32_f16 v[44:47], v[92:95], v[128:131], v[44:47]\n\t")
        "global_load_dwordx4 v[108:111], %[l16], s[96:97] offset:1024 " POL_DMA "\n\t"
        "ds_read_b128 v[128:131], v165 offset:0\n\t"
        "s_waitcnt lgkmcnt(9)\n\t"
        MF("v_mfma_f32_16x16x32_f16 v[12:15], v[88:91], v[132:135], v[12:15]\n\t")
        "s_add_u32 s88, s88, 9216\n\t"
        "s_cmp_eq_u32 s88, 147456\n\t"
        "s_cselect_b32 s88, 0, s88\n\t"
        MF("v_mfma_f32_16x16x32_f16 v[48:51], v[92:95], v[132:135], v[48:51]\n\t")
        "ds_read_b128 v[132:135], v165 offset:1024\n\t"
        "s_waitcnt lgkmcnt(9)\n\t"
        MF("v_mfma_f32_16x16x32_f16 v[16:19], v[88:91], v[136:139], v[16:19]\n\t")
        "s_add_u32 s96, s96, s95\n\t"
        "s_addc_u32 s97, s97, 0\n\t"
        MF("v_mfma_f32_16x16x32_f16 v[52:55], v[92:95], v[136:139], v[52:55]\n\t")
        "ds_read_b128 v[136:139], v165 offset:2048\n\t"
        "ds_read_b32 v164, %[vctl]\n\t"
        "s_waitcnt lgkmcnt(9)\n\t"
        MF("v_mfma_f32_16x16x32_f16 v[20:23], v[88:91], v[140:143], v[20:23]\n\t")
        MF("v_mfma_f32_16x16x32_f16 v[56:59], v[92:95], v[140:143], v[56:59]\n\t")
        "ds_read_b128 v[140:143], v165 offset:3072\n\t"
        "s_waitcnt lgkmcnt(9)\n\t"
        MF("v_mfma_f32_16x16x32_f16 v[24:27], v[88:91], v[144:147], v[24:27]\n\t")
        "s_add_u32 s89, s89, 1\n\t"
        MF("v_mfma_f32_16x16x32_f16 v[60:63], v[92:95], v[144:147], v[60:63]\n\t")
        "ds_read_b128 v[144:147], v165 offset:4096\n\t"
        "s_waitcnt lgkmcnt(9)\n\t"
        MF("v_mfma_f32_16x16x32_f16 v[28:31], v[88:91], v[148:151], v[28:31]\n\t")
        MF("v_mfma_f32_16x16x32_f16 v[64:67], v[92:95], v[148:151], v[64:67]\n\t")
        "ds_read_b128 v[148:151], v165 offset:5120\n\t"
        "s_waitcnt lgkmcnt(9)\n\t"
        "s_waitcnt vmcnt(2)\n\t"
        MF("v_mfma_f32_16x16x32_f16 v[32:35], v[88:91], v[152:155], v[32:35]\n\t")
        "v_bitop3_b32 v166, v96, v100, %[tag] bitop3:0x7e\n\t"
        MF("v_mfma_f32_16x16x32_f16 v[68:71], v[92:95], v[152:155], v[68:71]\n\t")
        "v_and_b32 v166, 1, v166\n\t"
        "ds_read_b128 v[152:155], v165 offset:6144\n\t"
        "s_waitcnt lgkmcnt(4)\n\t"
        MF("v_mfma_f32_16x16x32_f16 v[36:39], v[88:91], v[156:159], v[36:39]\n\t")
        "v_cmp_gt_u32_e64 s[86:87], s89, v164\n\t"
        TG("v_cmp_ne_u32_e64 s[84:85], 0, v166\n\t")
        NTG("s_mov_b64 s[84:85], 0\n\t")
        MF("v_mfma_f32_16x16x32_f16 v[72:75], v[92:95], v[156:159], v[72:75]\n\t")
        "ds_read_b128 v[156:159], v165 offset:7168\n\t"
        "s_mov_b64 exec, 1\n\t"
        "ds_add_u32 %[vctl], v167\n\t"
        "s_mov_b64 exec, -1\n\t"
        "s_or_b64 s[90:91], s[84:85], s[86:87]\n\t"
        "s_cbranch_scc1 SLOWP2_%=\n\t"
        "PREP2_%=:\n\t"
        "s_waitcnt lgkmcnt(9)\n\t"
        MF("v_mfma_f32_16x16x32_f16 v[8:11], v[96:99], v[128:131], v[8:11]\n\t")
        "v_add_u32 v165, s88, %[l16]\n\t"
        "global_load_dwordx4 v[112:115], %[l16], s[96:97] offset:0 " POL_DMA "\n\t"
        MF("v_mfma_f32_16x16x32_f16 v[44:47], v[100:103], v[128:131], v[44:47]\n\t")
        "global_load_dwordx4 v[116:119], %[l16], s[96:97] offset:1024 " POL_DMA "\n\t"
        "ds_read_b128 v[128:131], v165 offset:0\n\t"
        "s_waitcnt lgkmcnt(9)\n\t"
        MF("v_mfma_f32_16x16x32_f16 v[12:15], v[96:99], v[132:135], v[12:15]\n\t")
        "s_add_u32 s88, s88, 9216\n\t"
        "s_cmp_eq_u32 s88, 147456\n\t"
        "s_cselect_b32 s88, 0, s88\n\t"
        MF("v_mfma_f32_16x16x32_f16 v[48:51], v[100:103], v[132:135], v[48:51]\n\t")
        "ds_read_b128 v[132:135], v165 offset:1024\n\t"
        "s_waitcnt lgkmcnt(9)\n\t"
        MF("v_mfma_f32_16x16x32_f16 v[16:19], v[96:99], v[136:139], v[16:19]\n\t")
        "s_add_u32 s96, s96, s95\n\t"
        "s_addc_u32 s97, s97, 0\n\t"
        MF("v_mfma_f32_16x16x32_f16 v[52:55], v[100:103], v[136:139], v[52:55]\n\t")
        "ds_read_b128 v[136:139], v165 offset:2048\n\t"
        "ds_read_b32 v164, %[vctl]\n\t"
        "s_waitcnt lgkmcnt(9)\n\t"
        MF("v_mfma_f32_16x16x32_f16 v[20:23], v[96:99], v[140:143], v[20:23]\n\t")
        "s_cmp_eq_u32 s89, 16\n\t"
        "s_cselect_b64 s[96:97], %[xfar], s[96:97]\n\t"
        MF("v_mfma_f32_16x16x32_f16 v[56:59], v[100:103], v[140:143], v[56:59]\n\t")
        "ds_read_b128 v[140:143], v165 offset:3072\n\t"
        "s_waitcnt lgkmcnt(9)\n\t"
        MF("v_mfma_f32_16x16x32_f16 v[24:27], v[96:99], v[144:147], v[24:27]\n\t")
        "s_add_u32 s89, s89, 1\n\t"
        MF("v_mfma_f32_16x16x32_f16 v[60:63], v[100:103], v[144:147], v[60:63]\n\t")
        "ds_read_b128 v[144:147], v165 offset:4096\n\t"
        "s_waitcnt lgkmcnt(9)\n\t"
        MF("v_mfma_f32_16x16x32_f16 v[28:31], v[96:99], v[148:151], v[28:31]\n\t")
        MF("v_mfma_f32_16x16x32_f16 v[64:67], v[100:103], v[148:151], v[64:67]\n\t")
        "ds_read_b128 v[148:151], v165 offset:5120\n\t"
        "s_waitcnt lgkmcnt(9)\n\t"
        "s_waitcnt vmcnt(2)\n\t"
        MF("v_mfma_f32_16x16x32_f16 v[32:35], v[96:99], v[152:155], v[32:35]\n\t")
        "v_bitop3_b32 v166, v104, v108, %[tag] bitop3:0x7e\n\t"
        MF("v_mfma_f32_16x16x32_f16 v[68:71], v[100:103], v[152:155], v[68:71]\n\t")
        "v_and_b32 v166, 1, v166\n\t"
        "ds_read_b128 v[152:155], v165 offset:6144\n\t"
        "s_waitcnt lgkmcnt(4)\n\t"
        MF("v_mfma_f32_16x16x32_f16 v[36:39], v[96:99], v[156:159], v[36:39]\n\t")
        "v_cmp_gt_u32_e64 s[86:87], s89, v164\n\t"
        TG("v_cmp_ne_u32_e64 s[84:85], 0, v166\n\t")
        NTG("s_mov_b64 s[84:85], 0\n\t")
        MF("v_mfma_f32_16x16x32_f16 v[72:75], v[100:103], v[156:159], v[72:75]\n\t")
        "ds_read_b128 v[156:159], v165 offset:7168\n\t"
        "s_mov_b64 exec, 1\n\t"
        "ds_add_u32 %[vctl], v167\n\t"
        "s_mov_b64 exec, -1\n\t"
        "s_or_b64 s[90:91], s[84:85], s[86:87]\n\t"
        "s_cbranch_scc1 SLOWP3_%=\n\t"
        "PREP3_%=:\n\t"
        "s_cmp_eq_u32 s89, 17\n\t"
        "s_cbranch_scc1 PREF3_%=\n\t"
        "s_waitcnt lgkmcnt(9)\n\t"
        MF("v_mfma_f32_16x16x32_f16 v[8:11], v[104:107], v[128:131], v[8:11]\n\t")
        "v_add_u32 v165, s88, %[l16]\n\t"
        "global_load_dwordx4 v[120:123], %[l16], s[96:97] offset:0 " POL_DMA "\n\t"
        MF("v_mfma_f32_16x16x32_f16 v[44:47], v[108:111], v[128:131], v[44:47]\n\t")
        "global_load_dwordx4 v[124:127], %[l16], s[96:97] offset:1024 " POL_DMA "\n\t"
        "ds_read_b128 v[128:131], v165 offset:0\n\t"
        "s_waitcnt lgkmcnt(9)\n\t"
        MF("v_mfma_f32_16x16x32_f16 v[12:15], v[104:107], v[132:135], v[12:15]\n\t")
        "s_add_u32 s88, s88, 9216\n\t"
        "s_cmp_eq_u32 s88, 147456\n\t"
        "s_cselect_b32 s88, 0, s88\n\t"
        MF("v_mfma_f32_16x16x32_f16 v[48:51], v[108:111], v[132:135], v[48:51]\n\t")
        "ds_read_b128 v[132:135], v165 offset:1024\n\t"
        "s_waitcnt lgkmcnt(9)\n\t"
        MF("v_mfma_f32_16x16x32_f16 v[16:19], v[104:107], v[136:139], v[16:19]\n\t")
        "s_add_u32 s96, s96, s95\n\t"
        "s_addc_u32 s97, s97, 0\n\t"
        MF("v_mfma_f32_16x16x32_f16 v[52:55], v[108:111], v[136:139], v[52:55]\n\t")
        "ds_read_b128 v[136:139], v165 offset:2048\n\t"
        "ds_read_b32 v164, %[vctl]\n\t"
        "s_waitcnt lgkmcnt(9)\n\t"
        MF("v_mfma_f32_16x16x32_f16 v[20:23], v[104:107], v[140:143], v[20:23]\n\t")
        MF("v_mfma_f32_16x16x32_f16 v[56:59], v[108:111], v[140:143], v[56:59]\n\t")
        "ds_read_b128 v[140:143], v165 offset:3072\n\t"
        "s_waitcnt lgkmcnt(9)\n\t"
        MF("v_mfma_f32_16x16x32_f16 v[24:27], v[104:107], v[144:147], v[24:27]\n\t")
        "s_add_u32 s89, s89, 1\n\t"
        MF("v_mfma_f32_16x16x32_f16 v[60:63], v[108:111], v[144:147], v[60:63]\n\t")
        "ds_read_b128 v[144:147], v165 offset:4096\n\t"
        "s_waitcnt lgkmcnt(9)\n\t"
        MF("v_mfma_f32_16x16x32_f16 v[28:31], v[104:107], v[148:151], v[28:31]\n\t")
        MF("v_mfma_f32_16x16x32_f16 v[64:67], v[108:111], v[148:151], v[64:67]\n\t")
        "ds_read_b128 v[148:151], v165 offset:5120\n\t"
        "s_waitcnt lgkmcnt(9)\n\t"
        "s_waitcnt vmcnt(2)\n\t"
        MF("v_mfma_f32_16x16x32_f16 v[32:35], v[104:107], v[152:155], v[32:35]\n\t")
        "v_bitop3_b32 v166, v112, v116, %[tag] bitop3:0x7e\n\t"
        MF("v_mfma_f32_16x16x32_f16 v[68:71], v[108:111], v[152:155], v[68:71]\n\t")
        "v_and_b32 v166, 1, v166\n\t"
        "ds_read_b128 v[152:155], v165 offset:6144\n\t"
        "s_waitcnt lgkmcnt(4)\n\t"
        MF("v_mfma_f32_16x16x32_f16 v[36:39], v[104:107], v[156:159], v[36:39]\n\t")
        "v_cmp_gt_u32_e64 s[86:87], s89, v164\n\t"
        TG("v_cmp_ne_u32_e64 s[84:85], 0, v166\n\t")
        NTG("s_mov_b64 s[84:85], 0\n\t")
        MF("v_mfma_f32_16x16x32_f16 v[72:75], v[108:111], v[156:159], v[72:75]\n\t")
        "ds_read_b128 v[156:159], v165 offset:7168\n\t"
        "s_mov_b64 exec, 1\n\t"
        "ds_add_u32 %[vctl], v167\n\t"
        "s_mov_b64 exec, -1\n\t"
        "s_or_b64 s[90:91], s[84:85], s[86:87]\n\t"
        "s_cbranch_scc1 SLOWP4_%=\n\t"
        "PREP4_%=:\n\t"
        "s_waitcnt lgkmcnt(9)\n\t"
        MF("v_mfma_f32_16x16x32_f16 v[8:11], v[112:115], v[128:131], v[8:11]\n\t")
        "v_add_u32 v165, s88, %[l16]\n\t"
        "global_load_dwordx4 v[80:83], %[l16], s[96:97] offset:0 " POL_DMA "\n\t"
        MF("v_mfma_f32_16x16x32_f16 v[44:47], v[116:119], v[128:131], v[44:47]\n\t")
        "global_load_dwordx4 v[84:87], %[l16], s[96:97] offset:1024 " POL_DMA "\n\t"
        "ds_read_b128 v[128:131], v165 offset:0\n\t"
        "s_waitcnt lgkmcnt(9)\n\t"
        MF("v_mfma_f32_16x16x32_f16 v[12:15], v[112:115], v[132:135], v[12:15]\n\t")
        "s_add_u32 s88, s88, 9216\n\t"
        "s_cmp_eq_u32 s88, 147456\n\t"
        "s_cselect_b32 s88, 0, s88\n\t"
        MF("v_mfma_f32_16x16x32_f16 v[48:51], v[116:119], v[132:135], v[48:51]\n\t")
        "ds_read_b128 v[132:135], v165 offset:1024\n\t"
        "s_waitcnt lgkmcnt(9)\n\t"
        MF("v_mfma_f32_16x16x32_f16 v[16:19], v[112:115], v[136:139], v[16:19]\n\t")
        "s_add_u32 s96, s96, s95\n\t"
        "s_addc_u32 s97, s97, 0\n\t"
        MF("v_mfma_f32_16x16x32_f16 v[52:55], v[116:119], v[136:139], v[52:55]\n\t")
        "ds_read_b128 v[136:139], v165 offset:2048\n\t"
        "ds_read_b32 v164, %[vctl]\n\t"
        "s_waitcnt lgkmcnt(9)\n\t"
        MF("v_mfma_f32_16x16x32_f16 v[20:23], v[112:115], v[140:143], v[20:23]\n\t")
        MF("v_mfma_f32_16x16x32_f16 v[56:59], v[116:119], v[140:143], v[56:59]\n\t")
        "ds_read_b128 v[140:143], v165 offset:3072\n\t"
        "s_waitcnt lgkmcnt(9)\n\t"
        MF("v_mfma_f32_16x16x32_f16 v[24:27], v[112:115], v[144:147], v[24:27]\n\t")
        "s_add_u32 s89, s89, 1\n\t"
        MF("v_mfma_f32_16x16x32_f16 v[60:63], v[116:119], v[144:147], v[60:63]\n\t")
        "ds_read_b128 v[144:147], v165 offset:4096\n\t"
        "s_waitcnt lgkmcnt(9)\n\t"
        MF("v_mfma_f32_16x16x32_f16 v[28:31], v[112:115], v[148:151], v[28:31]\n\t")
        MF("v_mfma_f32_16x16x32_f16 v[64:67], v[116:119], v[148:151], v[64:67]\n\t")
        "ds_read_b128 v[148:151], v165 offset:5120\n\t"
        "s_waitcnt lgkmcnt(9)\n\t"
        "s_waitcnt vmcnt(2)\n\t"
        MF("v_mfma_f32_16x16x32_f16 v[32:35], v[112:115], v[152:155], v[32:35]\n\t")
        "v_bitop3_b32 v166, v120, v124, %[tag] bitop3:0x7e\n\t"
        MF("v_mfma_f32_16x16x32_f16 v[68:71], v[116:119], v[152:155], v[68:71]\n\t")
        "v_and_b32 v166, 1, v166\n\t"
        "ds_read_b128 v[152:155], v165 offset:6144\n\t"
        "s_waitcnt lgkmcnt(4)\n\t"
        MF("v_mfma_f32_16x16x32_f16 v[36:39], v[112:115], v[156:159], v[36:39]\n\t")
        "v_cmp_gt_u32_e64 s[86:87], s89, v164\n\t"
        TG("v_cmp_ne_u32_e64 s[84:85], 0, v166\n\t")
        NTG("s_mov_b64 s[84:85], 0\n\t")
        MF("v_mfma_f32_16x16x32_f16 v[72:75], v[116:119], v[156:159], v[72:75]\n\t")
        "ds_read_b128 v[156:159], v165 offset:7168\n\t"
        "s_mov_b64 exec, 1\n\t"
        "ds_add_u32 %[vctl], v167\n\t"
        "s_mov_b64 exec, -1\n\t"
        "s_or_b64 s[90:91], s[84:85], s[86:87]\n\t"
        "s_cbranch_scc1 SLOWP5_%=\n\t"
        "PREP5_%=:\n\t"
        "s_waitcnt lgkmcnt(9)\n\t"
        MF("v_mfma_f32_16x16x32_f16 v[8:11], v[120:123], v[128:131], v[8:11]\n\t")
        "v_add_u32 v165, s88, %[l16]\n\t"
        "global_load_dwordx4 v[88:91], %[l16], s[96:97] offset:0 " POL_DMA "\n\t"
        MF("v_mfma_f32_16x16x32_f16 v[44:47], v[124:127], v[128:131], v[44:47]\n\t")
        "global_load_dwordx4 v[92:95], %[l16], s[96:97] offset:1024 " POL_DMA "\n\t"
        "ds_read_b128 v[128:131], v165 offset:0\n\t"
        "s_waitcnt lgkmcnt(9)\n\t"
        MF("v_mfma_f32_16x16x32_f16 v[12:15], v[120:123], v[132:135], v[12:15]\n\t")
        "s_add_u32 s88, s88, 9216\n\t"
        "s_cmp_eq_u32 s88, 147456\n\t"
        "s_cselect_b32 s88, 0, s88\n\t"
        MF("v_mfma_f32_16x16x32_f16 v[48:51], v[124:127], v[132:135], v[48:51]\n\t")
        "ds_read_b128 v[132:135], v165 offset:1024\n\t"
        "s_waitcnt lgkmcnt(9)\n\t"
        MF("v_mfma_f32_16x16x32_f16 v[16:19], v[120:123], v[136:139], v[16:19]\n\t")
        "s_add_u32 s96, s96, s95\n\t"
        "s_addc_u32 s97, s97, 0\n\t"
        MF("v_mfma_f32_16x16x32_f16 v[52:55], v[124:127], v[136:139], v[52:55]\n\t")
        "ds_read_b128 v[136:139], v165 offset:2048\n\t"
        "ds_read_b32 v164, %[vctl]\n\t"
        "s_waitcnt lgkmcnt(9)\n\t"
        MF("v_mfma_f32_16x16x32_f16 v[20:23], v[120:123], v[140:143], v[20:23]\n\t")
        MF("v_mfma_f32_16x16x32_f16 v[56:59], v[124:127], v[140:143], v[56:59]\n\t")
        "ds_read_b128 v[140:143], v165 offset:3072\n\t"
        "s_waitcnt lgkmcnt(9)\n\t"
        MF("v_mfma_f32_16x16x32_f16 v[24:27], v[120:123], v[144:147], v[24:27]\n\t")
        "s_add_u32 s89, s89, 1\n\t"
        MF("v_mfma_f32_16x16x32_f16 v[60:63], v[124:127], v[144:147], v[60:63]\n\t")
        "ds_read_b128 v[144:147], v165 offset:4096\n\t"
        "s_waitcnt lgkmcnt(9)\n\t"
        MF("v_mfma_f32_16x16x32_f16 v[28:31], v[120:123], v[148:151], v[28:31]\n\t")
        MF("v_mfma_f32_16x16x32_f16 v[64:67], v[124:127], v[148:151], v[64:67]\n\t")
        "ds_read_b128 v[148:151], v165 offset:5120\n\t"
        "s_waitcnt lgkmcnt(9)\n\t"
        "s_waitcnt vmcnt(2)\n\t"
        MF("v_mfma_f32_16x16x32_f16 v[32:35], v[120:123], v[152:155], v[32:35]\n\t")
        "v_bitop3_b32 v166, v80, v84, %[tag] bitop3:0x7e\n\t"
        MF("v_mfma_f32_16x16x32_f16 v[68:71], v[124:127], v[152:155], v[68:71]\n\t")
        "v_and_b32 v166, 1, v166\n\t"
        "ds_read_b128 v[152:155], v165 offset:6144\n\t"
        "s_waitcnt lgkmcnt(4)\n\t"
        MF("v_mfma_f32_16x16x32_f16 v[36:39], v[120:123], v[156:159], v[36:39]\n\t")
        "v_cmp_gt_u32_e64 s[86:87], s89, v164\n\t"
        TG("v_cmp_ne_u32_e64 s[84:85], 0, v166\n\t")
        NTG("s_mov_b64 s[84:85], 0\n\t")
        MF("v_mfma_f32_16x16x32_f16 v[72:75], v[124:127], v[156:159], v[72:75]\n\t")
        "ds_read_b128 v[156:159], v165 offset:7168\n\t"
        "s_mov_b64 exec, 1\n\t"
        "ds_add_u32 %[vctl], v167\n\t"
        "s_mov_b64 exec, -1\n\t"
        "s_or_b64 s[90:91], s[84:85], s[86:87]\n\t"
        "s_cbranch_scc1 SLOWP0_%=\n\t"
        "s_branch PREP0_%=\n\t"
        "SLOWP0_%=:\n\t"
        SC1("s_add_u32 %[nfix], %[nfix], 1\n\t")
        "s_cmp_eq_u64 s[84:85], 0\n\t"
        "s_cbranch_scc1 NOFIXP0_%=\n\t"
        "s_add_u32 %[nfix], %[nfix], 1\n\t"
        "v_mov_b32 v166, %[bad]\n\t"
        "ds_read_b32 v164, v166\n\t"
        "s_sub_u32 s92, s89, 2\n\t"
        "s_cmp_lt_u32 s92, 17\n\t"
        "s_cselect_b64 s[86:87], %[xloc], %[xfar]\n\t"
        "s_cselect_b32 s93, 0, 17\n\t"
        "s_sub_u32 s93, s92, s93\n\t"
        "s_lshl_b32 s93, s93, 14\n\t"
        "s_cmp_lt_u32 s92, 112\n\t"
        "s_cselect_b64 s[86:87], s[86:87], %[xt0]\n\t"
        "s_cselect_b32 s92, s93, 0\n\t"
        "s_add_u32 s86, s86, s92\n\t"
        "s_addc_u32 s87, s87, 0\n\t"
        "s_memrealtime s[90:91]\n\t"
        "s_waitcnt lgkmcnt(0)\n\t"
        "v_cmp_ne_u32_e64 s[92:93], 0, v164\n\t"
        "s_cmp_lg_u64 s[92:93], 0\n\t"
        "s_cbranch_scc1 NOFIXP0_%=\n\t"
        "FIXLP0_%=:\n\t"
        "global_load_dwordx4 v[80:83], %[l16], s[86:87] offset:0 sc0 sc1\n\t"
        "global_load_dwordx4 v[84:87], %[l16], s[86:87] offset:1024 sc0 sc1\n\t"
        "s_waitcnt vmcnt(0)\n\t"
        "v_bitop3_b32 v166, v80, v84, %[tag] bitop3:0x7e\n\t"
        "v_and_b32 v166, 1, v166\n\t"
        "v_cmp_ne_u32_e64 s[92:93], 0, v166\n\t"
        "s_cmp_eq_u64 s[92:93], 0\n\t"
        "s_cbranch_scc1 NOFIXP0_%=\n\t"
        "s_sleep 1\n\t"
        "s_memrealtime s[92:93]\n\t"
        "s_waitcnt lgkmcnt(0)\n\t"
        "s_sub_u32 s92, s92, s90\n\t"
        "s_cmp_lt_u32 s92, " KDL "\n\t"
        "s_cbranch_scc1 FIXLP0_%=\n\t"
        "v_mov_b32 v166, %[bad]\n\t"
        "ds_write_b32 v166, v166\n\t"
        "NOFIXP0_%=:\n\t"
        "s_mov_b32 s94, 0\n\t"
        "SPINP0_%=:\n\t"
        "ds_read_b32 v164, %[vctl]\n\t"
        "s_waitcnt lgkmcnt(0)\n\t"
        "v_cmp_gt_u32_e64 s[86:87], s89, v164\n\t"
        "s_cmp_eq_u64 s[86:87], 0\n\t"
        "s_cbranch_scc1 PREP0_%=\n\t"
        SC3("s_add_u32 %[nfix], %[nfix], 1\n\t")
        "s_sleep 1\n\t"
        "s_add_u32 s94, s94, 1\n\t"
        "s_and_b32 s92, s94, 0x3ff\n\t"
        "s_cbranch_scc1 SPINP0_%=\n\t"
        "s_memrealtime s[92:93]\n\t"
        "s_waitcnt lgkmcnt(0)\n\t"
        "s_cmp_eq_u32 s94, 0x400\n\t"
        "s_cselect_b32 s90, s92, s90\n\t"
        "s_sub_u32 s92, s92, s90\n\t"
        "s_cmp_lt_u32 s92, " KDL3 "\n\t"
        "s_cbranch_scc1 SPINP0_%=\n\t"
        "v_mov_b32 v166, %[bad]\n\t"
        "ds_write_b32 v166, v166\n\t"
        "s_branch PREP0_%=\n\t"
        "SLOWP1_%=:\n\t"
        SC1("s_add_u32 %[nfix], %[nfix], 1\n\t")
        "s_cmp_eq_u64 s[84:85], 0\n\t"
        "s_cbranch_scc1 NOFIXP1_%=\n\t"
        "s_add_u32 %[nfix], %[nfix], 1\n\t"
        "v_mov_b32 v166, %[bad]\n\t"
        "ds_read_b32 v164, v166\n\t"
        "s_sub_u32 s92, s89, 2\n\t"
        "s_cmp_lt_u32 s92, 17\n\t"
        "s_cselect_b64 s[86:87], %[xloc], %[xfar]\n\t"
        "s_cselect_b32 s93, 0, 17\n\t"
        "s_sub_u32 s93, s92, s93\n\t"
        "s_lshl_b32 s93, s93, 14\n\t"
        "s_cmp_lt_u32 s92, 112\n\t"
        "s_cselect_b64 s[86:87], s[86:87], %[xt0]\n\t"
        "s_cselect_b32 s92, s93, 0\n\t"
        "s_add_u32 s86, s86, s92\n\t"
        "s_addc_u32 s87, s87, 0\n\t"
        "s_memrealtime s[90:91]\n\t"
        "s_waitcnt lgkmcnt(0)\n\t"
        "v_cmp_ne_u32_e64 s[92:93], 0, v164\n\t"
        "s_cmp_lg_u64 s[92:93], 0\n\t"
        "s_cbranch_scc1 NOFIXP1_%=\n\t"
        "FIXLP1_%=:\n\t"
        "global_load_dwordx4 v[88:91], %[l16], s[86:87] offset:0 sc0 sc1\n\t"
        "global_load_dwordx4 v[92:95], %[l16], s[86:87] offset:1024 sc0 sc1\n\t"
        "s_waitcnt vmcnt(0)\n\t"
        "v_bitop3_b32 v166, v88, v92, %[tag] bitop3:0x7e\n\t"
        "v_and_b32 v166, 1, v166\n\t"
        "v_cmp_ne_u32_e64 s[92:93], 0, v166\n\t"
        "s_cmp_eq_u64 s[92:93], 0\n\t"
        "s_cbranch_scc1 NOFIXP1_%=\n\t"
        "s_sleep 1\n\t"
        "s_memrealtime s[92:93]\n\t"
        "s_waitcnt lgkmcnt(0)\n\t"
        "s_sub_u32 s92, s92, s90\n\t"
        "s_cmp_lt_u32 s92, " KDL "\n\t"
        "s_cbranch_scc1 FIXLP1_%=\n\t"
        "v_mov_b32 v166, %[bad]\n\t"
        "ds_write_b32 v166, v166\n\t"
        "NOFIXP1_%=:\n\t"
        "s_mov_b32 s94, 0\n\t"
        "SPINP1_%=:\n\t"
        "ds_read_b32 v164, %[vctl]\n\t"
        "s_waitcnt lgkmcnt(0)\n\t"
        "v_cmp_gt_u32_e64 s[86:87], s89, v164\n\t"
        "s_cmp_eq_u64 s[86:87], 0\n\t"
        "s_cbranch_scc1 PREP1_%=\n\t"
        SC3("s_add_u32 %[nfix], %[nfix], 1\n\t")
        "s_sleep 1\n\t"
        "s_add_u32 s94, s94, 1\n\t"
        "s_and_b32 s92, s94, 0x3ff\n\t"
        "s_cbranch_scc1 SPINP1_%=\n\t"
        "s_memrealtime s[92:93]\n\t"
        "s_waitcnt lgkmcnt(0)\n\t"
        "s_cmp_eq_u32 s94, 0x400\n\t"
        "s_cselect_b32 s90, s92, s90\n\t"
        "s_sub_u32 s92, s92, s90\n\t"
        "s_cmp_lt_u32 s92, " KDL3 "\n\t"
        "s_cbranch_scc1 SPINP1_%=\n\t"
        "v_mov_b32 v166, %[bad]\n\t"
        "ds_write_b32 v166, v166\n\t"
        "s_branch PREP1_%=\n\t"
        "SLOWP2_%=:\n\t"
        SC1("s_add_u32 %[nfix], %[nfix], 1\n\t")
        "s_cmp_eq_u64 s[84:85], 0\n\t"
        "s_cbranch_scc1 NOFIXP2_%=\n\t"
        "s_add_u32 %[nfix], %[nfix], 1\n\t"
        "v_mov_b32 v166, %[bad]\n\t"
        "ds_read_b32 v164, v166\n\t"
        "s_sub_u32 s92, s89, 2\n\t"
        "s_cmp_lt_u32 s92, 17\n\t"
        "s_cselect_b64 s[86:87], %[xloc], %[xfar]\n\t"
        "s_cselect_b32 s93, 0, 17\n\t"
        "s_sub_u32 s93, s92, s93\n\t"
        "s_lshl_b32 s93, s93, 14\n\t"
        "s_cmp_lt_u32 s92, 112\n\t"
        "s_cselect_b64 s[86:87], s[86:87], %[xt0]\n\t"
        "s_cselect_b32 s92, s93, 0\n\t"
        "s_add_u32 s86, s86, s92\n\t"
        "s_addc_u32 s87, s87, 0\n\t"
        "s_memrealtime s[90:91]\n\t"
        "s_waitcnt lgkmcnt(0)\n\t"
        "v_cmp_ne_u32_e64 s[92:93], 0, v164\n\t"
        "s_cmp_lg_u64 s[92:93], 0\n\t"
        "s_cbranch_scc1 NOFIXP2_%=\n\t"
        "FIXLP2_%=:\n\t"
        "global_load_dwordx4 v[96:99], %[l16], s[86:87] offset:0 sc0 sc1\n\t"
        "global_load_dwordx4 v[100:103], %[l16], s[86:87] offset:1024 sc0 sc1\n\t"
        "s_waitcnt vmcnt(0)\n\t"
        "v_bitop3_b32 v166, v96, v100, %[tag] bitop3:0x7e\n\t"
        "v_and_b32 v166, 1, v166\n\t"
        "v_cmp_ne_u32_e64 s[92:93], 0, v166\n\t"
        "s_cmp_eq_u64 s[92:93], 0\n\t"
        "s_cbranch_scc1 NOFIXP2_%=\n\t"
        "s_sleep 1\n\t"
        "s_memrealtime s[92:93]\n\t"
        "s_waitcnt lgkmcnt(0)\n\t"
        "s_sub_u32 s92, s92, s90\n\t"
        "s_cmp_lt_u32 s92, " KDL "\n\t"
        "s_cbranch_scc1 FIXLP2_%=\n\t"
        "v_mov_b32 v166, %[bad]\n\t"
        "ds_write_b32 v166, v166\n\t"
        "NOFIXP2_%=:\n\t"
        "s_mov_b32 s94, 0\n\t"
        "SPINP2_%=:\n\t"
        "ds_read_b32 v164, %[vctl]\n\t"
        "s_waitcnt lgkmcnt(0)\n\t"
        "v_cmp_gt_u32_e64 s[86:87], s89, v164\n\t"
        "s_cmp_eq_u64 s[86:87], 0\n\t"
        "s_cbranch_scc1 PREP2_%=\n\t"
        SC3("s_add_u32 %[nfix], %[nfix], 1\n\t")
        "s_sleep 1\n\t"
        "s_add_u32 s94, s94, 1\n\t"
        "s_and_b32 s92, s94, 0x3ff\n\t"
        "s_cbranch_scc1 SPINP2_%=\n\t"
        "s_memrealtime s[92:93]\n\t"
        "s_waitcnt lgkmcnt(0)\n\t"
        "s_cmp_eq_u32 s94, 0x400\n\t"
        "s_cselect_b32 s90, s92, s90\n\t"
        "s_sub_u32 s92, s92, s90\n\t"
        "s_cmp_lt_u32 s92, " KDL3 "\n\t"
        "s_cbranch_scc1 SPINP2_%=\n\t"
        "v_mov_b32 v166, %[bad]\n\t"
        "ds_write_b32 v166, v166\n\t"
        "s_branch PREP2_%=\n\t"
        "SLOWP3_%=:\n\t"
        SC1("s_add_u32 %[nfix], %[nfix], 1\n\t")
        "s_cmp_eq_u64 s[84:85], 0\n\t"
        "s_cbranch_scc1 NOFIXP3_%=\n\t"
        "s_add_u32 %[nfix], %[nfix], 1\n\t"
        "v_mov_b32 v166, %[bad]\n\t"
        "ds_read_b32 v164, v166\n\t"
        "s_sub_u32 s92, s89, 2\n\t"
        "s_cmp_lt_u32 s92, 17\n\t"
        "s_cselect_b64 s[86:87], %[xloc], %[xfar]\n\t"
        "s_cselect_b32 s93, 0, 17\n\t"
        "s_sub_u32 s93, s92, s93\n\t"
        "s_lshl_b32 s93, s93, 14\n\t"
        "s_cmp_lt_u32 s92, 112\n\t"
        "s_cselect_b64 s[86:87], s[86:87], %[xt0]\n\t"
        "s_cselect_b32 s92, s93, 0\n\t"
        "s_add_u32 s86, s86, s92\n\t"
        "s_addc_u32 s87, s87, 0\n\t"
        "s_memrealtime s[90:91]\n\t"
        "s_waitcnt lgkmcnt(0)\n\t"
        "v_cmp_ne_u32_e64 s[92:93], 0, v164\n\t"
        "s_cmp_lg_u64 s[92:93], 0\n\t"
        "s_cbranch_scc1 NOFIXP3_%=\n\t"
        "FIXLP3_%=:\n\t"
        "global_load_dwordx4 v[104:107], %[l16], s[86:87] offset:0 sc0 sc1\n\t"
        "global_load_dwordx4 v[108:111], %[l16], s[86:87] offset:1024 sc0 sc1\n\t"
        "s_waitcnt vmcnt(0)\n\t"
        "v_bitop3_b32 v166, v104, v108, %[tag] bitop3:0x7e\n\t"
        "v_and_b32 v166, 1, v166\n\t"
        "v_cmp_ne_u32_e64 s[92:93], 0, v166\n\t"
        "s_cmp_eq_u64 s[92:93], 0\n\t"
        "s_cbranch_scc1 NOFIXP3_%=\n\t"
        "s_sleep 1\n\t"
        "s_memrealtime s[92:93]\n\t"
        "s_waitcnt lgkmcnt(0)\n\t"
        "s_sub_u32 s92, s92, s90\n\t"
        "s_cmp_lt_u32 s92, " KDL "\n\t"
        "s_cbranch_scc1 FIXLP3_%=\n\t"
        "v_mov_b32 v166, %[bad]\n\t"
        "ds_write_b32 v166, v166\n\t"
        "NOFIXP3_%=:\n\t"
        "s_mov_b32 s94, 0\n\t"
        "SPINP3_%=:\n\t"
        "ds_read_b32 v164, %[vctl]\n\t"
        "s_waitcnt lgkmcnt(0)\n\t"
        "v_cmp_gt_u32_e64 s[86:87], s89, v164\n\t"
        "s_cmp_eq_u64 s[86:87], 0\n\t"
        "s_cbranch_scc1 PREP3_%=\n\t"
        SC3("s_add_u32 %[nfix], %[nfix], 1\n\t")
        "s_sleep 1\n\t"
        "s_add_u32 s94, s94, 1\n\t"
        "s_and_b32 s92, s94, 0x3ff\n\t"
        "s_cbranch_scc1 SPINP3_%=\n\t"
        "s_memrealtime s[92:93]\n\t"
        "s_waitcnt lgkmcnt(0)\n\t"
        "s_cmp_eq_u32 s94, 0x400\n\t"
        "s_cselect_b32 s90, s92, s90\n\t"
        "s_sub_u32 s92, s92, s90\n\t"
        "s_cmp_lt_u32 s92, " KDL3 "\n\t"
        "s_cbranch_scc1 SPINP3_%=\n\t"
        "v_mov_b32 v166, %[bad]\n\t"
        "ds_write_b32 v166, v166\n\t"
        "s_branch PREP3_%=\n\t"
        "SLOWP4_%=:\n\t"
        SC1("s_add_u32 %[nfix], %[nfix], 1\n\t")
        "s_cmp_eq_u64 s[84:85], 0\n\t"
        "s_cbranch_scc1 NOFIXP4_%=\n\t"
        "s_add_u32 %[nfix], %[nfix], 1\n\t"
        "v_mov_b32 v166, %[bad]\n\t"
        "ds_read_b32 v164, v166\n\t"
        "s_sub_u32 s92, s89, 2\n\t"
        "s_cmp_lt_u32 s92, 17\n\t"
        "s_cselect_b64 s[86:87], %[xloc], %[xfar]\n\t"
        "s_cselect_b32 s93, 0, 17\n\t"
        "s_sub_u32 s93, s92, s93\n\t"
        "s_lshl_b32 s93, s93, 14\n\t"
        "s_cmp_lt_u32 s92, 112\n\t"
        "s_cselect_b64 s[86:87], s[86:87], %[xt0]\n\t"
        "s_cselect_b32 s92, s93, 0\n\t"
        "s_add_u32 s86, s86, s92\n\t"
        "s_addc_u32 s87, s87, 0\n\t"
        "s_memrealtime s[90:91]\n\t"
        "s_waitcnt lgkmcnt(0)\n\t"
        "v_cmp_ne_u32_e64 s[92:93], 0, v164\n\t"
        "s_cmp_lg_u64 s[92:93], 0\n\t"
        "s_cbranch_scc1 NOFIXP4_%=\n\t"
        "FIXLP4_%=:\n\t"
        "global_load_dwordx4 v[112:115], %[l16], s[86:87] offset:0 sc0 sc1\n\t"
        "global_load_dwordx4 v[116:119], %[l16], s[86:87] offset:1024 sc0 sc1\n\t"
        "s_waitcnt vmcnt(0)\n\t"
        "v_bitop3_b32 v166, v112, v116, %[tag] bitop3:0x7e\n\t"
        "v_and_b32 v166, 1, v166\n\t"
        "v_cmp_ne_u32_e64 s[92:93], 0, v166\n\t"
        "s_cmp_eq_u64 s[92:93], 0\n\t"
        "s_cbranch_scc1 NOFIXP4_%=\n\t"
        "s_sleep 1\n\t"
        "s_memrealtime s[92:93]\n\t"
        "s_waitcnt lgkmcnt(0)\n\t"
        "s_sub_u32 s92, s92, s90\n\t"
        "s_cmp_lt_u32 s92, " KDL "\n\t"
        "s_cbranch_scc1 FIXLP4_%=\n\t"
        "v_mov_b32 v166, %[bad]\n\t"
        "ds_write_b32 v166, v166\n\t"
        "NOFIXP4_%=:\n\t"
        "s_mov_b32 s94, 0\n\t"
        "SPINP4_%=:\n\t"
        "ds_read_b32 v164, %[vctl]\n\t"
        "s_waitcnt lgkmcnt(0)\n\t"
        "v_cmp_gt_u32_e64 s[86:87], s89, v164\n\t"
        "s_cmp_eq_u64 s[86:87], 0\n\t"
        "s_cbranch_scc1 PREP4_%=\n\t"
        SC3("s_add_u32 %[nfix], %[nfix], 1\n\t")
        "s_sleep 1\n\t"
        "s_add_u32 s94, s94, 1\n\t"
        "s_and_b32 s92, s94, 0x3ff\n\t"
        "s_cbranch_scc1 SPINP4_%=\n\t"
        "s_memrealtime s[92:93]\n\t"
        "s_waitcnt lgkmcnt(0)\n\t"
        "s_cmp_eq_u32 s94, 0x400\n\t"
        "s_cselect_b32 s90, s92, s90\n\t"
        "s_sub_u32 s92, s92, s90\n\t"
        "s_cmp_lt_u32 s92, " KDL3 "\n\t"
        "s_cbranch_scc1 SPINP4_%=\n\t"
        "v_mov_b32 v166, %[bad]\n\t"
        "ds_write_b32 v166, v166\n\t"
        "s_branch PREP4_%=\n\t"
        "SLOWP5_%=:\n\t"
        SC1("s_add_u32 %[nfix], %[nfix], 1\n\t")
        "s_cmp_eq_u64 s[84:85], 0\n\t"
        "s_cbranch_scc1 NOFIXP5_%=\n\t"
        "s_add_u32 %[nfix], %[nfix], 1\n\t"
        "v_mov_b32 v166, %[bad]\n\t"
        "ds_read_b32 v164, v166\n\t"
        "s_sub_u32 s92, s89, 2\n\t"
        "s_cmp_lt_u32 s92, 17\n\t"
        "s_cselect_b64 s[86:87], %[xloc], %[xfar]\n\t"
        "s_cselect_b32 s93, 0, 17\n\t"
        "s_sub_u32 s93, s92, s93\n\t"
        "s_lshl_b32 s93, s93, 14\n\t"
        "s_cmp_lt_u32 s92, 112\n\t"
        "s_cselect_b64 s[86:87], s[86:87], %[xt0]\n\t"
        "s_cselect_b32 s92, s93, 0\n\t"
        "s_add_u32 s86, s86, s92\n\t"
        "s_addc_u32 s87, s87, 0\n\t"
        "s_memrealtime s[90:91]\n\t"
        "s_waitcnt lgkmcnt(0)\n\t"
        "v_cmp_ne_u32_e64 s[92:93], 0, v164\n\t"
        "s_cmp_lg_u64 s[92:93], 0\n\t"
        "s_cbranch_scc1 NOFIXP5_%=\n\t"
        "FIXLP5_%=:\n\t"
        "global_load_dwordx4 v[120:123], %[l16], s[86:87] offset:0 sc0 sc1\n\t"
        "global_load_dwordx4 v[124:127], %[l16], s[86:87] offset:1024 sc0 sc1\n\t"
        "s_waitcnt vmcnt(0)\n\t"
        "v_bitop3_b32 v166, v120, v124, %[tag] bitop3:0x7e\n\t"
        "v_and_b32 v166, 1, v166\n\t"
        "v_cmp_ne_u32_e64 s[92:93], 0, v166\n\t"
        "s_cmp_eq_u64 s[92:93], 0\n\t"
        "s_cbranch_scc1 NOFIXP5_%=\n\t"
        "s_sleep 1\n\t"
        "s_memrealtime s[92:93]\n\t"
        "s_waitcnt lgkmcnt(0)\n\t"
        "s_sub_u32 s92, s92, s90\n\t"
        "s_cmp_lt_u32 s92, " KDL "\n\t"
        "s_cbranch_scc1 FIXLP5_%=\n\t"
        "v_mov_b32 v166, %[bad]\n\t"
        "ds_write_b32 v166, v166\n\t"
        "NOFIXP5_%=:\n\t"
        "s_mov_b32 s94, 0\n\t"
        "SPINP5_%=:\n\t"
        "ds_read_b32 v164, %[vctl]\n\t"
        "s_waitcnt lgkmcnt(0)\n\t"
        "v_cmp_gt_u32_e64 s[86:87], s89, v164\n\t"
        "s_cmp_eq_u64 s[86:87], 0\n\t"
        "s_cbranch_scc1 PREP5_%=\n\t"
        SC3("s_add_u32 %[nfix], %[nfix], 1\n\t")
        "s_sleep 1\n\t"
        "s_add_u32 s94, s94, 1\n\t"
        "s_and_b32 s92, s94, 0x3ff\n\t"
        "s_cbranch_scc1 SPINP5_%=\n\t"
        "s_memrealtime s[92:93]\n\t"
        "s_waitcnt lgkmcnt(0)\n\t"
        "s_cmp_eq_u32 s94, 0x400\n\t"
        "s_cselect_b32 s90, s92, s90\n\t"
        "s_sub_u32 s92, s92, s90\n\t"
        "s_cmp_lt_u32 s92, " KDL3 "\n\t"
        "s_cbranch_scc1 SPINP5_%=\n\t"
        "v_mov_b32 v166, %[bad]\n\t"
        "ds_write_b32 v166, v166\n\t"
        "s_branch PREP5_%=\n\t"
        "SLOWF0_%=:\n\t"
        SC1("s_add_u32 %[nfix], %[nfix], 1\n\t")
        "s_cmp_eq_u64 s[84:85], 0\n\t"
        "s_cbranch_scc1 NOFIXF0_%=\n\t"
        "s_add_u32 %[nfix], %[nfix], 1\n\t"
        "v_mov_b32 v166, %[bad]\n\t"
        "ds_read_b32 v164, v166\n\t"
        "s_sub_u32 s92, s89, 2\n\t"
        "s_cmp_lt_u32 s92, 17\n\t"
        "s_cselect_b64 s[86:87], %[xloc], %[xfar]\n\t"
        "s_cselect_b32 s93, 0, 17\n\t"
        "s_sub_u32 s93, s92, s93\n\t"
        "s_lshl_b32 s93, s93, 14\n\t"
        "s_cmp_lt_u32 s92, 112\n\t"
        "s_cselect_b64 s[86:87], s[86:87], %[xt0]\n\t"
        "s_cselect_b32 s92, s93, 0\n\t"
        "s_add_u32 s86, s86, s92\n\t"
        "s_addc_u32 s87, s87, 0\n\t"
        "s_memrealtime s[90:91]\n\t"
        "s_waitcnt lgkmcnt(0)\n\t"
        "v_cmp_ne_u32_e64 s[92:93], 0, v164\n\t"
        "s_cmp_lg_u64 s[92:93], 0\n\t"
        "s_cbranch_scc1 NOFIXF0_%=\n\t"
        "FIXLF0_%=:\n\t"
        "global_load_dwordx4 v[80:83], %[l16], s[86:87] offset:0 sc0 sc1\n\t"
        "global_load_dwordx4 v[84:87], %[l16], s[86:87] offset:1024 sc0 sc1\n\t"
        "s_waitcnt vmcnt(0)\n\t"
        "v_bitop3_b32 v166, v80, v84, %[tag] bitop3:0x7e\n\t"
        "v_and_b32 v166, 1, v166\n\t"
        "v_cmp_ne_u32_e64 s[92:93], 0, v166\n\t"
        "s_cmp_eq_u64 s[92:93], 0\n\t"
        "s_cbranch_scc1 NOFIXF0_%=\n\t"
        "s_sleep 1\n\t"
        "s_memrealtime s[92:93]\n\t"
        "s_waitcnt lgkmcnt(0)\n\t"
        "s_sub_u32 s92, s92, s90\n\t"
        "s_cmp_lt_u32 s92, " KDL "\n\t"
        "s_cbranch_scc1 FIXLF0_%=\n\t"
        "v_mov_b32 v166, %[bad]\n\t"
        "ds_write_b32 v166, v166\n\t"
        "NOFIXF0_%=:\n\t"
        "s_mov_b32 s94, 0\n\t"
        "SPINF0_%=:\n\t"
        "ds_read_b32 v164, %[vctl]\n\t"
        "s_waitcnt lgkmcnt(0)\n\t"
        "v_cmp_gt_u32_e64 s[86:87], s89, v164\n\t"
        "s_cmp_eq_u64 s[86:87], 0\n\t"
        "s_cbranch_scc1 PREF0_%=\n\t"
        SC3("s_add_u32 %[nfix], %[nfix], 1\n\t")
        "s_sleep 1\n\t"
        "s_add_u32 s94, s94, 1\n\t"
        "s_and_b32 s92, s94, 0x3ff\n\t"
        "s_cbranch_scc1 SPINF0_%=\n\t"
        "s_memrealtime s[92:93]\n\t"
        "s_waitcnt lgkmcnt(0)\n\t"
        "s_cmp_eq_u32 s94, 0x400\n\t"
        "s_cselect_b32 s90, s92, s90\n\t"
        "s_sub_u32 s92, s92, s90\n\t"
        "s_cmp_lt_u32 s92, " KDL3 "\n\t"
        "s_cbranch_scc1 SPINF0_%=\n\t"
        "v_mov_b32 v166, %[bad]\n\t"
        "ds_write_b32 v166, v166\n\t"
        "s_branch PREF0_%=\n\t"
        "SLOWF1_%=:\n\t"
        SC1("s_add_u32 %[nfix], %[nfix], 1\n\t")
        "s_cmp_eq_u64 s[84:85], 0\n\t"
        "s_cbranch_scc1 NOFIXF1_%=\n\t"
        "s_add_u32 %[nfix], %[nfix], 1\n\t"
        "v_mov_b32 v166, %[bad]\n\t"
        "ds_read_b32 v164, v166\n\t"
        "s_sub_u32 s92, s89, 2\n\t"
        "s_cmp_lt_u32 s92, 17\n\t"
        "s_cselect_b64 s[86:87], %[xloc], %[xfar]\n\t"
        "s_cselect_b32 s93, 0, 17\n\t"
        "s_sub_u32 s93, s92, s93\n\t"
        "s_lshl_b32 s93, s93, 14\n\t"
        "s_cmp_lt_u32 s92, 112\n\t"
        "s_cselect_b64 s[86:87], s[86:87], %[xt0]\n\t"
        "s_cselect_b32 s92, s93, 0\n\t"
        "s_add_u32 s86, s86, s92\n\t"
        "s_addc_u32 s87, s87, 0\n\t"
        "s_memrealtime s[90:91]\n\t"
        "s_waitcnt lgkmcnt(0)\n\t"
        "v_cmp_ne_u32_e64 s[92:93], 0, v164\n\t"
        "s_cmp_lg_u64 s[92:93], 0\n\t"
        "s_cbranch_scc1 NOFIXF1_%=\n\t"
        "FIXLF1_%=:\n\t"
        "global_load_dwordx4 v[88:91], %[l16], s[86:87] offset:0 sc0 sc1\n\t"
        "global_load_dwordx4 v[92:95], %[l16], s[86:87] offset:1024 sc0 sc1\n\t"
        "s_waitcnt vmcnt(0)\n\t"
        "v_bitop3_b32 v166, v88, v92, %[tag] bitop3:0x7e\n\t"
        "v_and_b32 v166, 1, v166\n\t"
        "v_cmp_ne_u32_e64 s[92:93], 0, v166\n\t"
        "s_cmp_eq_u64 s[92:93], 0\n\t"
        "s_cbranch_scc1 NOFIXF1_%=\n\t"
        "s_sleep 1\n\t"
        "s_memrealtime s[92:93]\n\t"
        "s_waitcnt lgkmcnt(0)\n\t"
        "s_sub_u32 s92, s92, s90\n\t"
        "s_cmp_lt_u32 s92, " KDL "\n\t"
        "s_cbranch_scc1 FIXLF1_%=\n\t"
        "v_mov_b32 v166, %[bad]\n\t"
        "ds_write_b32 v166, v166\n\t"
        "NOFIXF1_%=:\n\t"
        "s_mov_b32 s94, 0\n\t"
        "SPINF1_%=:\n\t"
        "ds_read_b32 v164, %[vctl]\n\t"
        "s_waitcnt lgkmcnt(0)\n\t"
        "v_cmp_gt_u32_e64 s[86:87], s89, v164\n\t"
        "s_cmp_eq_u64 s[86:87], 0\n\t"
        "s_cbranch_scc1 PREF1_%=\n\t"
        SC3("s_add_u32 %[nfix], %[nfix], 1\n\t")
        "s_sleep 1\n\t"
        "s_add_u32 s94, s94, 1\n\t"
        "s_and_b32 s92, s94, 0x3ff\n\t"
        "s_cbranch_scc1 SPINF1_%=\n\t"
        "s_memrealtime s[92:93]\n\t"
        "s_waitcnt lgkmcnt(0)\n\t"
        "s_cmp_eq_u32 s94, 0x400\n\t"
        "s_cselect_b32 s90, s92, s90\n\t"
        "s_sub_u32 s92, s92, s90\n\t"
        "s_cmp_lt_u32 s92, " KDL3 "\n\t"
        "s_cbranch_scc1 SPINF1_%=\n\t"
        "v_mov_b32 v166, %[bad]\n\t"
        "ds_write_b32 v166, v166\n\t"
        "s_branch PREF1_%=\n\t"
        "SLOWF2_%=:\n\t"
        SC1("s_add_u32 %[nfix], %[nfix], 1\n\t")
        "s_cmp_eq_u64 s[84:85], 0\n\t"
        "s_cbranch_scc1 NOFIXF2_%=\n\t"
        "s_add_u32 %[nfix], %[nfix], 1\n\t"
        "v_mov_b32 v166, %[bad]\n\t"
        "ds_read_b32 v164, v166\n\t"
        "s_sub_u32 s92, s89, 2\n\t"
        "s_cmp_lt_u32 s92, 17\n\t"
        "s_cselect_b64 s[86:87], %[xloc], %[xfar]\n\t"
        "s_cselect_b32 s93, 0, 17\n\t"
        "s_sub_u32 s93, s92, s93\n\t"
        "s_lshl_b32 s93, s93, 14\n\t"
        "s_cmp_lt_u32 s92, 112\n\t"
        "s_cselect_b64 s[86:87], s[86:87], %[xt0]\n\t"
        "s_cselect_b32 s92, s93, 0\n\t"
        "s_add_u32 s86, s86, s92\n\t"
        "s_addc_u32 s87, s87, 0\n\t"
        "s_memrealtime s[90:91]\n\t"
        "s_waitcnt lgkmcnt(0)\n\t"
        "v_cmp_ne_u32_e64 s[92:93], 0, v164\n\t"
        "s_cmp_lg_u64 s[92:93], 0\n\t"
        "s_cbranch_scc1 NOFIXF2_%=\n\t"
        "FIXLF2_%=:\n\t"
        "global_load_dwordx4 v[96:99], %[l16], s[86:87] offset:0 sc0 sc1\n\t"
        "global_load_dwordx4 v[100:103], %[l16], s[86:87] offset:1024 sc0 sc1\n\t"
        "s_waitcnt vmcnt(0)\n\t"
        "v_bitop3_b32 v166, v96, v100, %[tag] bitop3:0x7e\n\t"
        "v_and_b32 v166, 1, v166\n\t"
        "v_cmp_ne_u32_e64 s[92:93], 0, v166\n\t"
        "s_cmp_eq_u64 s[92:93], 0\n\t"
        "s_cbranch_scc1 NOFIXF2_%=\n\t"
        "s_sleep 1\n\t"
        "s_memrealtime s[92:93]\n\t"
        "s_waitcnt lgkmcnt(0)\n\t"
        "s_sub_u32 s92, s92, s90\n\t"
        "s_cmp_lt_u32 s92, " KDL "\n\t"
        "s_cbranch_scc1 FIXLF2_%=\n\t"
        "v_mov_b32 v166, %[bad]\n\t"
        "ds_write_b32 v166, v166\n\t"
        "NOFIXF2_%=:\n\t"
        "s_mov_b32 s94, 0\n\t"
        "SPINF2_%=:\n\t"
        "ds_read_b32 v164, %[vctl]\n\t"
        "s_waitcnt lgkmcnt(0)\n\t"
        "v_cmp_gt_u32_e64 s[86:87], s89, v164\n\t"
        "s_cmp_eq_u64 s[86:87], 0\n\t"
        "s_cbranch_scc1 PREF2_%=\n\t"
        SC3("s_add_u32 %[nfix], %[nfix], 1\n\t")
        "s_sleep 1\n\t"
        "s_add_u32 s94, s94, 1\n\t"
        "s_and_b32 s92, s94, 0x3ff\n\t"
        "s_cbranch_scc1 SPINF2_%=\n\t"
        "s_memrealtime s[92:93]\n\t"
        "s_waitcnt lgkmcnt(0)\n\t"
        "s_cmp_eq_u32 s94, 0x400\n\t"
        "s_cselect_b32 s90, s92, s90\n\t"
        "s_sub_u32 s92, s92, s90\n\t"
        "s_cmp_lt_u32 s92, " KDL3 "\n\t"
        "s_cbranch_scc1 SPINF2_%=\n\t"
        "v_mov_b32 v166, %[bad]\n\t"
        "ds_write_b32 v166, v166\n\t"
        "s_branch PREF2_%=\n\t"
        "SLOWF3_%=:\n\t"
        SC1("s_add_u32 %[nfix], %[nfix], 1\n\t")
        "s_cmp_eq_u64 s[84:85], 0\n\t"
        "s_cbranch_scc1 NOFIXF3_%=\n\t"
        "s_add_u32 %[nfix], %[nfix], 1\n\t"
        "v_mov_b32 v166, %[bad]\n\t"
        "ds_read_b32 v164, v166\n\t"
        "s_sub_u32 s92, s89, 2\n\t"
        "s_cmp_lt_u32 s92, 17\n\t"
        "s_cselect_b64 s[86:87], %[xloc], %[xfar]\n\t"
        "s_cselect_b32 s93, 0, 17\n\t"
        "s_sub_u32 s93, s92, s93\n\t"
        "s_lshl_b32 s93, s93, 14\n\t"
        "s_cmp_lt_u32 s92, 112\n\t"
        "s_cselect_b64 s[86:87], s[86:87], %[xt0]\n\t"
        "s_cselect_b32 s92, s93, 0\n\t"
        "s_add_u32 s86, s86, s92\n\t"
        "s_addc_u32 s87, s87, 0\n\t"
        "s_memrealtime s[90:91]\n\t"
        "s_waitcnt lgkmcnt(0)\n\t"
        "v_cmp_ne_u32_e64 s[92:93], 0, v164\n\t"
        "s_cmp_lg_u64 s[92:93], 0\n\t"
        "s_cbranch_scc1 NOFIXF3_%=\n\t"
        "FIXLF3_%=:\n\t"
        "global_load_dwordx4 v[104:107], %[l16], s[86:87] offset:0 sc0 sc1\n\t"
        "global_load_dwordx4 v[108:111], %[l16], s[86:87] offset:1024 sc0 sc1\n\t"
        "s_waitcnt vmcnt(0)\n\t"
        "v_bitop3_b32 v166, v104, v108, %[tag] bitop3:0x7e\n\t"
        "v_and_b32 v166, 1, v166\n\t"
        "v_cmp_ne_u32_e64 s[92:93], 0, v166\n\t"
        "s_cmp_eq_u64 s[92:93], 0\n\t"
        "s_cbranch_scc1 NOFIXF3_%=\n\t"
        "s_sleep 1\n\t"
        "s_memrealtime s[92:93]\n\t"
        "s_waitcnt lgkmcnt(0)\n\t"
        "s_sub_u32 s92, s92, s90\n\t"
        "s_cmp_lt_u32 s92, " KDL "\n\t"
        "s_cbranch_scc1 FIXLF3_%=\n\t"
        "v_mov_b32 v166, %[bad]\n\t"
        "ds_write_b32 v166, v166\n\t"
        "NOFIXF3_%=:\n\t"
        "s_mov_b32 s94, 0\n\t"
        "SPINF3_%=:\n\t"
        "ds_read_b32 v164, %[vctl]\n\t"
        "s_waitcnt lgkmcnt(0)\n\t"
        "v_cmp_gt_u32_e64 s[86:87], s89, v164\n\t"
        "s_cmp_eq_u64 s[86:87], 0\n\t"
        "s_cbranch_scc1 PREF3_%=\n\t"
        SC3("s_add_u32 %[nfix], %[nfix], 1\n\t")
        "s_sleep 1\n\t"
        "s_add_u32 s94, s94, 1\n\t"
        "s_and_b32 s92, s94, 0x3ff\n\t"
        "s_cbranch_scc1 SPINF3_%=\n\t"
        "s_memrealtime s[92:93]\n\t"
        "s_waitcnt lgkmcnt(0)\n\t"
        "s_cmp_eq_u32 s94, 0x400\n\t"
        "s_cselect_b32 s90, s92, s90\n\t"
        "s_sub_u32 s92, s92, s90\n\t"
        "s_cmp_lt_u32 s92, " KDL3 "\n\t"
        "s_cbranch_scc1 SPINF3_%=\n\t"
        "v_mov_b32 v166, %[bad]\n\t"
        "ds_write_b32 v166, v166\n\t"
        "s_branch PREF3_%=\n\t"
        "SLOWF4_%=:\n\t"
        SC1("s_add_u32 %[nfix], %[nfix], 1\n\t")
        "s_cmp_eq_u64 s[84:85], 0\n\t"
        "s_cbranch_scc1 NOFIXF4_%=\n\t"
        "s_add_u32 %[nfix], %[nfix], 1\n\t"
        "v_mov_b32 v166, %[bad]\n\t"
        "ds_read_b32 v164, v166\n\t"
        "s_sub_u32 s92, s89, 2\n\t"
        "s_cmp_lt_u32 s92, 17\n\t"
        "s_cselect_b64 s[86:87], %[xloc], %[xfar]\n\t"
        "s_cselect_b32 s93, 0, 17\n\t"
        "s_sub_u32 s93, s92, s93\n\t"
        "s_lshl_b32 s93, s93, 14\n\t"
        "s_cmp_lt_u32 s92, 112\n\t"
        "s_cselect_b64 s[86:87], s[86:87], %[xt0]\n\t"
        "s_cselect_b32 s92, s93, 0\n\t"
        "s_add_u32 s86, s86, s92\n\t"
        "s_addc_u32 s87, s87, 0\n\t"
        "s_memrealtime s[90:91]\n\t"
        "s_waitcnt lgkmcnt(0)\n\t"
        "v_cmp_ne_u32_e64 s[92:93], 0, v164\n\t"
        "s_cmp_lg_u64 s[92:93], 0\n\t"
        "s_cbranch_scc1 NOFIXF4_%=\n\t"
        "FIXLF4_%=:\n\t"
        "global_load_dwordx4 v[112:115], %[l16], s[86:87] offset:0 sc0 sc1\n\t"
        "global_load_dwordx4 v[116:119], %[l16], s[86:87] offset:1024 sc0 sc1\n\t"
        "s_waitcnt vmcnt(0)\n\t"
        "v_bitop3_b32 v166, v112, v116, %[tag] bitop3:0x7e\n\t"
        "v_and_b32 v166, 1, v166\n\t"
        "v_cmp_ne_u32_e64 s[92:93], 0, v166\n\t"
        "s_cmp_eq_u64 s[92:93], 0\n\t"
        "s_cbranch_scc1 NOFIXF4_%=\n\t"
        "s_sleep 1\n\t"
        "s_memrealtime s[92:93]\n\t"
        "s_waitcnt lgkmcnt(0)\n\t"
        "s_sub_u32 s92, s92, s90\n\t"
        "s_cmp_lt_u32 s92, " KDL "\n\t"
        "s_cbranch_scc1 FIXLF4_%=\n\t"
        "v_mov_b32 v166, %[bad]\n\t"
        "ds_write_b32 v166, v166\n\t"
        "NOFIXF4_%=:\n\t"
        "s_mov_b32 s94, 0\n\t"
        "SPINF4_%=:\n\t"
        "ds_read_b32 v164, %[vctl]\n\t"
        "s_waitcnt lgkmcnt(0)\n\t"
        "v_cmp_gt_u32_e64 s[86:87], s89, v164\n\t"
        "s_cmp_eq_u64 s[86:87], 0\n\t"
        "s_cbranch_scc1 PREF4_%=\n\t"
        SC3("s_add_u32 %[nfix], %[nfix], 1\n\t")
        "s_sleep 1\n\t"
        "s_add_u32 s94, s94, 1\n\t"
        "s_and_b32 s92, s94, 0x3ff\n\t"
        "s_cbranch_scc1 SPINF4_%=\n\t"
        "s_memrealtime s[92:93]\n\t"
        "s_waitcnt lgkmcnt(0)\n\t"
        "s_cmp_eq_u32 s94, 0x400\n\t"
        "s_cselect_b32 s90, s92, s90\n\t"
        "s_sub_u32 s92, s92, s90\n\t"
        "s_cmp_lt_u32 s92, " KDL3 "\n\t"
        "s_cbranch_scc1 SPINF4_%=\n\t"
        "v_mov_b32 v166, %[bad]\n\t"
        "ds_write_b32 v166, v166\n\t"
        "s_branch PREF4_%=\n\t"
        "SLOWF5_%=:\n\t"
        SC1("s_add_u32 %[nfix], %[nfix], 1\n\t")
        "s_cmp_eq_u64 s[84:85], 0\n\t"
        "s_cbranch_scc1 NOFIXF5_%=\n\t"
        "s_add_u32 %[nfix], %[nfix], 1\n\t"
        "v_mov_b32 v166, %[bad]\n\t"
        "ds_read_b32 v164, v166\n\t"
        "s_sub_u32 s92, s89, 2\n\t"
        "s_cmp_lt_u32 s92, 17\n\t"
        "s_cselect_b64 s[86:87], %[xloc], %[xfar]\n\t"
        "s_cselect_b32 s93, 0, 17\n\t"
        "s_sub_u32 s93, s92, s93\n\t"
        "s_lshl_b32 s93, s93, 14\n\t"
        "s_cmp_lt_u32 s92, 112\n\t"
        "s_cselect_b64 s[86:87], s[86:87], %[xt0]\n\t"
        "s_cselect_b32 s92, s93, 0\n\t"
        "s_add_u32 s86, s86, s92\n\t"
        "s_addc_u32 s87, s87, 0\n\t"
        "s_memrealtime s[90:91]\n\t"
        "s_waitcnt lgkmcnt(0)\n\t"
        "v_cmp_ne_u32_e64 s[92:93], 0, v164\n\t"
        "s_cmp_lg_u64 s[92:93], 0\n\t"
        "s_cbranch_scc1 NOFIXF5_%=\n\t"
        "FIXLF5_%=:\n\t"
        "global_load_dwordx4 v[120:123], %[l16], s[86:87] offset:0 sc0 sc1\n\t"
        "global_load_dwordx4 v[124:127], %[l16], s[86:87] offset:1024 sc0 sc1\n\t"
        "s_waitcnt vmcnt(0)\n\t"
        "v_bitop3_b32 v166, v120, v124, %[tag] bitop3:0x7e\n\t"
        "v_and_b32 v166, 1, v166\n\t"
        "v_cmp_ne_u32_e64 s[92:93], 0, v166\n\t"
        "s_cmp_eq_u64 s[92:93], 0\n\t"
        "s_cbranch_scc1 NOFIXF5_%=\n\t"
        "s_sleep 1\n\t"
        "s_memrealtime s[92:93]\n\t"
        "s_waitcnt lgkmcnt(0)\n\t"
        "s_sub_u32 s92, s92, s90\n\t"
        "s_cmp_lt_u32 s92, " KDL "\n\t"
        "s_cbranch_scc1 FIXLF5_%=\n\t"
        "v_mov_b32 v166, %[bad]\n\t"
        "ds_write_b32 v166, v166\n\t"
        "NOFIXF5_%=:\n\t"
        "s_mov_b32 s94, 0\n\t"
        "SPINF5_%=:\n\t"
        "ds_read_b32 v164, %[vctl]\n\t"
        "s_waitcnt lgkmcnt(0)\n\t"
        "v_cmp_gt_u32_e64 s[86:87], s89, v164\n\t"
        "s_cmp_eq_u64 s[86:87], 0\n\t"
        "s_cbranch_scc1 PREF5_%=\n\t"
        SC3("s_add_u32 %[nfix], %[nfix], 1\n\t")
        "s_sleep 1\n\t"
        "s_add_u32 s94, s94, 1\n\t"
        "s_and_b32 s92, s94, 0x3ff\n\t"
        "s_cbranch_scc1 SPINF5_%=\n\t"
        "s_memrealtime s[92:93]\n\t"
        "s_waitcnt lgkmcnt(0)\n\t"
        "s_cmp_eq_u32 s94, 0x400\n\t"
        "s_cselect_b32 s90, s92, s90\n\t"
        "s_sub_u32 s92, s92, s90\n\t"
        "s_cmp_lt_u32 s92, " KDL3 "\n\t"
        "s_cbranch_scc1 SPINF5_%=\n\t"
        "v_mov_b32 v166, %[bad]\n\t"
        "ds_write_b32 v166, v166\n\t"
        "s_branch PREF5_%=\n\t"
        ".p2align 6\n\t"
        "TOPF_%=:\n\t"
        TRM("s_memrealtime s[90:91]\n\t")
        TRM("s_mul_i32 s93, s89, 43\n\t")
        TRM("s_lshr_b32 s93, s93, 8\n\t")
        TRM("s_add_u32 s93, s93, 24\n\t")
        TRM("s_mov_b32 m0, s93\n\t")
        TRM("s_waitcnt lgkmcnt(0)\n\t")
        TRM("v_writelane_b32 %[tr], s90, m0\n\t")
        "s_waitcnt lgkmcnt(9)\n\t"
        MF("v_mfma_f32_16x16x32_f16 v[8:11], v[112:115], v[128:131], v[8:11]\n\t")
        "v_add_u32 v165, s88, %[l16]\n\t"
        "global_load_dwordx4 v[80:83], %[l16], s[96:97] offset:0 " POL_FAR "\n\t"
        MF("v_mfma_f32_16x16x32_f16 v[44:47], v[116:119], v[128:131], v[44:47]\n\t")
        "global_load_dwordx4 v[84:87], %[l16], s[96:97] offset:1024 " POL_FAR "\n\t"
        "ds_read_b128 v[128:131], v165 offset:0\n\t"
        "s_waitcnt lgkmcnt(9)\n\t"
        MF("v_mfma_f32_16x16x32_f16 v[12:15], v[112:115], v[132:135], v[12:15]\n\t")
        "s_add_u32 s88, s88, 9216\n\t"
        "s_cmp_eq_u32 s88, 147456\n\t"
        "s_cselect_b32 s88, 0, s88\n\t"
        MF("v_mfma_f32_16x16x32_f16 v[48:51], v[116:119], v[132:135], v[48:51]\n\t")
        "ds_read_b128 v[132:135], v165 offset:1024\n\t"
        "s_waitcnt lgkmcnt(9)\n\t"
        MF("v_mfma_f32_16x16x32_f16 v[16:19], v[112:115], v[136:139], v[16:19]\n\t")
        "s_add_u32 s96, s96, s95\n\t"
        "s_addc_u32 s97, s97, 0\n\t"
        MF("v_mfma_f32_16x16x32_f16 v[52:55], v[116:119], v[136:139], v[52:55]\n\t")
        "ds_read_b128 v[136:139], v165 offset:2048\n\t"
        "ds_read_b32 v164, %[vctl]\n\t"
        "s_waitcnt lgkmcnt(9)\n\t"
        MF("v_mfma_f32_16x16x32_f16 v[20:23], v[112:115], v[140:143], v[20:23]\n\t")
        MF("v_mfma_f32_16x16x32_f16 v[56:59], v[116:119], v[140:143], v[56:59]\n\t")
        "ds_read_b128 v[140:143], v165 offset:3072\n\t"
        "s_waitcnt lgkmcnt(9)\n\t"
        MF("v_mfma_f32_16x16x32_f16 v[24:27], v[112:115], v[144:147], v[24:27]\n\t")
        "s_add_u32 s89, s89, 1\n\t"
        MF("v_mfma_f32_16x16x32_f16 v[60:63], v[116:119], v[144:147], v[60:63]\n\t")
        "ds_read_b128 v[144:147], v165 offset:4096\n\t"
        "s_waitcnt lgkmcnt(9)\n\t"
        MF("v_mfma_f32_16x16x32_f16 v[28:31], v[112:115], v[148:151], v[28:31]\n\t")
        MF("v_mfma_f32_16x16x32_f16 v[64:67], v[116:119], v[148:151], v[64:67]\n\t")
        "ds_read_b128 v[148:151], v165 offset:5120\n\t"
        "s_waitcnt lgkmcnt(9)\n\t"
        "s_waitcnt vmcnt(2)\n\t"
        MF("v_mfma_f32_16x16x32_f16 v[32:35], v[112:115], v[152:155], v[32:35]\n\t")
        "v_bitop3_b32 v166, v120, v124, %[tag] bitop3:0x7e\n\t"
        MF("v_mfma_f32_16x16x32_f16 v[68:71], v[116:119], v[152:155], v[68:71]\n\t")
        "v_and_b32 v166, 1, v166\n\t"
        "ds_read_b128 v[152:155], v165 offset:6144\n\t"
        "s_waitcnt lgkmcnt(4)\n\t"
        MF("v_mfma_f32_16x16x32_f16 v[36:39], v[112:115], v[156:159], v[36:39]\n\t")
        "v_cmp_gt_u32_e64 s[86:87], s89, v164\n\t"
        TG("v_cmp_ne_u32_e64 s[84:85], 0, v166\n\t")
        NTG("s_mov_b64 s[84:85], 0\n\t")
        MF("v_mfma_f32_16x16x32_f16 v[72:75], v[116:119], v[156:159], v[72:75]\n\t")
        "ds_read_b128 v[156:159], v165 offset:7168\n\t"
        "s_mov_b64 exec, 1\n\t"
        "ds_add_u32 %[vctl], v167\n\t"
        "s_mov_b64 exec, -1\n\t"
        "s_or_b64 s[90:91], s[84:85], s[86:87]\n\t"
        "s_cbranch_scc1 SLOWF5_%=\n\t"
        "PREF5_%=:\n\t"
        "s_waitcnt lgkmcnt(9)\n\t"
        MF("v_mfma_f32_16x16x32_f16 v[8:11], v[120:123], v[128:131], v[8:11]\n\t")
        "v_add_u32 v165, s88, %[l16]\n\t"
        "global_load_dwordx4 v[88:91], %[l16], s[96:97] offset:0 " POL_FAR "\n\t"
        MF("v_mfma_f32_16x16x32_f16 v[44:47], v[124:127], v[128:131], v[44:47]\n\t")
        "global_load_dwordx4 v[92:95], %[l16], s[96:97] offset:1024 " POL_FAR "\n\t"
        "ds_read_b128 v[128:131], v165 offset:0\n\t"
        "s_waitcnt lgkmcnt(9)\n\t"
        MF("v_mfma_f32_16x16x32_f16 v[12:15], v[120:123], v[132:135], v[12:15]\n\t")
        "s_add_u32 s88, s88, 9216\n\t"
        "s_cmp_eq_u32 s88, 147456\n\t"
        "s_cselect_b32 s88, 0, s88\n\t"
        MF("v_mfma_f32_16x16x32_f16 v[48:51], v[124:127], v[132:135], v[48:51]\n\t")
        "ds_read_b128 v[132:135], v165 offset:1024\n\t"
        "s_waitcnt lgkmcnt(9)\n\t"
        MF("v_mfma_f32_16x16x32_f16 v[16:19], v[120:123], v[136:139], v[16:19]\n\t")
        "s_add_u32 s96, s96, s95\n\t"
        "s_addc_u32 s97, s97, 0\n\t"
        MF("v_mfma_f32_16x16x32_f16 v[52:55], v[124:127], v[136:139], v[52:55]\n\t")
        "ds_read_b128 v[136:139], v165 offset:2048\n\t"
        "ds_read_b32 v164, %[vctl]\n\t"
        "s_waitcnt lgkmcnt(9)\n\t"
        MF("v_mfma_f32_16x16x32_f16 v[20:23], v[120:123], v[140:143], v[20:23]\n\t")
        MF("v_mfma_f32_16x16x32_f16 v[56:59], v[124:127], v[140:143], v[56:59]\n\t")
        "ds_read_b128 v[140:143], v165 offset:3072\n\t"
        "s_waitcnt lgkmcnt(9)\n\t"
        MF("v_mfma_f32_16x16x32_f16 v[24:27], v[120:123], v[144:147], v[24:27]\n\t")
        "s_add_u32 s89, s89, 1\n\t"
        MF("v_mfma_f32_16x16x32_f16 v[60:63], v[124:127], v[144:147], v[60:63]\n\t")
        "ds_read_b128 v[144:147], v165 offset:4096\n\t"
        "s_waitcnt lgkmcnt(9)\n\t"
        MF("v_mfma_f32_16x16x32_f16 v[28:31], v[120:123], v[148:151], v[28:31]\n\t")
        MF("v_mfma_f32_16x16x32_f16 v[64:67], v[124:127], v[148:151], v[64:67]\n\t")
        "ds_read_b128 v[148:151], v165 offset:5120\n\t"
        "s_waitcnt lgkmcnt(9)\n\t"
        "s_waitcnt vmcnt(2)\n\t"
        MF("v_mfma_f32_16x16x32_f16 v[32:35], v[120:123], v[152:155], v[32:35]\n\t")
        "v_bitop3_b32 v166, v80, v84, %[tag] bitop3:0x7e\n\t"
        MF("v_mfma_f32_16x16x32_f16 v[68:71], v[124:127], v[152:155], v[68:71]\n\t")
        "v_and_b32 v166, 1, v166\n\t"
        "ds_read_b128 v[152:155], v165 offset:6144\n\t"
        "s_waitcnt lgkmcnt(4)\n\t"
        MF("v_mfma_f32_16x16x32_f16 v[36:39], v[120:123], v[156:159], v[36:39]\n\t")
        "v_cmp_gt_u32_e64 s[86:87], s89, v164\n\t"
        TG("v_cmp_ne_u32_e64 s[84:85], 0, v166\n\t")
        NTG("s_mov_b64 s[84:85], 0\n\t")
        MF("v_mfma_f32_16x16x32_f16 v[72:75], v[124:127], v[156:159], v[72:75]\n\t")
        "ds_read_b128 v[156:159], v165 offset:7168\n\t"
        "s_mov_b64 exec, 1\n\t"
        "ds_add_u32 %[vctl], v167\n\t"
        "s_mov_b64 exec, -1\n\t"
        "s_or_b64 s[90:91], s[84:85], s[86:87]\n\t"
        "s_cbranch_scc1 SLOWF0_%=\n\t"
        "PREF0_%=:\n\t"
        "s_waitcnt lgkmcnt(9)\n\t"
        MF("v_mfma_f32_16x16x32_f16 v[8:11], v[80:83], v[128:131], v[8:11]\n\t")
        "v_add_u32 v165, s88, %[l16]\n\t"
        "global_load_dwordx4 v[96:99], %[l16], s[96:97] offset:0 " POL_FAR "\n\t"
        MF("v_mfma_f32_16x16x32_f16 v[44:47], v[84:87], v[128:131], v[44:47]\n\t")
        "global_load_dwordx4 v[100:103], %[l16], s[96:97] offset:1024 " POL_FAR "\n\t"
        "ds_read_b128 v[128:131], v165 offset:0\n\t"
        "s_waitcnt lgkmcnt(9)\n\t"
        MF("v_mfma_f32_16x16x32_f16 v[12:15], v[80:83], v[132:135], v[12:15]\n\t")
        "s_add_u32 s88, s88, 9216\n\t"
        "s_cmp_eq_u32 s88, 147456\n\t"
        "s_cselect_b32 s88, 0, s88\n\t"
        MF("v_mfma_f32_16x16x32_f16 v[48:51], v[84:87], v[132:135], v[48:51]\n\t")
        "ds_read_b128 v[132:135], v165 offset:1024\n\t"
        "s_waitcnt lgkmcnt(9)\n\t"
        MF("v_mfma_f32_16x16x32_f16 v[16:19], v[80:83], v[136:139], v[16:19]\n\t")
        "s_add_u32 s96, s96, s95\n\t"
        "s_addc_u32 s97, s97, 0\n\t"
        MF("v_mfma_f32_16x16x32_f16 v[52:55], v[84:87], v[136:139], v[52:55]\n\t")
        "ds_read_b128 v[136:139], v165 offset:2048\n\t"
        "ds_read_b32 v164, %[vctl]\n\t"
        "s_waitcnt lgkmcnt(9)\n\t"
        MF("v_mfma_f32_16x16x32_f16 v[20:23], v[80:83], v[140:143], v[20:23]\n\t")
        MF("v_mfma_f32_16x16x32_f16 v[56:59], v[84:87], v[140:143], v[56:59]\n\t")
        "ds_read_b128 v[140:143], v165 offset:3072\n\t"
        "s_waitcnt lgkmcnt(9)\n\t"
        MF("v_mfma_f32_16x16x32_f16 v[24:27], v[80:83], v[144:147], v[24:27]\n\t")
        "s_add_u32 s89, s89, 1\n\t"
        MF("v_mfma_f32_16x16x32_f16 v[60:63], v[84:87], v[144:147], v[60:63]\n\t")
        "ds_read_b128 v[144:147], v165 offset:4096\n\t"
        "s_waitcnt lgkmcnt(9)\n\t"
        MF("v_mfma_f32_16x16x32_f16 v[28:31], v[80:83], v[148:151], v[28:31]\n\t")
        MF("v_mfma_f32_16x16x32_f16 v[64:67], v[84:87], v[148:151], v[64:67]\n\t")
        "ds_read_b128 v[148:151], v165 offset:5120\n\t"
        "s_waitcnt lgkmcnt(9)\n\t"
        "s_waitcnt vmcnt(2)\n\t"
        MF("v_mfma_f32_16x16x32_f16 v[32:35], v[80:83], v[152:155], v[32:35]\n\t")
        "v_bitop3_b32 v166, v88, v92, %[tag] bitop3:0x7e\n\t"
        MF("v_mfma_f32_16x16x32_f16 v[68:71], v[84:87], v[152:155], v[68:71]\n\t")
        "v_and_b32 v166, 1, v166\n\t"
        "ds_read_b128 v[152:155], v165 offset:6144\n\t"
        "s_waitcnt lgkmcnt(4)\n\t"
        MF("v_mfma_f32_16x16x32_f16 v[36:39], v[80:83], v[156:159], v[36:39]\n\t")
        "v_cmp_gt_u32_e64 s[86:87], s89, v164\n\t"
        TG("v_cmp_ne_u32_e64 s[84:85], 0, v166\n\t")
        NTG("s_mov_b64 s[84:85], 0\n\t")
        MF("v_mfma_f32_16x16x32_f16 v[72:75], v[84:87], v[156:159], v[72:75]\n\t")
        "ds_read_b128 v[156:159], v165 offset:7168\n\t"
        "s_mov_b64 exec, 1\n\t"
        "ds_add_u32 %[vctl], v167\n\t"
        "s_mov_b64 exec, -1\n\t"
        "s_or_b64 s[90:91], s[84:85], s[86:87]\n\t"
        "s_cbranch_scc1 SLOWF1_%=\n\t"
        "PREF1_%=:\n\t"
        "s_waitcnt lgkmcnt(9)\n\t"
        MF("v_mfma_f32_16x16x32_f16 v[8:11], v[88:91], v[128:131], v[8:11]\n\t")
        "v_add_u32 v165, s88, %[l16]\n\t"
        "global_load_dwordx4 v[104:107], %[l16], s[96:97] offset:0 " POL_FAR "\n\t"
        MF("v_mfma_f32_16x16x32_f16 v[44:47], v[92:95], v[128:131], v[44:47]\n\t")
        "global_load_dwordx4 v[108:111], %[l16], s[96:97] offset:1024 " POL_FAR "\n\t"
        "ds_read_b128 v[128:131], v165 offset:0\n\t"
        "s_waitcnt lgkmcnt(9)\n\t"
        MF("v_mfma_f32_16x16x32_f16 v[12:15], v[88:91], v[132:135], v[12:15]\n\t")
        "s_add_u32 s88, s88, 9216\n\t"
        "s_cmp_eq_u32 s88, 147456\n\t"
        "s_cselect_b32 s88, 0, s88\n\t"
        MF("v_mfma_f32_16x16x32_f16 v[48:51], v[92:95], v[132:135], v[48:51]\n\t")
        "ds_read_b128 v[132:135], v165 offset:1024\n\t"
        "s_waitcnt lgkmcnt(9)\n\t"
        MF("v_mfma_f32_16x16x32_f16 v[16:19], v[88:91], v[136:139], v[16:19]\n\t")
        "s_add_u32 s96, s96, s95\n\t"
        "s_addc_u32 s97, s97, 0\n\t"
        MF("v_mfma_f32_16x16x32_f16 v[52:55], v[92:95], v[136:139], v[52:55]\n\t")
        "ds_read_b128 v[136:139], v165 offset:2048\n\t"
        "ds_read_b32 v164, %[vctl]\n\t"
        "s_waitcnt lgkmcnt(9)\n\t"
        MF("v_mfma_f32_16x16x32_f16 v[20:23], v[88:91], v[140:143], v[20:23]\n\t")
        "s_cmp_eq_u32 s89, 111\n\t"
        "s_cselect_b64 s[96:97], %[xt0], s[96:97]\n\t"
        "s_cselect_b32 s95, s98, s95\n\t"
        MF("v_mfma_f32_16x16x32_f16 v[56:59], v[92:95], v[140:143], v[56:59]\n\t")
        "ds_read_b128 v[140:143], v165 offset:3072\n\t"
        "s_waitcnt lgkmcnt(9)\n\t"
        MF("v_mfma_f32_16x16x32_f16 v[24:27], v[88:91], v[144:147], v[24:27]\n\t")
        "s_add_u32 s89, s89, 1\n\t"
        MF("v_mfma_f32_16x16x32_f16 v[60:63], v[92:95], v[144:147], v[60:63]\n\t")
        "ds_read_b128 v[144:147], v165 offset:4096\n\t"
        "s_waitcnt lgkmcnt(9)\n\t"
        MF("v_mfma_f32_16x16x32_f16 v[28:31], v[88:91], v[148:151], v[28:31]\n\t")
        MF("v_mfma_f32_16x16x32_f16 v[64:67], v[92:95], v[148:151], v[64:67]\n\t")
        "ds_read_b128 v[148:151], v165 offset:5120\n\t"
        "s_waitcnt lgkmcnt(9)\n\t"
        "s_waitcnt vmcnt(2)\n\t"
        MF("v_mfma_f32_16x16x32_f16 v[32:35], v[88:91], v[152:155], v[32:35]\n\t")
        "v_bitop3_b32 v166, v96, v100, %[tag] bitop3:0x7e\n\t"
        MF("v_mfma_f32_16x16x32_f16 v[68:71], v[92:95], v[152:155], v[68:71]\n\t")
        "v_and_b32 v166, 1, v166\n\t"
        "ds_read_b128 v[152:155], v165 offset:6144\n\t"
        "s_waitcnt lgkmcnt(4)\n\t"
        MF("v_mfma_f32_16x16x32_f16 v[36:39], v[88:91], v[156:159], v[36:39]\n\t")
        "v_cmp_gt_u32_e64 s[86:87], s89, v164\n\t"
        TG("v_cmp_ne_u32_e64 s[84:85], 0, v166\n\t")
        NTG("s_mov_b64 s[84:85], 0\n\t")
        MF("v_mfma_f32_16x16x32_f16 v[72:75], v[92:95], v[156:159], v[72:75]\n\t")
        "ds_read_b128 v[156:159], v165 offset:7168\n\t"
        "s_mov_b64 exec, 1\n\t"
        "ds_add_u32 %[vctl], v167\n\t"
        "s_mov_b64 exec, -1\n\t"
        "s_or_b64 s[90:91], s[84:85], s[86:87]\n\t"
        "s_cbranch_scc1 SLOWF2_%=\n\t"
        "PREF2_%=:\n\t"
        "s_waitcnt lgkmcnt(9)\n\t"
        MF("v_mfma_f32_16x16x32_f16 v[8:11], v[96:99], v[128:131], v[8:11]\n\t")
        "v_add_u32 v165, s88, %[l16]\n\t"
        "global_load_dwordx4 v[112:115], %[l16], s[96:97] offset:0 " POL_FAR "\n\t"
        MF("v_mfma_f32_16x16x32_f16 v[44:47], v[100:103], v[128:131], v[44:47]\n\t")
        "global_load_dwordx4 v[116:119], %[l16], s[96:97] offset:1024 " POL_FAR "\n\t"
        "ds_read_b128 v[128:131], v165 offset:0\n\t"
        "s_waitcnt lgkmcnt(9)\n\t"
        MF("v_mfma_f32_16x16x32_f16 v[12:15], v[96:99], v[132:135], v[12:15]\n\t")
        "s_add_u32 s88, s88, 9216\n\t"
        "s_cmp_eq_u32 s88, 147456\n\t"
        "s_cselect_b32 s88, 0, s88\n\t"
        MF("v_mfma_f32_16x16x32_f16 v[48:51], v[100:103], v[132:135], v[48:51]\n\t")
        "ds_read_b128 v[132:135], v165 offset:1024\n\t"
        "s_waitcnt lgkmcnt(9)\n\t"
        MF("v_mfma_f32_16x16x32_f16 v[16:19], v[96:99], v[136:139], v[16:19]\n\t")
        "s_add_u32 s96, s96, s95\n\t"
        "s_addc_u32 s97, s97, 0\n\t"
        MF("v_mfma_f32_16x16x32_f16 v[52:55], v[100:103], v[136:139], v[52:55]\n\t")
        "ds_read_b128 v[136:139], v165 offset:2048\n\t"
        "ds_read_b32 v164, %[vctl]\n\t"
        "s_waitcnt lgkmcnt(9)\n\t"
        MF("v_mfma_f32_16x16x32_f16 v[20:23], v[96:99], v[140:143], v[20:23]\n\t")
        MF("v_mfma_f32_16x16x32_f16 v[56:59], v[100:103], v[140:143], v[56:59]\n\t")
        "ds_read_b128 v[140:143], v165 offset:3072\n\t"
        "s_waitcnt lgkmcnt(9)\n\t"
        MF("v_mfma_f32_16x16x32_f16 v[24:27], v[96:99], v[144:147], v[24:27]\n\t")
        "s_add_u32 s89, s89, 1\n\t"
        MF("v_mfma_f32_16x16x32_f16 v[60:63], v[100:103], v[144:147], v[60:63]\n\t")
        "ds_read_b128 v[144:147], v165 offset:4096\n\t"
        "s_waitcnt lgkmcnt(9)\n\t"
        MF("v_mfma_f32_16x16x32_f16 v[28:31], v[96:99], v[148:151], v[28:31]\n\t")
        MF("v_mfma_f32_16x16x32_f16 v[64:67], v[100:103], v[148:151], v[64:67]\n\t")
        "ds_read_b128 v[148:151], v165 offset:5120\n\t"
        "s_waitcnt lgkmcnt(9)\n\t"
        "s_waitcnt vmcnt(2)\n\t"
        MF("v_mfma_f32_16x16x32_f16 v[32:35], v[96:99], v[152:155], v[32:35]\n\t")
        "v_bitop3_b32 v166, v104, v108, %[tag] bitop3:0x7e\n\t"
        MF("v_mfma_f32_16x16x32_f16 v[68:71], v[100:103], v[152:155], v[68:71]\n\t")
        "v_and_b32 v166, 1, v166\n\t"
        "ds_read_b128 v[152:155], v165 offset:6144\n\t"
        "s_waitcnt lgkmcnt(4)\n\t"
        MF("v_mfma_f32_16x16x32_f16 v[36:39], v[96:99], v[156:159], v[36:39]\n\t")
        "v_cmp_gt_u32_e64 s[86:87], s89, v164\n\t"
        TG("v_cmp_ne_u32_e64 s[84:85], 0, v166\n\t")
        NTG("s_mov_b64 s[84:85], 0\n\t")
        MF("v_mfma_f32_16x16x32_f16 v[72:75], v[100:103], v[156:159], v[72:75]\n\t")
        "ds_read_b128 v[156:159], v165 offset:7168\n\t"
        "s_mov_b64 exec, 1\n\t"
        "ds_add_u32 %[vctl], v167\n\t"
        "s_mov_b64 exec, -1\n\t"
        "s_or_b64 s[90:91], s[84:85], s[86:87]\n\t"
        "s_cbranch_scc1 SLOWF3_%=\n\t"
        "PREF3_%=:\n\t"
        "s_waitcnt lgkmcnt(9)\n\t"
        MF("v_mfma_f32_16x16x32_f16 v[8:11], v[104:107], v[128:131], v[8:11]\n\t")
        "v_add_u32 v165, s88, %[l16]\n\t"
        "global_load_dwordx4 v[120:123], %[l16], s[96:97] offset:0 " POL_FAR "\n\t"
        MF("v_mfma_f32_16x16x32_f16 v[44:47], v[108:111], v[128:131], v[44:47]\n\t")
        "global_load_dwordx4 v[124:127], %[l16], s[96:97] offset:1024 " POL_FAR "\n\t"
        "ds_read_b128 v[128:131], v165 offset:0\n\t"
        "s_waitcnt lgkmcnt(9)\n\t"
        MF("v_mfma_f32_16x16x32_f16 v[12:15], v[104:107], v[132:135], v[12:15]\n\t")
        "s_add_u32 s88, s88, 9216\n\t"
        "s_cmp_eq_u32 s88, 147456\n\t"
        "s_cselect_b32 s88, 0, s88\n\t"
        MF("v_mfma_f32_16x16x32_f16 v[48:51], v[108:111], v[132:135], v[48:51]\n\t")
        "ds_read_b128 v[132:135], v165 offset:1024\n\t"
        "s_waitcnt lgkmcnt(9)\n\t"
        MF("v_mfma_f32_16x16x32_f16 v[16:19], v[104:107], v[136:139], v[16:19]\n\t")
        "s_add_u32 s96, s96, s95\n\t"
        "s_addc_u32 s97, s97, 0\n\t"
        MF("v_mfma_f32_16x16x32_f16 v[52:55], v[108:111], v[136:139], v[52:55]\n\t")
        "ds_read_b128 v[136:139], v165 offset:2048\n\t"
        "ds_read_b32 v164, %[vctl]\n\t"
        "s_waitcnt lgkmcnt(9)\n\t"
        MF("v_mfma_f32_16x16x32_f16 v[20:23], v[104:107], v[140:143], v[20:23]\n\t")
        MF("v_mfma_f32_16x16x32_f16 v[56:59], v[108:111], v[140:143], v[56:59]\n\t")
        "ds_read_b128 v[140:143], v165 offset:3072\n\t"
        "s_waitcnt lgkmcnt(9)\n\t"
        MF("v_mfma_f32_16x16x32_f16 v[24:27], v[104:107], v[144:147], v[24:27]\n\t")
        "s_add_u32 s89, s89, 1\n\t"
        MF("v_mfma_f32_16x16x32_f16 v[60:63], v[108:111], v[144:147], v[60:63]\n\t")
        "ds_read_b128 v[144:147], v165 offset:4096\n\t"
        "s_waitcnt lgkmcnt(9)\n\t"
        MF("v_mfma_f32_16x16x32_f16 v[28:31], v[104:107], v[148:151], v[28:31]\n\t")
        MF("v_mfma_f32_16x16x32_f16 v[64:67], v[108:111], v[148:151], v[64:67]\n\t")
        "ds_read_b128 v[148:151], v165 offset:5120\n\t"
        "s_waitcnt lgkmcnt(9)\n\t"
        "s_waitcnt vmcnt(2)\n\t"
        MF("v_mfma_f32_16x16x32_f16 v[32:35], v[104:107], v[152:155], v[32:35]\n\t")
        "v_bitop3_b32 v166, v112, v116, %[tag] bitop3:0x7e\n\t"
        MF("v_mfma_f32_16x16x32_f16 v[68:71], v[108:111], v[152:155], v[68:71]\n\t")
        "v_and_b32 v166, 1, v166\n\t"
        "ds_read_b128 v[152:155], v165 offset:6144\n\t"
        "s_waitcnt lgkmcnt(4)\n\t"
        MF("v_mfma_f32_16x16x32_f16 v[36:39], v[104:107], v[156:159], v[36:39]\n\t")
        "v_cmp_gt_u32_e64 s[86:87], s89, v164\n\t"
        TG("v_cmp_ne_u32_e64 s[84:85], 0, v166\n\t")
        NTG("s_mov_b64 s[84:85], 0\n\t")
        MF("v_mfma_f32_16x16x32_f16 v[72:75], v[108:111], v[156:159], v[72:75]\n\t")
        "ds_read_b128 v[156:159], v165 offset:7168\n\t"
        "s_mov_b64 exec, 1\n\t"
        "ds_add_u32 %[vctl], v167\n\t"
        "s_mov_b64 exec, -1\n\t"
        "s_or_b64 s[90:91], s[84:85], s[86:87]\n\t"
        "s_cbranch_scc1 SLOWF4_%=\n\t"
        "PREF4_%=:\n\t"
        "s_cmp_eq_u32 s89, 114\n\t"
        "s_cbranch_scc0 TOPF_%=\n\t"
        "s_cmp_lg_u32 %[stage], 0\n\t"
        "s_cbranch_scc1 ENTRY2_%=\n\t"
        "s_movk_i32 s89, 115\n\t"
        "v_cmp_gt_u32_e64 s[86:87], s89, v164\n\t"
        "s_add_u32 s92, 0, 0\n\t"
        "v_add_u32 v165, s92, %[l16]\n\t"
        "s_and_b64 s[86:87], s[86:87], 0x400\n\t"
        "s_cbranch_scc1 WAITB_%=\n\t"
        "TAILX_%=:\n\t"
        "TAIL0_%=:\n\t"
        TRM("s_memrealtime s[90:91]\n\t")
        TRM("s_mov_b32 s93, 8\n\t")
        TRM("s_mov_b32 m0, s93\n\t")
        TRM("s_waitcnt lgkmcnt(0)\n\t")
        TRM("v_writelane_b32 %[tr], s90, m0\n\t")
        "s_waitcnt lgkmcnt(9)\n\t"
        MF("v_mfma_f32_16x16x32_f16 v[8:11], v[112:115], v[128:131], v[8:11]\n\t")
        "global_load_dwordx4 v[80:83], %[l16], s[96:97] offset:0 " POL_FAR "\n\t"
        "ds_read_b128 v[128:131], v165 offset:9216\n\t"
        "s_waitcnt lgkmcnt(9)\n\t"
        MF("v_mfma_f32_16x16x32_f16 v[12:15], v[112:115], v[132:135], v[12:15]\n\t")
        "global_load_dwordx4 v[84:87], %[l16], s[96:97] offset:1024 " POL_FAR "\n\t"
        "ds_read_b128 v[132:135], v165 offset:10240\n\t"
        "s_waitcnt lgkmcnt(9)\n\t"
        MF("v_mfma_f32_16x16x32_f16 v[16:19], v[112:115], v[136:139], v[16:19]\n\t")
        "s_add_u32 s96, s96, s95\n\t"
        "s_addc_u32 s97, s97, 0\n\t"
        "ds_read_b128 v[136:139], v165 offset:11264\n\t"
        "s_waitcnt lgkmcnt(8)\n\t"
        MF("v_mfma_f32_16x16x32_f16 v[20:23], v[112:115], v[140:143], v[20:23]\n\t")
        "ds_read_b128 v[140:143], v165 offset:12288\n\t"
        "s_waitcnt lgkmcnt(8)\n\t"
        MF("v_mfma_f32_16x16x32_f16 v[24:27], v[112:115], v[144:147], v[24:27]\n\t")
        "ds_read_b128 v[144:147], v165 offset:13312\n\t"
        "s_waitcnt lgkmcnt(8)\n\t"
        MF("v_mfma_f32_16x16x32_f16 v[28:31], v[112:115], v[148:151], v[28:31]\n\t")
        "ds_read_b128 v[148:151], v165 offset:14336\n\t"
        "s_waitcnt lgkmcnt(8)\n\t"
        MF("v_mfma_f32_16x16x32_f16 v[32:35], v[112:115], v[152:155], v[32:35]\n\t")
        "ds_read_b128 v[152:155], v165 offset:15360\n\t"
        "s_waitcnt lgkmcnt(8)\n\t"
        MF("v_mfma_f32_16x16x32_f16 v[36:39], v[112:115], v[156:159], v[36:39]\n\t")
        "ds_read_b128 v[156:159], v165 offset:16384\n\t"
        "TAILH_%=:\n\t"
        "s_waitcnt lgkmcnt(7)\n\t"
        MF("v_mfma_f32_16x16x32_f16 v[8:11], v[116:119], v[128:131], v[8:11]\n\t")
        "ds_read_b32 v164, %[vctl]\n\t"
        "s_movk_i32 s89, 117\n\t"
        "ds_read_b128 v[128:131], v165 offset:18432\n\t"
        "s_waitcnt lgkmcnt(8)\n\t"
        MF("v_mfma_f32_16x16x32_f16 v[12:15], v[116:119], v[132:135], v[12:15]\n\t")
        "ds_read_b128 v[132:135], v165 offset:19456\n\t"
        "s_waitcnt lgkmcnt(8)\n\t"
        MF("v_mfma_f32_16x16x32_f16 v[16:19], v[116:119], v[136:139], v[16:19]\n\t")
        "ds_read_b128 v[136:139], v165 offset:20480\n\t"
        "s_waitcnt lgkmcnt(8)\n\t"
        MF("v_mfma_f32_16x16x32_f16 v[20:23], v[116:119], v[140:143], v[20:23]\n\t")
        "ds_read_b128 v[140:143], v165 offset:21504\n\t"
        "s_waitcnt lgkmcnt(8)\n\t"
        MF("v_mfma_f32_16x16x32_f16 v[24:27], v[116:119], v[144:147], v[24:27]\n\t")
        "ds_read_b128 v[144:147], v165 offset:22528\n\t"
        "s_waitcnt lgkmcnt(8)\n\t"
        "s_waitcnt vmcnt(2)\n\t"
        MF("v_mfma_f32_16x16x32_f16 v[28:31], v[116:119], v[148:151], v[28:31]\n\t")
        "ds_read_b128 v[148:151], v165 offset:23552\n\t"
        "v_bitop3_b32 v166, v120, v124, %[tag] bitop3:0x7e\n\t"
        "v_and_b32 v166, 1, v166\n\t"
        "s_waitcnt lgkmcnt(6)\n\t"
        MF("v_mfma_f32_16x16x32_f16 v[32:35], v[116:119], v[152:155], v[32:35]\n\t")
        "ds_read_b128 v[152:155], v165 offset:24576\n\t"
        TG("v_cmp_ne_u32_e64 s[84:85], 0, v166\n\t")
        NTG("s_mov_b64 s[84:85], 0\n\t")
        "v_cmp_gt_u32_e64 s[86:87], s89, v164\n\t"
        "s_waitcnt lgkmcnt(8)\n\t"
        MF("v_mfma_f32_16x16x32_f16 v[36:39], v[116:119], v[156:159], v[36:39]\n\t")
        "ds_read_b128 v[156:159], v165 offset:25600\n\t"
        "s_and_b64 s[86:87], s[86:87], 0x400\n\t"
        "s_or_b64 s[90:91], s[84:85], s[86:87]\n\t"
        "s_cbranch_scc1 FIXT1_%=\n\t"
        "TAIL1_%=:\n\t"
        TRM("s_memrealtime s[90:91]\n\t")
        TRM("s_lshl_b32 s93, %[stage], 2\n\t")
        TRM("s_and_b32 s93, s93, 8\n\t")
        TRM("s_add_u32 s93, s93, 9\n\t")
        TRM("s_mov_b32 m0, s93\n\t")
        TRM("s_waitcnt lgkmcnt(0)\n\t")
        TRM("v_writelane_b32 %[tr], s90, m0\n\t")
        "s_waitcnt lgkmcnt(7)\n\t"
        MF("v_mfma_f32_16x16x32_f16 v[8:11], v[120:123], v[128:131], v[8:11]\n\t")
        "global_load_dwordx4 v[88:91], %[l16], s[96:97] offset:0 " POL_FAR "\n\t"
        "ds_read_b128 v[128:131], v165 offset:27648\n\t"
        "s_waitcnt lgkmcnt(7)\n\t"
        MF("v_mfma_f32_16x16x32_f16 v[12:15], v[120:123], v[132:135], v[12:15]\n\t")
        "global_load_dwordx4 v[92:95], %[l16], s[96:97] offset:1024 " POL_FAR "\n\t"
        "ds_read_b128 v[132:135], v165 offset:28672\n\t"
        "s_waitcnt lgkmcnt(7)\n\t"
        MF("v_mfma_f32_16x16x32_f16 v[16:19], v[120:123], v[136:139], v[16:19]\n\t")
        "s_add_u32 s96, s96, s95\n\t"
        "s_addc_u32 s97, s97, 0\n\t"
        "ds_read_b128 v[136:139], v165 offset:29696\n\t"
        "s_waitcnt lgkmcnt(7)\n\t"
        MF("v_mfma_f32_16x16x32_f16 v[20:23], v[120:123], v[140:143], v[20:23]\n\t")
        "ds_read_b128 v[140:143], v165 offset:30720\n\t"
        "s_waitcnt lgkmcnt(7)\n\t"
        MF("v_mfma_f32_16x16x32_f16 v[24:27], v[120:123], v[144:147], v[24:27]\n\t")
        "ds_read_b128 v[144:147], v165 offset:31744\n\t"
        "s_waitcnt lgkmcnt(7)\n\t"
        MF("v_mfma_f32_16x16x32_f16 v[28:31], v[120:123], v[148:151], v[28:31]\n\t")
        "ds_read_b128 v[148:151], v165 offset:32768\n\t"
        "s_waitcnt lgkmcnt(7)\n\t"
        MF("v_mfma_f32_16x16x32_f16 v[32:35], v[120:123], v[152:155], v[32:35]\n\t")
        "ds_read_b128 v[152:155], v165 offset:33792\n\t"
        "s_waitcnt lgkmcnt(7)\n\t"
        MF("v_mfma_f32_16x16x32_f16 v[36:39], v[120:123], v[156:159], v[36:39]\n\t")
        "ds_read_b128 v[156:159], v165 offset:34816\n\t"
        "s_waitcnt lgkmcnt(7)\n\t"
        MF("v_mfma_f32_16x16x32_f16 v[8:11], v[124:127], v[128:131], v[8:11]\n\t")
        "ds_read_b32 v164, %[vctl]\n\t"
        "s_movk_i32 s89, 119\n\t"
        "ds_read_b128 v[128:131], v165 offset:36864\n\t"
        "s_waitcnt lgkmcnt(8)\n\t"
        MF("v_mfma_f32_16x16x32_f16 v[12:15], v[124:127], v[132:135], v[12:15]\n\t")
        "ds_read_b128 v[132:135], v165 offset:37888\n\t"
        "s_waitcnt lgkmcnt(8)\n\t"
        MF("v_mfma_f32_16x16x32_f16 v[16:19], v[124:127], v[136:139], v[16:19]\n\t")
        "ds_read_b128 v[136:139], v165 offset:38912\n\t"
        "s_waitcnt lgkmcnt(8)\n\t"
        MF("v_mfma_f32_16x16x32_f16 v[20:23], v[124:127], v[140:143], v[20:23]\n\t")
        "ds_read_b128 v[140:143], v165 offset:39936\n\t"
        "s_waitcnt lgkmcnt(8)\n\t"
        MF("v_mfma_f32_16x16x32_f16 v[24:27], v[124:127], v[144:147], v[24:27]\n\t")
        "ds_read_b128 v[144:147], v165 offset:40960\n\t"
        "s_waitcnt lgkmcnt(8)\n\t"
        "s_waitcnt vmcnt(2)\n\t"
        MF("v_mfma_f32_16x16x32_f16 v[28:31], v[124:127], v[148:151], v[28:31]\n\t")
        "ds_read_b128 v[148:151], v165 offset:41984\n\t"
        "v_bitop3_b32 v166, v80, v84, %[tag] bitop3:0x7e\n\t"
        "v_and_b32 v166, 1, v166\n\t"
        "s_waitcnt lgkmcnt(6)\n\t"
        MF("v_mfma_f32_16x16x32_f16 v[32:35], v[124:127], v[152:155], v[32:35]\n\t")
        "ds_read_b128 v[152:155], v165 offset:43008\n\t"
        TG("v_cmp_ne_u32_e64 s[84:85], 0, v166\n\t")
        NTG("s_mov_b64 s[84:85], 0\n\t")
        "v_cmp_gt_u32_e64 s[86:87], s89, v164\n\t"
        "s_waitcnt lgkmcnt(8)\n\t"
        MF("v_mfma_f32_16x16x32_f16 v[36:39], v[124:127], v[156:159], v[36:39]\n\t")
        "ds_read_b128 v[156:159], v165 offset:44032\n\t"
        "s_and_b64 s[86:87], s[86:87], 0x400\n\t"
        "s_or_b64 s[90:91], s[84:85], s[86:87]\n\t"
        "s_cbranch_scc1 FIXT2_%=\n\t"
        "TAIL2_%=:\n\t"
        TRM("s_memrealtime s[90:91]\n\t")
        TRM("s_lshl_b32 s93, %[stage], 2\n\t")
        TRM("s_and_b32 s93, s93, 8\n\t")
        TRM("s_add_u32 s93, s93, 10\n\t")
        TRM("s_mov_b32 m0, s93\n\t")
        TRM("s_waitcnt lgkmcnt(0)\n\t")
        TRM("v_writelane_b32 %[tr], s90, m0\n\t")
        "s_waitcnt lgkmcnt(7)\n\t"
        MF("v_mfma_f32_16x16x32_f16 v[8:11], v[80:83], v[128:131], v[8:11]\n\t")
        "global_load_dwordx4 v[96:99], %[l16], s[96:97] offset:0 " POL_FAR "\n\t"
        "ds_read_b128 v[128:131], v165 offset:46080\n\t"
        "s_waitcnt lgkmcnt(7)\n\t"
        MF("v_mfma_f32_16x16x32_f16 v[12:15], v[80:83], v[132:135], v[12:15]\n\t")
        "global_load_dwordx4 v[100:103], %[l16], s[96:97] offset:1024 " POL_FAR "\n\t"
        "ds_read_b128 v[132:135], v165 offset:47104\n\t"
        "s_waitcnt lgkmcnt(7)\n\t"
        MF("v_mfma_f32_16x16x32_f16 v[16:19], v[80:83], v[136:139], v[16:19]\n\t")
        "s_add_u32 s96, s96, s95\n\t"
        "s_addc_u32 s97, s97, 0\n\t"
        "ds_read_b128 v[136:139], v165 offset:48128\n\t"
        "s_waitcnt lgkmcnt(7)\n\t"
        MF("v_mfma_f32_16x16x32_f16 v[20:23], v[80:83], v[140:143], v[20:23]\n\t")
        "ds_read_b128 v[140:143], v165 offset:49152\n\t"
        "s_waitcnt lgkmcnt(7)\n\t"
        MF("v_mfma_f32_16x16x32_f16 v[24:27], v[80:83], v[144:147], v[24:27]\n\t")
        "ds_read_b128 v[144:147], v165 offset:50176\n\t"
        "s_waitcnt lgkmcnt(7)\n\t"
        MF("v_mfma_f32_16x16x32_f16 v[28:31], v[80:83], v[148:151], v[28:31]\n\t")
        "ds_read_b128 v[148:151], v165 offset:51200\n\t"
        "s_waitcnt lgkmcnt(7)\n\t"
        MF("v_mfma_f32_16x16x32_f16 v[32:35], v[80:83], v[152:155], v[32:35]\n\t")
        "ds_read_b128 v[152:155], v165 offset:52224\n\t"
        "s_waitcnt lgkmcnt(7)\n\t"
        MF("v_mfma_f32_16x16x32_f16 v[36:39], v[80:83], v[156:159], v[36:39]\n\t")
        "ds_read_b128 v[156:159], v165 offset:53248\n\t"
        "s_waitcnt lgkmcnt(7)\n\t"
        MF("v_mfma_f32_16x16x32_f16 v[8:11], v[84:87], v[128:131], v[8:11]\n\t")
        "ds_read_b32 v164, %[vctl]\n\t"
        "s_movk_i32 s89, 121\n\t"
        "v_add_u32 v165, 55296, v165\n\t"
        "ds_read_b128 v[128:131], v165 offset:0\n\t"
        "s_waitcnt lgkmcnt(8)\n\t"
        MF("v_mfma_f32_16x16x32_f16 v[12:15], v[84:87], v[132:135], v[12:15]\n\t")
        "ds_read_b128 v[132:135], v165 offset:1024\n\t"
        "s_waitcnt lgkmcnt(8)\n\t"
        MF("v_mfma_f32_16x16x32_f16 v[16:19], v[84:87], v[136:139], v[16:19]\n\t")
        "ds_read_b128 v[136:139], v165 offset:2048\n\t"
        "s_waitcnt lgkmcnt(8)\n\t"
        MF("v_mfma_f32_16x16x32_f16 v[20:23], v[84:87], v[140:143], v[20:23]\n\t")
        "ds_read_b128 v[140:143], v165 offset:3072\n\t"
        "s_waitcnt lgkmcnt(8)\n\t"
        MF("v_mfma_f32_16x16x32_f16 v[24:27], v[84:87], v[144:147], v[24:27]\n\t")
        "ds_read_b128 v[144:147], v165 offset:4096\n\t"
        "s_waitcnt lgkmcnt(8)\n\t"
        "s_waitcnt vmcnt(2)\n\t"
        MF("v_mfma_f32_16x16x32_f16 v[28:31], v[84:87], v[148:151], v[28:31]\n\t")
        "ds_read_b128 v[148:151], v165 offset:5120\n\t"
        "v_bitop3_b32 v166, v88, v92, %[tag] bitop3:0x7e\n\t"
        "v_and_b32 v166, 1, v166\n\t"
        "s_waitcnt lgkmcnt(6)\n\t"
        MF("v_mfma_f32_16x16x32_f16 v[32:35], v[84:87], v[152:155], v[32:35]\n\t")
        "ds_read_b128 v[152:155], v165 offset:6144\n\t"
        TG("v_cmp_ne_u32_e64 s[84:85], 0, v166\n\t")
        NTG("s_mov_b64 s[84:85], 0\n\t")
        "v_cmp_gt_u32_e64 s[86:87], s89, v164\n\t"
        "s_waitcnt lgkmcnt(8)\n\t"
        MF("v_mfma_f32_16x16x32_f16 v[36:39], v[84:87], v[156:159], v[36:39]\n\t")
        "ds_read_b128 v[156:159], v165 offset:7168\n\t"
        "s_and_b64 s[86:87], s[86:87], 0x400\n\t"
        "s_or_b64 s[90:91], s[84:85], s[86:87]\n\t"
        "s_cbranch_scc1 FIXT3_%=\n\t"
        "TAIL3_%=:\n\t"
        TRM("s_memrealtime s[90:91]\n\t")
        TRM("s_lshl_b32 s93, %[stage], 2\n\t")
        TRM("s_and_b32 s93, s93, 8\n\t")
        TRM("s_add_u32 s93, s93, 11\n\t")
        TRM("s_mov_b32 m0, s93\n\t")
        TRM("s_waitcnt lgkmcnt(0)\n\t")
        TRM("v_writelane_b32 %[tr], s90, m0\n\t")
        "s_waitcnt lgkmcnt(7)\n\t"
        MF("v_mfma_f32_16x16x32_f16 v[8:11], v[88:91], v[128:131], v[8:11]\n\t")
        "global_load_dwordx4 v[104:107], %[l16], s[96:97] offset:0 " POL_FAR "\n\t"
        "ds_read_b128 v[128:131], v165 offset:9216\n\t"
        "s_waitcnt lgkmcnt(7)\n\t"
        MF("v_mfma_f32_16x16x32_f16 v[12:15], v[88:91], v[132:135], v[12:15]\n\t")
        "global_load_dwordx4 v[108:111], %[l16], s[96:97] offset:1024 " POL_FAR "\n\t"
        "v_mov_b32 v166, %[bad]\n\t"
        "ds_read_b64 v[80:81], v166 offset:" KXEXP "\n\t"
        "ds_read_b128 v[132:135], v165 offset:10240\n\t"
        "s_waitcnt lgkmcnt(8)\n\t"
        MF("v_mfma_f32_16x16x32_f16 v[16:19], v[88:91], v[136:139], v[16:19]\n\t")
        "s_add_u32 s96, s96, s95\n\t"
        "s_addc_u32 s97, s97, 0\n\t"
        "ds_read_b128 v[136:139], v165 offset:11264\n\t"
        "s_waitcnt lgkmcnt(8)\n\t"
        MF("v_mfma_f32_16x16x32_f16 v[20:23], v[88:91], v[140:143], v[20:23]\n\t")
        "v_lshrrev_b32 v85, 8, %[l16]\n\t"
        "v_and_b32 v84, 0xff, %[l16]\n\t"
        "ds_read_b128 v[140:143], v165 offset:12288\n\t"
        "s_waitcnt lgkmcnt(8)\n\t"
        MF("v_mfma_f32_16x16x32_f16 v[24:27], v[88:91], v[144:147], v[24:27]\n\t")
        "v_lshl_or_b32 v84, v85, 15, v84\n\t"
        "ds_read_b128 v[144:147], v165 offset:13312\n\t"
        "s_waitcnt lgkmcnt(8)\n\t"
        MF("v_mfma_f32_16x16x32_f16 v[28:31], v[88:91], v[148:151], v[28:31]\n\t")
        "v_or_b32 v86, 0x4000, v84\n\t"
        "v_or_b32 v87, 0x6000, v84\n\t"
        "ds_read_b128 v[148:151], v165 offset:14336\n\t"
        "s_waitcnt lgkmcnt(8)\n\t"
        MF("v_mfma_f32_16x16x32_f16 v[32:35], v[88:91], v[152:155], v[32:35]\n\t")
        "ds_read_b128 v[152:155], v165 offset:15360\n\t"
        "s_waitcnt lgkmcnt(8)\n\t"
        MF("v_mfma_f32_16x16x32_f16 v[36:39], v[88:91], v[156:159], v[36:39]\n\t")
        "ds_read_b128 v[156:159], v165 offset:16384\n\t"
        "s_waitcnt lgkmcnt(8)\n\t"
        MF("v_mfma_f32_16x16x32_f16 v[8:11], v[92:95], v[128:131], v[8:11]\n\t")
        "ds_read_b32 v164, %[vctl]\n\t"
        "s_movk_i32 s89, 123\n\t"
        "ds_read_b128 v[128:131], v165 offset:18432\n\t"
        "s_waitcnt lgkmcnt(8)\n\t"
        MF("v_mfma_f32_16x16x32_f16 v[12:15], v[92:95], v[132:135], v[12:15]\n\t")
        "ds_read_b128 v[132:135], v165 offset:19456\n\t"
        "s_waitcnt lgkmcnt(8)\n\t"
        MF("v_mfma_f32_16x16x32_f16 v[16:19], v[92:95], v[136:139], v[16:19]\n\t")
        "ds_read_b128 v[136:139], v165 offset:20480\n\t"
        "s_waitcnt lgkmcnt(8)\n\t"
        MF("v_mfma_f32_16x16x32_f16 v[20:23], v[92:95], v[140:143], v[20:23]\n\t")
        "ds_read_b128 v[140:143], v165 offset:21504\n\t"
        "s_waitcnt lgkmcnt(8)\n\t"
        MF("v_mfma_f32_16x16x32_f16 v[24:27], v[92:95], v[144:147], v[24:27]\n\t")
        "ds_read_b128 v[144:147], v165 offset:22528\n\t"
        "s_waitcnt lgkmcnt(8)\n\t"
        "s_waitcnt vmcnt(2)\n\t"
        MF("v_mfma_f32_16x16x32_f16 v[28:31], v[92:95], v[148:151], v[28:31]\n\t")
        "ds_read_b128 v[148:151], v165 offset:23552\n\t"
        "v_bitop3_b32 v166, v96, v100, %[tag] bitop3:0x7e\n\t"
        "v_and_b32 v166, 1, v166\n\t"
        "s_waitcnt lgkmcnt(6)\n\t"
        MF("v_mfma_f32_16x16x32_f16 v[32:35], v[92:95], v[152:155], v[32:35]\n\t")
        "ds_read_b128 v[152:155], v165 offset:24576\n\t"
        TG("v_cmp_ne_u32_e64 s[84:85], 0, v166\n\t")
        NTG("s_mov_b64 s[84:85], 0\n\t")
        "v_cmp_gt_u32_e64 s[86:87], s89, v164\n\t"
        "s_waitcnt lgkmcnt(8)\n\t"
        MF("v_mfma_f32_16x16x32_f16 v[36:39], v[92:95], v[156:159], v[36:39]\n\t")
        "ds_read_b128 v[156:159], v165 offset:25600\n\t"
        "s_and_b64 s[86:87], s[86:87], 0x400\n\t"
        "s_or_b64 s[90:91], s[84:85], s[86:87]\n\t"
        "s_cbranch_scc1 FIXT4_%=\n\t"
        "TAIL4_%=:\n\t"
        TRM("s_memrealtime s[90:91]\n\t")
        TRM("s_lshl_b32 s93, %[stage], 2\n\t")
        TRM("s_and_b32 s93, s93, 8\n\t")
        TRM("s_add_u32 s93, s93, 12\n\t")
        TRM("s_mov_b32 m0, s93\n\t")
        TRM("s_waitcnt lgkmcnt(0)\n\t")
        TRM("v_writelane_b32 %[tr], s90, m0\n\t")
        "s_waitcnt lgkmcnt(7)\n\t"
        MF("v_mfma_f32_16x16x32_f16 v[8:11], v[96:99], v[128:131], v[8:11]\n\t")
        "global_load_dwordx4 v[112:115], %[l16], s[96:97] offset:0 " POL_FAR "\n\t"
        "ds_read_b128 v[128:131], v165 offset:27648\n\t"
        "s_waitcnt lgkmcnt(7)\n\t"
        MF("v_mfma_f32_16x16x32_f16 v[12:15], v[96:99], v[132:135], v[12:15]\n\t")
        "global_load_dwordx4 v[116:119], %[l16], s[96:97] offset:1024 " POL_FAR "\n\t"
        "ds_read_b128 v[132:135], v165 offset:28672\n\t"
        "s_waitcnt lgkmcnt(7)\n\t"
        MF("v_mfma_f32_16x16x32_f16 v[16:19], v[96:99], v[136:139], v[16:19]\n\t")
        "s_add_u32 s96, s96, s95\n\t"
        "s_addc_u32 s97, s97, 0\n\t"
        "ds_read_b128 v[136:139], v165 offset:29696\n\t"
        "s_waitcnt lgkmcnt(7)\n\t"
        MF("v_mfma_f32_16x16x32_f16 v[20:23], v[96:99], v[140:143], v[20:23]\n\t")
        "s_waitcnt lgkmcnt(15)\n\t"
        "v_add_u32 v80, v80, v81\n\t"
        "ds_read_b128 v[140:143], v165 offset:30720\n\t"
        "s_waitcnt lgkmcnt(7)\n\t"
        MF("v_mfma_f32_16x16x32_f16 v[24:27], v[96:99], v[144:147], v[24:27]\n\t")
        "v_readfirstlane_b32 s92, v80\n\t"
        "ds_read_b128 v[144:147], v165 offset:31744\n\t"
        "s_waitcnt lgkmcnt(7)\n\t"
        MF("v_mfma_f32_16x16x32_f16 v[28:31], v[96:99], v[148:151], v[28:31]\n\t")
        "s_add_u32 s93, s92, 63\n\t"
        "s_lshl_b32 s93, s93, 23\n\t"
        "s_sub_u32 s88, 0xff38aa3b, s93\n\t"
        "ds_read_b128 v[148:151], v165 offset:32768\n\t"
        "s_waitcnt lgkmcnt(7)\n\t"
        MF("v_mfma_f32_16x16x32_f16 v[32:35], v[96:99], v[152:155], v[32:35]\n\t")
        "v_mov_b32 v83, s93\n\t"
        "s_cmp_gt_i32 s92, 104\n\t"
        "s_mov_b32 s98, 0x2b3cedc7\n\t"
        "s_mov_b32 s92, 0xc2ad1f97\n\t"
        "s_cmov_b32 s98, 0x6000000\n\t"
        "s_cmov_b32 s92, 0xc1400000\n\t"
        "ds_read_b128 v[152:155], v165 offset:33792\n\t"
        "s_waitcnt lgkmcnt(7)\n\t"
        MF("v_mfma_f32_16x16x32_f16 v[36:39], v[96:99], v[156:159], v[36:39]\n\t")
        "s_add_u32 s98, s98, s93\n\t"
        "v_mov_b32 v82, s92\n\t"
        "ds_read_b128 v[156:159], v165 offset:34816\n\t"
        "s_waitcnt lgkmcnt(7)\n\t"
        MF("v_mfma_f32_16x16x32_f16 v[8:11], v[100:103], v[128:131], v[8:11]\n\t")
        "ds_read_b32 v164, %[vctl]\n\t"
        "s_movk_i32 s89, 125\n\t"
        "ds_read_b128 v[128:131], v165 offset:36864\n\t"
        "s_waitcnt lgkmcnt(8)\n\t"
        MF("v_mfma_f32_16x16x32_f16 v[12:15], v[100:103], v[132:135], v[12:15]\n\t")
        "ds_read_b128 v[132:135], v165 offset:37888\n\t"
        "s_waitcnt lgkmcnt(8)\n\t"
        MF("v_mfma_f32_16x16x32_f16 v[16:19], v[100:103], v[136:139], v[16:19]\n\t")
        "ds_read_b128 v[136:139], v165 offset:38912\n\t"
        "s_waitcnt lgkmcnt(8)\n\t"
        MF("v_mfma_f32_16x16x32_f16 v[20:23], v[100:103], v[140:143], v[20:23]\n\t")
        "ds_read_b128 v[140:143], v165 offset:39936\n\t"
        "s_waitcnt lgkmcnt(8)\n\t"
        MF("v_mfma_f32_16x16x32_f16 v[24:27], v[100:103], v[144:147], v[24:27]\n\t")
        "ds_read_b128 v[144:147], v165 offset:40960\n\t"
        "s_waitcnt lgkmcnt(8)\n\t"
        "s_waitcnt vmcnt(2)\n\t"
        MF("v_mfma_f32_16x16x32_f16 v[28:31], v[100:103], v[148:151], v[28:31]\n\t")
        "ds_read_b128 v[148:151], v165 offset:41984\n\t"
        "v_bitop3_b32 v166, v104, v108, %[tag] bitop3:0x7e\n\t"
        "v_and_b32 v166, 1, v166\n\t"
        "s_waitcnt lgkmcnt(6)\n\t"
        MF("v_mfma_f32_16x16x32_f16 v[32:35], v[100:103], v[152:155], v[32:35]\n\t")
        "ds_read_b128 v[152:155], v165 offset:43008\n\t"
        TG("v_cmp_ne_u32_e64 s[84:85], 0, v166\n\t")
        NTG("s_mov_b64 s[84:85], 0\n\t")
        "v_cmp_gt_u32_e64 s[86:87], s89, v164\n\t"
        "s_waitcnt lgkmcnt(8)\n\t"
        MF("v_mfma_f32_16x16x32_f16 v[36:39], v[100:103], v[156:159], v[36:39]\n\t")
        "ds_read_b128 v[156:159], v165 offset:44032\n\t"
        "s_and_b64 s[86:87], s[86:87], 0x400\n\t"
        "s_or_b64 s[90:91], s[84:85], s[86:87]\n\t"
        "s_cbranch_scc1 FIXT5_%=\n\t"
        "TAIL5_%=:\n\t"
        TRM("s_memrealtime s[90:91]\n\t")
        TRM("s_lshl_b32 s93, %[stage], 2\n\t")
        TRM("s_and_b32 s93, s93, 8\n\t")
        TRM("s_add_u32 s93, s93, 13\n\t")
        TRM("s_mov_b32 m0, s93\n\t")
        TRM("s_waitcnt lgkmcnt(0)\n\t")
        TRM("v_writelane_b32 %[tr], s90, m0\n\t")
        "s_waitcnt lgkmcnt(7)\n\t"
        MF("v_mfma_f32_16x16x32_f16 v[8:11], v[104:107], v[128:131], v[8:11]\n\t")
        "global_load_dwordx4 v[120:123], %[l16], s[96:97] offset:0 " POL_FAR "\n\t"
        "ds_read_b128 v[128:131], v165 offset:46080\n\t"
        "s_waitcnt lgkmcnt(7)\n\t"
        MF("v_mfma_f32_16x16x32_f16 v[12:15], v[104:107], v[132:135], v[12:15]\n\t")
        "global_load_dwordx4 v[124:127], %[l16], s[96:97] offset:1024 " POL_FAR "\n\t"
        "ds_read_b128 v[132:135], v165 offset:47104\n\t"
        "s_waitcnt lgkmcnt(7)\n\t"
        MF("v_mfma_f32_16x16x32_f16 v[16:19], v[104:107], v[136:139], v[16:19]\n\t")
        "s_add_u32 s96, s96, s95\n\t"
        "s_addc_u32 s97, s97, 0\n\t"
        "v_mov_b32 v166, %[bad]\n\t"
        "s_mov_b64 exec, 1\n\t"
        "ds_add_u32 v166, v167 offset:" KTOK "\n\t"
        "s_mov_b64 exec, -1\n\t"
        "ds_read_b128 v[136:139], v165 offset:48128\n\t"
        "s_waitcnt lgkmcnt(8)\n\t"
        MF("v_mfma_f32_16x16x32_f16 v[20:23], v[104:107], v[140:143], v[20:23]\n\t")
        "ds_read_b128 v[140:143], v165 offset:49152\n\t"
        "s_waitcnt lgkmcnt(8)\n\t"
        MF("v_mfma_f32_16x16x32_f16 v[24:27], v[104:107], v[144:147], v[24:27]\n\t")
        "ds_read_b128 v[144:147], v165 offset:50176\n\t"
        "s_waitcnt lgkmcnt(8)\n\t"
        MF("v_mfma_f32_16x16x32_f16 v[28:31], v[104:107], v[148:151], v[28:31]\n\t")
        "ds_read_b128 v[148:151], v165 offset:51200\n\t"
        "s_waitcnt lgkmcnt(8)\n\t"
        MF("v_mfma_f32_16x16x32_f16 v[32:35], v[104:107], v[152:155], v[32:35]\n\t")
        "v_or_b32 v85, 0x2000, v84\n\t"
        "ds_read_b128 v[152:155], v165 offset:52224\n\t"
        "s_waitcnt lgkmcnt(8)\n\t"
        MF("v_mfma_f32_16x16x32_f16 v[36:39], v[104:107], v[156:159], v[36:39]\n\t")
        "ds_read_b128 v[156:159], v165 offset:53248\n\t"
        "s_waitcnt lgkmcnt(8)\n\t"
        MF("v_mfma_f32_16x16x32_f16 v[8:11], v[108:111], v[128:131], v[8:11]\n\t")
        "ds_read_b32 v164, %[vctl]\n\t"
        "s_movk_i32 s89, 128\n\t"
        "v_add_u32 v165, 55296, v165\n\t"
        "ds_read_b128 v[128:131], v165 offset:0\n\t"
        "s_waitcnt lgkmcnt(9)\n\t"
        MF("v_mfma_f32_16x16x32_f16 v[12:15], v[108:111], v[132:135], v[12:15]\n\t")
        "ds_read_b128 v[132:135], v165 offset:1024\n\t"
        "s_waitcnt lgkmcnt(8)\n\t"
        MF("v_mfma_f32_16x16x32_f16 v[16:19], v[108:111], v[136:139], v[16:19]\n\t")
        "ds_read_b128 v[136:139], v165 offset:2048\n\t"
        "s_waitcnt lgkmcnt(8)\n\t"
        MF("v_mfma_f32_16x16x32_f16 v[20:23], v[108:111], v[140:143], v[20:23]\n\t")
        "ds_read_b128 v[140:143], v165 offset:3072\n\t"
        "s_waitcnt lgkmcnt(8)\n\t"
        MF("v_mfma_f32_16x16x32_f16 v[24:27], v[108:111], v[144:147], v[24:27]\n\t")
        "ds_read_b128 v[144:147], v165 offset:4096\n\t"
        "s_waitcnt lgkmcnt(8)\n\t"
        "s_waitcnt vmcnt(2)\n\t"
        MF("v_mfma_f32_16x16x32_f16 v[28:31], v[108:111], v[148:151], v[28:31]\n\t")
        "ds_read_b128 v[148:151], v165 offset:5120\n\t"
        "v_bitop3_b32 v166, v112, v116, %[tag] bitop3:0x7e\n\t"
        "v_and_b32 v166, 1, v166\n\t"
        "s_waitcnt lgkmcnt(6)\n\t"
        MF("v_mfma_f32_16x16x32_f16 v[32:35], v[108:111], v[152:155], v[32:35]\n\t")
        "ds_read_b128 v[152:155], v165 offset:6144\n\t"
        TG("v_cmp_ne_u32_e64 s[84:85], 0, v166\n\t")
        NTG("s_mov_b64 s[84:85], 0\n\t")
        "v_cmp_gt_u32_e64 s[86:87], s89, v164\n\t"
        "s_waitcnt lgkmcnt(8)\n\t"
        MF("v_mfma_f32_16x16x32_f16 v[36:39], v[108:111], v[156:159], v[36:39]\n\t")
        "ds_read_b128 v[156:159], v165 offset:7168\n\t"
        "s_and_b64 s[86:87], s[86:87], 0x400\n\t"
        "s_or_b64 s[90:91], s[84:85], s[86:87]\n\t"
        "s_cbranch_scc1 FIXT6_%=\n\t"
        "TAIL6_%=:\n\t"
        TRM("s_memrealtime s[90:91]\n\t")
        TRM("s_lshl_b32 s93, %[stage], 2\n\t")
        TRM("s_and_b32 s93, s93, 8\n\t")
        TRM("s_add_u32 s93, s93, 14\n\t")
        TRM("s_mov_b32 m0, s93\n\t")
        TRM("s_waitcnt lgkmcnt(0)\n\t")
        TRM("v_writelane_b32 %[tr], s90, m0\n\t")
        "s_waitcnt lgkmcnt(7)\n\t"
        MF("v_mfma_f32_16x16x32_f16 v[8:11], v[112:115], v[128:131], v[8:11]\n\t")
        "ds_read_b128 v[128:131], v165 offset:9216\n\t"
        "s_waitcnt lgkmcnt(7)\n\t"
        MF("v_mfma_f32_16x16x32_f16 v[12:15], v[112:115], v[132:135], v[12:15]\n\t")
        "ds_read_b128 v[132:135], v165 offset:10240\n\t"
        "s_waitcnt lgkmcnt(7)\n\t"
        MF("v_mfma_f32_16x16x32_f16 v[16:19], v[112:115], v[136:139], v[16:19]\n\t")
        "ds_read_b128 v[136:139], v165 offset:11264\n\t"
        "s_waitcnt lgkmcnt(7)\n\t"
        MF("v_mfma_f32_16x16x32_f16 v[20:23], v[112:115], v[140:143], v[20:23]\n\t")
        "ds_read_b128 v[140:143], v165 offset:12288\n\t"
        "s_waitcnt lgkmcnt(7)\n\t"
        MF("v_mfma_f32_16x16x32_f16 v[24:27], v[112:115], v[144:147], v[24:27]\n\t")
        "ds_read_b128 v[144:147], v165 offset:13312\n\t"
        "s_waitcnt lgkmcnt(7)\n\t"
        MF("v_mfma_f32_16x16x32_f16 v[28:31], v[112:115], v[148:151], v[28:31]\n\t")
        "ds_read_b128 v[148:151], v165 offset:14336\n\t"
        "s_waitcnt lgkmcnt(7)\n\t"
        MF("v_mfma_f32_16x16x32_f16 v[32:35], v[112:115], v[152:155], v[32:35]\n\t")
        "ds_read_b128 v[152:155], v165 offset:15360\n\t"
        "s_waitcnt lgkmcnt(7)\n\t"
        MF("v_mfma_f32_16x16x32_f16 v[36:39], v[112:115], v[156:159], v[36:39]\n\t")
        "ds_read_b128 v[156:159], v165 offset:16384\n\t"
        "s_waitcnt lgkmcnt(7)\n\t"
        MF("v_mfma_f32_16x16x32_f16 v[8:11], v[116:119], v[128:131], v[8:11]\n\t")
        "ds_read_b128 v[128:131], v165 offset:18432\n\t"
        "s_waitcnt lgkmcnt(7)\n\t"
        MF("v_mfma_f32_16x16x32_f16 v[12:15], v[116:119], v[132:135], v[12:15]\n\t")
        "ds_read_b128 v[132:135], v165 offset:27648\n\t"
        "s_waitcnt lgkmcnt(7)\n\t"
        MF("v_mfma_f32_16x16x32_f16 v[16:19], v[116:119], v[136:139], v[16:19]\n\t")
        "ds_read_b128 v[136:139], v165 offset:19456\n\t"
        "s_waitcnt lgkmcnt(7)\n\t"
        MF("v_mfma_f32_16x16x32_f16 v[20:23], v[116:119], v[140:143], v[20:23]\n\t")
        "ds_read_b128 v[140:143], v165 offset:28672\n\t"
        "s_waitcnt lgkmcnt(7)\n\t"
        MF("v_mfma_f32_16x16x32_f16 v[24:27], v[116:119], v[144:147], v[24:27]\n\t")
        "ds_read_b128 v[144:147], v165 offset:20480\n\t"
        "s_waitcnt lgkmcnt(7)\n\t"
        MF("v_mfma_f32_16x16x32_f16 v[28:31], v[116:119], v[148:151], v[28:31]\n\t")
        "ds_read_b128 v[148:151], v165 offset:29696\n\t"
        "s_waitcnt lgkmcnt(7)\n\t"
        MF("v_mfma_f32_16x16x32_f16 v[32:35], v[116:119], v[152:155], v[32:35]\n\t")
        "ds_read_b128 v[152:155], v165 offset:21504\n\t"
        "s_waitcnt lgkmcnt(7)\n\t"
        MF("v_mfma_f32_16x16x32_f16 v[36:39], v[116:119], v[156:159], v[36:39]\n\t")
        "ds_read_b128 v[156:159], v165 offset:30720\n\t"
        "TAIL7_%=:\n\t"
        TRM("s_memrealtime s[90:91]\n\t")
        TRM("s_lshl_b32 s93, %[stage], 2\n\t")
        TRM("s_and_b32 s93, s93, 8\n\t")
        TRM("s_add_u32 s93, s93, 15\n\t")
        TRM("s_mov_b32 m0, s93\n\t")
        TRM("s_waitcnt lgkmcnt(0)\n\t")
        TRM("v_writelane_b32 %[tr], s90, m0\n\t")
        "v_mov_b32 v164, %[bad]\n\t"
        "s_waitcnt lgkmcnt(6)\n\t"
        "s_waitcnt vmcnt(0)\n\t"
        TRM("s_memrealtime s[90:91]\n\t")
        TRM("s_and_b32 s93, %[stage], 2\n\t")
        TRM("s_add_u32 s93, s93, 1\n\t")
        TRM("s_mov_b32 m0, s93\n\t")
        TRM("s_waitcnt lgkmcnt(0)\n\t")
        TRM("v_writelane_b32 %[tr], s90, m0\n\t")
        "v_bitop3_b32 v166, v120, v124, %[tag] bitop3:0x7e\n\t"
        MF("v_mfma_f32_16x16x32_f16 v[116:119], v[120:123], v[128:131], v[8:11]\n\t")
        "v_and_b32 v166, 1, v166\n\t"
        TG("v_cmp_ne_u32_e64 s[84:85], 0, v166\n\t")
        NTG("s_mov_b64 s[84:85], 0\n\t")
        MF("v_mfma_f32_16x16x32_f16 v[116:119], v[124:127], v[132:135], v[116:119]\n\t")
        "s_cmp_eq_u64 s[84:85], 0\n\t"
        "s_cbranch_scc0 FIXT7_%=\n\t"
        "TAILS_%=:\n\t"
        "ds_read_b32 v81, v164\n\t"
        "ds_read_b128 v[128:131], v165 offset:22528\n\t"
        "ds_read_b128 v[132:135], v165 offset:31744\n\t"
        "s_waitcnt lgkmcnt(8)\n\t"
        MF("v_mfma_f32_16x16x32_f16 v[12:15], v[120:123], v[136:139], v[12:15]\n\t")
        "ds_read_b128 v[136:139], v165 offset:23552\n\t"
        "s_waitcnt lgkmcnt(8)\n\t"
        MF("v_mfma_f32_16x16x32_f16 v[12:15], v[124:127], v[140:143], v[12:15]\n\t")
        "ds_read_b128 v[140:143], v165 offset:32768\n\t"
        "v_fma_f32 v88, v116, s88, v82\n\t"
        "v_fma_f32 v92, v117, s88, v82\n\t"
        "s_waitcnt lgkmcnt(8)\n\t"
        MF("v_mfma_f32_16x16x32_f16 v[16:19], v[120:123], v[144:147], v[16:19]\n\t")
        "ds_read_b128 v[144:147], v165 offset:24576\n\t"
        "v_fma_f32 v96, v118, s88, v82\n\t"
        "v_fma_f32 v100, v119, s88, v82\n\t"
        "s_waitcnt lgkmcnt(8)\n\t"
        MF("v_mfma_f32_16x16x32_f16 v[16:19], v[124:127], v[148:151], v[16:19]\n\t")
        "ds_read_b128 v[148:151], v165 offset:33792\n\t"
        "v_exp_f32_e64 v88, v88 clamp\n\t"
        "s_waitcnt lgkmcnt(8)\n\t"
        MF("v_mfma_f32_16x16x32_f16 v[20:23], v[120:123], v[152:155], v[20:23]\n\t")
        "ds_read_b128 v[152:155], v165 offset:25600\n\t"
        "v_exp_f32_e64 v92, v92 clamp\n\t"
        "s_waitcnt lgkmcnt(8)\n\t"
        MF("v_mfma_f32_16x16x32_f16 v[20:23], v[124:127], v[156:159], v[20:23]\n\t")
        "ds_read_b128 v[156:159], v165 offset:34816\n\t"
        "s_waitcnt lgkmcnt(8)\n\t"
        "v_cmp_ne_u32_e64 s[90:91], 0, v81\n\t"
        "v_exp_f32_e64 v96, v96 clamp\n\t"
        "v_exp_f32_e64 v100, v100 clamp\n\t"
        "s_waitcnt lgkmcnt(7)\n\t"
        MF("v_mfma_f32_16x16x32_f16 v[24:27], v[120:123], v[128:131], v[24:27]\n\t")
        "s_or_b32 s98, s98, s90\n\t"
        "v_fma_f32 v88, v88, s98, v83\n\t"
        "v_fma_f32 v92, v92, s98, v83\n\t"
        "v_fma_f32 v96, v96, s98, v83\n\t"
        "v_fma_f32 v100, v100, s98, v83\n\t"
        "s_waitcnt lgkmcnt(6)\n\t"
        MF("v_mfma_f32_16x16x32_f16 v[24:27], v[124:127], v[132:135], v[24:27]\n\t")
        "v_rcp_f32 v88, v88\n\t"
        "v_rcp_f32 v92, v92\n\t"
        "s_waitcnt lgkmcnt(5)\n\t"
        MF("v_mfma_f32_16x16x32_f16 v[28:31], v[120:123], v[136:139], v[28:31]\n\t")
        "v_rcp_f32 v96, v96\n\t"
        "v_rcp_f32 v100, v100\n\t"
        "s_waitcnt lgkmcnt(4)\n\t"
        MF("v_mfma_f32_16x16x32_f16 v[28:31], v[124:127], v[140:143], v[28:31]\n\t")
        "v_mul_f32 v88, v116, v88\n\t"
        "v_mul_f32 v92, v117, v92\n\t"
        "v_mul_f32 v96, v118, v96\n\t"
        "v_mul_f32 v100, v119, v100\n\t"
        "s_waitcnt lgkmcnt(3)\n\t"
        MF("v_mfma_f32_16x16x32_f16 v[32:35], v[120:123], v[144:147], v[32:35]\n\t")
        "v_fma_f32 v89, v12, s88, v82\n\t"
        "v_fma_f32 v90, v16, s88, v82\n\t"
        "v_fma_f32 v91, v20, s88, v82\n\t"
        "v_fma_f32 v93, v13, s88, v82\n\t"
        "s_waitcnt lgkmcnt(2)\n\t"
        MF("v_mfma_f32_16x16x32_f16 v[32:35], v[124:127], v[148:151], v[32:35]\n\t")
        "v_fma_f32 v94, v17, s88, v82\n\t"
        "v_fma_f32 v95, v21, s88, v82\n\t"
        "v_exp_f32_e64 v89, v89 clamp\n\t"
        "s_waitcnt lgkmcnt(1)\n\t"
        MF("v_mfma_f32_16x16x32_f16 v[36:39], v[120:123], v[152:155], v[36:39]\n\t")
        "v_exp_f32_e64 v90, v90 clamp\n\t"
        "v_exp_f32_e64 v91, v91 clamp\n\t"
        "s_waitcnt lgkmcnt(0)\n\t"
        MF("v_mfma_f32_16x16x32_f16 v[36:39], v[124:127], v[156:159], v[36:39]\n\t")
        "s_setprio " KPRE "\n\t"
        "v_exp_f32_e64 v93, v93 clamp\n\t"
        "v_exp_f32_e64 v94, v94 clamp\n\t"
        "v_exp_f32_e64 v95, v95 clamp\n\t"
        "v_fma_f32 v89, v89, s98, v83\n\t"
        "v_fma_f32 v90, v90, s98, v83\n\t"
        "v_fma_f32 v91, v91, s98, v83\n\t"
        "v_fma_f32 v93, v93, s98, v83\n\t"
        "v_fma_f32 v94, v94, s98, v83\n\t"
        "v_fma_f32 v95, v95, s98, v83\n\t"
        "v_rcp_f32 v89, v89\n\t"
        "v_rcp_f32 v90, v90\n\t"
        "v_rcp_f32 v91, v91\n\t"
        "v_rcp_f32 v93, v93\n\t"
        "v_rcp_f32 v94, v94\n\t"
        "v_rcp_f32 v95, v95\n\t"
        "v_mul_f32 v89, v12, v89\n\t"
        "v_mul_f32 v90, v16, v90\n\t"
        "v_mul_f32 v91, v20, v91\n\t"
        "global_store_dwordx4 v84, v[88:91], %[outp] offset:0 " POL_OUT "\n\t"
        "v_mul_f32 v93, v13, v93\n\t"
        "v_mul_f32 v94, v17, v94\n\t"
        "v_mul_f32 v95, v21, v95\n\t"
        "global_store_dwordx4 v85, v[92:95], %[outp] offset:0 " POL_OUT "\n\t"
        "v_fma_f32 v97, v14, s88, v82\n\t"
        "v_fma_f32 v98, v18, s88, v82\n\t"
        "v_fma_f32 v99, v22, s88, v82\n\t"
        "v_fma_f32 v101, v15, s88, v82\n\t"
        "v_fma_f32 v102, v19, s88, v82\n\t"
        "v_fma_f32 v103, v23, s88, v82\n\t"
        "v_exp_f32_e64 v97, v97 clamp\n\t"
        "v_exp_f32_e64 v98, v98 clamp\n\t"
        "v_exp_f32_e64 v99, v99 clamp\n\t"
        "v_exp_f32_e64 v101, v101 clamp\n\t"
        "v_exp_f32_e64 v102, v102 clamp\n\t"
        "v_exp_f32_e64 v103, v103 clamp\n\t"
        "v_fma_f32 v97, v97, s98, v83\n\t"
        "v_fma_f32 v98, v98, s98, v83\n\t"
        "v_fma_f32 v99, v99, s98, v83\n\t"
        "v_fma_f32 v101, v101, s98, v83\n\t"
        "v_fma_f32 v102, v102, s98, v83\n\t"
        "v_fma_f32 v103, v103, s98, v83\n\t"
        "v_rcp_f32 v97, v97\n\t"
        "v_rcp_f32 v98, v98\n\t"
        "v_rcp_f32 v99, v99\n\t"
        "v_rcp_f32 v101, v101\n\t"
        "v_rcp_f32 v102, v102\n\t"
        "v_rcp_f32 v103, v103\n\t"
        "v_mul_f32 v97, v14, v97\n\t"
        "v_mul_f32 v98, v18, v98\n\t"
        "v_mul_f32 v99, v22, v99\n\t"
        "global_store_dwordx4 v86, v[96:99], %[outp] offset:0 " POL_OUT "\n\t"
        "v_mul_f32 v101, v15, v101\n\t"
        "v_mul_f32 v102, v19, v102\n\t"
        "v_mul_f32 v103, v23, v103\n\t"
        "global_store_dwordx4 v87, v[100:103], %[outp] offset:0 " POL_OUT "\n\t"
        "v_fma_f32 v104, v24, s88, v82\n\t"
        "v_fma_f32 v105, v28, s88, v82\n\t"
        "v_fma_f32 v106, v32, s88, v82\n\t"
        "v_fma_f32 v107, v36, s88, v82\n\t"
        "v_fma_f32 v108, v25, s88, v82\n\t"
        "v_fma_f32 v109, v29, s88, v82\n\t"
        "v_fma_f32 v110, v33, s88, v82\n\t"
        "v_fma_f32 v111, v37, s88, v82\n\t"
        "v_exp_f32_e64 v104, v104 clamp\n\t"
        "v_exp_f32_e64 v105, v105 clamp\n\t"
        "v_exp_f32_e64 v106, v106 clamp\n\t"
        "v_exp_f32_e64 v107, v107 clamp\n\t"
        "v_exp_f32_e64 v108, v108 clamp\n\t"
        "v_exp_f32_e64 v109, v109 clamp\n\t"
        "v_exp_f32_e64 v110, v110 clamp\n\t"
        "v_exp_f32_e64 v111, v111 clamp\n\t"
        "v_fma_f32 v104, v104, s98, v83\n\t"
        "v_fma_f32 v105, v105, s98, v83\n\t"
        "v_fma_f32 v106, v106, s98, v83\n\t"
        "v_fma_f32 v107, v107, s98, v83\n\t"
        "v_fma_f32 v108, v108, s98, v83\n\t"
        "v_fma_f32 v109, v109, s98, v83\n\t"
        "v_fma_f32 v110, v110, s98, v83\n\t"
        "v_fma_f32 v111, v111, s98, v83\n\t"
        "v_rcp_f32 v104, v104\n\t"
        "v_rcp_f32 v105, v105\n\t"
        "v_rcp_f32 v106, v106\n\t"
        "v_rcp_f32 v107, v107\n\t"
        "v_rcp_f32 v108, v108\n\t"
        "v_rcp_f32 v109, v109\n\t"
        "v_rcp_f32 v110, v110\n\t"
        "v_rcp_f32 v111, v111\n\t"
        "v_mul_f32 v104, v24, v104\n\t"
        "v_mul_f32 v105, v28, v105\n\t"
        "v_mul_f32 v106, v32, v106\n\t"
        "v_mul_f32 v107, v36, v107\n\t"
        "global_store_dwordx4 v84, v[104:107], %[outp] offset:256 " POL_OUT "\n\t"
        "v_mul_f32 v108, v25, v108\n\t"
        "v_mul_f32 v109, v29, v109\n\t"
        "v_mul_f32 v110, v33, v110\n\t"
        "v_mul_f32 v111, v37, v111\n\t"
        "global_store_dwordx4 v85, v[108:111], %[outp] offset:256 " POL_OUT "\n\t"
        "v_fma_f32 v112, v26, s88, v82\n\t"
        "v_fma_f32 v113, v30, s88, v82\n\t"
        "v_fma_f32 v114, v34, s88, v82\n\t"
        "v_fma_f32 v115, v38, s88, v82\n\t"
        "v_fma_f32 v116, v27, s88, v82\n\t"
        "v_fma_f32 v117, v31, s88, v82\n\t"
        "v_fma_f32 v118, v35, s88, v82\n\t"
        "v_fma_f32 v119, v39, s88, v82\n\t"
        "v_exp_f32_e64 v112, v112 clamp\n\t"
        "v_exp_f32_e64 v113, v113 clamp\n\t"
        "v_exp_f32_e64 v114, v114 clamp\n\t"
        "v_exp_f32_e64 v115, v115 clamp\n\t"
        "v_exp_f32_e64 v116, v116 clamp\n\t"
        "v_exp_f32_e64 v117, v117 clamp\n\t"
        "v_exp_f32_e64 v118, v118 clamp\n\t"
        "v_exp_f32_e64 v119, v119 clamp\n\t"
        "v_fma_f32 v112, v112, s98, v83\n\t"
        "v_fma_f32 v113, v113, s98, v83\n\t"
        "v_fma_f32 v114, v114, s98, v83\n\t"
        "v_fma_f32 v115, v115, s98, v83\n\t"
        "v_fma_f32 v116, v116, s98, v83\n\t"
        "v_fma_f32 v117, v117, s98, v83\n\t"
        "v_fma_f32 v118, v118, s98, v83\n\t"
        "v_fma_f32 v119, v119, s98, v83\n\t"
        "v_rcp_f32 v112, v112\n\t"
        "v_rcp_f32 v113, v113\n\t"
        "v_rcp_f32 v114, v114\n\t"
        "v_rcp_f32 v115, v115\n\t"
        "v_rcp_f32 v116, v116\n\t"
        "v_rcp_f32 v117, v117\n\t"
        "v_rcp_f32 v118, v118\n\t"
        "v_rcp_f32 v119, v119\n\t"
        "v_mul_f32 v112, v26, v112\n\t"
        "v_mul_f32 v113, v30, v113\n\t"
        "v_mul_f32 v114, v34, v114\n\t"
        "v_mul_f32 v115, v38, v115\n\t"
        "global_store_dwordx4 v86, v[112:115], %[outp] offset:256 " POL_OUT "\n\t"
        "v_mul_f32 v116, v27, v116\n\t"
        "v_mul_f32 v117, v31, v117\n\t"
        "v_mul_f32 v118, v35, v118\n\t"
        "v_mul_f32 v119, v39, v119\n\t"
        "global_store_dwordx4 v87, v[116:119], %[outp] offset:256 " POL_OUT "\n\t"
        TRM("s_memrealtime s[90:91]\n\t")
        TRM("s_lshl_b32 s93, %[stage], 1\n\t")
        TRM("s_and_b32 s93, s93, 4\n\t")
        TRM("s_lshr_b32 s93, s93, 1\n\t")
        TRM("s_add_u32 s93, s93, 2\n\t")
        TRM("s_mov_b32 m0, s93\n\t")
        TRM("s_waitcnt lgkmcnt(0)\n\t")
        TRM("v_writelane_b32 %[tr], s90, m0\n\t")
        "s_cmp_gt_u32 %[stage], 1\n\t"
        "s_cbranch_scc1 DONE_%=\n\t"
        "s_add_u32 %[stage], %[stage], 2\n\t"
        "s_mov_b64 %[outp], %[out2]\n\t"
        "s_mov_b64 %[xtail], %[xtl2]\n\t"
        "s_branch ENTRY2_%=\n\t"
        "WAITB_%=:\n\t"
        "s_mov_b32 s94, 0\n\t"
        "SPINWB_%=:\n\t"
        "ds_read_b32 v164, %[vctl]\n\t"
        "s_waitcnt lgkmcnt(0)\n\t"
        "v_cmp_gt_u32_e64 s[86:87], s89, v164\n\t"
        "s_and_b64 s[86:87], s[86:87], 0x400\n\t"
        "s_cbranch_scc0 TAILX_%=\n\t"
        SC3("s_add_u32 %[nfix], %[nfix], 1\n\t")
        "s_sleep 1\n\t"
        "s_add_u32 s94, s94, 1\n\t"
        "s_and_b32 s92, s94, 0x3ff\n\t"
        "s_cbranch_scc1 SPINWB_%=\n\t"
        "s_memrealtime s[92:93]\n\t"
        "s_waitcnt lgkmcnt(0)\n\t"
        "s_cmp_eq_u32 s94, 0x400\n\t"
        "s_cselect_b32 s90, s92, s90\n\t"
        "s_sub_u32 s92, s92, s90\n\t"
        "s_cmp_lt_u32 s92, " KDL3 "\n\t"
        "s_cbranch_scc1 SPINWB_%=\n\t"
        "v_mov_b32 v166, %[bad]\n\t"
        "ds_write_b32 v166, v166\n\t"
        "s_branch TAILX_%=\n\t"
        "FIXT0_%=:\n\t"
        "s_cmp_eq_u64 s[84:85], 0\n\t"
        "s_cbranch_scc1 NOFIXT0_%=\n\t"
        "s_add_u32 %[nfix], %[nfix], 1\n\t"
        "v_mov_b32 v166, %[bad]\n\t"
        "ds_read_b32 v164, v166\n\t"
        "s_mov_b64 s[86:87], %[xtail]\n\t"
        "s_mov_b32 s92, 0\n\t"
        "s_add_u32 s86, s86, s92\n\t"
        "s_addc_u32 s87, s87, 0\n\t"
        "s_memrealtime s[90:91]\n\t"
        "s_waitcnt lgkmcnt(0)\n\t"
        "v_cmp_ne_u32_e64 s[92:93], 0, v164\n\t"
        "s_cmp_lg_u64 s[92:93], 0\n\t"
        "s_cbranch_scc1 NOFIXT0_%=\n\t"
        "FIXLT0_%=:\n\t"
        "global_load_dwordx4 v[112:115], %[l16], s[86:87] offset:0 sc0 sc1\n\t"
        "global_load_dwordx4 v[116:119], %[l16], s[86:87] offset:1024 sc0 sc1\n\t"
        "s_waitcnt vmcnt(0)\n\t"
        "v_bitop3_b32 v166, v112, v116, %[tag] bitop3:0x7e\n\t"
        "v_and_b32 v166, 1, v166\n\t"
        "v_cmp_ne_u32_e64 s[92:93], 0, v166\n\t"
        "s_cmp_eq_u64 s[92:93], 0\n\t"
        "s_cbranch_scc1 NOFIXT0_%=\n\t"
        "s_sleep 1\n\t"
        "s_memrealtime s[92:93]\n\t"
        "s_waitcnt lgkmcnt(0)\n\t"
        "s_sub_u32 s92, s92, s90\n\t"
        "s_cmp_lt_u32 s92, " KDL "\n\t"
        "s_cbranch_scc1 FIXLT0_%=\n\t"
        "v_mov_b32 v166, %[bad]\n\t"
        "ds_write_b32 v166, v166\n\t"
        "NOFIXT0_%=:\n\t"
        "s_mov_b32 s94, 0\n\t"
        "SPINT0_%=:\n\t"
        "ds_read_b32 v164, %[vctl]\n\t"
        "s_waitcnt lgkmcnt(0)\n\t"
        "v_cmp_gt_u32_e64 s[86:87], s89, v164\n\t"
        "s_and_b64 s[86:87], s[86:87], 0x400\n\t"
        "s_cbranch_scc0 GO1_%=\n\t"
        SC3("s_add_u32 %[nfix], %[nfix], 1\n\t")
        "s_sleep 1\n\t"
        "s_add_u32 s94, s94, 1\n\t"
        "s_and_b32 s92, s94, 0x3ff\n\t"
        "s_cbranch_scc1 SPINT0_%=\n\t"
        "s_memrealtime s[92:93]\n\t"
        "s_waitcnt lgkmcnt(0)\n\t"
        "s_cmp_eq_u32 s94, 0x400\n\t"
        "s_cselect_b32 s90, s92, s90\n\t"
        "s_sub_u32 s92, s92, s90\n\t"
        "s_cmp_lt_u32 s92, " KDL3 "\n\t"
        "s_cbranch_scc1 SPINT0_%=\n\t"
        "v_mov_b32 v166, %[bad]\n\t"
        "ds_write_b32 v166, v166\n\t"
        "s_branch GO1_%=\n\t"
        "FIXT1_%=:\n\t"
        "s_cmp_eq_u64 s[84:85], 0\n\t"
        "s_cbranch_scc1 NOFIXT1_%=\n\t"
        "s_add_u32 %[nfix], %[nfix], 1\n\t"
        "v_mov_b32 v166, %[bad]\n\t"
        "ds_read_b32 v164, v166\n\t"
        "s_mov_b64 s[86:87], %[xtail]\n\t"
        "s_mov_b32 s92, 8192\n\t"
        "s_add_u32 s86, s86, s92\n\t"
        "s_addc_u32 s87, s87, 0\n\t"
        "s_memrealtime s[90:91]\n\t"
        "s_waitcnt lgkmcnt(0)\n\t"
        "v_cmp_ne_u32_e64 s[92:93], 0, v164\n\t"
        "s_cmp_lg_u64 s[92:93], 0\n\t"
        "s_cbranch_scc1 NOFIXT1_%=\n\t"
        "FIXLT1_%=:\n\t"
        "global_load_dwordx4 v[120:123], %[l16], s[86:87] offset:0 sc0 sc1\n\t"
        "global_load_dwordx4 v[124:127], %[l16], s[86:87] offset:1024 sc0 sc1\n\t"
        "s_waitcnt vmcnt(0)\n\t"
        "v_bitop3_b32 v166, v120, v124, %[tag] bitop3:0x7e\n\t"
        "v_and_b32 v166, 1, v166\n\t"
        "v_cmp_ne_u32_e64 s[92:93], 0, v166\n\t"
        "s_cmp_eq_u64 s[92:93], 0\n\t"
        "s_cbranch_scc1 NOFIXT1_%=\n\t"
        "s_sleep 1\n\t"
        "s_memrealtime s[92:93]\n\t"
        "s_waitcnt lgkmcnt(0)\n\t"
        "s_sub_u32 s92, s92, s90\n\t"
        "s_cmp_lt_u32 s92, " KDL "\n\t"
        "s_cbranch_scc1 FIXLT1_%=\n\t"
        "v_mov_b32 v166, %[bad]\n\t"
        "ds_write_b32 v166, v166\n\t"
        "NOFIXT1_%=:\n\t"
        "s_mov_b32 s94, 0\n\t"
        "SPINT1_%=:\n\t"
        "ds_read_b32 v164, %[vctl]\n\t"
        "s_waitcnt lgkmcnt(0)\n\t"
        "v_cmp_gt_u32_e64 s[86:87], s89, v164\n\t"
        "s_and_b64 s[86:87], s[86:87], 0x400\n\t"
        "s_cbranch_scc0 TAIL1_%=\n\t"
        SC3("s_add_u32 %[nfix], %[nfix], 1\n\t")
        "s_sleep 1\n\t"
        "s_add_u32 s94, s94, 1\n\t"
        "s_and_b32 s92, s94, 0x3ff\n\t"
        "s_cbranch_scc1 SPINT1_%=\n\t"
        "s_memrealtime s[92:93]\n\t"
        "s_waitcnt lgkmcnt(0)\n\t"
        "s_cmp_eq_u32 s94, 0x400\n\t"
        "s_cselect_b32 s90, s92, s90\n\t"
        "s_sub_u32 s92, s92, s90\n\t"
        "s_cmp_lt_u32 s92, " KDL3 "\n\t"
        "s_cbranch_scc1 SPINT1_%=\n\t"
        "v_mov_b32 v166, %[bad]\n\t"
        "ds_write_b32 v166, v166\n\t"
        "s_branch TAIL1_%=\n\t"
        "FIXT2_%=:\n\t"
        "s_cmp_eq_u64 s[84:85], 0\n\t"
        "s_cbranch_scc1 NOFIXT2_%=\n\t"
        "s_add_u32 %[nfix], %[nfix], 1\n\t"
        "v_mov_b32 v166, %[bad]\n\t"
        "ds_read_b32 v164, v166\n\t"
        "s_mov_b64 s[86:87], %[xtail]\n\t"
        "s_mov_b32 s92, 16384\n\t"
        "s_add_u32 s86, s86, s92\n\t"
        "s_addc_u32 s87, s87, 0\n\t"
        "s_memrealtime s[90:91]\n\t"
        "s_waitcnt lgkmcnt(0)\n\t"
        "v_cmp_ne_u32_e64 s[92:93], 0, v164\n\t"
        "s_cmp_lg_u64 s[92:93], 0\n\t"
        "s_cbranch_scc1 NOFIXT2_%=\n\t"
        "FIXLT2_%=:\n\t"
        "global_load_dwordx4 v[80:83], %[l16], s[86:87] offset:0 sc0 sc1\n\t"
        "global_load_dwordx4 v[84:87], %[l16], s[86:87] offset:1024 sc0 sc1\n\t"
        "s_waitcnt vmcnt(0)\n\t"
        "v_bitop3_b32 v166, v80, v84, %[tag] bitop3:0x7e\n\t"
        "v_and_b32 v166, 1, v166\n\t"
        "v_cmp_ne_u32_e64 s[92:93], 0, v166\n\t"
        "s_cmp_eq_u64 s[92:93], 0\n\t"
        "s_cbranch_scc1 NOFIXT2_%=\n\t"
        "s_sleep 1\n\t"
        "s_memrealtime s[92:93]\n\t"
        "s_waitcnt lgkmcnt(0)\n\t"
        "s_sub_u32 s92, s92, s90\n\t"
        "s_cmp_lt_u32 s92, " KDL "\n\t"
        "s_cbranch_scc1 FIXLT2_%=\n\t"
        "v_mov_b32 v166, %[bad]\n\t"
        "ds_write_b32 v166, v166\n\t"
        "NOFIXT2_%=:\n\t"
        "s_mov_b32 s94, 0\n\t"
        "SPINT2_%=:\n\t"
        "ds_read_b32 v164, %[vctl]\n\t"
        "s_waitcnt lgkmcnt(0)\n\t"
        "v_cmp_gt_u32_e64 s[86:87], s89, v164\n\t"
        "s_and_b64 s[86:87], s[86:87], 0x400\n\t"
        "s_cbranch_scc0 TAIL2_%=\n\t"
        SC3("s_add_u32 %[nfix], %[nfix], 1\n\t")
        "s_sleep 1\n\t"
        "s_add_u32 s94, s94, 1\n\t"
        "s_and_b32 s92, s94, 0x3ff\n\t"
        "s_cbranch_scc1 SPINT2_%=\n\t"
        "s_memrealtime s[92:93]\n\t"
        "s_waitcnt lgkmcnt(0)\n\t"
        "s_cmp_eq_u32 s94, 0x400\n\t"
        "s_cselect_b32 s90, s92, s90\n\t"
        "s_sub_u32 s92, s92, s90\n\t"
        "s_cmp_lt_u32 s92, " KDL3 "\n\t"
        "s_cbranch_scc1 SPINT2_%=\n\t"
        "v_mov_b32 v166, %[bad]\n\t"
        "ds_write_b32 v166, v166\n\t"
        "s_branch TAIL2_%=\n\t"
        "FIXT3_%=:\n\t"
        "s_cmp_eq_u64 s[84:85], 0\n\t"
        "s_cbranch_scc1 NOFIXT3_%=\n\t"
        "s_add_u32 %[nfix], %[nfix], 1\n\t"
        "v_mov_b32 v166, %[bad]\n\t"
        "ds_read_b32 v164, v166\n\t"
        "s_mov_b64 s[86:87], %[xtail]\n\t"
        "s_mov_b32 s92, 24576\n\t"
        "s_add_u32 s86, s86, s92\n\t"
        "s_addc_u32 s87, s87, 0\n\t"
        "s_memrealtime s[90:91]\n\t"
        "s_waitcnt lgkmcnt(0)\n\t"
        "v_cmp_ne_u32_e64 s[92:93], 0, v164\n\t"
        "s_cmp_lg_u64 s[92:93], 0\n\t"
        "s_cbranch_scc1 NOFIXT3_%=\n\t"
        "FIXLT3_%=:\n\t"
        "global_load_dwordx4 v[88:91], %[l16], s[86:87] offset:0 sc0 sc1\n\t"
        "global_load_dwordx4 v[92:95], %[l16], s[86:87] offset:1024 sc0 sc1\n\t"
        "s_waitcnt vmcnt(0)\n\t"
        "v_bitop3_b32 v166, v88, v92, %[tag] bitop3:0x7e\n\t"
        "v_and_b32 v166, 1, v166\n\t"
        "v_cmp_ne_u32_e64 s[92:93], 0, v166\n\t"
        "s_cmp_eq_u64 s[92:93], 0\n\t"
        "s_cbranch_scc1 NOFIXT3_%=\n\t"
        "s_sleep 1\n\t"
        "s_memrealtime s[92:93]\n\t"
        "s_waitcnt lgkmcnt(0)\n\t"
        "s_sub_u32 s92, s92, s90\n\t"
        "s_cmp_lt_u32 s92, " KDL "\n\t"
        "s_cbranch_scc1 FIXLT3_%=\n\t"
        "v_mov_b32 v166, %[bad]\n\t"
        "ds_write_b32 v166, v166\n\t"
        "NOFIXT3_%=:\n\t"
        "s_mov_b32 s94, 0\n\t"
        "SPINT3_%=:\n\t"
        "ds_read_b32 v164, %[vctl]\n\t"
        "s_waitcnt lgkmcnt(0)\n\t"
        "v_cmp_gt_u32_e64 s[86:87], s89, v164\n\t"
        "s_and_b64 s[86:87], s[86:87], 0x400\n\t"
        "s_cbranch_scc0 TAIL3_%=\n\t"
        SC3("s_add_u32 %[nfix], %[nfix], 1\n\t")
        "s_sleep 1\n\t"
        "s_add_u32 s94, s94, 1\n\t"
        "s_and_b32 s92, s94, 0x3ff\n\t"
        "s_cbranch_scc1 SPINT3_%=\n\t"
        "s_memrealtime s[92:93]\n\t"
        "s_waitcnt lgkmcnt(0)\n\t"
        "s_cmp_eq_u32 s94, 0x400\n\t"
        "s_cselect_b32 s90, s92, s90\n\t"
        "s_sub_u32 s92, s92, s90\n\t"
        "s_cmp_lt_u32 s92, " KDL3 "\n\t"
        "s_cbranch_scc1 SPINT3_%=\n\t"
        "v_mov_b32 v166, %[bad]\n\t"
        "ds_write_b32 v166, v166\n\t"
        "s_branch TAIL3_%=\n\t"
        "FIXT4_%=:\n\t"
        "s_cmp_eq_u64 s[84:85], 0\n\t"
        "s_cbranch_scc1 NOFIXT4_%=\n\t"
        "s_add_u32 %[nfix], %[nfix], 1\n\t"
        "v_mov_b32 v166, %[bad]\n\t"
        "ds_read_b32 v164, v166\n\t"
        "s_mov_b64 s[86:87], %[xtail]\n\t"
        "s_mov_b32 s92, 32768\n\t"
        "s_add_u32 s86, s86, s92\n\t"
        "s_addc_u32 s87, s87, 0\n\t"
        "s_memrealtime s[90:91]\n\t"
        "s_waitcnt lgkmcnt(0)\n\t"
        "v_cmp_ne_u32_e64 s[92:93], 0, v164\n\t"
        "s_cmp_lg_u64 s[92:93], 0\n\t"
        "s_cbranch_scc1 NOFIXT4_%=\n\t"
        "FIXLT4_%=:\n\t"
        "global_load_dwordx4 v[96:99], %[l16], s[86:87] offset:0 sc0 sc1\n\t"
        "global_load_dwordx4 v[100:103], %[l16], s[86:87] offset:1024 sc0 sc1\n\t"
        "s_waitcnt vmcnt(0)\n\t"
        "v_bitop3_b32 v166, v96, v100, %[tag] bitop3:0x7e\n\t"
        "v_and_b32 v166, 1, v166\n\t"
        "v_cmp_ne_u32_e64 s[92:93], 0, v166\n\t"
        "s_cmp_eq_u64 s[92:93], 0\n\t"
        "s_cbranch_scc1 NOFIXT4_%=\n\t"
        "s_sleep 1\n\t"
        "s_memrealtime s[92:93]\n\t"
        "s_waitcnt lgkmcnt(0)\n\t"
        "s_sub_u32 s92, s92, s90\n\t"
        "s_cmp_lt_u32 s92, " KDL "\n\t"
        "s_cbranch_scc1 FIXLT4_%=\n\t"
        "v_mov_b32 v166, %[bad]\n\t"
        "ds_write_b32 v166, v166\n\t"
        "NOFIXT4_%=:\n\t"
        "s_mov_b32 s94, 0\n\t"
        "SPINT4_%=:\n\t"
        "ds_read_b32 v164, %[vctl]\n\t"
        "s_waitcnt lgkmcnt(0)\n\t"
        "v_cmp_gt_u32_e64 s[86:87], s89, v164\n\t"
        "s_and_b64 s[86:87], s[86:87], 0x400\n\t"
        "s_cbranch_scc0 TAIL4_%=\n\t"
        SC3("s_add_u32 %[nfix], %[nfix], 1\n\t")
        "s_sleep 1\n\t"
        "s_add_u32 s94, s94, 1\n\t"
        "s_and_b32 s92, s94, 0x3ff\n\t"
        "s_cbranch_scc1 SPINT4_%=\n\t"
        "s_memrealtime s[92:93]\n\t"
        "s_waitcnt lgkmcnt(0)\n\t"
        "s_cmp_eq_u32 s94, 0x400\n\t"
        "s_cselect_b32 s90, s92, s90\n\t"
        "s_sub_u32 s92, s92, s90\n\t"
        "s_cmp_lt_u32 s92, " KDL3 "\n\t"
        "s_cbranch_scc1 SPINT4_%=\n\t"
        "v_mov_b32 v166, %[bad]\n\t"
        "ds_write_b32 v166, v166\n\t"
        "s_branch TAIL4_%=\n\t"
        "FIXT5_%=:\n\t"
        "s_cmp_eq_u64 s[84:85], 0\n\t"
        "s_cbranch_scc1 NOFIXT5_%=\n\t"
        "s_add_u32 %[nfix], %[nfix], 1\n\t"
        "v_mov_b32 v166, %[bad]\n\t"
        "ds_read_b32 v164, v166\n\t"
        "s_mov_b64 s[86:87], %[xtail]\n\t"
        "s_mov_b32 s92, 40960\n\t"
        "s_add_u32 s86, s86, s92\n\t"
        "s_addc_u32 s87, s87, 0\n\t"
        "s_memrealtime s[90:91]\n\t"
        "s_waitcnt lgkmcnt(0)\n\t"
        "v_cmp_ne_u32_e64 s[92:93], 0, v164\n\t"
        "s_cmp_lg_u64 s[92:93], 0\n\t"
        "s_cbranch_scc1 NOFIXT5_%=\n\t"
        "FIXLT5_%=:\n\t"
        "global_load_dwordx4 v[104:107], %[l16], s[86:87] offset:0 sc0 sc1\n\t"
        "global_load_dwordx4 v[108:111], %[l16], s[86:87] offset:1024 sc0 sc1\n\t"
        "s_waitcnt vmcnt(0)\n\t"
        "v_bitop3_b32 v166, v104, v108, %[tag] bitop3:0x7e\n\t"
        "v_and_b32 v166, 1, v166\n\t"
        "v_cmp_ne_u32_e64 s[92:93], 0, v166\n\t"
        "s_cmp_eq_u64 s[92:93], 0\n\t"
        "s_cbranch_scc1 NOFIXT5_%=\n\t"
        "s_sleep 1\n\t"
        "s_memrealtime s[92:93]\n\t"
        "s_waitcnt lgkmcnt(0)\n\t"
        "s_sub_u32 s92, s92, s90\n\t"
        "s_cmp_lt_u32 s92, " KDL "\n\t"
        "s_cbranch_scc1 FIXLT5_%=\n\t"
        "v_mov_b32 v166, %[bad]\n\t"
        "ds_write_b32 v166, v166\n\t"
        "NOFIXT5_%=:\n\t"
        "s_mov_b32 s94, 0\n\t"
        "SPINT5_%=:\n\t"
        "ds_read_b32 v164, %[vctl]\n\t"
        "s_waitcnt lgkmcnt(0)\n\t"
        "v_cmp_gt_u32_e64 s[86:87], s89, v164\n\t"
        "s_and_b64 s[86:87], s[86:87], 0x400\n\t"
        "s_cbranch_scc0 TAIL5_%=\n\t"
        SC3("s_add_u32 %[nfix], %[nfix], 1\n\t")
        "s_sleep 1\n\t"
        "s_add_u32 s94, s94, 1\n\t"
        "s_and_b32 s92, s94, 0x3ff\n\t"
        "s_cbranch_scc1 SPINT5_%=\n\t"
        "s_memrealtime s[92:93]\n\t"
        "s_waitcnt lgkmcnt(0)\n\t"
        "s_cmp_eq_u32 s94, 0x400\n\t"
        "s_cselect_b32 s90, s92, s90\n\t"
        "s_sub_u32 s92, s92, s90\n\t"
        "s_cmp_lt_u32 s92, " KDL3 "\n\t"
        "s_cbranch_scc1 SPINT5_%=\n\t"
        "v_mov_b32 v166, %[bad]\n\t"
        "ds_write_b32 v166, v166\n\t"
        "s_branch TAIL5_%=\n\t"
        "FIXT6_%=:\n\t"
        "s_cmp_eq_u64 s[84:85], 0\n\t"
        "s_cbranch_scc1 NOFIXT6_%=\n\t"
        "s_add_u32 %[nfix], %[nfix], 1\n\t"
        "v_mov_b32 v166, %[bad]\n\t"
        "ds_read_b32 v164, v166\n\t"
        "s_mov_b64 s[86:87], %[xtail]\n\t"
        "s_mov_b32 s92, 49152\n\t"
        "s_add_u32 s86, s86, s92\n\t"
        "s_addc_u32 s87, s87, 0\n\t"
        "s_memrealtime s[90:91]\n\t"
        "s_waitcnt lgkmcnt(0)\n\t"
        "v_cmp_ne_u32_e64 s[92:93], 0, v164\n\t"
        "s_cmp_lg_u64 s[92:93], 0\n\t"
        "s_cbranch_scc1 NOFIXT6_%=\n\t"
        "FIXLT6_%=:\n\t"
        "global_load_dwordx4 v[112:115], %[l16], s[86:87] offset:0 sc0 sc1\n\t"
        "global_load_dwordx4 v[116:119], %[l16], s[86:87] offset:1024 sc0 sc1\n\t"
        "s_waitcnt vmcnt(0)\n\t"
        "v_bitop3_b32 v166, v112, v116, %[tag] bitop3:0x7e\n\t"
        "v_and_b32 v166, 1, v166\n\t"
        "v_cmp_ne_u32_e64 s[92:93], 0, v166\n\t"
        "s_cmp_eq_u64 s[92:93], 0\n\t"
        "s_cbranch_scc1 NOFIXT6_%=\n\t"
        "s_sleep 1\n\t"
        "s_memrealtime s[92:93]\n\t"
        "s_waitcnt lgkmcnt(0)\n\t"
        "s_sub_u32 s92, s92, s90\n\t"
        "s_cmp_lt_u32 s92, " KDL "\n\t"
        "s_cbranch_scc1 FIXLT6_%=\n\t"
        "v_mov_b32 v166, %[bad]\n\t"
        "ds_write_b32 v166, v166\n\t"
        "NOFIXT6_%=:\n\t"
        "s_mov_b32 s94, 0\n\t"
        "SPINT6_%=:\n\t"
        "ds_read_b32 v164, %[vctl]\n\t"
        "s_waitcnt lgkmcnt(0)\n\t"
        "v_cmp_gt_u32_e64 s[86:87], s89, v164\n\t"
        "s_and_b64 s[86:87], s[86:87], 0x400\n\t"
        "s_cbranch_scc0 TAIL6_%=\n\t"
        SC3("s_add_u32 %[nfix], %[nfix], 1\n\t")
        "s_sleep 1\n\t"
        "s_add_u32 s94, s94, 1\n\t"
        "s_and_b32 s92, s94, 0x3ff\n\t"
        "s_cbranch_scc1 SPINT6_%=\n\t"
        "s_memrealtime s[92:93]\n\t"
        "s_waitcnt lgkmcnt(0)\n\t"
        "s_cmp_eq_u32 s94, 0x400\n\t"
        "s_cselect_b32 s90, s92, s90\n\t"
        "s_sub_u32 s92, s92, s90\n\t"
        "s_cmp_lt_u32 s92, " KDL3 "\n\t"
        "s_cbranch_scc1 SPINT6_%=\n\t"
        "v_mov_b32 v166, %[bad]\n\t"
        "ds_write_b32 v166, v166\n\t"
        "s_branch TAIL6_%=\n\t"
        "FIXT7_%=:\n\t"
        "s_cmp_eq_u64 s[84:85], 0\n\t"
        "s_cbranch_scc1 NOFIXT7_%=\n\t"
        "s_add_u32 %[nfix], %[nfix], 1\n\t"
        "v_mov_b32 v166, %[bad]\n\t"
        "ds_read_b32 v164, v166\n\t"
        "s_mov_b64 s[86:87], %[xtail]\n\t"
        "s_mov_b32 s92, 57344\n\t"
        "s_add_u32 s86, s86, s92\n\t"
        "s_addc_u32 s87, s87, 0\n\t"
        "s_memrealtime s[90:91]\n\t"
        "s_waitcnt lgkmcnt(0)\n\t"
        "v_cmp_ne_u32_e64 s[92:93], 0, v164\n\t"
        "s_cmp_lg_u64 s[92:93], 0\n\t"
        "s_cbranch_scc1 NOFIXT7_%=\n\t"
        "FIXLT7_%=:\n\t"
        "global_load_dwordx4 v[120:123], %[l16], s[86:87] offset:0 sc0 sc1\n\t"
        "global_load_dwordx4 v[124:127], %[l16], s[86:87] offset:1024 sc0 sc1\n\t"
        "s_waitcnt vmcnt(0)\n\t"
        "v_bitop3_b32 v166, v120, v124, %[tag] bitop3:0x7e\n\t"
        "v_and_b32 v166, 1, v166\n\t"
        "v_cmp_ne_u32_e64 s[92:93], 0, v166\n\t"
        "s_cmp_eq_u64 s[92:93], 0\n\t"
        "s_cbranch_scc1 NOFIXT7_%=\n\t"
        "s_sleep 1\n\t"
        "s_memrealtime s[92:93]\n\t"
        "s_waitcnt lgkmcnt(0)\n\t"
        "s_sub_u32 s92, s92, s90\n\t"
        "s_cmp_lt_u32 s92, " KDL "\n\t"
        "s_cbranch_scc1 FIXLT7_%=\n\t"
        "v_mov_b32 v166, %[bad]\n\t"
        "ds_write_b32 v166, v166\n\t"
        "NOFIXT7_%=:\n\t"
        "s_mov_b32 s94, 0\n\t"
        "SPINT7_%=:\n\t"
        "ds_read_b32 v164, %[vctl]\n\t"
        "s_waitcnt lgkmcnt(0)\n\t"
        "v_cmp_gt_u32_e64 s[86:87], s89, v164\n\t"
        "s_and_b64 s[86:87], s[86:87], 0x400\n\t"
        "s_cbranch_scc0 RESPEC_%=\n\t"
        SC3("s_add_u32 %[nfix], %[nfix], 1\n\t")
        "s_sleep 1\n\t"
        "s_add_u32 s94, s94, 1\n\t"
        "s_and_b32 s92, s94, 0x3ff\n\t"
        "s_cbranch_scc1 SPINT7_%=\n\t"
        "s_memrealtime s[92:93]\n\t"
        "s_waitcnt lgkmcnt(0)\n\t"
        "s_cmp_eq_u32 s94, 0x400\n\t"
        "s_cselect_b32 s90, s92, s90\n\t"
        "s_sub_u32 s92, s92, s90\n\t"
        "s_cmp_lt_u32 s92, " KDL3 "\n\t"
        "s_cbranch_scc1 SPINT7_%=\n\t"
        "v_mov_b32 v166, %[bad]\n\t"
        "ds_write_b32 v166, v166\n\t"
        "s_branch RESPEC_%=\n\t"
        "RESPEC_%=:\n\t"
        "v_mov_b32 v164, %[bad]\n\t"
        MF("v_mfma_f32_16x16x32_f16 v[116:119], v[120:123], v[128:131], v[8:11]\n\t")
        MF("v_mfma_f32_16x16x32_f16 v[116:119], v[124:127], v[132:135], v[116:119]\n\t")
        "s_branch TAILS_%=\n\t"
        "ENTRY2_%=:\n\t"
        "s_mov_b64 s[96:97], %[xtail]\n\t"
        "s_movk_i32 s95, 0x2000\n\t"
        "s_lshl_b32 s89, %[stage], 2\n\t"
        "v_mov_b32 v165, %[bad]\n\t"
        "s_setprio " KPRM "\n\t"
        "s_mov_b32 s94, 0\n\t"
        "SPINTK_%=:\n\t"
        "ds_read_b32 v164, v165 offset:" KTOK "\n\t"
        "s_waitcnt lgkmcnt(0)\n\t"
        "v_cmp_gt_u32_e64 s[86:87], s89, v164\n\t"
        "s_cmp_eq_u64 s[86:87], 0\n\t"
        "s_cbranch_scc1 GOTK_%=\n\t"
        SC3("s_add_u32 %[nfix], %[nfix], 1\n\t")
        "s_sleep 1\n\t"
        "s_add_u32 s94, s94, 1\n\t"
        "s_and_b32 s92, s94, 0x3ff\n\t"
        "s_cbranch_scc1 SPINTK_%=\n\t"
        "s_memrealtime s[92:93]\n\t"
        "s_waitcnt lgkmcnt(0)\n\t"
        "s_cmp_eq_u32 s94, 0x400\n\t"
        "s_cselect_b32 s90, s92, s90\n\t"
        "s_sub_u32 s92, s92, s90\n\t"
        "s_cmp_lt_u32 s92, " KDL3 "\n\t"
        "s_cbranch_scc1 SPINTK_%=\n\t"
        "v_mov_b32 v166, %[bad]\n\t"
        "ds_write_b32 v166, v166\n\t"
        "s_branch GOTK_%=\n\t"
        "GOTK_%=:\n\t"
        "global_load_dwordx4 v[112:115], %[l16], s[96:97] offset:0 " POL_FAR "\n\t"
        "global_load_dwordx4 v[116:119], %[l16], s[96:97] offset:1024 " POL_FAR "\n\t"
        "s_add_u32 s96, s96, s95\n\t"
        "s_addc_u32 s97, s97, 0\n\t"
        "s_add_u32 s92, 0, 0\n\t"
        "v_add_u32 v165, s92, %[l16]\n\t"
        "ds_read_b128 v[128:131], v165 offset:0\n\t"
        "ds_read_b128 v[132:135], v165 offset:1024\n\t"
        "ds_read_b128 v[136:139], v165 offset:2048\n\t"
        "ds_read_b128 v[140:143], v165 offset:3072\n\t"
        "ds_read_b128 v[144:147], v165 offset:4096\n\t"
        "ds_read_b128 v[148:151], v165 offset:5120\n\t"
        "ds_read_b128 v[152:155], v165 offset:6144\n\t"
        "ds_read_b128 v[156:159], v165 offset:7168\n\t"
        "s_waitcnt lgkmcnt(0)\n\t"
        "global_load_dwordx4 v[120:123], %[l16], s[96:97] offset:0 " POL_FAR "\n\t"
        "global_load_dwordx4 v[124:127], %[l16], s[96:97] offset:1024 " POL_FAR "\n\t"
        "s_add_u32 s96, s96, s95\n\t"
        "s_addc_u32 s97, s97, 0\n\t"
        "s_waitcnt vmcnt(2)\n\t"
        "s_setprio " KPRC "\n\t"
        "v_bitop3_b32 v166, v112, v116, %[tag] bitop3:0x7e\n\t"
        "v_and_b32 v166, 1, v166\n\t"
        TG("v_cmp_ne_u32_e64 s[84:85], 0, v166\n\t")
        NTG("s_mov_b64 s[84:85], 0\n\t")
        "s_waitcnt lgkmcnt(0)\n\t"
        "s_cmp_eq_u64 s[84:85], 0\n\t"
        "s_cbranch_scc0 FIXT0_%=\n\t"
        "GO1_%=:\n\t"
        "s_cmp_eq_u32 %[stage], 1\n\t"
        "s_cbranch_scc1 TAIL0_%=\n\t"
        TRM("s_memrealtime s[90:91]\n\t")
        TRM("s_mov_b32 s93, 16\n\t")
        TRM("s_mov_b32 m0, s93\n\t")
        TRM("s_waitcnt lgkmcnt(0)\n\t")
        TRM("v_writelane_b32 %[tr], s90, m0\n\t")
        MF("v_mfma_f32_16x16x32_f16 v[8:11], v[112:115], v[128:131], v[44:47]\n\t")
        "global_load_dwordx4 v[80:83], %[l16], s[96:97] offset:0 " POL_FAR "\n\t"
        "ds_read_b128 v[128:131], v165 offset:9216\n\t"
        MF("v_mfma_f32_16x16x32_f16 v[12:15], v[112:115], v[132:135], v[48:51]\n\t")
        "global_load_dwordx4 v[84:87], %[l16], s[96:97] offset:1024 " POL_FAR "\n\t"
        "ds_read_b128 v[132:135], v165 offset:10240\n\t"
        MF("v_mfma_f32_16x16x32_f16 v[16:19], v[112:115], v[136:139], v[52:55]\n\t")
        "s_add_u32 s96, s96, s95\n\t"
        "s_addc_u32 s97, s97, 0\n\t"
        "ds_read_b128 v[136:139], v165 offset:11264\n\t"
        MF("v_mfma_f32_16x16x32_f16 v[20:23], v[112:115], v[140:143], v[56:59]\n\t")
        "ds_read_b128 v[140:143], v165 offset:12288\n\t"
        MF("v_mfma_f32_16x16x32_f16 v[24:27], v[112:115], v[144:147], v[60:63]\n\t")
        "ds_read_b128 v[144:147], v165 offset:13312\n\t"
        MF("v_mfma_f32_16x16x32_f16 v[28:31], v[112:115], v[148:151], v[64:67]\n\t")
        "ds_read_b128 v[148:151], v165 offset:14336\n\t"
        MF("v_mfma_f32_16x16x32_f16 v[32:35], v[112:115], v[152:155], v[68:71]\n\t")
        "ds_read_b128 v[152:155], v165 offset:15360\n\t"
        MF("v_mfma_f32_16x16x32_f16 v[36:39], v[112:115], v[156:159], v[72:75]\n\t")
        "ds_read_b128 v[156:159], v165 offset:16384\n\t"
        "s_branch TAILH_%=\n\t"
        "DONE_%=:\n\t"
        : [nfix] "+s"(nfix), [stage] "+s"(k.stage), [xtail] "+s"(k.xtail), [outp] "+s"(k.outp) TRO
        : [vctl] "v"(k.vctl), [l16] "v"(k.l16), [tag] "s"(k.tag), [bad] "s"(k.bad), [xc] "s"(k.xc),
          [xloc] "s"(k.xloc), [xfar] "s"(k.xfar), [xt0] "s"(k.xt0), [out2] "s"(k.out2), [xtl2] "s"(k.xtl2)
        : "memory", "scc", "v8", "v9", "v10", "v11", "v12", "v13", "v14", "v15", "v16", "v17", "v18", "v19", "v20", "v21", "v22", "v23", "v24", "v25", "v26", "v27", "v28", "v29", "v30", "v31", "v32", "v33", "v34", "v35", "v36", "v37", "v38", "v39", "v40", "v41", "v42", "v43", "v44", "v45", "v46", "v47", "v48", "v49", "v50", "v51", "v52", "v53", "v54", "v55", "v56", "v57", "v58", "v59", "v60", "v61", "v62", "v63", "v64", "v65", "v66", "v67", "v68", "v69", "v70", "v71", "v72", "v73", "v74", "v75", "v80", "v81", "v82", "v83", "v84", "v85", "v86", "v87", "v88", "v89", "v90", "v91", "v92", "v93", "v94", "v95", "v96", "v97", "v98", "v99", "v100", "v101", "v102", "v103", "v104", "v105", "v106", "v107", "v108", "v109", "v110", "v111", "v112", "v113", "v114", "v115", "v116", "v117", "v118", "v119", "v120", "v121", "v122", "v123", "v124", "v125", "v126", "v127", "v128", "v129", "v130", "v131", "v132", "v133", "v134", "v135", "v136", "v137", "v138", "v139", "v140", "v141", "v142", "v143", "v144", "v145", "v146", "v147", "v148", "v149", "v150", "v151", "v152", "v153", "v154", "v155", "v156", "v157", "v158", "v159", "v160", "v161", "v162", "v163", "v164", "v165", "v166", "v167", "s84", "s85", "s86", "s87", "s88", "s89", "s90", "s91", "s92", "s93", "s94", "s95", "s96", "s97", "s98", "s99");
}
__device__ __forceinline__ void kall9(KArgs& k, unsigned& nfix, unsigned& tr) {
    asm volatile(
        "v_mov_b32 v167, 1\n\t"
        "v_mov_b32 v8, 0\n\t"
        "v_mov_b32 v9, 0\n\t"
        "v_mov_b32 v10, 0\n\t"
        "v_mov_b32 v11, 0\n\t"
        "v_mov_b32 v12, 0\n\t"
        "v_mov_b32 v13, 0\n\t"
        "v_mov_b32 v14, 0\n\t"
        "v_mov_b32 v15, 0\n\t"
        "v_mov_b32 v16, 0\n\t"
        "v_mov_b32 v17, 0\n\t"
        "v_mov_b32 v18, 0\n\t"
        "v_mov_b32 v19, 0\n\t"
        "v_mov_b32 v20, 0\n\t"
        "v_mov_b32 v21, 0\n\t"
        "v_mov_b32 v22, 0\n\t"
        "v_mov_b32 v23, 0\n\t"
        "v_mov_b32 v24, 0\n\t"
        "v_mov_b32 v25, 0\n\t"
        "v_mov_b32 v26, 0\n\t"
        "v_mov_b32 v27, 0\n\t"
        "v_mov_b32 v28, 0\n\t"
        "v_mov_b32 v29, 0\n\t"
        "v_mov_b32 v30, 0\n\t"
        "v_mov_b32 v31, 0\n\t"
        "v_mov_b32 v32, 0\n\t"
        "v_mov_b32 v33, 0\n\t"
        "v_mov_b32 v34, 0\n\t"
        "v_mov_b32 v35, 0\n\t"
        "v_mov_b32 v36, 0\n\t"
        "v_mov_b32 v37, 0\n\t"
        "v_mov_b32 v38, 0\n\t"
        "v_mov_b32 v39, 0\n\t"
        "v_mov_b32 v40, 0\n\t"
        "v_mov_b32 v41, 0\n\t"
        "v_mov_b32 v42, 0\n\t"
        "v_mov_b32 v43, 0\n\t"
        "v_mov_b32 v44, 0\n\t"
        "v_mov_b32 v45, 0\n\t"
        "v_mov_b32 v46, 0\n\t"
        "v_mov_b32 v47, 0\n\t"
        "v_mov_b32 v48, 0\n\t"
        "v_mov_b32 v49, 0\n\t"
        "v_mov_b32 v50, 0\n\t"
        "v_mov_b32 v51, 0\n\t"
        "v_mov_b32 v52, 0\n\t"
        "v_mov_b32 v53, 0\n\t"
        "v_mov_b32 v54, 0\n\t"
        "v_mov_b32 v55, 0\n\t"
        "v_mov_b32 v56, 0\n\t"
        "v_mov_b32 v57, 0\n\t"
        "v_mov_b32 v58, 0\n\t"
        "v_mov_b32 v59, 0\n\t"
        "v_mov_b32 v60, 0\n\t"
        "v_mov_b32 v61, 0\n\t"
        "v_mov_b32 v62, 0\n\t"
        "v_mov_b32 v63, 0\n\t"
        "v_mov_b32 v64, 0\n\t"
        "v_mov_b32 v65, 0\n\t"
        "v_mov_b32 v66, 0\n\t"
        "v_mov_b32 v67, 0\n\t"
        "v_mov_b32 v68, 0\n\t"
        "v_mov_b32 v69, 0\n\t"
        "v_mov_b32 v70, 0\n\t"
        "v_mov_b32 v71, 0\n\t"
        "v_mov_b32 v72, 0\n\t"
        "v_mov_b32 v73, 0\n\t"
        "v_mov_b32 v74, 0\n\t"
        "v_mov_b32 v75, 0\n\t"
        "v_mov_b32 v76, 0\n\t"
        "v_mov_b32 v77, 0\n\t"
        "v_mov_b32 v78, 0\n\t"
        "v_mov_b32 v79, 0\n\t"
        "s_cmp_eq_u32 %[stage], 0\n\t"
        "s_cselect_b32 s98, 0x2000, 0\n\t"
        "s_mov_b32 s89, 1\n\t"
        "s_mov_b64 s[96:97], %[xloc]\n\t"
        "s_movk_i32 s95, 0x4000\n\t"
        "s_add_u32 s88, 0, 9216\n\t"
        "v_add_u32 v165, 0, %[l16]\n\t"
        "s_mov_b32 s94, 0\n\t"
        "SPINI0_%=:\n\t"
        "ds_read_b32 v164, %[vctl]\n\t"
        "s_waitcnt lgkmcnt(0)\n\t"
        "v_cmp_gt_u32_e64 s[86:87], s89, v164\n\t"
        "s_and_b64 s[86:87], s[86:87], 2\n\t"
        "s_cbranch_scc0 GOI0_%=\n\t"
        SC3("s_add_u32 %[nfix], %[nfix], 1\n\t")
        "s_sleep 1\n\t"
        "s_add_u32 s94, s94, 1\n\t"
        "s_and_b32 s92, s94, 0x3ff\n\t"
        "s_cbranch_scc1 SPINI0_%=\n\t"
        "s_memrealtime s[92:93]\n\t"
        "s_waitcnt lgkmcnt(0)\n\t"
        "s_cmp_eq_u32 s94, 0x400\n\t"
        "s_cselect_b32 s90, s92, s90\n\t"
        "s_sub_u32 s92, s92, s90\n\t"
        "s_cmp_lt_u32 s92, " KDL3 "\n\t"
        "s_cbranch_scc1 SPINI0_%=\n\t"
        "v_mov_b32 v166, %[bad]\n\t"
        "ds_write_b32 v166, v166\n\t"
        "s_branch GOI0_%=\n\t"
        "GOI0_%=:\n\t"
        "global_load_dwordx4 v[80:83], %[l16], s[96:97] offset:0 " POL_DMA "\n\t"
        "global_load_dwordx4 v[84:87], %[l16], s[96:97] offset:1024 " POL_DMA "\n\t"
        "s_add_u32 s96, s96, s95\n\t"
        "s_addc_u32 s97, s97, 0\n\t"
        "global_load_dwordx4 v[88:91], %[l16], s[96:97] offset:0 " POL_DMA "\n\t"
        "global_load_dwordx4 v[92:95], %[l16], s[96:97] offset:1024 " POL_DMA "\n\t"
        "s_add_u32 s96, s96, s95\n\t"
        "s_addc_u32 s97, s97, 0\n\t"
        "s_mov_b32 s89, 2\n\t"
        "s_mov_b32 s94, 0\n\t"
        "SPINI1_%=:\n\t"
        "ds_read_b32 v164, %[vctl]\n\t"
        "s_waitcnt lgkmcnt(0)\n\t"
        "v_cmp_gt_u32_e64 s[86:87], s89, v164\n\t"
        "s_cmp_eq_u64 s[86:87], 0\n\t"
        "s_cbranch_scc1 GOI1_%=\n\t"
        SC3("s_add_u32 %[nfix], %[nfix], 1\n\t")
        "s_sleep 1\n\t"
        "s_add_u32 s94, s94, 1\n\t"
        "s_and_b32 s92, s94, 0x3ff\n\t"
        "s_cbranch_scc1 SPINI1_%=\n\t"
        "s_memrealtime s[92:93]\n\t"
        "s_waitcnt lgkmcnt(0)\n\t"
        "s_cmp_eq_u32 s94, 0x400\n\t"
        "s_cselect_b32 s90, s92, s90\n\t"
        "s_sub_u32 s92, s92, s90\n\t"
        "s_cmp_lt_u32 s92, " KDL3 "\n\t"
        "s_cbranch_scc1 SPINI1_%=\n\t"
        "v_mov_b32 v166, %[bad]\n\t"
        "ds_write_b32 v166, v166\n\t"
        "s_branch GOI1_%=\n\t"
        "GOI1_%=:\n\t"
        "ds_read_b128 v[128:131], v165 offset:0\n\t"
        "ds_read_b128 v[132:135], v165 offset:1024\n\t"
        "ds_read_b128 v[136:139], v165 offset:2048\n\t"
        "ds_read_b128 v[140:143], v165 offset:3072\n\t"
        "ds_read_b128 v[144:147], v165 offset:4096\n\t"
        "ds_read_b128 v[148:151], v165 offset:5120\n\t"
        "ds_read_b128 v[152:155], v165 offset:6144\n\t"
        "ds_read_b128 v[156:159], v165 offset:7168\n\t"
        "ds_read_b128 v[160:163], v165 offset:8192\n\t"
        "s_mov_b64 exec, 1\n\t"
        "ds_add_u32 %[vctl], v167\n\t"
        "s_mov_b64 exec, -1\n\t"
        "s_waitcnt vmcnt(2)\n\t"
        "v_bitop3_b32 v166, v80, v84, %[tag] bitop3:0x7e\n\t"
        "v_and_b32 v166, 1, v166\n\t"
        TG("v_cmp_ne_u32_e64 s[84:85], 0, v166\n\t")
        NTG("s_mov_b64 s[84:85], 0\n\t")
        "s_waitcnt lgkmcnt(0)\n\t"
        "s_cmp_eq_u64 s[84:85], 0\n\t"
        "s_cbranch_scc0 SLOWP0_%=\n\t"
        "PREP0_%=:\n\t"
        "s_waitcnt lgkmcnt(10)\n\t"
        MF("v_mfma_f32_16x16x32_f16 v[8:11], v[80:83], v[128:131], v[8:11]\n\t")
        "v_add_u32 v165, s88, %[l16]\n\t"
        "global_load_dwordx4 v[96:99], %[l16], s[96:97] offset:0 " POL_DMA "\n\t"
        MF("v_mfma_f32_16x16x32_f16 v[44:47], v[84:87], v[128:131], v[44:47]\n\t")
        "global_load_dwordx4 v[100:103], %[l16], s[96:97] offset:1024 " POL_DMA "\n\t"
        "ds_read_b128 v[128:131], v165 offset:0\n\t"
        "s_waitcnt lgkmcnt(10)\n\t"
        MF("v_mfma_f32_16x16x32_f16 v[12:15], v[80:83], v[132:135], v[12:15]\n\t")
        "s_add_u32 s88, s88, 9216\n\t"
        "s_cmp_eq_u32 s88, 147456\n\t"
        "s_cselect_b32 s88, 0, s88\n\t"
        MF("v_mfma_f32_16x16x32_f16 v[48:51], v[84:87], v[132:135], v[48:51]\n\t")
        "ds_read_b128 v[132:135], v165 offset:1024\n\t"
        "s_waitcnt lgkmcnt(10)\n\t"
        MF("v_mfma_f32_16x16x32_f16 v[16:19], v[80:83], v[136:139], v[16:19]\n\t")
        "s_add_u32 s96, s96, s95\n\t"
        "s_addc_u32 s97, s97, 0\n\t"
        MF("v_mfma_f32_16x16x32_f16 v[52:55], v[84:87], v[136:139], v[52:55]\n\t")
        "ds_read_b128 v[136:139], v165 offset:2048\n\t"
        "ds_read_b32 v164, %[vctl]\n\t"
        "s_waitcnt lgkmcnt(10)\n\t"
        MF("v_mfma_f32_16x16x32_f16 v[20:23], v[80:83], v[140:143], v[20:23]\n\t")
        MF("v_mfma_f32_16x16x32_f16 v[56:59], v[84:87], v[140:143], v[56:59]\n\t")
        "ds_read_b128 v[140:143], v165 offset:3072\n\t"
        "s_waitcnt lgkmcnt(10)\n\t"
        MF("v_mfma_f32_16x16x32_f16 v[24:27], v[80:83], v[144:147], v[24:27]\n\t")
        "s_add_u32 s89, s89, 1\n\t"
        MF("v_mfma_f32_16x16x32_f16 v[60:63], v[84:87], v[144:147], v[60:63]\n\t")
        "ds_read_b128 v[144:147], v165 offset:4096\n\t"
        "s_waitcnt lgkmcnt(10)\n\t"
        MF("v_mfma_f32_16x16x32_f16 v[28:31], v[80:83], v[148:151], v[28:31]\n\t")
        MF("v_mfma_f32_16x16x32_f16 v[64:67], v[84:87], v[148:151], v[64:67]\n\t")
        "ds_read_b128 v[148:151], v165 offset:5120\n\t"
        "s_waitcnt lgkmcnt(10)\n\t"
        MF("v_mfma_f32_16x16x32_f16 v[32:35], v[80:83], v[152:155], v[32:35]\n\t")
        MF("v_mfma_f32_16x16x32_f16 v[68:71], v[84:87], v[152:155], v[68:71]\n\t")
        "ds_read_b128 v[152:155], v165 offset:6144\n\t"
        "s_waitcnt lgkmcnt(10)\n\t"
        "s_waitcnt vmcnt(2)\n\t"
        MF("v_mfma_f32_16x16x32_f16 v[36:39], v[80:83], v[156:159], v[36:39]\n\t")
        "v_bitop3_b32 v166, v88, v92, %[tag] bitop3:0x7e\n\t"
        MF("v_mfma_f32_16x16x32_f16 v[72:75], v[84:87], v[156:159], v[72:75]\n\t")
        "v_and_b32 v166, 1, v166\n\t"
        "ds_read_b128 v[156:159], v165 offset:7168\n\t"
        "s_waitcnt lgkmcnt(5)\n\t"
        MF("v_mfma_f32_16x16x32_f16 v[40:43], v[160:163], v[80:83], v[40:43]\n\t")
        "v_cmp_gt_u32_e64 s[86:87], s89, v164\n\t"
        TG("v_cmp_ne_u32_e64 s[84:85], 0, v166\n\t")
        NTG("s_mov_b64 s[84:85], 0\n\t")
        MF("v_mfma_f32_16x16x32_f16 v[76:79], v[160:163], v[84:87], v[76:79]\n\t")
        "ds_read_b128 v[160:163], v165 offset:8192\n\t"
        "s_mov_b64 exec, 1\n\t"
        "ds_add_u32 %[vctl], v167\n\t"
        "s_mov_b64 exec, -1\n\t"
        "s_or_b64 s[90:91], s[84:85], s[86:87]\n\t"
        "s_cbranch_scc1 SLOWP1_%=\n\t"
        "PREP1_%=:\n\t"
        "s_waitcnt lgkmcnt(10)\n\t"
        MF("v_mfma_f32_16x16x32_f16 v[8:11], v[88:91], v[128:131], v[8:11]\n\t")
        "v_add_u32 v165, s88, %[l16]\n\t"
        "global_load_dwordx4 v[104:107], %[l16], s[96:97] offset:0 " POL_DMA "\n\t"
        MF("v_mfma_f32_16x16x32_f16 v[44:47], v[92:95], v[128:131], v[44:47]\n\t")
        "global_load_dwordx4 v[108:111], %[l16], s[96:97] offset:1024 " POL_DMA "\n\t"
        "ds_read_b128 v[128:131], v165 offset:0\n\t"
        "s_waitcnt lgkmcnt(10)\n\t"
        MF("v_mfma_f32_16x16x32_f16 v[12:15], v[88:91], v[132:135], v[12:15]\n\t")
        "s_add_u32 s88, s88, 9216\n\t"
        "s_cmp_eq_u32 s88, 147456\n\t"
        "s_cselect_b32 s88, 0, s88\n\t"
        MF("v_mfma_f32_16x16x32_f16 v[48:51], v[92:95], v[132:135], v[48:51]\n\t")
        "ds_read_b128 v[132:135], v165 offset:1024\n\t"
        "s_waitcnt lgkmcnt(10)\n\t"
        MF("v_mfma_f32_16x16x32_f16 v[16:19], v[88:91], v[136:139], v[16:19]\n\t")
        "s_add_u32 s96, s96, s95\n\t"
        "s_addc_u32 s97, s97, 0\n\t"
        MF("v_mfma_f32_16x16x32_f16 v[52:55], v[92:95], v[136:139], v[52:55]\n\t")
        "ds_read_b128 v[136:139], v165 offset:2048\n\t"
        "ds_read_b32 v164, %[vctl]\n\t"
        "s_waitcnt lgkmcnt(10)\n\t"
        MF("v_mfma_f32_16x16x32_f16 v[20:23], v[88:91], v[140:143], v[20:23]\n\t")
        MF("v_mfma_f32_16x16x32_f16 v[56:59], v[92:95], v[140:143], v[56:59]\n\t")
        "ds_read_b128 v[140:143], v165 offset:3072\n\t"
        "s_waitcnt lgkmcnt(10)\n\t"
        MF("v_mfma_f32_16x16x32_f16 v[24:27], v[88:91], v[144:147], v[24:27]\n\t")
        "s_add_u32 s89, s89, 1\n\t"
        MF("v_mfma_f32_16x16x32_f16 v[60:63], v[92:95], v[144:147], v[60:63]\n\t")
        "ds_read_b128 v[144:147], v165 offset:4096\n\t"
        "s_waitcnt lgkmcnt(10)\n\t"
        MF("v_mfma_f32_16x16x32_f16 v[28:31], v[88:91], v[148:151], v[28:31]\n\t")
        MF("v_mfma_f32_16x16x32_f16 v[64:67], v[92:95], v[148:151], v[64:67]\n\t")
        "ds_read_b128 v[148:151], v165 offset:5120\n\t"
        "s_waitcnt lgkmcnt(10)\n\t"
        MF("v_mfma_f32_16x16x32_f16 v[32:35], v[88:91], v[152:155], v[32:35]\n\t")
        MF("v_mfma_f32_16x16x32_f16 v[68:71], v[92:95], v[152:155], v[68:71]\n\t")
        "ds_read_b128 v[152:155], v165 offset:6144\n\t"
        "s_waitcnt lgkmcnt(10)\n\t"
        "s_waitcnt vmcnt(2)\n\t"
        MF("v_mfma_f32_16x16x32_f16 v[36:39], v[88:91], v[156:159], v[36:39]\n\t")
        "v_bitop3_b32 v166, v96, v100, %[tag] bitop3:0x7e\n\t"
        MF("v_mfma_f32_16x16x32_f16 v[72:75], v[92:95], v[156:159], v[72:75]\n\t")
        "v_and_b32 v166, 1, v166\n\t"
        "ds_read_b128 v[156:159], v165 offset:7168\n\t"
        "s_waitcnt lgkmcnt(5)\n\t"
        MF("v_mfma_f32_16x16x32_f16 v[40:43], v[160:163], v[88:91], v[40:43]\n\t")
        "v_cmp_gt_u32_e64 s[86:87], s89, v164\n\t"
        TG("v_cmp_ne_u32_e64 s[84:85], 0, v166\n\t")
        NTG("s_mov_b64 s[84:85], 0\n\t")
        MF("v_mfma_f32_16x16x32_f16 v[76:79], v[160:163], v[92:95], v[76:79]\n\t")
        "ds_read_b128 v[160:163], v165 offset:8192\n\t"
        "s_mov_b64 exec, 1\n\t"
        "ds_add_u32 %[vctl], v167\n\t"
        "s_mov_b64 exec, -1\n\t"
        "s_or_b64 s[90:91], s[84:85], s[86:87]\n\t"
        "s_cbranch_scc1 SLOWP2_%=\n\t"
        "PREP2_%=:\n\t"
        "s_waitcnt lgkmcnt(10)\n\t"
        MF("v_mfma_f32_16x16x32_f16 v[8:11], v[96:99], v[128:131], v[8:11]\n\t")
        "v_add_u32 v165, s88, %[l16]\n\t"
        "global_load_dwordx4 v[112:115], %[l16], s[96:97] offset:0 " POL_DMA "\n\t"
        MF("v_mfma_f32_16x16x32_f16 v[44:47], v[100:103], v[128:131], v[44:47]\n\t")
        "global_load_dwordx4 v[116:119], %[l16], s[96:97] offset:1024 " POL_DMA "\n\t"
        "ds_read_b128 v[128:131], v165 offset:0\n\t"
        "s_waitcnt lgkmcnt(10)\n\t"
        MF("v_mfma_f32_16x16x32_f16 v[12:15], v[96:99], v[132:135], v[12:15]\n\t")
        "s_add_u32 s88, s88, 9216\n\t"
        "s_cmp_eq_u32 s88, 147456\n\t"
        "s_cselect_b32 s88, 0, s88\n\t"
        MF("v_mfma_f32_16x16x32_f16 v[48:51], v[100:103], v[132:135], v[48:51]\n\t")
        "ds_read_b128 v[132:135], v165 offset:1024\n\t"
        "s_waitcnt lgkmcnt(10)\n\t"
        MF("v_mfma_f32_16x16x32_f16 v[16:19], v[96:99], v[136:139], v[16:19]\n\t")
        "s_add_u32 s96, s96, s95\n\t"
        "s_addc_u32 s97, s97, 0\n\t"
        MF("v_mfma_f32_16x16x32_f16 v[52:55], v[100:103], v[136:139], v[52:55]\n\t")
        "ds_read_b128 v[136:139], v165 offset:2048\n\t"
        "ds_read_b32 v164, %[vctl]\n\t"
        "s_waitcnt lgkmcnt(10)\n\t"
        MF("v_mfma_f32_16x16x32_f16 v[20:23], v[96:99], v[140:143], v[20:23]\n\t")
        "s_cmp_eq_u32 s89, 16\n\t"
        "s_cselect_b64 s[96:97], %[xfar], s[96:97]\n\t"
        MF("v_mfma_f32_16x16x32_f16 v[56:59], v[100:103], v[140:143], v[56:59]\n\t")
        "ds_read_b128 v[140:143], v165 offset:3072\n\t"
        "s_waitcnt lgkmcnt(10)\n\t"
        MF("v_mfma_f32_16x16x32_f16 v[24:27], v[96:99], v[144:147], v[24:27]\n\t")
        "s_add_u32 s89, s89, 1\n\t"
        MF("v_mfma_f32_16x16x32_f16 v[60:63], v[100:103], v[144:147], v[60:63]\n\t")
        "ds_read_b128 v[144:147], v165 offset:4096\n\t"
        "s_waitcnt lgkmcnt(10)\n\t"
        MF("v_mfma_f32_16x16x32_f16 v[28:31], v[96:99], v[148:151], v[28:31]\n\t")
        MF("v_mfma_f32_16x16x32_f16 v[64:67], v[100:103], v[148:151], v[64:67]\n\t")
        "ds_read_b128 v[148:151], v165 offset:5120\n\t"
        "s_waitcnt lgkmcnt(10)\n\t"
        MF("v_mfma_f32_16x16x32_f16 v[32:35], v[96:99], v[152:155], v[32:35]\n\t")
        MF("v_mfma_f32_16x16x32_f16 v[68:71], v[100:103], v[152:155], v[68:71]\n\t")
        "ds_read_b128 v[152:155], v165 offset:6144\n\t"
        "s_waitcnt lgkmcnt(10)\n\t"
        "s_waitcnt vmcnt(2)\n\t"
        MF("v_mfma_f32_16x16x32_f16 v[36:39], v[96:99], v[156:159], v[36:39]\n\t")
        "v_bitop3_b32 v166, v104, v108, %[tag] bitop3:0x7e\n\t"
        MF("v_mfma_f32_16x16x32_f16 v[72:75], v[100:103], v[156:159], v[72:75]\n\t")
        "v_and_b32 v166, 1, v166\n\t"
        "ds_read_b128 v[156:159], v165 offset:7168\n\t"
        "s_waitcnt lgkmcnt(5)\n\t"
        MF("v_mfma_f32_16x16x32_f16 v[40:43], v[160:163], v[96:99], v[40:43]\n\t")
        "v_cmp_gt_u32_e64 s[86:87], s89, v164\n\t"
        TG("v_cmp_ne_u32_e64 s[84:85], 0, v166\n\t")
        NTG("s_mov_b64 s[84:85], 0\n\t")
        MF("v_mfma_f32_16x16x32_f16 v[76:79], v[160:163], v[100:103], v[76:79]\n\t")
        "ds_read_b128 v[160:163], v165 offset:8192\n\t"
        "s_mov_b64 exec, 1\n\t"
        "ds_add_u32 %[vctl], v167\n\t"
        "s_mov_b64 exec, -1\n\t"
        "s_or_b64 s[90:91], s[84:85], s[86:87]\n\t"
        "s_cbranch_scc1 SLOWP3_%=\n\t"
        "PREP3_%=:\n\t"
        "s_cmp_eq_u32 s89, 17\n\t"
        "s_cbranch_scc1 PREF3_%=\n\t"
        "s_waitcnt lgkmcnt(10)\n\t"
        MF("v_mfma_f32_16x16x32_f16 v[8:11], v[104:107], v[128:131], v[8:11]\n\t")
        "v_add_u32 v165, s88, %[l16]\n\t"
        "global_load_dwordx4 v[120:123], %[l16], s[96:97] offset:0 " POL_DMA "\n\t"
        MF("v_mfma_f32_16x16x32_f16 v[44:47], v[108:111], v[128:131], v[44:47]\n\t")
        "global_load_dwordx4 v[124:127], %[l16], s[96:97] offset:1024 " POL_DMA "\n\t"
        "ds_read_b128 v[128:131], v165 offset:0\n\t"
        "s_waitcnt lgkmcnt(10)\n\t"
        MF("v_mfma_f32_16x16x32_f16 v[12:15], v[104:107], v[132:135], v[12:15]\n\t")
        "s_add_u32 s88, s88, 9216\n\t"
        "s_cmp_eq_u32 s88, 147456\n\t"
        "s_cselect_b32 s88, 0, s88\n\t"
        MF("v_mfma_f32_16x16x32_f16 v[48:51], v[108:111], v[132:135], v[48:51]\n\t")
        "ds_read_b128 v[132:135], v165 offset:1024\n\t"
        "s_waitcnt lgkmcnt(10)\n\t"
        MF("v_mfma_f32_16x16x32_f16 v[16:19], v[104:107], v[136:139], v[16:19]\n\t")
        "s_add_u32 s96, s96, s95\n\t"
        "s_addc_u32 s97, s97, 0\n\t"
        MF("v_mfma_f32_16x16x32_f16 v[52:55], v[108:111], v[136:139], v[52:55]\n\t")
        "ds_read_b128 v[136:139], v165 offset:2048\n\t"
        "ds_read_b32 v164, %[vctl]\n\t"
        "s_waitcnt lgkmcnt(10)\n\t"
        MF("v_mfma_f32_16x16x32_f16 v[20:23], v[104:107], v[140:143], v[20:23]\n\t")
        MF("v_mfma_f32_16x16x32_f16 v[56:59], v[108:111], v[140:143], v[56:59]\n\t")
        "ds_read_b128 v[140:143], v165 offset:3072\n\t"
        "s_waitcnt lgkmcnt(10)\n\t"
        MF("v_mfma_f32_16x16x32_f16 v[24:27], v[104:107], v[144:147], v[24:27]\n\t")
        "s_add_u32 s89, s89, 1\n\t"
        MF("v_mfma_f32_16x16x32_f16 v[60:63], v[108:111], v[144:147], v[60:63]\n\t")
        "ds_read_b128 v[144:147], v165 offset:4096\n\t"
        "s_waitcnt lgkmcnt(10)\n\t"
        MF("v_mfma_f32_16x16x32_f16 v[28:31], v[104:107], v[148:151], v[28:31]\n\t")
        MF("v_mfma_f32_16x16x32_f16 v[64:67], v[108:111], v[148:151], v[64:67]\n\t")
        "ds_read_b128 v[148:151], v165 offset:5120\n\t"
        "s_waitcnt lgkmcnt(10)\n\t"
        MF("v_mfma_f32_16x16x32_f16 v[32:35], v[104:107], v[152:155], v[32:35]\n\t")
        MF("v_mfma_f32_16x16x32_f16 v[68:71], v[108:111], v[152:155], v[68:71]\n\t")
        "ds_read_b128 v[152:155], v165 offset:6144\n\t"
        "s_waitcnt lgkmcnt(10)\n\t"
        "s_waitcnt vmcnt(2)\n\t"
        MF("v_mfma_f32_16x16x32_f16 v[36:39], v[104:107], v[156:159], v[36:39]\n\t")
        "v_bitop3_b32 v166, v112, v116, %[tag] bitop3:0x7e\n\t"
        MF("v_mfma_f32_16x16x32_f16 v[72:75], v[108:111], v[156:159], v[72:75]\n\t")
        "v_and_b32 v166, 1, v166\n\t"
        "ds_read_b128 v[156:159], v165 offset:7168\n\t"
        "s_waitcnt lgkmcnt(5)\n\t"
        MF("v_mfma_f32_16x16x32_f16 v[40:43], v[160:163], v[104:107], v[40:43]\n\t")
        "v_cmp_gt_u32_e64 s[86:87], s89, v164\n\t"
        TG("v_cmp_ne_u32_e64 s[84:85], 0, v166\n\t")
        NTG("s_mov_b64 s[84:85], 0\n\t")
        MF("v_mfma_f32_16x16x32_f16 v[76:79], v[160:163], v[108:111], v[76:79]\n\t")
        "ds_read_b128 v[160:163], v165 offset:8192\n\t"
        "s_mov_b64 exec, 1\n\t"
        "ds_add_u32 %[vctl], v167\n\t"
        "s_mov_b64 exec, -1\n\t"
        "s_or_b64 s[90:91], s[84:85], s[86:87]\n\t"
        "s_cbranch_scc1 SLOWP4_%=\n\t"
        "PREP4_%=:\n\t"
        "s_waitcnt lgkmcnt(10)\n\t"
        MF("v_mfma_f32_16x16x32_f16 v[8:11], v[112:115], v[128:131], v[8:11]\n\t")
        "v_add_u32 v165, s88, %[l16]\n\t"
        "global_load_dwordx4 v[80:83], %[l16], s[96:97] offset:0 " POL_DMA "\n\t"
        MF("v_mfma_f32_16x16x32_f16 v[44:47], v[116:119], v[128:131], v[44:47]\n\t")
        "global_load_dwordx4 v[84:87], %[l16], s[96:97] offset:1024 " POL_DMA "\n\t"
        "ds_read_b128 v[128:131], v165 offset:0\n\t"
        "s_waitcnt lgkmcnt(10)\n\t"
        MF("v_mfma_f32_16x16x32_f16 v[12:15], v[112:115], v[132:135], v[12:15]\n\t")
        "s_add_u32 s88, s88, 9216\n\t"
        "s_cmp_eq_u32 s88, 147456\n\t"
        "s_cselect_b32 s88, 0, s88\n\t"
        MF("v_mfma_f32_16x16x32_f16 v[48:51], v[116:119], v[132:135], v[48:51]\n\t")
        "ds_read_b128 v[132:135], v165 offset:1024\n\t"
        "s_waitcnt lgkmcnt(10)\n\t"
        MF("v_mfma_f32_16x16x32_f16 v[16:19], v[112:115], v[136:139], v[16:19]\n\t")
        "s_add_u32 s96, s96, s95\n\t"
        "s_addc_u32 s97, s97, 0\n\t"
        MF("v_mfma_f32_16x16x32_f16 v[52:55], v[116:119], v[136:139], v[52:55]\n\t")
        "ds_read_b128 v[136:139], v165 offset:2048\n\t"
        "ds_read_b32 v164, %[vctl]\n\t"
        "s_waitcnt lgkmcnt(10)\n\t"
        MF("v_mfma_f32_16x16x32_f16 v[20:23], v[112:115], v[140:143], v[20:23]\n\t")
        MF("v_mfma_f32_16x16x32_f16 v[56:59], v[116:119], v[140:143], v[56:59]\n\t")
        "ds_read_b128 v[140:143], v165 offset:3072\n\t"
        "s_waitcnt lgkmcnt(10)\n\t"
        MF("v_mfma_f32_16x16x32_f16 v[24:27], v[112:115], v[144:147], v[24:27]\n\t")
        "s_add_u32 s89, s89, 1\n\t"
        MF("v_mfma_f32_16x16x32_f16 v[60:63], v[116:119], v[144:147], v[60:63]\n\t")
        "ds_read_b128 v[144:147], v165 offset:4096\n\t"
        "s_waitcnt lgkmcnt(10)\n\t"
        MF("v_mfma_f32_16x16x32_f16 v[28:31], v[112:115], v[148:151], v[28:31]\n\t")
        MF("v_mfma_f32_16x16x32_f16 v[64:67], v[116:119], v[148:151], v[64:67]\n\t")
        "ds_read_b128 v[148:151], v165 offset:5120\n\t"
        "s_waitcnt lgkmcnt(10)\n\t"
        MF("v_mfma_f32_16x16x32_f16 v[32:35], v[112:115], v[152:155], v[32:35]\n\t")
        MF("v_mfma_f32_16x16x32_f16 v[68:71], v[116:119], v[152:155], v[68:71]\n\t")
        "ds_read_b128 v[152:155], v165 offset:6144\n\t"
        "s_waitcnt lgkmcnt(10)\n\t"
        "s_waitcnt vmcnt(2)\n\t"
        MF("v_mfma_f32_16x16x32_f16 v[36:39], v[112:115], v[156:159], v[36:39]\n\t")
        "v_bitop3_b32 v166, v120, v124, %[tag] bitop3:0x7e\n\t"
        MF("v_mfma_f32_16x16x32_f16 v[72:75], v[116:119], v[156:159], v[72:75]\n\t")
        "v_and_b32 v166, 1, v166\n\t"
        "ds_read_b128 v[156:159], v165 offset:7168\n\t"
        "s_waitcnt lgkmcnt(5)\n\t"
        MF("v_mfma_f32_16x16x32_f16 v[40:43], v[160:163], v[112:115], v[40:43]\n\t")
        "v_cmp_gt_u32_e64 s[86:87], s89, v164\n\t"
        TG("v_cmp_ne_u32_e64 s[84:85], 0, v166\n\t")
        NTG("s_mov_b64 s[84:85], 0\n\t")
        MF("v_mfma_f32_16x16x32_f16 v[76:79], v[160:163], v[116:119], v[76:79]\n\t")
        "ds_read_b128 v[160:163], v165 offset:8192\n\t"
        "s_mov_b64 exec, 1\n\t"
        "ds_add_u32 %[vctl], v167\n\t"
        "s_mov_b64 exec, -1\n\t"
        "s_or_b64 s[90:91], s[84:85], s[86:87]\n\t"
        "s_cbranch_scc1 SLOWP5_%=\n\t"
        "PREP5_%=:\n\t"
        "s_waitcnt lgkmcnt(10)\n\t"
        MF("v_mfma_f32_16x16x32_f16 v[8:11], v[120:123], v[128:131], v[8:11]\n\t")
        "v_add_u32 v165, s88, %[l16]\n\t"
        "global_load_dwordx4 v[88:91], %[l16], s[96:97] offset:0 " POL_DMA "\n\t"
        MF("v_mfma_f32_16x16x32_f16 v[44:47], v[124:127], v[128:131], v[44:47]\n\t")
        "global_load_dwordx4 v[92:95], %[l16], s[96:97] offset:1024 " POL_DMA "\n\t"
        "ds_read_b128 v[128:131], v165 offset:0\n\t"
        "s_waitcnt lgkmcnt(10)\n\t"
        MF("v_mfma_f32_16x16x32_f16 v[12:15], v[120:123], v[132:135], v[12:15]\n\t")
        "s_add_u32 s88, s88, 9216\n\t"
        "s_cmp_eq_u32 s88, 147456\n\t"
        "s_cselect_b32 s88, 0, s88\n\t"
        MF("v_mfma_f32_16x16x32_f16 v[48:51], v[124:127], v[132:135], v[48:51]\n\t")
        "ds_read_b128 v[132:135], v165 offset:1024\n\t"
        "s_waitcnt lgkmcnt(10)\n\t"
        MF("v_mfma_f32_16x16x32_f16 v[16:19], v[120:123], v[136:139], v[16:19]\n\t")
        "s_add_u32 s96, s96, s95\n\t"
        "s_addc_u32 s97, s97, 0\n\t"
        MF("v_mfma_f32_16x16x32_f16 v[52:55], v[124:127], v[136:139], v[52:55]\n\t")
        "ds_read_b128 v[136:139], v165 offset:2048\n\t"
        "ds_read_b32 v164, %[vctl]\n\t"
        "s_waitcnt lgkmcnt(10)\n\t"
        MF("v_mfma_f32_16x16x32_f16 v[20:23], v[120:123], v[140:143], v[20:23]\n\t")
        MF("v_mfma_f32_16x16x32_f16 v[56:59], v[124:127], v[140:143], v[56:59]\n\t")
        "ds_read_b128 v[140:143], v165 offset:3072\n\t"
        "s_waitcnt lgkmcnt(10)\n\t"
        MF("v_mfma_f32_16x16x32_f16 v[24:27], v[120:123], v[144:147], v[24:27]\n\t")
        "s_add_u32 s89, s89, 1\n\t"
        MF("v_mfma_f32_16x16x32_f16 v[60:63], v[124:127], v[144:147], v[60:63]\n\t")
        "ds_read_b128 v[144:147], v165 offset:4096\n\t"
        "s_waitcnt lgkmcnt(10)\n\t"
        MF("v_mfma_f32_16x16x32_f16 v[28:31], v[120:123], v[148:151], v[28:31]\n\t")
        MF("v_mfma_f32_16x16x32_f16 v[64:67], v[124:127], v[148:151], v[64:67]\n\t")
        "ds_read_b128 v[148:151], v165 offset:5120\n\t"
        "s_waitcnt lgkmcnt(10)\n\t"
        MF("v_mfma_f32_16x16x32_f16 v[32:35], v[120:123], v[152:155], v[32:35]\n\t")
        MF("v_mfma_f32_16x16x32_f16 v[68:71], v[124:127], v[152:155], v[68:71]\n\t")
        "ds_read_b128 v[152:155], v165 offset:6144\n\t"
        "s_waitcnt lgkmcnt(10)\n\t"
        "s_waitcnt vmcnt(2)\n\t"
        MF("v_mfma_f32_16x16x32_f16 v[36:39], v[120:123], v[156:159], v[36:39]\n\t")
        "v_bitop3_b32 v166, v80, v84, %[tag] bitop3:0x7e\n\t"
        MF("v_mfma_f32_16x16x32_f16 v[72:75], v[124:127], v[156:159], v[72:75]\n\t")
        "v_and_b32 v166, 1, v166\n\t"
        "ds_read_b128 v[156:159], v165 offset:7168\n\t"
        "s_waitcnt lgkmcnt(5)\n\t"
        MF("v_mfma_f32_16x16x32_f16 v[40:43], v[160:163], v[120:123], v[40:43]\n\t")
        "v_cmp_gt_u32_e64 s[86:87], s89, v164\n\t"
        TG("v_cmp_ne_u32_e64 s[84:85], 0, v166\n\t")
        NTG("s_mov_b64 s[84:85], 0\n\t")
        MF("v_mfma_f32_16x16x32_f16 v[76:79], v[160:163], v[124:127], v[76:79]\n\t")
        "ds_read_b128 v[160:163], v165 offset:8192\n\t"
        "s_mov_b64 exec, 1\n\t"
        "ds_add_u32 %[vctl], v167\n\t"
        "s_mov_b64 exec, -1\n\t"
        "s_or_b64 s[90:91], s[84:85], s[86:87]\n\t"
        "s_cbranch_scc1 SLOWP0_%=\n\t"
        "s_branch PREP0_%=\n\t"
        "SLOWP0_%=:\n\t"
        SC1("s_add_u32 %[nfix], %[nfix], 1\n\t")
        "s_cmp_eq_u64 s[84:85], 0\n\t"
        "s_cbranch_scc1 NOFIXP0_%=\n\t"
        "s_add_u32 %[nfix], %[nfix], 1\n\t"
        "v_mov_b32 v166, %[bad]\n\t"
        "ds_read_b32 v164, v166\n\t"
        "s_sub_u32 s92, s89, 2\n\t"
        "s_cmp_lt_u32 s92, 17\n\t"
        "s_cselect_b64 s[86:87], %[xloc], %[xfar]\n\t"
        "s_cselect_b32 s93, 0, 17\n\t"
        "s_sub_u32 s93, s92, s93\n\t"
        "s_lshl_b32 s93, s93, 14\n\t"
        "s_cmp_lt_u32 s92, 112\n\t"
        "s_cselect_b64 s[86:87], s[86:87], %[xt0]\n\t"
        "s_cselect_b32 s92, s93, 0\n\t"
        "s_add_u32 s86, s86, s92\n\t"
        "s_addc_u32 s87, s87, 0\n\t"
        "s_memrealtime s[90:91]\n\t"
        "s_waitcnt lgkmcnt(0)\n\t"
        "v_cmp_ne_u32_e64 s[92:93], 0, v164\n\t"
        "s_cmp_lg_u64 s[92:93], 0\n\t"
        "s_cbranch_scc1 NOFIXP0_%=\n\t"
        "FIXLP0_%=:\n\t"
        "global_load_dwordx4 v[80:83], %[l16], s[86:87] offset:0 sc0 sc1\n\t"
        "global_load_dwordx4 v[84:87], %[l16], s[86:87] offset:1024 sc0 sc1\n\t"
        "s_waitcnt vmcnt(0)\n\t"
        "v_bitop3_b32 v166, v80, v84, %[tag] bitop3:0x7e\n\t"
        "v_and_b32 v166, 1, v166\n\t"
        "v_cmp_ne_u32_e64 s[92:93], 0, v166\n\t"
        "s_cmp_eq_u64 s[92:93], 0\n\t"
        "s_cbranch_scc1 NOFIXP0_%=\n\t"
        "s_sleep 1\n\t"
        "s_memrealtime s[92:93]\n\t"
        "s_waitcnt lgkmcnt(0)\n\t"
        "s_sub_u32 s92, s92, s90\n\t"
        "s_cmp_lt_u32 s92, " KDL "\n\t"
        "s_cbranch_scc1 FIXLP0_%=\n\t"
        "v_mov_b32 v166, %[bad]\n\t"
        "ds_write_b32 v166, v166\n\t"
        "NOFIXP0_%=:\n\t"
        "s_mov_b32 s94, 0\n\t"
        "SPINP0_%=:\n\t"
        "ds_read_b32 v164, %[vctl]\n\t"
        "s_waitcnt lgkmcnt(0)\n\t"
        "v_cmp_gt_u32_e64 s[86:87], s89, v164\n\t"
        "s_cmp_eq_u64 s[86:87], 0\n\t"
        "s_cbranch_scc1 PREP0_%=\n\t"
        SC3("s_add_u32 %[nfix], %[nfix], 1\n\t")
        "s_sleep 1\n\t"
        "s_add_u32 s94, s94, 1\n\t"
        "s_and_b32 s92, s94, 0x3ff\n\t"
        "s_cbranch_scc1 SPINP0_%=\n\t"
        "s_memrealtime s[92:93]\n\t"
        "s_waitcnt lgkmcnt(0)\n\t"
        "s_cmp_eq_u32 s94, 0x400\n\t"
        "s_cselect_b32 s90, s92, s90\n\t"
        "s_sub_u32 s92, s92, s90\n\t"
        "s_cmp_lt_u32 s92, " KDL3 "\n\t"
        "s_cbranch_scc1 SPINP0_%=\n\t"
        "v_mov_b32 v166, %[bad]\n\t"
        "ds_write_b32 v166, v166\n\t"
        "s_branch PREP0_%=\n\t"
        "SLOWP1_%=:\n\t"
        SC1("s_add_u32 %[nfix], %[nfix], 1\n\t")
        "s_cmp_eq_u64 s[84:85], 0\n\t"
        "s_cbranch_scc1 NOFIXP1_%=\n\t"
        "s_add_u32 %[nfix], %[nfix], 1\n\t"
        "v_mov_b32 v166, %[bad]\n\t"
        "ds_read_b32 v164, v166\n\t"
        "s_sub_u32 s92, s89, 2\n\t"
        "s_cmp_lt_u32 s92, 17\n\t"
        "s_cselect_b64 s[86:87], %[xloc], %[xfar]\n\t"
        "s_cselect_b32 s93, 0, 17\n\t"
        "s_sub_u32 s93, s92, s93\n\t"
        "s_lshl_b32 s93, s93, 14\n\t"
        "s_cmp_lt_u32 s92, 112\n\t"
        "s_cselect_b64 s[86:87], s[86:87], %[xt0]\n\t"
        "s_cselect_b32 s92, s93, 0\n\t"
        "s_add_u32 s86, s86, s92\n\t"
        "s_addc_u32 s87, s87, 0\n\t"
        "s_memrealtime s[90:91]\n\t"
        "s_waitcnt lgkmcnt(0)\n\t"
        "v_cmp_ne_u32_e64 s[92:93], 0, v164\n\t"
        "s_cmp_lg_u64 s[92:93], 0\n\t"
        "s_cbranch_scc1 NOFIXP1_%=\n\t"
        "FIXLP1_%=:\n\t"
        "global_load_dwordx4 v[88:91], %[l16], s[86:87] offset:0 sc0 sc1\n\t"
        "global_load_dwordx4 v[92:95], %[l16], s[86:87] offset:1024 sc0 sc1\n\t"
        "s_waitcnt vmcnt(0)\n\t"
        "v_bitop3_b32 v166, v88, v92, %[tag] bitop3:0x7e\n\t"
        "v_and_b32 v166, 1, v166\n\t"
        "v_cmp_ne_u32_e64 s[92:93], 0, v166\n\t"
        "s_cmp_eq_u64 s[92:93], 0\n\t"
        "s_cbranch_scc1 NOFIXP1_%=\n\t"
        "s_sleep 1\n\t"
        "s_memrealtime s[92:93]\n\t"
        "s_waitcnt lgkmcnt(0)\n\t"
        "s_sub_u32 s92, s92, s90\n\t"
        "s_cmp_lt_u32 s92, " KDL "\n\t"
        "s_cbranch_scc1 FIXLP1_%=\n\t"
        "v_mov_b32 v166, %[bad]\n\t"
        "ds_write_b32 v166, v166\n\t"
        "NOFIXP1_%=:\n\t"
        "s_mov_b32 s94, 0\n\t"
        "SPINP1_%=:\n\t"
        "ds_read_b32 v164, %[vctl]\n\t"
        "s_waitcnt lgkmcnt(0)\n\t"
        "v_cmp_gt_u32_e64 s[86:87], s89, v164\n\t"
        "s_cmp_eq_u64 s[86:87], 0\n\t"
        "s_cbranch_scc1 PREP1_%=\n\t"
        SC3("s_add_u32 %[nfix], %[nfix], 1\n\t")
        "s_sleep 1\n\t"
        "s_add_u32 s94, s94, 1\n\t"
        "s_and_b32 s92, s94, 0x3ff\n\t"
        "s_cbranch_scc1 SPINP1_%=\n\t"
        "s_memrealtime s[92:93]\n\t"
        "s_waitcnt lgkmcnt(0)\n\t"
        "s_cmp_eq_u32 s94, 0x400\n\t"
        "s_cselect_b32 s90, s92, s90\n\t"
        "s_sub_u32 s92, s92, s90\n\t"
        "s_cmp_lt_u32 s92, " KDL3 "\n\t"
        "s_cbranch_scc1 SPINP1_%=\n\t"
        "v_mov_b32 v166, %[bad]\n\t"
        "ds_write_b32 v166, v166\n\t"
        "s_branch PREP1_%=\n\t"
        "SLOWP2_%=:\n\t"
        SC1("s_add_u32 %[nfix], %[nfix], 1\n\t")
        "s_cmp_eq_u64 s[84:85], 0\n\t"
        "s_cbranch_scc1 NOFIXP2_%=\n\t"
        "s_add_u32 %[nfix], %[nfix], 1\n\t"
        "v_mov_b32 v166, %[bad]\n\t"
        "ds_read_b32 v164, v166\n\t"
        "s_sub_u32 s92, s89, 2\n\t"
        "s_cmp_lt_u32 s92, 17\n\t"
        "s_cselect_b64 s[86:87], %[xloc], %[xfar]\n\t"
        "s_cselect_b32 s93, 0, 17\n\t"
        "s_sub_u32 s93, s92, s93\n\t"
        "s_lshl_b32 s93, s93, 14\n\t"
        "s_cmp_lt_u32 s92, 112\n\t"
        "s_cselect_b64 s[86:87], s[86:87], %[xt0]\n\t"
        "s_cselect_b32 s92, s93, 0\n\t"
        "s_add_u32 s86, s86, s92\n\t"
        "s_addc_u32 s87, s87, 0\n\t"
        "s_memrealtime s[90:91]\n\t"
        "s_waitcnt lgkmcnt(0)\n\t"
        "v_cmp_ne_u32_e64 s[92:93], 0, v164\n\t"
        "s_cmp_lg_u64 s[92:93], 0\n\t"
        "s_cbranch_scc1 NOFIXP2_%=\n\t"
        "FIXLP2_%=:\n\t"
        "global_load_dwordx4 v[96:99], %[l16], s[86:87] offset:0 sc0 sc1\n\t"
        "global_load_dwordx4 v[100:103], %[l16], s[86:87] offset:1024 sc0 sc1\n\t"
        "s_waitcnt vmcnt(0)\n\t"
        "v_bitop3_b32 v166, v96, v100, %[tag] bitop3:0x7e\n\t"
        "v_and_b32 v166, 1, v166\n\t"
        "v_cmp_ne_u32_e64 s[92:93], 0, v166\n\t"
        "s_cmp_eq_u64 s[92:93], 0\n\t"
        "s_cbranch_scc1 NOFIXP2_%=\n\t"
        "s_sleep 1\n\t"
        "s_memrealtime s[92:93]\n\t"
        "s_waitcnt lgkmcnt(0)\n\t"
        "s_sub_u32 s92, s92, s90\n\t"
        "s_cmp_lt_u32 s92, " KDL "\n\t"
        "s_cbranch_scc1 FIXLP2_%=\n\t"
        "v_mov_b32 v166, %[bad]\n\t"
        "ds_write_b32 v166, v166\n\t"
        "NOFIXP2_%=:\n\t"
        "s_mov_b32 s94, 0\n\t"
        "SPINP2_%=:\n\t"
        "ds_read_b32 v164, %[vctl]\n\t"
        "s_waitcnt lgkmcnt(0)\n\t"
        "v_cmp_gt_u32_e64 s[86:87], s89, v164\n\t"
        "s_cmp_eq_u64 s[86:87], 0\n\t"
        "s_cbranch_scc1 PREP2_%=\n\t"
        SC3("s_add_u32 %[nfix], %[nfix], 1\n\t")
        "s_sleep 1\n\t"
        "s_add_u32 s94, s94, 1\n\t"
        "s_and_b32 s92, s94, 0x3ff\n\t"
        "s_cbranch_scc1 SPINP2_%=\n\t"
        "s_memrealtime s[92:93]\n\t"
        "s_waitcnt lgkmcnt(0)\n\t"
        "s_cmp_eq_u32 s94, 0x400\n\t"
        "s_cselect_b32 s90, s92, s90\n\t"
        "s_sub_u32 s92, s92, s90\n\t"
        "s_cmp_lt_u32 s92, " KDL3 "\n\t"
        "s_cbranch_scc1 SPINP2_%=\n\t"
        "v_mov_b32 v166, %[bad]\n\t"
        "ds_write_b32 v166, v166\n\t"
        "s_branch PREP2_%=\n\t"
        "SLOWP3_%=:\n\t"
        SC1("s_add_u32 %[nfix], %[nfix], 1\n\t")
        "s_cmp_eq_u64 s[84:85], 0\n\t"
        "s_cbranch_scc1 NOFIXP3_%=\n\t"
        "s_add_u32 %[nfix], %[nfix], 1\n\t"
        "v_mov_b32 v166, %[bad]\n\t"
        "ds_read_b32 v164, v166\n\t"
        "s_sub_u32 s92, s89, 2\n\t"
        "s_cmp_lt_u32 s92, 17\n\t"
        "s_cselect_b64 s[86:87], %[xloc], %[xfar]\n\t"
        "s_cselect_b32 s93, 0, 17\n\t"
        "s_sub_u32 s93, s92, s93\n\t"
        "s_lshl_b32 s93, s93, 14\n\t"
        "s_cmp_lt_u32 s92, 112\n\t"
        "s_cselect_b64 s[86:87], s[86:87], %[xt0]\n\t"
        "s_cselect_b32 s92, s93, 0\n\t"
        "s_add_u32 s86, s86, s92\n\t"
        "s_addc_u32 s87, s87, 0\n\t"
        "s_memrealtime s[90:91]\n\t"
        "s_waitcnt lgkmcnt(0)\n\t"
        "v_cmp_ne_u32_e64 s[92:93], 0, v164\n\t"
        "s_cmp_lg_u64 s[92:93], 0\n\t"
        "s_cbranch_scc1 NOFIXP3_%=\n\t"
        "FIXLP3_%=:\n\t"
        "global_load_dwordx4 v[104:107], %[l16], s[86:87] offset:0 sc0 sc1\n\t"
        "global_load_dwordx4 v[108:111], %[l16], s[86:87] offset:1024 sc0 sc1\n\t"
        "s_waitcnt vmcnt(0)\n\t"
        "v_bitop3_b32 v166, v104, v108, %[tag] bitop3:0x7e\n\t"
        "v_and_b32 v166, 1, v166\n\t"
        "v_cmp_ne_u32_e64 s[92:93], 0, v166\n\t"
        "s_cmp_eq_u64 s[92:93], 0\n\t"
        "s_cbranch_scc1 NOFIXP3_%=\n\t"
        "s_sleep 1\n\t"
        "s_memrealtime s[92:93]\n\t"
        "s_waitcnt lgkmcnt(0)\n\t"
        "s_sub_u32 s92, s92, s90\n\t"
        "s_cmp_lt_u32 s92, " KDL "\n\t"
        "s_cbranch_scc1 FIXLP3_%=\n\t"
        "v_mov_b32 v166, %[bad]\n\t"
        "ds_write_b32 v166, v166\n\t"
        "NOFIXP3_%=:\n\t"
        "s_mov_b32 s94, 0\n\t"
        "SPINP3_%=:\n\t"
        "ds_read_b32 v164, %[vctl]\n\t"
        "s_waitcnt lgkmcnt(0)\n\t"
        "v_cmp_gt_u32_e64 s[86:87], s89, v164\n\t"
        "s_cmp_eq_u64 s[86:87], 0\n\t"
        "s_cbranch_scc1 PREP3_%=\n\t"
        SC3("s_add_u32 %[nfix], %[nfix], 1\n\t")
        "s_sleep 1\n\t"
        "s_add_u32 s94, s94, 1\n\t"
        "s_and_b32 s92, s94, 0x3ff\n\t"
        "s_cbranch_scc1 SPINP3_%=\n\t"
        "s_memrealtime s[92:93]\n\t"
        "s_waitcnt lgkmcnt(0)\n\t"
        "s_cmp_eq_u32 s94, 0x400\n\t"
        "s_cselect_b32 s90, s92, s90\n\t"
        "s_sub_u32 s92, s92, s90\n\t"
        "s_cmp_lt_u32 s92, " KDL3 "\n\t"
        "s_cbranch_scc1 SPINP3_%=\n\t"
        "v_mov_b32 v166, %[bad]\n\t"
        "ds_write_b32 v166, v166\n\t"
        "s_branch PREP3_%=\n\t"
        "SLOWP4_%=:\n\t"
        SC1("s_add_u32 %[nfix], %[nfix], 1\n\t")
        "s_cmp_eq_u64 s[84:85], 0\n\t"
        "s_cbranch_scc1 NOFIXP4_%=\n\t"
        "s_add_u32 %[nfix], %[nfix], 1\n\t"
        "v_mov_b32 v166, %[bad]\n\t"
        "ds_read_b32 v164, v166\n\t"
        "s_sub_u32 s92, s89, 2\n\t"
        "s_cmp_lt_u32 s92, 17\n\t"
        "s_cselect_b64 s[86:87], %[xloc], %[xfar]\n\t"
        "s_cselect_b32 s93, 0, 17\n\t"
        "s_sub_u32 s93, s92, s93\n\t"
        "s_lshl_b32 s93, s93, 14\n\t"
        "s_cmp_lt_u32 s92, 112\n\t"
        "s_cselect_b64 s[86:87], s[86:87], %[xt0]\n\t"
        "s_cselect_b32 s92, s93, 0\n\t"
        "s_add_u32 s86, s86, s92\n\t"
        "s_addc_u32 s87, s87, 0\n\t"
        "s_memrealtime s[90:91]\n\t"
        "s_waitcnt lgkmcnt(0)\n\t"
        "v_cmp_ne_u32_e64 s[92:93], 0, v164\n\t"
        "s_cmp_lg_u64 s[92:93], 0\n\t"
        "s_cbranch_scc1 NOFIXP4_%=\n\t"
        "FIXLP4_%=:\n\t"
        "global_load_dwordx4 v[112:115], %[l16], s[86:87] offset:0 sc0 sc1\n\t"
        "global_load_dwordx4 v[116:119], %[l16], s[86:87] offset:1024 sc0 sc1\n\t"
        "s_waitcnt vmcnt(0)\n\t"
        "v_bitop3_b32 v166, v112, v116, %[tag] bitop3:0x7e\n\t"
        "v_and_b32 v166, 1, v166\n\t"
        "v_cmp_ne_u32_e64 s[92:93], 0, v166\n\t"
        "s_cmp_eq_u64 s[92:93], 0\n\t"
        "s_cbranch_scc1 NOFIXP4_%=\n\t"
        "s_sleep 1\n\t"
        "s_memrealtime s[92:93]\n\t"
        "s_waitcnt lgkmcnt(0)\n\t"
        "s_sub_u32 s92, s92, s90\n\t"
        "s_cmp_lt_u32 s92, " KDL "\n\t"
        "s_cbranch_scc1 FIXLP4_%=\n\t"
        "v_mov_b32 v166, %[bad]\n\t"
        "ds_write_b32 v166, v166\n\t"
        "NOFIXP4_%=:\n\t"
        "s_mov_b32 s94, 0\n\t"
        "SPINP4_%=:\n\t"
        "ds_read_b32 v164, %[vctl]\n\t"
        "s_waitcnt lgkmcnt(0)\n\t"
        "v_cmp_gt_u32_e64 s[86:87], s89, v164\n\t"
        "s_cmp_eq_u64 s[86:87], 0\n\t"
        "s_cbranch_scc1 PREP4_%=\n\t"
        SC3("s_add_u32 %[nfix], %[nfix], 1\n\t")
        "s_sleep 1\n\t"
        "s_add_u32 s94, s94, 1\n\t"
        "s_and_b32 s92, s94, 0x3ff\n\t"
        "s_cbranch_scc1 SPINP4_%=\n\t"
        "s_memrealtime s[92:93]\n\t"
        "s_waitcnt lgkmcnt(0)\n\t"
        "s_cmp_eq_u32 s94, 0x400\n\t"
        "s_cselect_b32 s90, s92, s90\n\t"
        "s_sub_u32 s92, s92, s90\n\t"
        "s_cmp_lt_u32 s92, " KDL3 "\n\t"
        "s_cbranch_scc1 SPINP4_%=\n\t"
        "v_mov_b32 v166, %[bad]\n\t"
        "ds_write_b32 v166, v166\n\t"
        "s_branch PREP4_%=\n\t"
        "SLOWP5_%=:\n\t"
        SC1("s_add_u32 %[nfix], %[nfix], 1\n\t")
        "s_cmp_eq_u64 s[84:85], 0\n\t"
        "s_cbranch_scc1 NOFIXP5_%=\n\t"
        "s_add_u32 %[nfix], %[nfix], 1\n\t"
        "v_mov_b32 v166, %[bad]\n\t"
        "ds_read_b32 v164, v166\n\t"
        "s_sub_u32 s92, s89, 2\n\t"
        "s_cmp_lt_u32 s92, 17\n\t"
        "s_cselect_b64 s[86:87], %[xloc], %[xfar]\n\t"
        "s_cselect_b32 s93, 0, 17\n\t"
        "s_sub_u32 s93, s92, s93\n\t"
        "s_lshl_b32 s93, s93, 14\n\t"
        "s_cmp_lt_u32 s92, 112\n\t"
        "s_cselect_b64 s[86:87], s[86:87], %[xt0]\n\t"
        "s_cselect_b32 s92, s93, 0\n\t"
        "s_add_u32 s86, s86, s92\n\t"
        "s_addc_u32 s87, s87, 0\n\t"
        "s_memrealtime s[90:91]\n\t"
        "s_waitcnt lgkmcnt(0)\n\t"
        "v_cmp_ne_u32_e64 s[92:93], 0, v164\n\t"
        "s_cmp_lg_u64 s[92:93], 0\n\t"
        "s_cbranch_scc1 NOFIXP5_%=\n\t"
        "FIXLP5_%=:\n\t"
        "global_load_dwordx4 v[120:123], %[l16], s[86:87] offset:0 sc0 sc1\n\t"
        "global_load_dwordx4 v[124:127], %[l16], s[86:87] offset:1024 sc0 sc1\n\t"
        "s_waitcnt vmcnt(0)\n\t"
        "v_bitop3_b32 v166, v120, v124, %[tag] bitop3:0x7e\n\t"
        "v_and_b32 v166, 1, v166\n\t"
        "v_cmp_ne_u32_e64 s[92:93], 0, v166\n\t"
        "s_cmp_eq_u64 s[92:93], 0\n\t"
        "s_cbranch_scc1 NOFIXP5_%=\n\t"
        "s_sleep 1\n\t"
        "s_memrealtime s[92:93]\n\t"
        "s_waitcnt lgkmcnt(0)\n\t"
        "s_sub_u32 s92, s92, s90\n\t"
        "s_cmp_lt_u32 s92, " KDL "\n\t"
        "s_cbranch_scc1 FIXLP5_%=\n\t"
        "v_mov_b32 v166, %[bad]\n\t"
        "ds_write_b32 v166, v166\n\t"
        "NOFIXP5_%=:\n\t"
        "s_mov_b32 s94, 0\n\t"
        "SPINP5_%=:\n\t"
        "ds_read_b32 v164, %[vctl]\n\t"
        "s_waitcnt lgkmcnt(0)\n\t"
        "v_cmp_gt_u32_e64 s[86:87], s89, v164\n\t"
        "s_cmp_eq_u64 s[86:87], 0\n\t"
        "s_cbranch_scc1 PREP5_%=\n\t"
        SC3("s_add_u32 %[nfix], %[nfix], 1\n\t")
        "s_sleep 1\n\t"
        "s_add_u32 s94, s94, 1\n\t"
        "s_and_b32 s92, s94, 0x3ff\n\t"
        "s_cbranch_scc1 SPINP5_%=\n\t"
        "s_memrealtime s[92:93]\n\t"
        "s_waitcnt lgkmcnt(0)\n\t"
        "s_cmp_eq_u32 s94, 0x400\n\t"
        "s_cselect_b32 s90, s92, s90\n\t"
        "s_sub_u32 s92, s92, s90\n\t"
        "s_cmp_lt_u32 s92, " KDL3 "\n\t"
        "s_cbranch_scc1 SPINP5_%=\n\t"
        "v_mov_b32 v166, %[bad]\n\t"
        "ds_write_b32 v166, v166\n\t"
        "s_branch PREP5_%=\n\t"
        "SLOWF0_%=:\n\t"
        SC1("s_add_u32 %[nfix], %[nfix], 1\n\t")
        "s_cmp_eq_u64 s[84:85], 0\n\t"
        "s_cbranch_scc1 NOFIXF0_%=\n\t"
        "s_add_u32 %[nfix], %[nfix], 1\n\t"
        "v_mov_b32 v166, %[bad]\n\t"
        "ds_read_b32 v164, v166\n\t"
        "s_sub_u32 s92, s89, 2\n\t"
        "s_cmp_lt_u32 s92, 17\n\t"
        "s_cselect_b64 s[86:87], %[xloc], %[xfar]\n\t"
        "s_cselect_b32 s93, 0, 17\n\t"
        "s_sub_u32 s93, s92, s93\n\t"
        "s_lshl_b32 s93, s93, 14\n\t"
        "s_cmp_lt_u32 s92, 112\n\t"
        "s_cselect_b64 s[86:87], s[86:87], %[xt0]\n\t"
        "s_cselect_b32 s92, s93, 0\n\t"
        "s_add_u32 s86, s86, s92\n\t"
        "s_addc_u32 s87, s87, 0\n\t"
        "s_memrealtime s[90:91]\n\t"
        "s_waitcnt lgkmcnt(0)\n\t"
        "v_cmp_ne_u32_e64 s[92:93], 0, v164\n\t"
        "s_cmp_lg_u64 s[92:93], 0\n\t"
        "s_cbranch_scc1 NOFIXF0_%=\n\t"
        "FIXLF0_%=:\n\t"
        "global_load_dwordx4 v[80:83], %[l16], s[86:87] offset:0 sc0 sc1\n\t"
        "global_load_dwordx4 v[84:87], %[l16], s[86:87] offset:1024 sc0 sc1\n\t"
        "s_waitcnt vmcnt(0)\n\t"
        "v_bitop3_b32 v166, v80, v84, %[tag] bitop3:0x7e\n\t"
        "v_and_b32 v166, 1, v166\n\t"
        "v_cmp_ne_u32_e64 s[92:93], 0, v166\n\t"
        "s_cmp_eq_u64 s[92:93], 0\n\t"
        "s_cbranch_scc1 NOFIXF0_%=\n\t"
        "s_sleep 1\n\t"
        "s_memrealtime s[92:93]\n\t"
        "s_waitcnt lgkmcnt(0)\n\t"
        "s_sub_u32 s92, s92, s90\n\t"
        "s_cmp_lt_u32 s92, " KDL "\n\t"
        "s_cbranch_scc1 FIXLF0_%=\n\t"
        "v_mov_b32 v166, %[bad]\n\t"
        "ds_write_b32 v166, v166\n\t"
        "NOFIXF0_%=:\n\t"
        "s_mov_b32 s94, 0\n\t"
        "SPINF0_%=:\n\t"
        "ds_read_b32 v164, %[vctl]\n\t"
        "s_waitcnt lgkmcnt(0)\n\t"
        "v_cmp_gt_u32_e64 s[86:87], s89, v164\n\t"
        "s_cmp_eq_u64 s[86:87], 0\n\t"
        "s_cbranch_scc1 PREF0_%=\n\t"
        SC3("s_add_u32 %[nfix], %[nfix], 1\n\t")
        "s_sleep 1\n\t"
        "s_add_u32 s94, s94, 1\n\t"
        "s_and_b32 s92, s94, 0x3ff\n\t"
        "s_cbranch_scc1 SPINF0_%=\n\t"
        "s_memrealtime s[92:93]\n\t"
        "s_waitcnt lgkmcnt(0)\n\t"
        "s_cmp_eq_u32 s94, 0x400\n\t"
        "s_cselect_b32 s90, s92, s90\n\t"
        "s_sub_u32 s92, s92, s90\n\t"
        "s_cmp_lt_u32 s92, " KDL3 "\n\t"
        "s_cbranch_scc1 SPINF0_%=\n\t"
        "v_mov_b32 v166, %[bad]\n\t"
        "ds_write_b32 v166, v166\n\t"
        "s_branch PREF0_%=\n\t"
        "SLOWF1_%=:\n\t"
        SC1("s_add_u32 %[nfix], %[nfix], 1\n\t")
        "s_cmp_eq_u64 s[84:85], 0\n\t"
        "s_cbranch_scc1 NOFIXF1_%=\n\t"
        "s_add_u32 %[nfix], %[nfix], 1\n\t"
        "v_mov_b32 v166, %[bad]\n\t"
        "ds_read_b32 v164, v166\n\t"
        "s_sub_u32 s92, s89, 2\n\t"
        "s_cmp_lt_u32 s92, 17\n\t"
        "s_cselect_b64 s[86:87], %[xloc], %[xfar]\n\t"
        "s_cselect_b32 s93, 0, 17\n\t"
        "s_sub_u32 s93, s92, s93\n\t"
        "s_lshl_b32 s93, s93, 14\n\t"
        "s_cmp_lt_u32 s92, 112\n\t"
        "s_cselect_b64 s[86:87], s[86:87], %[xt0]\n\t"
        "s_cselect_b32 s92, s93, 0\n\t"
        "s_add_u32 s86, s86, s92\n\t"
        "s_addc_u32 s87, s87, 0\n\t"
        "s_memrealtime s[90:91]\n\t"
        "s_waitcnt lgkmcnt(0)\n\t"
        "v_cmp_ne_u32_e64 s[92:93], 0, v164\n\t"
        "s_cmp_lg_u64 s[92:93], 0\n\t"
        "s_cbranch_scc1 NOFIXF1_%=\n\t"
        "FIXLF1_%=:\n\t"
        "global_load_dwordx4 v[88:91], %[l16], s[86:87] offset:0 sc0 sc1\n\t"
        "global_load_dwordx4 v[92:95], %[l16], s[86:87] offset:1024 sc0 sc1\n\t"
        "s_waitcnt vmcnt(0)\n\t"
        "v_bitop3_b32 v166, v88, v92, %[tag] bitop3:0x7e\n\t"
        "v_and_b32 v166, 1, v166\n\t"
        "v_cmp_ne_u32_e64 s[92:93], 0, v166\n\t"
        "s_cmp_eq_u64 s[92:93], 0\n\t"
        "s_cbranch_scc1 NOFIXF1_%=\n\t"
        "s_sleep 1\n\t"
        "s_memrealtime s[92:93]\n\t"
        "s_waitcnt lgkmcnt(0)\n\t"
        "s_sub_u32 s92, s92, s90\n\t"
        "s_cmp_lt_u32 s92, " KDL "\n\t"
        "s_cbranch_scc1 FIXLF1_%=\n\t"
        "v_mov_b32 v166, %[bad]\n\t"
        "ds_write_b32 v166, v166\n\t"
        "NOFIXF1_%=:\n\t"
        "s_mov_b32 s94, 0\n\t"
        "SPINF1_%=:\n\t"
        "ds_read_b32 v164, %[vctl]\n\t"
        "s_waitcnt lgkmcnt(0)\n\t"
        "v_cmp_gt_u32_e64 s[86:87], s89, v164\n\t"
        "s_cmp_eq_u64 s[86:87], 0\n\t"
        "s_cbranch_scc1 PREF1_%=\n\t"
        SC3("s_add_u32 %[nfix], %[nfix], 1\n\t")
        "s_sleep 1\n\t"
        "s_add_u32 s94, s94, 1\n\t"
        "s_and_b32 s92, s94, 0x3ff\n\t"
        "s_cbranch_scc1 SPINF1_%=\n\t"
        "s_memrealtime s[92:93]\n\t"
        "s_waitcnt lgkmcnt(0)\n\t"
        "s_cmp_eq_u32 s94, 0x400\n\t"
        "s_cselect_b32 s90, s92, s90\n\t"
        "s_sub_u32 s92, s92, s90\n\t"
        "s_cmp_lt_u32 s92, " KDL3 "\n\t"
        "s_cbranch_scc1 SPINF1_%=\n\t"
        "v_mov_b32 v166, %[bad]\n\t"
        "ds_write_b32 v166, v166\n\t"
        "s_branch PREF1_%=\n\t"
        "SLOWF2_%=:\n\t"
        SC1("s_add_u32 %[nfix], %[nfix], 1\n\t")
        "s_cmp_eq_u64 s[84:85], 0\n\t"
        "s_cbranch_scc1 NOFIXF2_%=\n\t"
        "s_add_u32 %[nfix], %[nfix], 1\n\t"
        "v_mov_b32 v166, %[bad]\n\t"
        "ds_read_b32 v164, v166\n\t"
        "s_sub_u32 s92, s89, 2\n\t"
        "s_cmp_lt_u32 s92, 17\n\t"
        "s_cselect_b64 s[86:87], %[xloc], %[xfar]\n\t"
        "s_cselect_b32 s93, 0, 17\n\t"
        "s_sub_u32 s93, s92, s93\n\t"
        "s_lshl_b32 s93, s93, 14\n\t"
        "s_cmp_lt_u32 s92, 112\n\t"
        "s_cselect_b64 s[86:87], s[86:87], %[xt0]\n\t"
        "s_cselect_b32 s92, s93, 0\n\t"
        "s_add_u32 s86, s86, s92\n\t"
        "s_addc_u32 s87, s87, 0\n\t"
        "s_memrealtime s[90:91]\n\t"
        "s_waitcnt lgkmcnt(0)\n\t"
        "v_cmp_ne_u32_e64 s[92:93], 0, v164\n\t"
        "s_cmp_lg_u64 s[92:93], 0\n\t"
        "s_cbranch_scc1 NOFIXF2_%=\n\t"
        "FIXLF2_%=:\n\t"
        "global_load_dwordx4 v[96:99], %[l16], s[86:87] offset:0 sc0 sc1\n\t"
        "global_load_dwordx4 v[100:103], %[l16], s[86:87] offset:1024 sc0 sc1\n\t"
        "s_waitcnt vmcnt(0)\n\t"
        "v_bitop3_b32 v166, v96, v100, %[tag] bitop3:0x7e\n\t"
        "v_and_b32 v166, 1, v166\n\t"
        "v_cmp_ne_u32_e64 s[92:93], 0, v166\n\t"
        "s_cmp_eq_u64 s[92:93], 0\n\t"
        "s_cbranch_scc1 NOFIXF2_%=\n\t"
        "s_sleep 1\n\t"
        "s_memrealtime s[92:93]\n\t"
        "s_waitcnt lgkmcnt(0)\n\t"
        "s_sub_u32 s92, s92, s90\n\t"
        "s_cmp_lt_u32 s92, " KDL "\n\t"
        "s_cbranch_scc1 FIXLF2_%=\n\t"
        "v_mov_b32 v166, %[bad]\n\t"
        "ds_write_b32 v166, v166\n\t"
        "NOFIXF2_%=:\n\t"
        "s_mov_b32 s94, 0\n\t"
        "SPINF2_%=:\n\t"
        "ds_read_b32 v164, %[vctl]\n\t"
        "s_waitcnt lgkmcnt(0)\n\t"
        "v_cmp_gt_u32_e64 s[86:87], s89, v164\n\t"
        "s_cmp_eq_u64 s[86:87], 0\n\t"
        "s_cbranch_scc1 PREF2_%=\n\t"
        SC3("s_add_u32 %[nfix], %[nfix], 1\n\t")
        "s_sleep 1\n\t"
        "s_add_u32 s94, s94, 1\n\t"
        "s_and_b32 s92, s94, 0x3ff\n\t"
        "s_cbranch_scc1 SPINF2_%=\n\t"
        "s_memrealtime s[92:93]\n\t"
        "s_waitcnt lgkmcnt(0)\n\t"
        "s_cmp_eq_u32 s94, 0x400\n\t"
        "s_cselect_b32 s90, s92, s90\n\t"
        "s_sub_u32 s92, s92, s90\n\t"
        "s_cmp_lt_u32 s92, " KDL3 "\n\t"
        "s_cbranch_scc1 SPINF2_%=\n\t"
        "v_mov_b32 v166, %[bad]\n\t"
        "ds_write_b32 v166, v166\n\t"
        "s_branch PREF2_%=\n\t"
        "SLOWF3_%=:\n\t"
        SC1("s_add_u32 %[nfix], %[nfix], 1\n\t")
        "s_cmp_eq_u64 s[84:85], 0\n\t"
        "s_cbranch_scc1 NOFIXF3_%=\n\t"
        "s_add_u32 %[nfix], %[nfix], 1\n\t"
        "v_mov_b32 v166, %[bad]\n\t"
        "ds_read_b32 v164, v166\n\t"
        "s_sub_u32 s92, s89, 2\n\t"
        "s_cmp_lt_u32 s92, 17\n\t"
        "s_cselect_b64 s[86:87], %[xloc], %[xfar]\n\t"
        "s_cselect_b32 s93, 0, 17\n\t"
        "s_sub_u32 s93, s92, s93\n\t"
        "s_lshl_b32 s93, s93, 14\n\t"
        "s_cmp_lt_u32 s92, 112\n\t"
        "s_cselect_b64 s[86:87], s[86:87], %[xt0]\n\t"
        "s_cselect_b32 s92, s93, 0\n\t"
        "s_add_u32 s86, s86, s92\n\t"
        "s_addc_u32 s87, s87, 0\n\t"
        "s_memrealtime s[90:91]\n\t"
        "s_waitcnt lgkmcnt(0)\n\t"
        "v_cmp_ne_u32_e64 s[92:93], 0, v164\n\t"
        "s_cmp_lg_u64 s[92:93], 0\n\t"
        "s_cbranch_scc1 NOFIXF3_%=\n\t"
        "FIXLF3_%=:\n\t"
        "global_load_dwordx4 v[104:107], %[l16], s[86:87] offset:0 sc0 sc1\n\t"
        "global_load_dwordx4 v[108:111], %[l16], s[86:87] offset:1024 sc0 sc1\n\t"
        "s_waitcnt vmcnt(0)\n\t"
        "v_bitop3_b32 v166, v104, v108, %[tag] bitop3:0x7e\n\t"
        "v_and_b32 v166, 1, v166\n\t"
        "v_cmp_ne_u32_e64 s[92:93], 0, v166\n\t"
        "s_cmp_eq_u64 s[92:93], 0\n\t"
        "s_cbranch_scc1 NOFIXF3_%=\n\t"
        "s_sleep 1\n\t"
        "s_memrealtime s[92:93]\n\t"
        "s_waitcnt lgkmcnt(0)\n\t"
        "s_sub_u32 s92, s92, s90\n\t"
        "s_cmp_lt_u32 s92, " KDL "\n\t"
        "s_cbranch_scc1 FIXLF3_%=\n\t"
        "v_mov_b32 v166, %[bad]\n\t"
        "ds_write_b32 v166, v166\n\t"
        "NOFIXF3_%=:\n\t"
        "s_mov_b32 s94, 0\n\t"
        "SPINF3_%=:\n\t"
        "ds_read_b32 v164, %[vctl]\n\t"
        "s_waitcnt lgkmcnt(0)\n\t"
        "v_cmp_gt_u32_e64 s[86:87], s89, v164\n\t"
        "s_cmp_eq_u64 s[86:87], 0\n\t"
        "s_cbranch_scc1 PREF3_%=\n\t"
        SC3("s_add_u32 %[nfix], %[nfix], 1\n\t")
        "s_sleep 1\n\t"
        "s_add_u32 s94, s94, 1\n\t"
        "s_and_b32 s92, s94, 0x3ff\n\t"
        "s_cbranch_scc1 SPINF3_%=\n\t"
        "s_memrealtime s[92:93]\n\t"
        "s_waitcnt lgkmcnt(0)\n\t"
        "s_cmp_eq_u32 s94, 0x400\n\t"
        "s_cselect_b32 s90, s92, s90\n\t"
        "s_sub_u32 s92, s92, s90\n\t"
        "s_cmp_lt_u32 s92, " KDL3 "\n\t"
        "s_cbranch_scc1 SPINF3_%=\n\t"
        "v_mov_b32 v166, %[bad]\n\t"
        "ds_write_b32 v166, v166\n\t"
        "s_branch PREF3_%=\n\t"
        "SLOWF4_%=:\n\t"
        SC1("s_add_u32 %[nfix], %[nfix], 1\n\t")
        "s_cmp_eq_u64 s[84:85], 0\n\t"
        "s_cbranch_scc1 NOFIXF4_%=\n\t"
        "s_add_u32 %[nfix], %[nfix], 1\n\t"
        "v_mov_b32 v166, %[bad]\n\t"
        "ds_read_b32 v164, v166\n\t"
        "s_sub_u32 s92, s89, 2\n\t"
        "s_cmp_lt_u32 s92, 17\n\t"
        "s_cselect_b64 s[86:87], %[xloc], %[xfar]\n\t"
        "s_cselect_b32 s93, 0, 17\n\t"
        "s_sub_u32 s93, s92, s93\n\t"
        "s_lshl_b32 s93, s93, 14\n\t"
        "s_cmp_lt_u32 s92, 112\n\t"
        "s_cselect_b64 s[86:87], s[86:87], %[xt0]\n\t"
        "s_cselect_b32 s92, s93, 0\n\t"
        "s_add_u32 s86, s86, s92\n\t"
        "s_addc_u32 s87, s87, 0\n\t"
        "s_memrealtime s[90:91]\n\t"
        "s_waitcnt lgkmcnt(0)\n\t"
        "v_cmp_ne_u32_e64 s[92:93], 0, v164\n\t"
        "s_cmp_lg_u64 s[92:93], 0\n\t"
        "s_cbranch_scc1 NOFIXF4_%=\n\t"
        "FIXLF4_%=:\n\t"
        "global_load_dwordx4 v[112:115], %[l16], s[86:87] offset:0 sc0 sc1\n\t"
        "global_load_dwordx4 v[116:119], %[l16], s[86:87] offset:1024 sc0 sc1\n\t"
        "s_waitcnt vmcnt(0)\n\t"
        "v_bitop3_b32 v166, v112, v116, %[tag] bitop3:0x7e\n\t"
        "v_and_b32 v166, 1, v166\n\t"
        "v_cmp_ne_u32_e64 s[92:93], 0, v166\n\t"
        "s_cmp_eq_u64 s[92:93], 0\n\t"
        "s_cbranch_scc1 NOFIXF4_%=\n\t"
        "s_sleep 1\n\t"
        "s_memrealtime s[92:93]\n\t"
        "s_waitcnt lgkmcnt(0)\n\t"
        "s_sub_u32 s92, s92, s90\n\t"
        "s_cmp_lt_u32 s92, " KDL "\n\t"
        "s_cbranch_scc1 FIXLF4_%=\n\t"
        "v_mov_b32 v166, %[bad]\n\t"
        "ds_write_b32 v166, v166\n\t"
        "NOFIXF4_%=:\n\t"
        "s_mov_b32 s94, 0\n\t"
        "SPINF4_%=:\n\t"
        "ds_read_b32 v164, %[vctl]\n\t"
        "s_waitcnt lgkmcnt(0)\n\t"
        "v_cmp_gt_u32_e64 s[86:87], s89, v164\n\t"
        "s_cmp_eq_u64 s[86:87], 0\n\t"
        "s_cbranch_scc1 PREF4_%=\n\t"
        SC3("s_add_u32 %[nfix], %[nfix], 1\n\t")
        "s_sleep 1\n\t"
        "s_add_u32 s94, s94, 1\n\t"
        "s_and_b32 s92, s94, 0x3ff\n\t"
        "s_cbranch_scc1 SPINF4_%=\n\t"
        "s_memrealtime s[92:93]\n\t"
        "s_waitcnt lgkmcnt(0)\n\t"
        "s_cmp_eq_u32 s94, 0x400\n\t"
        "s_cselect_b32 s90, s92, s90\n\t"
        "s_sub_u32 s92, s92, s90\n\t"
        "s_cmp_lt_u32 s92, " KDL3 "\n\t"
        "s_cbranch_scc1 SPINF4_%=\n\t"
        "v_mov_b32 v166, %[bad]\n\t"
        "ds_write_b32 v166, v166\n\t"
        "s_branch PREF4_%=\n\t"
        "SLOWF5_%=:\n\t"
        SC1("s_add_u32 %[nfix], %[nfix], 1\n\t")
        "s_cmp_eq_u64 s[84:85], 0\n\t"
        "s_cbranch_scc1 NOFIXF5_%=\n\t"
        "s_add_u32 %[nfix], %[nfix], 1\n\t"
        "v_mov_b32 v166, %[bad]\n\t"
        "ds_read_b32 v164, v166\n\t"
        "s_sub_u32 s92, s89, 2\n\t"
        "s_cmp_lt_u32 s92, 17\n\t"
        "s_cselect_b64 s[86:87], %[xloc], %[xfar]\n\t"
        "s_cselect_b32 s93, 0, 17\n\t"
        "s_sub_u32 s93, s92, s93\n\t"
        "s_lshl_b32 s93, s93, 14\n\t"
        "s_cmp_lt_u32 s92, 112\n\t"
        "s_cselect_b64 s[86:87], s[86:87], %[xt0]\n\t"
        "s_cselect_b32 s92, s93, 0\n\t"
        "s_add_u32 s86, s86, s92\n\t"
        "s_addc_u32 s87, s87, 0\n\t"
        "s_memrealtime s[90:91]\n\t"
        "s_waitcnt lgkmcnt(0)\n\t"
        "v_cmp_ne_u32_e64 s[92:93], 0, v164\n\t"
        "s_cmp_lg_u64 s[92:93], 0\n\t"
        "s_cbranch_scc1 NOFIXF5_%=\n\t"
        "FIXLF5_%=:\n\t"
        "global_load_dwordx4 v[120:123], %[l16], s[86:87] offset:0 sc0 sc1\n\t"
        "global_load_dwordx4 v[124:127], %[l16], s[86:87] offset:1024 sc0 sc1\n\t"
        "s_waitcnt vmcnt(0)\n\t"
        "v_bitop3_b32 v166, v120, v124, %[tag] bitop3:0x7e\n\t"
        "v_and_b32 v166, 1, v166\n\t"
        "v_cmp_ne_u32_e64 s[92:93], 0, v166\n\t"
        "s_cmp_eq_u64 s[92:93], 0\n\t"
        "s_cbranch_scc1 NOFIXF5_%=\n\t"
        "s_sleep 1\n\t"
        "s_memrealtime s[92:93]\n\t"
        "s_waitcnt lgkmcnt(0)\n\t"
        "s_sub_u32 s92, s92, s90\n\t"
        "s_cmp_lt_u32 s92, " KDL "\n\t"
        "s_cbranch_scc1 FIXLF5_%=\n\t"
        "v_mov_b32 v166, %[bad]\n\t"
        "ds_write_b32 v166, v166\n\t"
        "NOFIXF5_%=:\n\t"
        "s_mov_b32 s94, 0\n\t"
        "SPINF5_%=:\n\t"
        "ds_read_b32 v164, %[vctl]\n\t"
        "s_waitcnt lgkmcnt(0)\n\t"
        "v_cmp_gt_u32_e64 s[86:87], s89, v164\n\t"
        "s_cmp_eq_u64 s[86:87], 0\n\t"
        "s_cbranch_scc1 PREF5_%=\n\t"
        SC3("s_add_u32 %[nfix], %[nfix], 1\n\t")
        "s_sleep 1\n\t"
        "s_add_u32 s94, s94, 1\n\t"
        "s_and_b32 s92, s94, 0x3ff\n\t"
        "s_cbranch_scc1 SPINF5_%=\n\t"
        "s_memrealtime s[92:93]\n\t"
        "s_waitcnt lgkmcnt(0)\n\t"
        "s_cmp_eq_u32 s94, 0x400\n\t"
        "s_cselect_b32 s90, s92, s90\n\t"
        "s_sub_u32 s92, s92, s90\n\t"
        "s_cmp_lt_u32 s92, " KDL3 "\n\t"
        "s_cbranch_scc1 SPINF5_%=\n\t"
        "v_mov_b32 v166, %[bad]\n\t"
        "ds_write_b32 v166, v166\n\t"
        "s_branch PREF5_%=\n\t"
        ".p2align 6\n\t"
        "TOPF_%=:\n\t"
        TRM("s_memrealtime s[90:91]\n\t")
        TRM("s_mul_i32 s93, s89, 43\n\t")
        TRM("s_lshr_b32 s93, s93, 8\n\t")
        TRM("s_add_u32 s93, s93, 24\n\t")
        TRM("s_mov_b32 m0, s93\n\t")
        TRM("s_waitcnt lgkmcnt(0)\n\t")
        TRM("v_writelane_b32 %[tr], s90, m0\n\t")
        "s_waitcnt lgkmcnt(10)\n\t"
        MF("v_mfma_f32_16x16x32_f16 v[8:11], v[112:115], v[128:131], v[8:11]\n\t")
        "v_add_u32 v165, s88, %[l16]\n\t"
        "global_load_dwordx4 v[80:83], %[l16], s[96:97] offset:0 " POL_FAR "\n\t"
        MF("v_mfma_f32_16x16x32_f16 v[44:47], v[116:119], v[128:131], v[44:47]\n\t")
        "global_load_dwordx4 v[84:87], %[l16], s[96:97] offset:1024 " POL_FAR "\n\t"
        "ds_read_b128 v[128:131], v165 offset:0\n\t"
        "s_waitcnt lgkmcnt(10)\n\t"
        MF("v_mfma_f32_16x16x32_f16 v[12:15], v[112:115], v[132:135], v[12:15]\n\t")
        "s_add_u32 s88, s88, 9216\n\t"
        "s_cmp_eq_u32 s88, 147456\n\t"
        "s_cselect_b32 s88, 0, s88\n\t"
        MF("v_mfma_f32_16x16x32_f16 v[48:51], v[116:119], v[132:135], v[48:51]\n\t")
        "ds_read_b128 v[132:135], v165 offset:1024\n\t"
        "s_waitcnt lgkmcnt(10)\n\t"
        MF("v_mfma_f32_16x16x32_f16 v[16:19], v[112:115], v[136:139], v[16:19]\n\t")
        "s_add_u32 s96, s96, s95\n\t"
        "s_addc_u32 s97, s97, 0\n\t"
        MF("v_mfma_f32_16x16x32_f16 v[52:55], v[116:119], v[136:139], v[52:55]\n\t")
        "ds_read_b128 v[136:139], v165 offset:2048\n\t"
        "ds_read_b32 v164, %[vctl]\n\t"
        "s_waitcnt lgkmcnt(10)\n\t"
        MF("v_mfma_f32_16x16x32_f16 v[20:23], v[112:115], v[140:143], v[20:23]\n\t")
        MF("v_mfma_f32_16x16x32_f16 v[56:59], v[116:119], v[140:143], v[56:59]\n\t")
        "ds_read_b128 v[140:143], v165 offset:3072\n\t"
        "s_waitcnt lgkmcnt(10)\n\t"
        MF("v_mfma_f32_16x16x32_f16 v[24:27], v[112:115], v[144:147], v[24:27]\n\t")
        "s_add_u32 s89, s89, 1\n\t"
        MF("v_mfma_f32_16x16x32_f16 v[60:63], v[116:119], v[144:147], v[60:63]\n\t")
        "ds_read_b128 v[144:147], v165 offset:4096\n\t"
        "s_waitcnt lgkmcnt(10)\n\t"
        MF("v_mfma_f32_16x16x32_f16 v[28:31], v[112:115], v[148:151], v[28:31]\n\t")
        MF("v_mfma_f32_16x16x32_f16 v[64:67], v[116:119], v[148:151], v[64:67]\n\t")
        "ds_read_b128 v[148:151], v165 offset:5120\n\t"
        "s_waitcnt lgkmcnt(10)\n\t"
        MF("v_mfma_f32_16x16x32_f16 v[32:35], v[112:115], v[152:155], v[32:35]\n\t")
        MF("v_mfma_f32_16x16x32_f16 v[68:71], v[116:119], v[152:155], v[68:71]\n\t")
        "ds_read_b128 v[152:155], v165 offset:6144\n\t"
        "s_waitcnt lgkmcnt(10)\n\t"
        "s_waitcnt vmcnt(2)\n\t"
        MF("v_mfma_f32_16x16x32_f16 v[36:39], v[112:115], v[156:159], v[36:39]\n\t")
        "v_bitop3_b32 v166, v120, v124, %[tag] bitop3:0x7e\n\t"
        MF("v_mfma_f32_16x16x32_f16 v[72:75], v[116:119], v[156:159], v[72:75]\n\t")
        "v_and_b32 v166, 1, v166\n\t"
        "ds_read_b128 v[156:159], v165 offset:7168\n\t"
        "s_waitcnt lgkmcnt(5)\n\t"
        MF("v_mfma_f32_16x16x32_f16 v[40:43], v[160:163], v[112:115], v[40:43]\n\t")
        "v_cmp_gt_u32_e64 s[86:87], s89, v164\n\t"
        TG("v_cmp_ne_u32_e64 s[84:85], 0, v166\n\t")
        NTG("s_mov_b64 s[84:85], 0\n\t")
        MF("v_mfma_f32_16x16x32_f16 v[76:79], v[160:163], v[116:119], v[76:79]\n\t")
        "ds_read_b128 v[160:163], v165 offset:8192\n\t"
        "s_mov_b64 exec, 1\n\t"
        "ds_add_u32 %[vctl], v167\n\t"
        "s_mov_b64 exec, -1\n\t"
        "s_or_b64 s[90:91], s[84:85], s[86:87]\n\t"
        "s_cbranch_scc1 SLOWF5_%=\n\t"
        "PREF5_%=:\n\t"
        "s_waitcnt lgkmcnt(10)\n\t"
        MF("v_mfma_f32_16x16x32_f16 v[8:11], v[120:123], v[128:131], v[8:11]\n\t")
        "v_add_u32 v165, s88, %[l16]\n\t"
        "global_load_dwordx4 v[88:91], %[l16], s[96:97] offset:0 " POL_FAR "\n\t"
        MF("v_mfma_f32_16x16x32_f16 v[44:47], v[124:127], v[128:131], v[44:47]\n\t")
        "global_load_dwordx4 v[92:95], %[l16], s[96:97] offset:1024 " POL_FAR "\n\t"
        "ds_read_b128 v[128:131], v165 offset:0\n\t"
        "s_waitcnt lgkmcnt(10)\n\t"
        MF("v_mfma_f32_16x16x32_f16 v[12:15], v[120:123], v[132:135], v[12:15]\n\t")
        "s_add_u32 s88, s88, 9216\n\t"
        "s_cmp_eq_u32 s88, 147456\n\t"
        "s_cselect_b32 s88, 0, s88\n\t"
        MF("v_mfma_f32_16x16x32_f16 v[48:51], v[124:127], v[132:135], v[48:51]\n\t")
        "ds_read_b128 v[132:135], v165 offset:1024\n\t"
        "s_waitcnt lgkmcnt(10)\n\t"
        MF("v_mfma_f32_16x16x32_f16 v[16:19], v[120:123], v[136:139], v[16:19]\n\t")
        "s_add_u32 s96, s96, s95\n\t"
        "s_addc_u32 s97, s97, 0\n\t"
        MF("v_mfma_f32_16x16x32_f16 v[52:55], v[124:127], v[136:139], v[52:55]\n\t")
        "ds_read_b128 v[136:139], v165 offset:2048\n\t"
        "ds_read_b32 v164, %[vctl]\n\t"
        "s_waitcnt lgkmcnt(10)\n\t"
        MF("v_mfma_f32_16x16x32_f16 v[20:23], v[120:123], v[140:143], v[20:23]\n\t")
        MF("v_mfma_f32_16x16x32_f16 v[56:59], v[124:127], v[140:143], v[56:59]\n\t")
        "ds_read_b128 v[140:143], v165 offset:3072\n\t"
        "s_waitcnt lgkmcnt(10)\n\t"
        MF("v_mfma_f32_16x16x32_f16 v[24:27], v[120:123], v[144:147], v[24:27]\n\t")
        "s_add_u32 s89, s89, 1\n\t"
        MF("v_mfma_f32_16x16x32_f16 v[60:63], v[124:127], v[144:147], v[60:63]\n\t")
        "ds_read_b128 v[144:147], v165 offset:4096\n\t"
        "s_waitcnt lgkmcnt(10)\n\t"
        MF("v_mfma_f32_16x16x32_f16 v[28:31], v[120:123], v[148:151], v[28:31]\n\t")
        MF("v_mfma_f32_16x16x32_f16 v[64:67], v[124:127], v[148:151], v[64:67]\n\t")
        "ds_read_b128 v[148:151], v165 offset:5120\n\t"
        "s_waitcnt lgkmcnt(10)\n\t"
        MF("v_mfma_f32_16x16x32_f16 v[32:35], v[120:123], v[152:155], v[32:35]\n\t")
        MF("v_mfma_f32_16x16x32_f16 v[68:71], v[124:127], v[152:155], v[68:71]\n\t")
        "ds_read_b128 v[152:155], v165 offset:6144\n\t"
        "s_waitcnt lgkmcnt(10)\n\t"
        "s_waitcnt vmcnt(2)\n\t"
        MF("v_mfma_f32_16x16x32_f16 v[36:39], v[120:123], v[156:159], v[36:39]\n\t")
        "v_bitop3_b32 v166, v80, v84, %[tag] bitop3:0x7e\n\t"
        MF("v_mfma_f32_16x16x32_f16 v[72:75], v[124:127], v[156:159], v[72:75]\n\t")
        "v_and_b32 v166, 1, v166\n\t"
        "ds_read_b128 v[156:159], v165 offset:7168\n\t"
        "s_waitcnt lgkmcnt(5)\n\t"
        MF("v_mfma_f32_16x16x32_f16 v[40:43], v[160:163], v[120:123], v[40:43]\n\t")
        "v_cmp_gt_u32_e64 s[86:87], s89, v164\n\t"
        TG("v_cmp_ne_u32_e64 s[84:85], 0, v166\n\t")
        NTG("s_mov_b64 s[84:85], 0\n\t")
        MF("v_mfma_f32_16x16x32_f16 v[76:79], v[160:163], v[124:127], v[76:79]\n\t")
        "ds_read_b128 v[160:163], v165 offset:8192\n\t"
        "s_mov_b64 exec, 1\n\t"
        "ds_add_u32 %[vctl], v167\n\t"
        "s_mov_b64 exec, -1\n\t"
        "s_or_b64 s[90:91], s[84:85], s[86:87]\n\t"
        "s_cbranch_scc1 SLOWF0_%=\n\t"
        "PREF0_%=:\n\t"
        "s_waitcnt lgkmcnt(10)\n\t"
        MF("v_mfma_f32_16x16x32_f16 v[8:11], v[80:83], v[128:131], v[8:11]\n\t")
        "v_add_u32 v165, s88, %[l16]\n\t"
        "global_load_dwordx4 v[96:99], %[l16], s[96:97] offset:0 " POL_FAR "\n\t"
        MF("v_mfma_f32_16x16x32_f16 v[44:47], v[84:87], v[128:131], v[44:47]\n\t")
        "global_load_dwordx4 v[100:103], %[l16], s[96:97] offset:1024 " POL_FAR "\n\t"
        "ds_read_b128 v[128:131], v165 offset:0\n\t"
        "s_waitcnt lgkmcnt(10)\n\t"
        MF("v_mfma_f32_16x16x32_f16 v[12:15], v[80:83], v[132:135], v[12:15]\n\t")
        "s_add_u32 s88, s88, 9216\n\t"
        "s_cmp_eq_u32 s88, 147456\n\t"
        "s_cselect_b32 s88, 0, s88\n\t"
        MF("v_mfma_f32_16x16x32_f16 v[48:51], v[84:87], v[132:135], v[48:51]\n\t")
        "ds_read_b128 v[132:135], v165 offset:1024\n\t"
        "s_waitcnt lgkmcnt(10)\n\t"
        MF("v_mfma_f32_16x16x32_f16 v[16:19], v[80:83], v[136:139], v[16:19]\n\t")
        "s_add_u32 s96, s96, s95\n\t"
        "s_addc_u32 s97, s97, 0\n\t"
        MF("v_mfma_f32_16x16x32_f16 v[52:55], v[84:87], v[136:139], v[52:55]\n\t")
        "ds_read_b128 v[136:139], v165 offset:2048\n\t"
        "ds_read_b32 v164, %[vctl]\n\t"
        "s_waitcnt lgkmcnt(10)\n\t"
        MF("v_mfma_f32_16x16x32_f16 v[20:23], v[80:83], v[140:143], v[20:23]\n\t")
        MF("v_mfma_f32_16x16x32_f16 v[56:59], v[84:87], v[140:143], v[56:59]\n\t")
        "ds_read_b128 v[140:143], v165 offset:3072\n\t"
        "s_waitcnt lgkmcnt(10)\n\t"
        MF("v_mfma_f32_16x16x32_f16 v[24:27], v[80:83], v[144:147], v[24:27]\n\t")
        "s_add_u32 s89, s89, 1\n\t"
        MF("v_mfma_f32_16x16x32_f16 v[60:63], v[84:87], v[144:147], v[60:63]\n\t")
        "ds_read_b128 v[144:147], v165 offset:4096\n\t"
        "s_waitcnt lgkmcnt(10)\n\t"
        MF("v_mfma_f32_16x16x32_f16 v[28:31], v[80:83], v[148:151], v[28:31]\n\t")
        MF("v_mfma_f32_16x16x32_f16 v[64:67], v[84:87], v[148:151], v[64:67]\n\t")
        "ds_read_b128 v[148:151], v165 offset:5120\n\t"
        "s_waitcnt lgkmcnt(10)\n\t"
        MF("v_mfma_f32_16x16x32_f16 v[32:35], v[80:83], v[152:155], v[32:35]\n\t")
        MF("v_mfma_f32_16x16x32_f16 v[68:71], v[84:87], v[152:155], v[68:71]\n\t")
        "ds_read_b128 v[152:155], v165 offset:6144\n\t"
        "s_waitcnt lgkmcnt(10)\n\t"
        "s_waitcnt vmcnt(2)\n\t"
        MF("v_mfma_f32_16x16x32_f16 v[36:39], v[80:83], v[156:159], v[36:39]\n\t")
        "v_bitop3_b32 v166, v88, v92, %[tag] bitop3:0x7e\n\t"
        MF("v_mfma_f32_16x16x32_f16 v[72:75], v[84:87], v[156:159], v[72:75]\n\t")
        "v_and_b32 v166, 1, v166\n\t"
        "ds_read_b128 v[156:159], v165 offset:7168\n\t"
        "s_waitcnt lgkmcnt(5)\n\t"
        MF("v_mfma_f32_16x16x32_f16 v[40:43], v[160:163], v[80:83], v[40:43]\n\t")
        "v_cmp_gt_u32_e64 s[86:87], s89, v164\n\t"
        TG("v_cmp_ne_u32_e64 s[84:85], 0, v166\n\t")
        NTG("s_mov_b64 s[84:85], 0\n\t")
        MF("v_mfma_f32_16x16x32_f16 v[76:79], v[160:163], v[84:87], v[76:79]\n\t")
        "ds_read_b128 v[160:163], v165 offset:8192\n\t"
        "s_mov_b64 exec, 1\n\t"
        "ds_add_u32 %[vctl], v167\n\t"
        "s_mov_b64 exec, -1\n\t"
        "s_or_b64 s[90:91], s[84:85], s[86:87]\n\t"
        "s_cbranch_scc1 SLOWF1_%=\n\t"
        "PREF1_%=:\n\t"
        "s_waitcnt lgkmcnt(10)\n\t"
        MF("v_mfma_f32_16x16x32_f16 v[8:11], v[88:91], v[128:131], v[8:11]\n\t")
        "v_add_u32 v165, s88, %[l16]\n\t"
        "global_load_dwordx4 v[104:107], %[l16], s[96:97] offset:0 " POL_FAR "\n\t"
        MF("v_mfma_f32_16x16x32_f16 v[44:47], v[92:95], v[128:131], v[44:47]\n\t")
        "global_load_dwordx4 v[108:111], %[l16], s[96:97] offset:1024 " POL_FAR "\n\t"
        "ds_read_b128 v[128:131], v165 offset:0\n\t"
        "s_waitcnt lgkmcnt(10)\n\t"
        MF("v_mfma_f32_16x16x32_f16 v[12:15], v[88:91], v[132:135], v[12:15]\n\t")
        "s_add_u32 s88, s88, 9216\n\t"
        "s_cmp_eq_u32 s88, 147456\n\t"
        "s_cselect_b32 s88, 0, s88\n\t"
        MF("v_mfma_f32_16x16x32_f16 v[48:51], v[92:95], v[132:135], v[48:51]\n\t")
        "ds_read_b128 v[132:135], v165 offset:1024\n\t"
        "s_waitcnt lgkmcnt(10)\n\t"
        MF("v_mfma_f32_16x16x32_f16 v[16:19], v[88:91], v[136:139], v[16:19]\n\t")
        "s_add_u32 s96, s96, s95\n\t"
        "s_addc_u32 s97, s97, 0\n\t"
        MF("v_mfma_f32_16x16x32_f16 v[52:55], v[92:95], v[136:139], v[52:55]\n\t")
        "ds_read_b128 v[136:139], v165 offset:2048\n\t"
        "ds_read_b32 v164, %[vctl]\n\t"
        "s_waitcnt lgkmcnt(10)\n\t"
        MF("v_mfma_f32_16x16x32_f16 v[20:23], v[88:91], v[140:143], v[20:23]\n\t")
        "s_cmp_eq_u32 s89, 111\n\t"
        "s_cselect_b64 s[96:97], %[xt0], s[96:97]\n\t"
        "s_cselect_b32 s95, s98, s95\n\t"
        MF("v_mfma_f32_16x16x32_f16 v[56:59], v[92:95], v[140:143], v[56:59]\n\t")
        "ds_read_b128 v[140:143], v165 offset:3072\n\t"
        "s_waitcnt lgkmcnt(10)\n\t"
        MF("v_mfma_f32_16x16x32_f16 v[24:27], v[88:91], v[144:147], v[24:27]\n\t")
        "s_add_u32 s89, s89, 1\n\t"
        MF("v_mfma_f32_16x16x32_f16 v[60:63], v[92:95], v[144:147], v[60:63]\n\t")
        "ds_read_b128 v[144:147], v165 offset:4096\n\t"
        "s_waitcnt lgkmcnt(10)\n\t"
        MF("v_mfma_f32_16x16x32_f16 v[28:31], v[88:91], v[148:151], v[28:31]\n\t")
        MF("v_mfma_f32_16x16x32_f16 v[64:67], v[92:95], v[148:151], v[64:67]\n\t")
        "ds_read_b128 v[148:151], v165 offset:5120\n\t"
        "s_waitcnt lgkmcnt(10)\n\t"
        MF("v_mfma_f32_16x16x32_f16 v[32:35], v[88:91], v[152:155], v[32:35]\n\t")
        MF("v_mfma_f32_16x16x32_f16 v[68:71], v[92:95], v[152:155], v[68:71]\n\t")
        "ds_read_b128 v[152:155], v165 offset:6144\n\t"
        "s_waitcnt lgkmcnt(10)\n\t"
        "s_waitcnt vmcnt(2)\n\t"
        MF("v_mfma_f32_16x16x32_f16 v[36:39], v[88:91], v[156:159], v[36:39]\n\t")
        "v_bitop3_b32 v166, v96, v100, %[tag] bitop3:0x7e\n\t"
        MF("v_mfma_f32_16x16x32_f16 v[72:75], v[92:95], v[156:159], v[72:75]\n\t")
        "v_and_b32 v166, 1, v166\n\t"
        "ds_read_b128 v[156:159], v165 offset:7168\n\t"
        "s_waitcnt lgkmcnt(5)\n\t"
        MF("v_mfma_f32_16x16x32_f16 v[40:43], v[160:163], v[88:91], v[40:43]\n\t")
        "v_cmp_gt_u32_e64 s[86:87], s89, v164\n\t"
        TG("v_cmp_ne_u32_e64 s[84:85], 0, v166\n\t")
        NTG("s_mov_b64 s[84:85], 0\n\t")
        MF("v_mfma_f32_16x16x32_f16 v[76:79], v[160:163], v[92:95], v[76:79]\n\t")
        "ds_read_b128 v[160:163], v165 offset:8192\n\t"
        "s_mov_b64 exec, 1\n\t"
        "ds_add_u32 %[vctl], v167\n\t"
        "s_mov_b64 exec, -1\n\t"
        "s_or_b64 s[90:91], s[84:85], s[86:87]\n\t"
        "s_cbranch_scc1 SLOWF2_%=\n\t"
        "PREF2_%=:\n\t"
        "s_waitcnt lgkmcnt(10)\n\t"
        MF("v_mfma_f32_16x16x32_f16 v[8:11], v[96:99], v[128:131], v[8:11]\n\t")
        "v_add_u32 v165, s88, %[l16]\n\t"
        "global_load_dwordx4 v[112:115], %[l16], s[96:97] offset:0 " POL_FAR "\n\t"
        MF("v_mfma_f32_16x16x32_f16 v[44:47], v[100:103], v[128:131], v[44:47]\n\t")
        "global_load_dwordx4 v[116:119], %[l16], s[96:97] offset:1024 " POL_FAR "\n\t"
        "ds_read_b128 v[128:131], v165 offset:0\n\t"
        "s_waitcnt lgkmcnt(10)\n\t"
        MF("v_mfma_f32_16x16x32_f16 v[12:15], v[96:99], v[132:135], v[12:15]\n\t")
        "s_add_u32 s88, s88, 9216\n\t"
        "s_cmp_eq_u32 s88, 147456\n\t"
        "s_cselect_b32 s88, 0, s88\n\t"
        MF("v_mfma_f32_16x16x32_f16 v[48:51], v[100:103], v[132:135], v[48:51]\n\t")
        "ds_read_b128 v[132:135], v165 offset:1024\n\t"
        "s_waitcnt lgkmcnt(10)\n\t"
        MF("v_mfma_f32_16x16x32_f16 v[16:19], v[96:99], v[136:139], v[16:19]\n\t")
        "s_add_u32 s96, s96, s95\n\t"
        "s_addc_u32 s97, s97, 0\n\t"
        MF("v_mfma_f32_16x16x32_f16 v[52:55], v[100:103], v[136:139], v[52:55]\n\t")
        "ds_read_b128 v[136:139], v165 offset:2048\n\t"
        "ds_read_b32 v164, %[vctl]\n\t"
        "s_waitcnt lgkmcnt(10)\n\t"
        MF("v_mfma_f32_16x16x32_f16 v[20:23], v[96:99], v[140:143], v[20:23]\n\t")
        MF("v_mfma_f32_16x16x32_f16 v[56:59], v[100:103], v[140:143], v[56:59]\n\t")
        "ds_read_b128 v[140:143], v165 offset:3072\n\t"
        "s_waitcnt lgkmcnt(10)\n\t"
        MF("v_mfma_f32_16x16x32_f16 v[24:27], v[96:99], v[144:147], v[24:27]\n\t")
        "s_add_u32 s89, s89, 1\n\t"
        MF("v_mfma_f32_16x16x32_f16 v[60:63], v[100:103], v[144:147], v[60:63]\n\t")
        "ds_read_b128 v[144:147], v165 offset:4096\n\t"
        "s_waitcnt lgkmcnt(10)\n\t"
        MF("v_mfma_f32_16x16x32_f16 v[28:31], v[96:99], v[148:151], v[28:31]\n\t")
        MF("v_mfma_f32_16x16x32_f16 v[64:67], v[100:103], v[148:151], v[64:67]\n\t")
        "ds_read_b128 v[148:151], v165 offset:5120\n\t"
        "s_waitcnt lgkmcnt(10)\n\t"
        MF("v_mfma_f32_16x16x32_f16 v[32:35], v[96:99], v[152:155], v[32:35]\n\t")
        MF("v_mfma_f32_16x16x32_f16 v[68:71], v[100:103], v[152:155], v[68:71]\n\t")
        "ds_read_b128 v[152:155], v165 offset:6144\n\t"
        "s_waitcnt lgkmcnt(10)\n\t"
        "s_waitcnt vmcnt(2)\n\t"
        MF("v_mfma_f32_16x16x32_f16 v[36:39], v[96:99], v[156:159], v[36:39]\n\t")
        "v_bitop3_b32 v166, v104, v108, %[tag] bitop3:0x7e\n\t"
        MF("v_mfma_f32_16x16x32_f16 v[72:75], v[100:103], v[156:159], v[72:75]\n\t")
        "v_and_b32 v166, 1, v166\n\t"
        "ds_read_b128 v[156:159], v165 offset:7168\n\t"
        "s_waitcnt lgkmcnt(5)\n\t"
        MF("v_mfma_f32_16x16x32_f16 v[40:43], v[160:163], v[96:99], v[40:43]\n\t")
        "v_cmp_gt_u32_e64 s[86:87], s89, v164\n\t"
        TG("v_cmp_ne_u32_e64 s[84:85], 0, v166\n\t")
        NTG("s_mov_b64 s[84:85], 0\n\t")
        MF("v_mfma_f32_16x16x32_f16 v[76:79], v[160:163], v[100:103], v[76:79]\n\t")
        "ds_read_b128 v[160:163], v165 offset:8192\n\t"
        "s_mov_b64 exec, 1\n\t"
        "ds_add_u32 %[vctl], v167\n\t"
        "s_mov_b64 exec, -1\n\t"
        "s_or_b64 s[90:91], s[84:85], s[86:87]\n\t"
        "s_cbranch_scc1 SLOWF3_%=\n\t"
        "PREF3_%=:\n\t"
        "s_waitcnt lgkmcnt(10)\n\t"
        MF("v_mfma_f32_16x16x32_f16 v[8:11], v[104:107], v[128:131], v[8:11]\n\t")
        "v_add_u32 v165, s88, %[l16]\n\t"
        "global_load_dwordx4 v[120:123], %[l16], s[96:97] offset:0 " POL_FAR "\n\t"
        MF("v_mfma_f32_16x16x32_f16 v[44:47], v[108:111], v[128:131], v[44:47]\n\t")
        "global_load_dwordx4 v[124:127], %[l16], s[96:97] offset:1024 " POL_FAR "\n\t"
        "ds_read_b128 v[128:131], v165 offset:0\n\t"
        "s_waitcnt lgkmcnt(10)\n\t"
        MF("v_mfma_f32_16x16x32_f16 v[12:15], v[104:107], v[132:135], v[12:15]\n\t")
        "s_add_u32 s88, s88, 9216\n\t"
        "s_cmp_eq_u32 s88, 147456\n\t"
        "s_cselect_b32 s88, 0, s88\n\t"
        MF("v_mfma_f32_16x16x32_f16 v[48:51], v[108:111], v[132:135], v[48:51]\n\t")
        "ds_read_b128 v[132:135], v165 offset:1024\n\t"
        "s_waitcnt lgkmcnt(10)\n\t"
        MF("v_mfma_f32_16x16x32_f16 v[16:19], v[104:107], v[136:139], v[16:19]\n\t")
        "s_add_u32 s96, s96, s95\n\t"
        "s_addc_u32 s97, s97, 0\n\t"
        MF("v_mfma_f32_16x16x32_f16 v[52:55], v[108:111], v[136:139], v[52:55]\n\t")
        "ds_read_b128 v[136:139], v165 offset:2048\n\t"
        "ds_read_b32 v164, %[vctl]\n\t"
        "s_waitcnt lgkmcnt(10)\n\t"
        MF("v_mfma_f32_16x16x32_f16 v[20:23], v[104:107], v[140:143], v[20:23]\n\t")
        MF("v_mfma_f32_16x16x32_f16 v[56:59], v[108:111], v[140:143], v[56:59]\n\t")
        "ds_read_b128 v[140:143], v165 offset:3072\n\t"
        "s_waitcnt lgkmcnt(10)\n\t"
        MF("v_mfma_f32_16x16x32_f16 v[24:27], v[104:107], v[144:147], v[24:27]\n\t")
        "s_add_u32 s89, s89, 1\n\t"
        MF("v_mfma_f32_16x16x32_f16 v[60:63], v[108:111], v[144:147], v[60:63]\n\t")
        "ds_read_b128 v[144:147], v165 offset:4096\n\t"
        "s_waitcnt lgkmcnt(10)\n\t"
        MF("v_mfma_f32_16x16x32_f16 v[28:31], v[104:107], v[148:151], v[28:31]\n\t")
        MF("v_mfma_f32_16x16x32_f16 v[64:67], v[108:111], v[148:151], v[64:67]\n\t")
        "ds_read_b128 v[148:151], v165 offset:5120\n\t"
        "s_waitcnt lgkmcnt(10)\n\t"
        MF("v_mfma_f32_16x16x32_f16 v[32:35], v[104:107], v[152:155], v[32:35]\n\t")
        MF("v_mfma_f32_16x16x32_f16 v[68:71], v[108:111], v[152:155], v[68:71]\n\t")
        "ds_read_b128 v[152:155], v165 offset:6144\n\t"
        "s_waitcnt lgkmcnt(10)\n\t"
        "s_waitcnt vmcnt(2)\n\t"
        MF("v_mfma_f32_16x16x32_f16 v[36:39], v[104:107], v[156:159], v[36:39]\n\t")
        "v_bitop3_b32 v166, v112, v116, %[tag] bitop3:0x7e\n\t"
        MF("v_mfma_f32_16x16x32_f16 v[72:75], v[108:111], v[156:159], v[72:75]\n\t")
        "v_and_b32 v166, 1, v166\n\t"
        "ds_read_b128 v[156:159], v165 offset:7168\n\t"
        "s_waitcnt lgkmcnt(5)\n\t"
        MF("v_mfma_f32_16x16x32_f16 v[40:43], v[160:163], v[104:107], v[40:43]\n\t")
        "v_cmp_gt_u32_e64 s[86:87], s89, v164\n\t"
        TG("v_cmp_ne_u32_e64 s[84:85], 0, v166\n\t")
        NTG("s_mov_b64 s[84:85], 0\n\t")
        MF("v_mfma_f32_16x16x32_f16 v[76:79], v[160:163], v[108:111], v[76:79]\n\t")
        "ds_read_b128 v[160:163], v165 offset:8192\n\t"
        "s_mov_b64 exec, 1\n\t"
        "ds_add_u32 %[vctl], v167\n\t"
        "s_mov_b64 exec, -1\n\t"
        "s_or_b64 s[90:91], s[84:85], s[86:87]\n\t"
        "s_cbranch_scc1 SLOWF4_%=\n\t"
        "PREF4_%=:\n\t"
        "s_cmp_eq_u32 s89, 114\n\t"
        "s_cbranch_scc0 TOPF_%=\n\t"
        "s_cmp_lg_u32 %[stage], 0\n\t"
        "s_cbranch_scc1 ENTRY2_%=\n\t"
        "s_movk_i32 s89, 115\n\t"
        "v_cmp_gt_u32_e64 s[86:87], s89, v164\n\t"
        "s_add_u32 s92, 0, 0\n\t"
        "v_add_u32 v165, s92, %[l16]\n\t"
        "s_and_b64 s[86:87], s[86:87], 0x400\n\t"
        "s_cbranch_scc1 WAITB_%=\n\t"
        "TAILX_%=:\n\t"
        "TAIL0_%=:\n\t"
        TRM("s_memrealtime s[90:91]\n\t")
        TRM("s_mov_b32 s93, 8\n\t")
        TRM("s_mov_b32 m0, s93\n\t")
        TRM("s_waitcnt lgkmcnt(0)\n\t")
        TRM("v_writelane_b32 %[tr], s90, m0\n\t")
        "s_waitcnt lgkmcnt(10)\n\t"
        MF("v_mfma_f32_16x16x32_f16 v[8:11], v[112:115], v[128:131], v[8:11]\n\t")
        "global_load_dwordx4 v[80:83], %[l16], s[96:97] offset:0 " POL_FAR "\n\t"
        "ds_read_b128 v[128:131], v165 offset:9216\n\t"
        "s_waitcnt lgkmcnt(10)\n\t"
        MF("v_mfma_f32_16x16x32_f16 v[12:15], v[112:115], v[132:135], v[12:15]\n\t")
        "global_load_dwordx4 v[84:87], %[l16], s[96:97] offset:1024 " POL_FAR "\n\t"
        "ds_read_b128 v[132:135], v165 offset:10240\n\t"
        "s_waitcnt lgkmcnt(10)\n\t"
        MF("v_mfma_f32_16x16x32_f16 v[16:19], v[112:115], v[136:139], v[16:19]\n\t")
        "s_add_u32 s96, s96, s95\n\t"
        "s_addc_u32 s97, s97, 0\n\t"
        "ds_read_b128 v[136:139], v165 offset:11264\n\t"
        "s_waitcnt lgkmcnt(9)\n\t"
        MF("v_mfma_f32_16x16x32_f16 v[20:23], v[112:115], v[140:143], v[20:23]\n\t")
        "ds_read_b128 v[140:143], v165 offset:12288\n\t"
        "s_waitcnt lgkmcnt(9)\n\t"
        MF("v_mfma_f32_16x16x32_f16 v[24:27], v[112:115], v[144:147], v[24:27]\n\t")
        "ds_read_b128 v[144:147], v165 offset:13312\n\t"
        "s_waitcnt lgkmcnt(9)\n\t"
        MF("v_mfma_f32_16x16x32_f16 v[28:31], v[112:115], v[148:151], v[28:31]\n\t")
        "ds_read_b128 v[148:151], v165 offset:14336\n\t"
        "s_waitcnt lgkmcnt(9)\n\t"
        MF("v_mfma_f32_16x16x32_f16 v[32:35], v[112:115], v[152:155], v[32:35]\n\t")
        "ds_read_b128 v[152:155], v165 offset:15360\n\t"
        "s_waitcnt lgkmcnt(9)\n\t"
        MF("v_mfma_f32_16x16x32_f16 v[36:39], v[112:115], v[156:159], v[36:39]\n\t")
        "ds_read_b128 v[156:159], v165 offset:16384\n\t"
        "s_waitcnt lgkmcnt(9)\n\t"
        MF("v_mfma_f32_16x16x32_f16 v[40:43], v[160:163], v[112:115], v[40:43]\n\t")
        "ds_read_b128 v[160:163], v165 offset:17408\n\t"
        "TAILH_%=:\n\t"
        "s_waitcnt lgkmcnt(8)\n\t"
        MF("v_mfma_f32_16x16x32_f16 v[8:11], v[116:119], v[128:131], v[8:11]\n\t")
        "ds_read_b32 v164, %[vctl]\n\t"
        "s_movk_i32 s89, 117\n\t"
        "ds_read_b128 v[128:131], v165 offset:18432\n\t"
        "s_waitcnt lgkmcnt(9)\n\t"
        MF("v_mfma_f32_16x16x32_f16 v[12:15], v[116:119], v[132:135], v[12:15]\n\t")
        "ds_read_b128 v[132:135], v165 offset:19456\n\t"
        "s_waitcnt lgkmcnt(9)\n\t"
        MF("v_mfma_f32_16x16x32_f16 v[16:19], v[116:119], v[136:139], v[16:19]\n\t")
        "ds_read_b128 v[136:139], v165 offset:20480\n\t"
        "s_waitcnt lgkmcnt(9)\n\t"
        MF("v_mfma_f32_16x16x32_f16 v[20:23], v[116:119], v[140:143], v[20:23]\n\t")
        "ds_read_b128 v[140:143], v165 offset:21504\n\t"
        "s_waitcnt lgkmcnt(9)\n\t"
        MF("v_mfma_f32_16x16x32_f16 v[24:27], v[116:119], v[144:147], v[24:27]\n\t")
        "ds_read_b128 v[144:147], v165 offset:22528\n\t"
        "s_waitcnt lgkmcnt(9)\n\t"
        MF("v_mfma_f32_16x16x32_f16 v[28:31], v[116:119], v[148:151], v[28:31]\n\t")
        "ds_read_b128 v[148:151], v165 offset:23552\n\t"
        "s_waitcnt lgkmcnt(9)\n\t"
        "s_waitcnt vmcnt(2)\n\t"
        MF("v_mfma_f32_16x16x32_f16 v[32:35], v[116:119], v[152:155], v[32:35]\n\t")
        "ds_read_b128 v[152:155], v165 offset:24576\n\t"
        "v_bitop3_b32 v166, v120, v124, %[tag] bitop3:0x7e\n\t"
        "v_and_b32 v166, 1, v166\n\t"
        "s_waitcnt lgkmcnt(7)\n\t"
        MF("v_mfma_f32_16x16x32_f16 v[36:39], v[116:119], v[156:159], v[36:39]\n\t")
        "ds_read_b128 v[156:159], v165 offset:25600\n\t"
        TG("v_cmp_ne_u32_e64 s[84:85], 0, v166\n\t")
        NTG("s_mov_b64 s[84:85], 0\n\t")
        "v_cmp_gt_u32_e64 s[86:87], s89, v164\n\t"
        "s_waitcnt lgkmcnt(9)\n\t"
        MF("v_mfma_f32_16x16x32_f16 v[40:43], v[160:163], v[116:119], v[40:43]\n\t")
        "ds_read_b128 v[160:163], v165 offset:26624\n\t"
        "s_and_b64 s[86:87], s[86:87], 0x400\n\t"
        "s_or_b64 s[90:91], s[84:85], s[86:87]\n\t"
        "s_cbranch_scc1 FIXT1_%=\n\t"
        "TAIL1_%=:\n\t"
        TRM("s_memrealtime s[90:91]\n\t")
        TRM("s_lshl_b32 s93, %[stage], 2\n\t")
        TRM("s_and_b32 s93, s93, 8\n\t")
        TRM("s_add_u32 s93, s93, 9\n\t")
        TRM("s_mov_b32 m0, s93\n\t")
        TRM("s_waitcnt lgkmcnt(0)\n\t")
        TRM("v_writelane_b32 %[tr], s90, m0\n\t")
        "s_waitcnt lgkmcnt(8)\n\t"
        MF("v_mfma_f32_16x16x32_f16 v[8:11], v[120:123], v[128:131], v[8:11]\n\t")
        "global_load_dwordx4 v[88:91], %[l16], s[96:97] offset:0 " POL_FAR "\n\t"
        "ds_read_b128 v[128:131], v165 offset:27648\n\t"
        "s_waitcnt lgkmcnt(8)\n\t"
        MF("v_mfma_f32_16x16x32_f16 v[12:15], v[120:123], v[132:135], v[12:15]\n\t")
        "global_load_dwordx4 v[92:95], %[l16], s[96:97] offset:1024 " POL_FAR "\n\t"
        "ds_read_b128 v[132:135], v165 offset:28672\n\t"
        "s_waitcnt lgkmcnt(8)\n\t"
        MF("v_mfma_f32_16x16x32_f16 v[16:19], v[120:123], v[136:139], v[16:19]\n\t")
        "s_add_u32 s96, s96, s95\n\t"
        "s_addc_u32 s97, s97, 0\n\t"
        "ds_read_b128 v[136:139], v165 offset:29696\n\t"
        "s_waitcnt lgkmcnt(8)\n\t"
        MF("v_mfma_f32_16x16x32_f16 v[20:23], v[120:123], v[140:143], v[20:23]\n\t")
        "ds_read_b128 v[140:143], v165 offset:30720\n\t"
        "s_waitcnt lgkmcnt(8)\n\t"
        MF("v_mfma_f32_16x16x32_f16 v[24:27], v[120:123], v[144:147], v[24:27]\n\t")
        "ds_read_b128 v[144:147], v165 offset:31744\n\t"
        "s_waitcnt lgkmcnt(8)\n\t"
        MF("v_mfma_f32_16x16x32_f16 v[28:31], v[120:123], v[148:151], v[28:31]\n\t")
        "ds_read_b128 v[148:151], v165 offset:32768\n\t"
        "s_waitcnt lgkmcnt(8)\n\t"
        MF("v_mfma_f32_16x16x32_f16 v[32:35], v[120:123], v[152:155], v[32:35]\n\t")
        "ds_read_b128 v[152:155], v165 offset:33792\n\t"
        "s_waitcnt lgkmcnt(8)\n\t"
        MF("v_mfma_f32_16x16x32_f16 v[36:39], v[120:123], v[156:159], v[36:39]\n\t")
        "ds_read_b128 v[156:159], v165 offset:34816\n\t"
        "s_waitcnt lgkmcnt(8)\n\t"
        MF("v_mfma_f32_16x16x32_f16 v[40:43], v[160:163], v[120:123], v[40:43]\n\t")
        "ds_read_b128 v[160:163], v165 offset:35840\n\t"
        "s_waitcnt lgkmcnt(8)\n\t"
        MF("v_mfma_f32_16x16x32_f16 v[8:11], v[124:127], v[128:131], v[8:11]\n\t")
        "ds_read_b32 v164, %[vctl]\n\t"
        "s_movk_i32 s89, 119\n\t"
        "ds_read_b128 v[128:131], v165 offset:36864\n\t"
        "s_waitcnt lgkmcnt(9)\n\t"
        MF("v_mfma_f32_16x16x32_f16 v[12:15], v[124:127], v[132:135], v[12:15]\n\t")
        "ds_read_b128 v[132:135], v165 offset:37888\n\t"
        "s_waitcnt lgkmcnt(9)\n\t"
        MF("v_mfma_f32_16x16x32_f16 v[16:19], v[124:127], v[136:139], v[16:19]\n\t")
        "ds_read_b128 v[136:139], v165 offset:38912\n\t"
        "s_waitcnt lgkmcnt(9)\n\t"
        MF("v_mfma_f32_16x16x32_f16 v[20:23], v[124:127], v[140:143], v[20:23]\n\t")
        "ds_read_b128 v[140:143], v165 offset:39936\n\t"
        "s_waitcnt lgkmcnt(9)\n\t"
        MF("v_mfma_f32_16x16x32_f16 v[24:27], v[124:127], v[144:147], v[24:27]\n\t")
        "ds_read_b128 v[144:147], v165 offset:40960\n\t"
        "s_waitcnt lgkmcnt(9)\n\t"
        MF("v_mfma_f32_16x16x32_f16 v[28:31], v[124:127], v[148:151], v[28:31]\n\t")
        "ds_read_b128 v[148:151], v165 offset:41984\n\t"
        "s_waitcnt lgkmcnt(9)\n\t"
        "s_waitcnt vmcnt(2)\n\t"
        MF("v_mfma_f32_16x16x32_f16 v[32:35], v[124:127], v[152:155], v[32:35]\n\t")
        "ds_read_b128 v[152:155], v165 offset:43008\n\t"
        "v_bitop3_b32 v166, v80, v84, %[tag] bitop3:0x7e\n\t"
        "v_and_b32 v166, 1, v166\n\t"
        "s_waitcnt lgkmcnt(7)\n\t"
        MF("v_mfma_f32_16x16x32_f16 v[36:39], v[124:127], v[156:159], v[36:39]\n\t")
        "ds_read_b128 v[156:159], v165 offset:44032\n\t"
        TG("v_cmp_ne_u32_e64 s[84:85], 0, v166\n\t")
        NTG("s_mov_b64 s[84:85], 0\n\t")
        "v_cmp_gt_u32_e64 s[86:87], s89, v164\n\t"
        "s_waitcnt lgkmcnt(9)\n\t"
        MF("v_mfma_f32_16x16x32_f16 v[40:43], v[160:163], v[124:127], v[40:43]\n\t")
        "ds_read_b128 v[160:163], v165 offset:45056\n\t"
        "s_and_b64 s[86:87], s[86:87], 0x400\n\t"
        "s_or_b64 s[90:91], s[84:85], s[86:87]\n\t"
        "s_cbranch_scc1 FIXT2_%=\n\t"
        "TAIL2_%=:\n\t"
        TRM("s_memrealtime s[90:91]\n\t")
        TRM("s_lshl_b32 s93, %[stage], 2\n\t")
        TRM("s_and_b32 s93, s93, 8\n\t")
        TRM("s_add_u32 s93, s93, 10\n\t")
        TRM("s_mov_b32 m0, s93\n\t")
        TRM("s_waitcnt lgkmcnt(0)\n\t")
        TRM("v_writelane_b32 %[tr], s90, m0\n\t")
        "s_waitcnt lgkmcnt(8)\n\t"
        MF("v_mfma_f32_16x16x32_f16 v[8:11], v[80:83], v[128:131], v[8:11]\n\t")
        "global_load_dwordx4 v[96:99], %[l16], s[96:97] offset:0 " POL_FAR "\n\t"
        "ds_read_b128 v[128:131], v165 offset:46080\n\t"
        "s_waitcnt lgkmcnt(8)\n\t"
        MF("v_mfma_f32_16x16x32_f16 v[12:15], v[80:83], v[132:135], v[12:15]\n\t")
        "global_load_dwordx4 v[100:103], %[l16], s[96:97] offset:1024 " POL_FAR "\n\t"
        "ds_read_b128 v[132:135], v165 offset:47104\n\t"
        "s_waitcnt lgkmcnt(8)\n\t"
        MF("v_mfma_f32_16x16x32_f16 v[16:19], v[80:83], v[136:139], v[16:19]\n\t")
        "s_add_u32 s96, s96, s95\n\t"
        "s_addc_u32 s97, s97, 0\n\t"
        "ds_read_b128 v[136:139], v165 offset:48128\n\t"
        "s_waitcnt lgkmcnt(8)\n\t"
        MF("v_mfma_f32_16x16x32_f16 v[20:23], v[80:83], v[140:143], v[20:23]\n\t")
        "ds_read_b128 v[140:143], v165 offset:49152\n\t"
        "s_waitcnt lgkmcnt(8)\n\t"
        MF("v_mfma_f32_16x16x32_f16 v[24:27], v[80:83], v[144:147], v[24:27]\n\t")
        "ds_read_b128 v[144:147], v165 offset:50176\n\t"
        "s_waitcnt lgkmcnt(8)\n\t"
        MF("v_mfma_f32_16x16x32_f16 v[28:31], v[80:83], v[148:151], v[28:31]\n\t")
        "ds_read_b128 v[148:151], v165 offset:51200\n\t"
        "s_waitcnt lgkmcnt(8)\n\t"
        MF("v_mfma_f32_16x16x32_f16 v[32:35], v[80:83], v[152:155], v[32:35]\n\t")
        "ds_read_b128 v[152:155], v165 offset:52224\n\t"
        "s_waitcnt lgkmcnt(8)\n\t"
        MF("v_mfma_f32_16x16x32_f16 v[36:39], v[80:83], v[156:159], v[36:39]\n\t")
        "ds_read_b128 v[156:159], v165 offset:53248\n\t"
        "s_waitcnt lgkmcnt(8)\n\t"
        MF("v_mfma_f32_16x16x32_f16 v[40:43], v[160:163], v[80:83], v[40:43]\n\t")
        "ds_read_b128 v[160:163], v165 offset:54272\n\t"
        "s_waitcnt lgkmcnt(8)\n\t"
        MF("v_mfma_f32_16x16x32_f16 v[8:11], v[84:87], v[128:131], v[8:11]\n\t")
        "ds_read_b32 v164, %[vctl]\n\t"
        "s_movk_i32 s89, 121\n\t"
        "v_add_u32 v165, 55296, v165\n\t"
        "ds_read_b128 v[128:131], v165 offset:0\n\t"
        "s_waitcnt lgkmcnt(9)\n\t"
        MF("v_mfma_f32_16x16x32_f16 v[12:15], v[84:87], v[132:135], v[12:15]\n\t")
        "ds_read_b128 v[132:135], v165 offset:1024\n\t"
        "s_waitcnt lgkmcnt(9)\n\t"
        MF("v_mfma_f32_16x16x32_f16 v[16:19], v[84:87], v[136:139], v[16:19]\n\t")
        "ds_read_b128 v[136:139], v165 offset:2048\n\t"
        "s_waitcnt lgkmcnt(9)\n\t"
        MF("v_mfma_f32_16x16x32_f16 v[20:23], v[84:87], v[140:143], v[20:23]\n\t")
        "ds_read_b128 v[140:143], v165 offset:3072\n\t"
        "s_waitcnt lgkmcnt(9)\n\t"
        MF("v_mfma_f32_16x16x32_f16 v[24:27], v[84:87], v[144:147], v[24:27]\n\t")
        "ds_read_b128 v[144:147], v165 offset:4096\n\t"
        "s_waitcnt lgkmcnt(9)\n\t"
        MF("v_mfma_f32_16x16x32_f16 v[28:31], v[84:87], v[148:151], v[28:31]\n\t")
        "ds_read_b128 v[148:151], v165 offset:5120\n\t"
        "s_waitcnt lgkmcnt(9)\n\t"
        "s_waitcnt vmcnt(2)\n\t"
        MF("v_mfma_f32_16x16x32_f16 v[32:35], v[84:87], v[152:155], v[32:35]\n\t")
        "ds_read_b128 v[152:155], v165 offset:6144\n\t"
        "v_bitop3_b32 v166, v88, v92, %[tag] bitop3:0x7e\n\t"
        "v_and_b32 v166, 1, v166\n\t"
        "s_waitcnt lgkmcnt(7)\n\t"
        MF("v_mfma_f32_16x16x32_f16 v[36:39], v[84:87], v[156:159], v[36:39]\n\t")
        "ds_read_b128 v[156:159], v165 offset:7168\n\t"
        TG("v_cmp_ne_u32_e64 s[84:85], 0, v166\n\t")
        NTG("s_mov_b64 s[84:85], 0\n\t")
        "v_cmp_gt_u32_e64 s[86:87], s89, v164\n\t"
        "s_waitcnt lgkmcnt(9)\n\t"
        MF("v_mfma_f32_16x16x32_f16 v[40:43], v[160:163], v[84:87], v[40:43]\n\t")
        "ds_read_b128 v[160:163], v165 offset:8192\n\t"
        "s_and_b64 s[86:87], s[86:87], 0x400\n\t"
        "s_or_b64 s[90:91], s[84:85], s[86:87]\n\t"
        "s_cbranch_scc1 FIXT3_%=\n\t"
        "TAIL3_%=:\n\t"
        TRM("s_memrealtime s[90:91]\n\t")
        TRM("s_lshl_b32 s93, %[stage], 2\n\t")
        TRM("s_and_b32 s93, s93, 8\n\t")
        TRM("s_add_u32 s93, s93, 11\n\t")
        TRM("s_mov_b32 m0, s93\n\t")
        TRM("s_waitcnt lgkmcnt(0)\n\t")
        TRM("v_writelane_b32 %[tr], s90, m0\n\t")
        "s_waitcnt lgkmcnt(8)\n\t"
        MF("v_mfma_f32_16x16x32_f16 v[8:11], v[88:91], v[128:131], v[8:11]\n\t")
        "global_load_dwordx4 v[104:107], %[l16], s[96:97] offset:0 " POL_FAR "\n\t"
        "ds_read_b128 v[128:131], v165 offset:9216\n\t"
        "s_waitcnt lgkmcnt(8)\n\t"
        MF("v_mfma_f32_16x16x32_f16 v[12:15], v[88:91], v[132:135], v[12:15]\n\t")
        "global_load_dwordx4 v[108:111], %[l16], s[96:97] offset:1024 " POL_FAR "\n\t"
        "v_mov_b32 v166, %[bad]\n\t"
        "ds_read_b64 v[80:81], v166 offset:" KXEXP "\n\t"
        "ds_read_b128 v[132:135], v165 offset:10240\n\t"
        "s_waitcnt lgkmcnt(9)\n\t"
        MF("v_mfma_f32_16x16x32_f16 v[16:19], v[88:91], v[136:139], v[16:19]\n\t")
        "s_add_u32 s96, s96, s95\n\t"
        "s_addc_u32 s97, s97, 0\n\t"
        "ds_read_b128 v[136:139], v165 offset:11264\n\t"
        "s_waitcnt lgkmcnt(9)\n\t"
        MF("v_mfma_f32_16x16x32_f16 v[20:23], v[88:91], v[140:143], v[20:23]\n\t")
        "v_lshrrev_b32 v85, 8, %[l16]\n\t"
        "v_and_b32 v84, 0xff, %[l16]\n\t"
        "ds_read_b128 v[140:143], v165 offset:12288\n\t"
        "s_waitcnt lgkmcnt(9)\n\t"
        MF("v_mfma_f32_16x16x32_f16 v[24:27], v[88:91], v[144:147], v[24:27]\n\t")
        "v_lshl_or_b32 v84, v85, 15, v84\n\t"
        "ds_read_b128 v[144:147], v165 offset:13312\n\t"
        "s_waitcnt lgkmcnt(9)\n\t"
        MF("v_mfma_f32_16x16x32_f16 v[28:31], v[88:91], v[148:151], v[28:31]\n\t")
        "v_or_b32 v86, 0x4000, v84\n\t"
        "v_or_b32 v87, 0x6000, v84\n\t"
        "ds_read_b128 v[148:151], v165 offset:14336\n\t"
        "s_waitcnt lgkmcnt(9)\n\t"
        MF("v_mfma_f32_16x16x32_f16 v[32:35], v[88:91], v[152:155], v[32:35]\n\t")
        "ds_read_b128 v[152:155], v165 offset:15360\n\t"
        "s_waitcnt lgkmcnt(9)\n\t"
        MF("v_mfma_f32_16x16x32_f16 v[36:39], v[88:91], v[156:159], v[36:39]\n\t")
        "ds_read_b128 v[156:159], v165 offset:16384\n\t"
        "s_waitcnt lgkmcnt(9)\n\t"
        MF("v_mfma_f32_16x16x32_f16 v[40:43], v[160:163], v[88:91], v[40:43]\n\t")
        "ds_read_b128 v[160:163], v165 offset:17408\n\t"
        "s_waitcnt lgkmcnt(9)\n\t"
        MF("v_mfma_f32_16x16x32_f16 v[8:11], v[92:95], v[128:131], v[8:11]\n\t")
        "ds_read_b32 v164, %[vctl]\n\t"
        "s_movk_i32 s89, 123\n\t"
        "ds_read_b128 v[128:131], v165 offset:18432\n\t"
        "s_waitcnt lgkmcnt(9)\n\t"
        MF("v_mfma_f32_16x16x32_f16 v[12:15], v[92:95], v[132:135], v[12:15]\n\t")
        "ds_read_b128 v[132:135], v165 offset:19456\n\t"
        "s_waitcnt lgkmcnt(9)\n\t"
        MF("v_mfma_f32_16x16x32_f16 v[16:19], v[92:95], v[136:139], v[16:19]\n\t")
        "ds_read_b128 v[136:139], v165 offset:20480\n\t"
        "s_waitcnt lgkmcnt(9)\n\t"
        MF("v_mfma_f32_16x16x32_f16 v[20:23], v[92:95], v[140:143], v[20:23]\n\t")
        "ds_read_b128 v[140:143], v165 offset:21504\n\t"
        "s_waitcnt lgkmcnt(9)\n\t"
        MF("v_mfma_f32_16x16x32_f16 v[24:27], v[92:95], v[144:147], v[24:27]\n\t")
        "ds_read_b128 v[144:147], v165 offset:22528\n\t"
        "s_waitcnt lgkmcnt(9)\n\t"
        MF("v_mfma_f32_16x16x32_f16 v[28:31], v[92:95], v[148:151], v[28:31]\n\t")
        "ds_read_b128 v[148:151], v165 offset:23552\n\t"
        "s_waitcnt lgkmcnt(9)\n\t"
        "s_waitcnt vmcnt(2)\n\t"
        MF("v_mfma_f32_16x16x32_f16 v[32:35], v[92:95], v[152:155], v[32:35]\n\t")
        "ds_read_b128 v[152:155], v165 offset:24576\n\t"
        "v_bitop3_b32 v166, v96, v100, %[tag] bitop3:0x7e\n\t"
        "v_and_b32 v166, 1, v166\n\t"
        "s_waitcnt lgkmcnt(7)\n\t"
        MF("v_mfma_f32_16x16x32_f16 v[36:39], v[92:95], v[156:159], v[36:39]\n\t")
        "ds_read_b128 v[156:159], v165 offset:25600\n\t"
        TG("v_cmp_ne_u32_e64 s[84:85], 0, v166\n\t")
        NTG("s_mov_b64 s[84:85], 0\n\t")
        "v_cmp_gt_u32_e64 s[86:87], s89, v164\n\t"
        "s_waitcnt lgkmcnt(9)\n\t"
        MF("v_mfma_f32_16x16x32_f16 v[40:43], v[160:163], v[92:95], v[40:43]\n\t")
        "ds_read_b128 v[160:163], v165 offset:26624\n\t"
        "s_and_b64 s[86:87], s[86:87], 0x400\n\t"
        "s_or_b64 s[90:91], s[84:85], s[86:87]\n\t"
        "s_cbranch_scc1 FIXT4_%=\n\t"
        "TAIL4_%=:\n\t"
        TRM("s_memrealtime s[90:91]\n\t")
        TRM("s_lshl_b32 s93, %[stage], 2\n\t")
        TRM("s_and_b32 s93, s93, 8\n\t")
        TRM("s_add_u32 s93, s93, 12\n\t")
        TRM("s_mov_b32 m0, s93\n\t")
        TRM("s_waitcnt lgkmcnt(0)\n\t")
        TRM("v_writelane_b32 %[tr], s90, m0\n\t")
        "s_waitcnt lgkmcnt(8)\n\t"
        MF("v_mfma_f32_16x16x32_f16 v[8:11], v[96:99], v[128:131], v[8:11]\n\t")
        "global_load_dwordx4 v[112:115], %[l16], s[96:97] offset:0 " POL_FAR "\n\t"
        "ds_read_b128 v[128:131], v165 offset:27648\n\t"
        "s_waitcnt lgkmcnt(8)\n\t"
        MF("v_mfma_f32_16x16x32_f16 v[12:15], v[96:99], v[132:135], v[12:15]\n\t")
        "global_load_dwordx4 v[116:119], %[l16], s[96:97] offset:1024 " POL_FAR "\n\t"
        "ds_read_b128 v[132:135], v165 offset:28672\n\t"
        "s_waitcnt lgkmcnt(8)\n\t"
        MF("v_mfma_f32_16x16x32_f16 v[16:19], v[96:99], v[136:139], v[16:19]\n\t")
        "s_add_u32 s96, s96, s95\n\t"
        "s_addc_u32 s97, s97, 0\n\t"
        "ds_read_b128 v[136:139], v165 offset:29696\n\t"
        "s_waitcnt lgkmcnt(8)\n\t"
        MF("v_mfma_f32_16x16x32_f16 v[20:23], v[96:99], v[140:143], v[20:23]\n\t")
        "s_waitcnt lgkmcnt(15)\n\t"
        "v_add_u32 v80, v80, v81\n\t"
        "ds_read_b128 v[140:143], v165 offset:30720\n\t"
        "s_waitcnt lgkmcnt(8)\n\t"
        MF("v_mfma_f32_16x16x32_f16 v[24:27], v[96:99], v[144:147], v[24:27]\n\t")
        "v_readfirstlane_b32 s92, v80\n\t"
        "ds_read_b128 v[144:147], v165 offset:31744\n\t"
        "s_waitcnt lgkmcnt(8)\n\t"
        MF("v_mfma_f32_16x16x32_f16 v[28:31], v[96:99], v[148:151], v[28:31]\n\t")
        "s_add_u32 s93, s92, 63\n\t"
        "s_lshl_b32 s93, s93, 23\n\t"
        "s_sub_u32 s88, 0xff38aa3b, s93\n\t"
        "ds_read_b128 v[148:151], v165 offset:32768\n\t"
        "s_waitcnt lgkmcnt(8)\n\t"
        MF("v_mfma_f32_16x16x32_f16 v[32:35], v[96:99], v[152:155], v[32:35]\n\t")
        "v_mov_b32 v83, s93\n\t"
        "s_cmp_gt_i32 s92, 104\n\t"
        "s_mov_b32 s98, 0x2b3cedc7\n\t"
        "s_mov_b32 s92, 0xc2ad1f97\n\t"
        "s_cmov_b32 s98, 0x6000000\n\t"
        "s_cmov_b32 s92, 0xc1400000\n\t"
        "ds_read_b128 v[152:155], v165 offset:33792\n\t"
        "s_waitcnt lgkmcnt(8)\n\t"
        MF("v_mfma_f32_16x16x32_f16 v[36:39], v[96:99], v[156:159], v[36:39]\n\t")
        "s_add_u32 s98, s98, s93\n\t"
        "v_mov_b32 v82, s92\n\t"
        "ds_read_b128 v[156:159], v165 offset:34816\n\t"
        "s_waitcnt lgkmcnt(8)\n\t"
        MF("v_mfma_f32_16x16x32_f16 v[40:43], v[160:163], v[96:99], v[40:43]\n\t")
        "ds_read_b128 v[160:163], v165 offset:35840\n\t"
        "s_waitcnt lgkmcnt(8)\n\t"
        MF("v_mfma_f32_16x16x32_f16 v[8:11], v[100:103], v[128:131], v[8:11]\n\t")
        "ds_read_b32 v164, %[vctl]\n\t"
        "s_movk_i32 s89, 125\n\t"
        "ds_read_b128 v[128:131], v165 offset:36864\n\t"
        "s_waitcnt lgkmcnt(9)\n\t"
        MF("v_mfma_f32_16x16x32_f16 v[12:15], v[100:103], v[132:135], v[12:15]\n\t")
        "ds_read_b128 v[132:135], v165 offset:37888\n\t"
        "s_waitcnt lgkmcnt(9)\n\t"
        MF("v_mfma_f32_16x16x32_f16 v[16:19], v[100:103], v[136:139], v[16:19]\n\t")
        "ds_read_b128 v[136:139], v165 offset:38912\n\t"
        "s_waitcnt lgkmcnt(9)\n\t"
        MF("v_mfma_f32_16x16x32_f16 v[20:23], v[100:103], v[140:143], v[20:23]\n\t")
        "ds_read_b128 v[140:143], v165 offset:39936\n\t"
        "s_waitcnt lgkmcnt(9)\n\t"
        MF("v_mfma_f32_16x16x32_f16 v[24:27], v[100:103], v[144:147], v[24:27]\n\t")
        "ds_read_b128 v[144:147], v165 offset:40960\n\t"
        "s_waitcnt lgkmcnt(9)\n\t"
        MF("v_mfma_f32_16x16x32_f16 v[28:31], v[100:103], v[148:151], v[28:31]\n\t")
        "ds_read_b128 v[148:151], v165 offset:41984\n\t"
        "s_waitcnt lgkmcnt(9)\n\t"
        "s_waitcnt vmcnt(2)\n\t"
        MF("v_mfma_f32_16x16x32_f16 v[32:35], v[100:103], v[152:155], v[32:35]\n\t")
        "ds_read_b128 v[152:155], v165 offset:43008\n\t"
        "v_bitop3_b32 v166, v104, v108, %[tag] bitop3:0x7e\n\t"
        "v_and_b32 v166, 1, v166\n\t"
        "s_waitcnt lgkmcnt(7)\n\t"
        MF("v_mfma_f32_16x16x32_f16 v[36:39], v[100:103], v[156:159], v[36:39]\n\t")
        "ds_read_b128 v[156:159], v165 offset:44032\n\t"
        TG("v_cmp_ne_u32_e64 s[84:85], 0, v166\n\t")
        NTG("s_mov_b64 s[84:85], 0\n\t")
        "v_cmp_gt_u32_e64 s[86:87], s89, v164\n\t"
        "s_waitcnt lgkmcnt(9)\n\t"
        MF("v_mfma_f32_16x16x32_f16 v[40:43], v[160:163], v[100:103], v[40:43]\n\t")
        "ds_read_b128 v[160:163], v165 offset:45056\n\t"
        "s_and_b64 s[86:87], s[86:87], 0x400\n\t"
        "s_or_b64 s[90:91], s[84:85], s[86:87]\n\t"
        "s_cbranch_scc1 FIXT5_%=\n\t"
        "TAIL5_%=:\n\t"
        TRM("s_memrealtime s[90:91]\n\t")
        TRM("s_lshl_b32 s93, %[stage], 2\n\t")
        TRM("s_and_b32 s93, s93, 8\n\t")
        TRM("s_add_u32 s93, s93, 13\n\t")
        TRM("s_mov_b32 m0, s93\n\t")
        TRM("s_waitcnt lgkmcnt(0)\n\t")
        TRM("v_writelane_b32 %[tr], s90, m0\n\t")
        "s_waitcnt lgkmcnt(8)\n\t"
        MF("v_mfma_f32_16x16x32_f16 v[8:11], v[104:107], v[128:131], v[8:11]\n\t")
        "global_load_dwordx4 v[120:123], %[l16], s[96:97] offset:0 " POL_FAR "\n\t"
        "ds_read_b128 v[128:131], v165 offset:46080\n\t"
        "s_waitcnt lgkmcnt(8)\n\t"
        MF("v_mfma_f32_16x16x32_f16 v[12:15], v[104:107], v[132:135], v[12:15]\n\t")
        "global_load_dwordx4 v[124:127], %[l16], s[96:97] offset:1024 " POL_FAR "\n\t"
        "ds_read_b128 v[132:135], v165 offset:47104\n\t"
        "s_waitcnt lgkmcnt(8)\n\t"
        MF("v_mfma_f32_16x16x32_f16 v[16:19], v[104:107], v[136:139], v[16:19]\n\t")
        "s_add_u32 s96, s96, s95\n\t"
        "s_addc_u32 s97, s97, 0\n\t"
        "v_mov_b32 v166, %[bad]\n\t"
        "s_mov_b64 exec, 1\n\t"
        "ds_add_u32 v166, v167 offset:" KTOK "\n\t"
        "s_mov_b64 exec, -1\n\t"
        "ds_read_b128 v[136:139], v165 offset:48128\n\t"
        "s_waitcnt lgkmcnt(9)\n\t"
        MF("v_mfma_f32_16x16x32_f16 v[20:23], v[104:107], v[140:143], v[20:23]\n\t")
        "v_bfe_u32 v80, %[l16], 4, 4\n\t"
        "v_lshlrev_b32 v85, 4, v85\n\t"
        "ds_read_b128 v[140:143], v165 offset:49152\n\t"
        "s_waitcnt lgkmcnt(9)\n\t"
        MF("v_mfma_f32_16x16x32_f16 v[24:27], v[104:107], v[144:147], v[24:27]\n\t")
        "v_lshl_or_b32 v80, v80, 13, v85\n\t"
        "ds_read_b128 v[144:147], v165 offset:50176\n\t"
        "s_waitcnt lgkmcnt(9)\n\t"
        MF("v_mfma_f32_16x16x32_f16 v[28:31], v[104:107], v[148:151], v[28:31]\n\t")
        "v_add_u32 v80, %[xc], v80\n\t"
        "ds_read_b128 v[148:151], v165 offset:51200\n\t"
        "s_waitcnt lgkmcnt(9)\n\t"
        MF("v_mfma_f32_16x16x32_f16 v[32:35], v[104:107], v[152:155], v[32:35]\n\t")
        "v_or_b32 v85, 0x2000, v84\n\t"
        "ds_read_b128 v[152:155], v165 offset:52224\n\t"
        "s_waitcnt lgkmcnt(9)\n\t"
        MF("v_mfma_f32_16x16x32_f16 v[36:39], v[104:107], v[156:159], v[36:39]\n\t")
        "ds_read_b128 v[156:159], v165 offset:53248\n\t"
        "s_waitcnt lgkmcnt(9)\n\t"
        MF("v_mfma_f32_16x16x32_f16 v[40:43], v[160:163], v[104:107], v[40:43]\n\t")
        "ds_read_b128 v[160:163], v165 offset:54272\n\t"
        "s_waitcnt lgkmcnt(9)\n\t"
        MF("v_mfma_f32_16x16x32_f16 v[8:11], v[108:111], v[128:131], v[8:11]\n\t")
        "ds_read_b32 v164, %[vctl]\n\t"
        "s_movk_i32 s89, 128\n\t"
        "v_add_u32 v165, 55296, v165\n\t"
        "ds_read_b128 v[128:131], v165 offset:0\n\t"
        "s_waitcnt lgkmcnt(10)\n\t"
        MF("v_mfma_f32_16x16x32_f16 v[12:15], v[108:111], v[132:135], v[12:15]\n\t")
        "ds_read_b128 v[132:135], v165 offset:1024\n\t"
        "s_waitcnt lgkmcnt(9)\n\t"
        MF("v_mfma_f32_16x16x32_f16 v[16:19], v[108:111], v[136:139], v[16:19]\n\t")
        "ds_read_b128 v[136:139], v165 offset:2048\n\t"
        "s_waitcnt lgkmcnt(9)\n\t"
        MF("v_mfma_f32_16x16x32_f16 v[20:23], v[108:111], v[140:143], v[20:23]\n\t")
        "ds_read_b128 v[140:143], v165 offset:3072\n\t"
        "s_waitcnt lgkmcnt(9)\n\t"
        MF("v_mfma_f32_16x16x32_f16 v[24:27], v[108:111], v[144:147], v[24:27]\n\t")
        "ds_read_b128 v[144:147], v165 offset:4096\n\t"
        "s_waitcnt lgkmcnt(9)\n\t"
        MF("v_mfma_f32_16x16x32_f16 v[28:31], v[108:111], v[148:151], v[28:31]\n\t")
        "ds_read_b128 v[148:151], v165 offset:5120\n\t"
        "s_waitcnt lgkmcnt(9)\n\t"
        "s_waitcnt vmcnt(2)\n\t"
        MF("v_mfma_f32_16x16x32_f16 v[32:35], v[108:111], v[152:155], v[32:35]\n\t")
        "ds_read_b128 v[152:155], v165 offset:6144\n\t"
        "v_bitop3_b32 v166, v112, v116, %[tag] bitop3:0x7e\n\t"
        "v_and_b32 v166, 1, v166\n\t"
        "s_waitcnt lgkmcnt(7)\n\t"
        MF("v_mfma_f32_16x16x32_f16 v[36:39], v[108:111], v[156:159], v[36:39]\n\t")
        "ds_read_b128 v[156:159], v165 offset:7168\n\t"
        TG("v_cmp_ne_u32_e64 s[84:85], 0, v166\n\t")
        NTG("s_mov_b64 s[84:85], 0\n\t")
        "v_cmp_gt_u32_e64 s[86:87], s89, v164\n\t"
        "s_waitcnt lgkmcnt(9)\n\t"
        MF("v_mfma_f32_16x16x32_f16 v[40:43], v[160:163], v[108:111], v[40:43]\n\t")
        "ds_read_b128 v[160:163], v165 offset:8192\n\t"
        "s_and_b64 s[86:87], s[86:87], 0x400\n\t"
        "s_or_b64 s[90:91], s[84:85], s[86:87]\n\t"
        "s_cbranch_scc1 FIXT6_%=\n\t"
        "TAIL6_%=:\n\t"
        TRM("s_memrealtime s[90:91]\n\t")
        TRM("s_lshl_b32 s93, %[stage], 2\n\t")
        TRM("s_and_b32 s93, s93, 8\n\t")
        TRM("s_add_u32 s93, s93, 14\n\t")
        TRM("s_mov_b32 m0, s93\n\t")
        TRM("s_waitcnt lgkmcnt(0)\n\t")
        TRM("v_writelane_b32 %[tr], s90, m0\n\t")
        "s_waitcnt lgkmcnt(8)\n\t"
        MF("v_mfma_f32_16x16x32_f16 v[8:11], v[112:115], v[128:131], v[8:11]\n\t")
        "ds_read_b128 v[128:131], v165 offset:9216\n\t"
        "s_waitcnt lgkmcnt(8)\n\t"
        MF("v_mfma_f32_16x16x32_f16 v[12:15], v[112:115], v[132:135], v[12:15]\n\t")
        "ds_read_b128 v[132:135], v165 offset:10240\n\t"
        "s_waitcnt lgkmcnt(8)\n\t"
        MF("v_mfma_f32_16x16x32_f16 v[16:19], v[112:115], v[136:139], v[16:19]\n\t")
        "ds_read_b128 v[136:139], v165 offset:11264\n\t"
        "s_waitcnt lgkmcnt(8)\n\t"
        MF("v_mfma_f32_16x16x32_f16 v[20:23], v[112:115], v[140:143], v[20:23]\n\t")
        "ds_read_b128 v[140:143], v165 offset:12288\n\t"
        "s_waitcnt lgkmcnt(8)\n\t"
        MF("v_mfma_f32_16x16x32_f16 v[24:27], v[112:115], v[144:147], v[24:27]\n\t")
        "ds_read_b128 v[144:147], v165 offset:13312\n\t"
        "s_waitcnt lgkmcnt(8)\n\t"
        MF("v_mfma_f32_16x16x32_f16 v[28:31], v[112:115], v[148:151], v[28:31]\n\t")
        "ds_read_b128 v[148:151], v165 offset:14336\n\t"
        "s_waitcnt lgkmcnt(8)\n\t"
        MF("v_mfma_f32_16x16x32_f16 v[32:35], v[112:115], v[152:155], v[32:35]\n\t")
        "ds_read_b128 v[152:155], v165 offset:15360\n\t"
        "s_waitcnt lgkmcnt(8)\n\t"
        MF("v_mfma_f32_16x16x32_f16 v[36:39], v[112:115], v[156:159], v[36:39]\n\t")
        "ds_read_b128 v[156:159], v165 offset:16384\n\t"
        "s_waitcnt lgkmcnt(8)\n\t"
        MF("v_mfma_f32_16x16x32_f16 v[40:43], v[160:163], v[112:115], v[40:43]\n\t")
        "ds_read_b128 v[160:163], v165 offset:17408\n\t"
        "s_waitcnt lgkmcnt(8)\n\t"
        MF("v_mfma_f32_16x16x32_f16 v[8:11], v[116:119], v[128:131], v[8:11]\n\t")
        "ds_read_b128 v[128:131], v165 offset:26624\n\t"
        "s_waitcnt lgkmcnt(8)\n\t"
        MF("v_mfma_f32_16x16x32_f16 v[12:15], v[116:119], v[132:135], v[12:15]\n\t")
        "ds_read_b128 v[132:135], v165 offset:35840\n\t"
        "s_waitcnt lgkmcnt(8)\n\t"
        MF("v_mfma_f32_16x16x32_f16 v[16:19], v[116:119], v[136:139], v[16:19]\n\t")
        "ds_read_b128 v[136:139], v165 offset:18432\n\t"
        "s_waitcnt lgkmcnt(8)\n\t"
        MF("v_mfma_f32_16x16x32_f16 v[20:23], v[116:119], v[140:143], v[20:23]\n\t")
        "ds_read_b128 v[140:143], v165 offset:27648\n\t"
        "s_waitcnt lgkmcnt(8)\n\t"
        MF("v_mfma_f32_16x16x32_f16 v[24:27], v[116:119], v[144:147], v[24:27]\n\t")
        "ds_read_b128 v[144:147], v165 offset:19456\n\t"
        "s_waitcnt lgkmcnt(8)\n\t"
        MF("v_mfma_f32_16x16x32_f16 v[28:31], v[116:119], v[148:151], v[28:31]\n\t")
        "ds_read_b128 v[148:151], v165 offset:28672\n\t"
        "s_waitcnt lgkmcnt(8)\n\t"
        MF("v_mfma_f32_16x16x32_f16 v[32:35], v[116:119], v[152:155], v[32:35]\n\t")
        "ds_read_b128 v[152:155], v165 offset:20480\n\t"
        "s_waitcnt lgkmcnt(8)\n\t"
        MF("v_mfma_f32_16x16x32_f16 v[36:39], v[116:119], v[156:159], v[36:39]\n\t")
        "ds_read_b128 v[156:159], v165 offset:29696\n\t"
        "s_waitcnt lgkmcnt(8)\n\t"
        MF("v_mfma_f32_16x16x32_f16 v[40:43], v[160:163], v[116:119], v[40:43]\n\t")
        "ds_read_b128 v[160:163], v165 offset:21504\n\t"
        "TAIL7_%=:\n\t"
        TRM("s_memrealtime s[90:91]\n\t")
        TRM("s_lshl_b32 s93, %[stage], 2\n\t")
        TRM("s_and_b32 s93, s93, 8\n\t")
        TRM("s_add_u32 s93, s93, 15\n\t")
        TRM("s_mov_b32 m0, s93\n\t")
        TRM("s_waitcnt lgkmcnt(0)\n\t")
        TRM("v_writelane_b32 %[tr], s90, m0\n\t")
        "v_mov_b32 v164, %[bad]\n\t"
        "s_waitcnt lgkmcnt(7)\n\t"
        "s_waitcnt vmcnt(0)\n\t"
        TRM("s_memrealtime s[90:91]\n\t")
        TRM("s_and_b32 s93, %[stage], 2\n\t")
        TRM("s_add_u32 s93, s93, 1\n\t")
        TRM("s_mov_b32 m0, s93\n\t")
        TRM("s_waitcnt lgkmcnt(0)\n\t")
        TRM("v_writelane_b32 %[tr], s90, m0\n\t")
        "v_bitop3_b32 v166, v120, v124, %[tag] bitop3:0x7e\n\t"
        MF("v_mfma_f32_16x16x32_f16 v[116:119], v[128:131], v[120:123], v[40:43]\n\t")
        "v_and_b32 v166, 1, v166\n\t"
        TG("v_cmp_ne_u32_e64 s[84:85], 0, v166\n\t")
        NTG("s_mov_b64 s[84:85], 0\n\t")
        MF("v_mfma_f32_16x16x32_f16 v[116:119], v[132:135], v[124:127], v[116:119]\n\t")
        "s_cmp_eq_u64 s[84:85], 0\n\t"
        "s_cbranch_scc0 FIXT7_%=\n\t"
        "TAILS_%=:\n\t"
        "ds_read_b32 v81, v164\n\t"
        "ds_read_b128 v[128:131], v165 offset:30720\n\t"
        "ds_read_b128 v[132:135], v165 offset:22528\n\t"
        "s_waitcnt lgkmcnt(9)\n\t"
        MF("v_mfma_f32_16x16x32_f16 v[8:11], v[120:123], v[136:139], v[8:11]\n\t")
        "ds_read_b128 v[136:139], v165 offset:31744\n\t"
        "s_waitcnt lgkmcnt(9)\n\t"
        MF("v_mfma_f32_16x16x32_f16 v[8:11], v[124:127], v[140:143], v[8:11]\n\t")
        "ds_read_b128 v[140:143], v165 offset:23552\n\t"
        "v_fma_f32 v112, v116, s88, v82\n\t"
        "v_fma_f32 v113, v117, s88, v82\n\t"
        "s_waitcnt lgkmcnt(9)\n\t"
        MF("v_mfma_f32_16x16x32_f16 v[12:15], v[120:123], v[144:147], v[12:15]\n\t")
        "ds_read_b128 v[144:147], v165 offset:32768\n\t"
        "v_fma_f32 v114, v118, s88, v82\n\t"
        "v_fma_f32 v115, v119, s88, v82\n\t"
        "s_waitcnt lgkmcnt(9)\n\t"
        MF("v_mfma_f32_16x16x32_f16 v[12:15], v[124:127], v[148:151], v[12:15]\n\t")
        "ds_read_b128 v[148:151], v165 offset:24576\n\t"
        "v_exp_f32_e64 v112, v112 clamp\n\t"
        "s_waitcnt lgkmcnt(9)\n\t"
        MF("v_mfma_f32_16x16x32_f16 v[16:19], v[120:123], v[152:155], v[16:19]\n\t")
        "ds_read_b128 v[152:155], v165 offset:33792\n\t"
        "v_exp_f32_e64 v113, v113 clamp\n\t"
        "s_waitcnt lgkmcnt(9)\n\t"
        MF("v_mfma_f32_16x16x32_f16 v[16:19], v[124:127], v[156:159], v[16:19]\n\t")
        "ds_read_b128 v[156:159], v165 offset:25600\n\t"
        "s_waitcnt lgkmcnt(8)\n\t"
        "v_cmp_ne_u32_e64 s[90:91], 0, v81\n\t"
        "v_exp_f32_e64 v114, v114 clamp\n\t"
        "s_waitcnt lgkmcnt(9)\n\t"
        MF("v_mfma_f32_16x16x32_f16 v[20:23], v[120:123], v[160:163], v[20:23]\n\t")
        "ds_read_b128 v[160:163], v165 offset:34816\n\t"
        "v_exp_f32_e64 v115, v115 clamp\n\t"
        "s_waitcnt lgkmcnt(8)\n\t"
        MF("v_mfma_f32_16x16x32_f16 v[20:23], v[124:127], v[128:131], v[20:23]\n\t")
        "s_or_b32 s98, s98, s90\n\t"
        "v_fma_f32 v112, v112, s98, v83\n\t"
        "v_fma_f32 v113, v113, s98, v83\n\t"
        "v_fma_f32 v114, v114, s98, v83\n\t"
        "v_fma_f32 v115, v115, s98, v83\n\t"
        "s_waitcnt lgkmcnt(7)\n\t"
        MF("v_mfma_f32_16x16x32_f16 v[24:27], v[120:123], v[132:135], v[24:27]\n\t")
        "v_rcp_f32 v112, v112\n\t"
        "v_rcp_f32 v113, v113\n\t"
        "s_waitcnt lgkmcnt(6)\n\t"
        MF("v_mfma_f32_16x16x32_f16 v[24:27], v[124:127], v[136:139], v[24:27]\n\t")
        "v_rcp_f32 v114, v114\n\t"
        "v_rcp_f32 v115, v115\n\t"
        "s_waitcnt lgkmcnt(5)\n\t"
        MF("v_mfma_f32_16x16x32_f16 v[28:31], v[120:123], v[140:143], v[28:31]\n\t")
        "v_mul_f32 v112, v116, v112\n\t"
        "v_mul_f32 v113, v117, v113\n\t"
        "v_mul_f32 v114, v118, v114\n\t"
        "v_mul_f32 v115, v119, v115\n\t"
        "s_waitcnt lgkmcnt(4)\n\t"
        MF("v_mfma_f32_16x16x32_f16 v[28:31], v[124:127], v[144:147], v[28:31]\n\t")
        "global_store_dwordx4 v80, v[112:115], %[outp] " POL_OUT "\n\t"
        "v_fma_f32 v88, v8, s88, v82\n\t"
        "v_fma_f32 v89, v12, s88, v82\n\t"
        "v_fma_f32 v90, v16, s88, v82\n\t"
        "s_waitcnt lgkmcnt(3)\n\t"
        MF("v_mfma_f32_16x16x32_f16 v[32:35], v[120:123], v[148:151], v[32:35]\n\t")
        "v_fma_f32 v91, v20, s88, v82\n\t"
        "v_fma_f32 v92, v9, s88, v82\n\t"
        "v_fma_f32 v93, v13, s88, v82\n\t"
        "v_fma_f32 v94, v17, s88, v82\n\t"
        "s_waitcnt lgkmcnt(2)\n\t"
        MF("v_mfma_f32_16x16x32_f16 v[32:35], v[124:127], v[152:155], v[32:35]\n\t")
        "v_fma_f32 v95, v21, s88, v82\n\t"
        "v_exp_f32_e64 v88, v88 clamp\n\t"
        "v_exp_f32_e64 v89, v89 clamp\n\t"
        "s_waitcnt lgkmcnt(1)\n\t"
        MF("v_mfma_f32_16x16x32_f16 v[36:39], v[120:123], v[156:159], v[36:39]\n\t")
        "v_exp_f32_e64 v90, v90 clamp\n\t"
        "v_exp_f32_e64 v91, v91 clamp\n\t"
        "s_waitcnt lgkmcnt(0)\n\t"
        MF("v_mfma_f32_16x16x32_f16 v[36:39], v[124:127], v[160:163], v[36:39]\n\t")
        "s_setprio " KPRE "\n\t"
        "v_exp_f32_e64 v92, v92 clamp\n\t"
        "v_exp_f32_e64 v93, v93 clamp\n\t"
        "v_exp_f32_e64 v94, v94 clamp\n\t"
        "v_exp_f32_e64 v95, v95 clamp\n\t"
        "v_fma_f32 v88, v88, s98, v83\n\t"
        "v_fma_f32 v89, v89, s98, v83\n\t"
        "v_fma_f32 v90, v90, s98, v83\n\t"
        "v_fma_f32 v91, v91, s98, v83\n\t"
        "v_fma_f32 v92, v92, s98, v83\n\t"
        "v_fma_f32 v93, v93, s98, v83\n\t"
        "v_fma_f32 v94, v94, s98, v83\n\t"
        "v_fma_f32 v95, v95, s98, v83\n\t"
        "v_rcp_f32 v88, v88\n\t"
        "v_rcp_f32 v89, v89\n\t"
        "v_rcp_f32 v90, v90\n\t"
        "v_rcp_f32 v91, v91\n\t"
        "v_rcp_f32 v92, v92\n\t"
        "v_rcp_f32 v93, v93\n\t"
        "v_rcp_f32 v94, v94\n\t"
        "v_rcp_f32 v95, v95\n\t"
        "v_mul_f32 v88, v8, v88\n\t"
        "v_mul_f32 v89, v12, v89\n\t"
        "v_mul_f32 v90, v16, v90\n\t"
        "v_mul_f32 v91, v20, v91\n\t"
        "global_store_dwordx4 v84, v[88:91], %[outp] offset:0 " POL_OUT "\n\t"
        "v_mul_f32 v92, v9, v92\n\t"
        "v_mul_f32 v93, v13, v93\n\t"
        "v_mul_f32 v94, v17, v94\n\t"
        "v_mul_f32 v95, v21, v95\n\t"
        "global_store_dwordx4 v85, v[92:95], %[outp] offset:0 " POL_OUT "\n\t"
        "v_fma_f32 v96, v10, s88, v82\n\t"
        "v_fma_f32 v97, v14, s88, v82\n\t"
        "v_fma_f32 v98, v18, s88, v82\n\t"
        "v_fma_f32 v99, v22, s88, v82\n\t"
        "v_fma_f32 v100, v11, s88, v82\n\t"
        "v_fma_f32 v101, v15, s88, v82\n\t"
        "v_fma_f32 v102, v19, s88, v82\n\t"
        "v_fma_f32 v103, v23, s88, v82\n\t"
        "v_exp_f32_e64 v96, v96 clamp\n\t"
        "v_exp_f32_e64 v97, v97 clamp\n\t"
        "v_exp_f32_e64 v98, v98 clamp\n\t"
        "v_exp_f32_e64 v99, v99 clamp\n\t"
        "v_exp_f32_e64 v100, v100 clamp\n\t"
        "v_exp_f32_e64 v101, v101 clamp\n\t"
        "v_exp_f32_e64 v102, v102 clamp\n\t"
        "v_exp_f32_e64 v103, v103 clamp\n\t"
        "v_fma_f32 v96, v96, s98, v83\n\t"
        "v_fma_f32 v97, v97, s98, v83\n\t"
        "v_fma_f32 v98, v98, s98, v83\n\t"
        "v_fma_f32 v99, v99, s98, v83\n\t"
        "v_fma_f32 v100, v100, s98, v83\n\t"
        "v_fma_f32 v101, v101, s98, v83\n\t"
        "v_fma_f32 v102, v102, s98, v83\n\t"
        "v_fma_f32 v103, v103, s98, v83\n\t"
        "v_rcp_f32 v96, v96\n\t"
        "v_rcp_f32 v97, v97\n\t"
        "v_rcp_f32 v98, v98\n\t"
        "v_rcp_f32 v99, v99\n\t"
        "v_rcp_f32 v100, v100\n\t"
        "v_rcp_f32 v101, v101\n\t"
        "v_rcp_f32 v102, v102\n\t"
        "v_rcp_f32 v103, v103\n\t"
        "v_mul_f32 v96, v10, v96\n\t"
        "v_mul_f32 v97, v14, v97\n\t"
        "v_mul_f32 v98, v18, v98\n\t"
        "v_mul_f32 v99, v22, v99\n\t"
        "global_store_dwordx4 v86, v[96:99], %[outp] offset:0 " POL_OUT "\n\t"
        "v_mul_f32 v100, v11, v100\n\t"
        "v_mul_f32 v101, v15, v101\n\t"
        "v_mul_f32 v102, v19, v102\n\t"
        "v_mul_f32 v103, v23, v103\n\t"
        "global_store_dwordx4 v87, v[100:103], %[outp] offset:0 " POL_OUT "\n\t"
        "v_fma_f32 v104, v24, s88, v82\n\t"
        "v_fma_f32 v105, v28, s88, v82\n\t"
        "v_fma_f32 v106, v32, s88, v82\n\t"
        "v_fma_f32 v107, v36, s88, v82\n\t"
        "v_fma_f32 v108, v25, s88, v82\n\t"
        "v_fma_f32 v109, v29, s88, v82\n\t"
        "v_fma_f32 v110, v33, s88, v82\n\t"
        "v_fma_f32 v111, v37, s88, v82\n\t"
        "v_exp_f32_e64 v104, v104 clamp\n\t"
        "v_exp_f32_e64 v105, v105 clamp\n\t"
        "v_exp_f32_e64 v106, v106 clamp\n\t"
        "v_exp_f32_e64 v107, v107 clamp\n\t"
        "v_exp_f32_e64 v108, v108 clamp\n\t"
        "v_exp_f32_e64 v109, v109 clamp\n\t"
        "v_exp_f32_e64 v110, v110 clamp\n\t"
        "v_exp_f32_e64 v111, v111 clamp\n\t"
        "v_fma_f32 v104, v104, s98, v83\n\t"
        "v_fma_f32 v105, v105, s98, v83\n\t"
        "v_fma_f32 v106, v106, s98, v83\n\t"
        "v_fma_f32 v107, v107, s98, v83\n\t"
        "v_fma_f32 v108, v108, s98, v83\n\t"
        "v_fma_f32 v109, v109, s98, v83\n\t"
        "v_fma_f32 v110, v110, s98, v83\n\t"
        "v_fma_f32 v111, v111, s98, v83\n\t"
        "v_rcp_f32 v104, v104\n\t"
        "v_rcp_f32 v105, v105\n\t"
        "v_rcp_f32 v106, v106\n\t"
        "v_rcp_f32 v107, v107\n\t"
        "v_rcp_f32 v108, v108\n\t"
        "v_rcp_f32 v109, v109\n\t"
        "v_rcp_f32 v110, v110\n\t"
        "v_rcp_f32 v111, v111\n\t"
        "v_mul_f32 v104, v24, v104\n\t"
        "v_mul_f32 v105, v28, v105\n\t"
        "v_mul_f32 v106, v32, v106\n\t"
        "v_mul_f32 v107, v36, v107\n\t"
        "global_store_dwordx4 v84, v[104:107], %[outp] offset:256 " POL_OUT "\n\t"
        "v_mul_f32 v108, v25, v108\n\t"
        "v_mul_f32 v109, v29, v109\n\t"
        "v_mul_f32 v110, v33, v110\n\t"
        "v_mul_f32 v111, v37, v111\n\t"
        "global_store_dwordx4 v85, v[108:111], %[outp] offset:256 " POL_OUT "\n\t"
        "v_fma_f32 v112, v26, s88, v82\n\t"
        "v_fma_f32 v113, v30, s88, v82\n\t"
        "v_fma_f32 v114, v34, s88, v82\n\t"
        "v_fma_f32 v115, v38, s88, v82\n\t"
        "v_fma_f32 v116, v27, s88, v82\n\t"
        "v_fma_f32 v117, v31, s88, v82\n\t"
        "v_fma_f32 v118, v35, s88, v82\n\t"
        "v_fma_f32 v119, v39, s88, v82\n\t"
        "v_exp_f32_e64 v112, v112 clamp\n\t"
        "v_exp_f32_e64 v113, v113 clamp\n\t"
        "v_exp_f32_e64 v114, v114 clamp\n\t"
        "v_exp_f32_e64 v115, v115 clamp\n\t"
        "v_exp_f32_e64 v116, v116 clamp\n\t"
        "v_exp_f32_e64 v117, v117 clamp\n\t"
        "v_exp_f32_e64 v118, v118 clamp\n\t"
        "v_exp_f32_e64 v119, v119 clamp\n\t"
        "v_fma_f32 v112, v112, s98, v83\n\t"
        "v_fma_f32 v113, v113, s98, v83\n\t"
        "v_fma_f32 v114, v114, s98, v83\n\t"
        "v_fma_f32 v115, v115, s98, v83\n\t"
        "v_fma_f32 v116, v116, s98, v83\n\t"
        "v_fma_f32 v117, v117, s98, v83\n\t"
        "v_fma_f32 v118, v118, s98, v83\n\t"
        "v_fma_f32 v119, v119, s98, v83\n\t"
        "v_rcp_f32 v112, v112\n\t"
        "v_rcp_f32 v113, v113\n\t"
        "v_rcp_f32 v114, v114\n\t"
        "v_rcp_f32 v115, v115\n\t"
        "v_rcp_f32 v116, v116\n\t"
        "v_rcp_f32 v117, v117\n\t"
        "v_rcp_f32 v118, v118\n\t"
        "v_rcp_f32 v119, v119\n\t"
        "v_mul_f32 v112, v26, v112\n\t"
        "v_mul_f32 v113, v30, v113\n\t"
        "v_mul_f32 v114, v34, v114\n\t"
        "v_mul_f32 v115, v38, v115\n\t"
        "global_store_dwordx4 v86, v[112:115], %[outp] offset:256 " POL_OUT "\n\t"
        "v_mul_f32 v116, v27, v116\n\t"
        "v_mul_f32 v117, v31, v117\n\t"
        "v_mul_f32 v118, v35, v118\n\t"
        "v_mul_f32 v119, v39, v119\n\t"
        "global_store_dwordx4 v87, v[116:119], %[outp] offset:256 " POL_OUT "\n\t"
        TRM("s_memrealtime s[90:91]\n\t")
        TRM("s_lshl_b32 s93, %[stage], 1\n\t")
        TRM("s_and_b32 s93, s93, 4\n\t")
        TRM("s_lshr_b32 s93, s93, 1\n\t")
        TRM("s_add_u32 s93, s93, 2\n\t")
        TRM("s_mov_b32 m0, s93\n\t")
        TRM("s_waitcnt lgkmcnt(0)\n\t")
        TRM("v_writelane_b32 %[tr], s90, m0\n\t")
        "s_cmp_gt_u32 %[stage], 1\n\t"
        "s_cbranch_scc1 DONE_%=\n\t"
        "s_add_u32 %[stage], %[stage], 2\n\t"
        "s_mov_b64 %[outp], %[out2]\n\t"
        "s_mov_b64 %[xtail], %[xtl2]\n\t"
        "s_branch ENTRY2_%=\n\t"
        "WAITB_%=:\n\t"
        "s_mov_b32 s94, 0\n\t"
        "SPINWB_%=:\n\t"
        "ds_read_b32 v164, %[vctl]\n\t"
        "s_waitcnt lgkmcnt(0)\n\t"
        "v_cmp_gt_u32_e64 s[86:87], s89, v164\n\t"
        "s_and_b64 s[86:87], s[86:87], 0x400\n\t"
        "s_cbranch_scc0 TAILX_%=\n\t"
        SC3("s_add_u32 %[nfix], %[nfix], 1\n\t")
        "s_sleep 1\n\t"
        "s_add_u32 s94, s94, 1\n\t"
        "s_and_b32 s92, s94, 0x3ff\n\t"
        "s_cbranch_scc1 SPINWB_%=\n\t"
        "s_memrealtime s[92:93]\n\t"
        "s_waitcnt lgkmcnt(0)\n\t"
        "s_cmp_eq_u32 s94, 0x400\n\t"
        "s_cselect_b32 s90, s92, s90\n\t"
        "s_sub_u32 s92, s92, s90\n\t"
        "s_cmp_lt_u32 s92, " KDL3 "\n\t"
        "s_cbranch_scc1 SPINWB_%=\n\t"
        "v_mov_b32 v166, %[bad]\n\t"
        "ds_write_b32 v166, v166\n\t"
        "s_branch TAILX_%=\n\t"
        "FIXT0_%=:\n\t"
        "s_cmp_eq_u64 s[84:85], 0\n\t"
        "s_cbranch_scc1 NOFIXT0_%=\n\t"
        "s_add_u32 %[nfix], %[nfix], 1\n\t"
        "v_mov_b32 v166, %[bad]\n\t"
        "ds_read_b32 v164, v166\n\t"
        "s_mov_b64 s[86:87], %[xtail]\n\t"
        "s_mov_b32 s92, 0\n\t"
        "s_add_u32 s86, s86, s92\n\t"
        "s_addc_u32 s87, s87, 0\n\t"
        "s_memrealtime s[90:91]\n\t"
        "s_waitcnt lgkmcnt(0)\n\t"
        "v_cmp_ne_u32_e64 s[92:93], 0, v164\n\t"
        "s_cmp_lg_u64 s[92:93], 0\n\t"
        "s_cbranch_scc1 NOFIXT0_%=\n\t"
        "FIXLT0_%=:\n\t"
        "global_load_dwordx4 v[112:115], %[l16], s[86:87] offset:0 sc0 sc1\n\t"
        "global_load_dwordx4 v[116:119], %[l16], s[86:87] offset:1024 sc0 sc1\n\t"
        "s_waitcnt vmcnt(0)\n\t"
        "v_bitop3_b32 v166, v112, v116, %[tag] bitop3:0x7e\n\t"
        "v_and_b32 v166, 1, v166\n\t"
        "v_cmp_ne_u32_e64 s[92:93], 0, v166\n\t"
        "s_cmp_eq_u64 s[92:93], 0\n\t"
        "s_cbranch_scc1 NOFIXT0_%=\n\t"
        "s_sleep 1\n\t"
        "s_memrealtime s[92:93]\n\t"
        "s_waitcnt lgkmcnt(0)\n\t"
        "s_sub_u32 s92, s92, s90\n\t"
        "s_cmp_lt_u32 s92, " KDL "\n\t"
        "s_cbranch_scc1 FIXLT0_%=\n\t"
        "v_mov_b32 v166, %[bad]\n\t"
        "ds_write_b32 v166, v166\n\t"
        "NOFIXT0_%=:\n\t"
        "s_mov_b32 s94, 0\n\t"
        "SPINT0_%=:\n\t"
        "ds_read_b32 v164, %[vctl]\n\t"
        "s_waitcnt lgkmcnt(0)\n\t"
        "v_cmp_gt_u32_e64 s[86:87], s89, v164\n\t"
        "s_and_b64 s[86:87], s[86:87], 0x400\n\t"
        "s_cbranch_scc0 GO1_%=\n\t"
        SC3("s_add_u32 %[nfix], %[nfix], 1\n\t")
        "s_sleep 1\n\t"
        "s_add_u32 s94, s94, 1\n\t"
        "s_and_b32 s92, s94, 0x3ff\n\t"
        "s_cbranch_scc1 SPINT0_%=\n\t"
        "s_memrealtime s[92:93]\n\t"
        "s_waitcnt lgkmcnt(0)\n\t"
        "s_cmp_eq_u32 s94, 0x400\n\t"
        "s_cselect_b32 s90, s92, s90\n\t"
        "s_sub_u32 s92, s92, s90\n\t"
        "s_cmp_lt_u32 s92, " KDL3 "\n\t"
        "s_cbranch_scc1 SPINT0_%=\n\t"
        "v_mov_b32 v166, %[bad]\n\t"
        "ds_write_b32 v166, v166\n\t"
        "s_branch GO1_%=\n\t"
        "FIXT1_%=:\n\t"
        "s_cmp_eq_u64 s[84:85], 0\n\t"
        "s_cbranch_scc1 NOFIXT1_%=\n\t"
        "s_add_u32 %[nfix], %[nfix], 1\n\t"
        "v_mov_b32 v166, %[bad]\n\t"
        "ds_read_b32 v164, v166\n\t"
        "s_mov_b64 s[86:87], %[xtail]\n\t"
        "s_mov_b32 s92, 8192\n\t"
        "s_add_u32 s86, s86, s92\n\t"
        "s_addc_u32 s87, s87, 0\n\t"
        "s_memrealtime s[90:91]\n\t"
        "s_waitcnt lgkmcnt(0)\n\t"
        "v_cmp_ne_u32_e64 s[92:93], 0, v164\n\t"
        "s_cmp_lg_u64 s[92:93], 0\n\t"
        "s_cbranch_scc1 NOFIXT1_%=\n\t"
        "FIXLT1_%=:\n\t"
        "global_load_dwordx4 v[120:123], %[l16], s[86:87] offset:0 sc0 sc1\n\t"
        "global_load_dwordx4 v[124:127], %[l16], s[86:87] offset:1024 sc0 sc1\n\t"
        "s_waitcnt vmcnt(0)\n\t"
        "v_bitop3_b32 v166, v120, v124, %[tag] bitop3:0x7e\n\t"
        "v_and_b32 v166, 1, v166\n\t"
        "v_cmp_ne_u32_e64 s[92:93], 0, v166\n\t"
        "s_cmp_eq_u64 s[92:93], 0\n\t"
        "s_cbranch_scc1 NOFIXT1_%=\n\t"
        "s_sleep 1\n\t"
        "s_memrealtime s[92:93]\n\t"
        "s_waitcnt lgkmcnt(0)\n\t"
        "s_sub_u32 s92, s92, s90\n\t"
        "s_cmp_lt_u32 s92, " KDL "\n\t"
        "s_cbranch_scc1 FIXLT1_%=\n\t"
        "v_mov_b32 v166, %[bad]\n\t"
        "ds_write_b32 v166, v166\n\t"
        "NOFIXT1_%=:\n\t"
        "s_mov_b32 s94, 0\n\t"
        "SPINT1_%=:\n\t"
        "ds_read_b32 v164, %[vctl]\n\t"
        "s_waitcnt lgkmcnt(0)\n\t"
        "v_cmp_gt_u32_e64 s[86:87], s89, v164\n\t"
        "s_and_b64 s[86:87], s[86:87], 0x400\n\t"
        "s_cbranch_scc0 TAIL1_%=\n\t"
        SC3("s_add_u32 %[nfix], %[nfix], 1\n\t")
        "s_sleep 1\n\t"
        "s_add_u32 s94, s94, 1\n\t"
        "s_and_b32 s92, s94, 0x3ff\n\t"
        "s_cbranch_scc1 SPINT1_%=\n\t"
        "s_memrealtime s[92:93]\n\t"
        "s_waitcnt lgkmcnt(0)\n\t"
        "s_cmp_eq_u32 s94, 0x400\n\t"
        "s_cselect_b32 s90, s92, s90\n\t"
        "s_sub_u32 s92, s92, s90\n\t"
        "s_cmp_lt_u32 s92, " KDL3 "\n\t"
        "s_cbranch_scc1 SPINT1_%=\n\t"
        "v_mov_b32 v166, %[bad]\n\t"
        "ds_write_b32 v166, v166\n\t"
        "s_branch TAIL1_%=\n\t"
        "FIXT2_%=:\n\t"
        "s_cmp_eq_u64 s[84:85], 0\n\t"
        "s_cbranch_scc1 NOFIXT2_%=\n\t"
        "s_add_u32 %[nfix], %[nfix], 1\n\t"
        "v_mov_b32 v166, %[bad]\n\t"
        "ds_read_b32 v164, v166\n\t"
        "s_mov_b64 s[86:87], %[xtail]\n\t"
        "s_mov_b32 s92, 16384\n\t"
        "s_add_u32 s86, s86, s92\n\t"
        "s_addc_u32 s87, s87, 0\n\t"
        "s_memrealtime s[90:91]\n\t"
        "s_waitcnt lgkmcnt(0)\n\t"
        "v_cmp_ne_u32_e64 s[92:93], 0, v164\n\t"
        "s_cmp_lg_u64 s[92:93], 0\n\t"
        "s_cbranch_scc1 NOFIXT2_%=\n\t"
        "FIXLT2_%=:\n\t"
        "global_load_dwordx4 v[80:83], %[l16], s[86:87] offset:0 sc0 sc1\n\t"
        "global_load_dwordx4 v[84:87], %[l16], s[86:87] offset:1024 sc0 sc1\n\t"
        "s_waitcnt vmcnt(0)\n\t"
        "v_bitop3_b32 v166, v80, v84, %[tag] bitop3:0x7e\n\t"
        "v_and_b32 v166, 1, v166\n\t"
        "v_cmp_ne_u32_e64 s[92:93], 0, v166\n\t"
        "s_cmp_eq_u64 s[92:93], 0\n\t"
        "s_cbranch_scc1 NOFIXT2_%=\n\t"
        "s_sleep 1\n\t"
        "s_memrealtime s[92:93]\n\t"
        "s_waitcnt lgkmcnt(0)\n\t"
        "s_sub_u32 s92, s92, s90\n\t"
        "s_cmp_lt_u32 s92, " KDL "\n\t"
        "s_cbranch_scc1 FIXLT2_%=\n\t"
        "v_mov_b32 v166, %[bad]\n\t"
        "ds_write_b32 v166, v166\n\t"
        "NOFIXT2_%=:\n\t"
        "s_mov_b32 s94, 0\n\t"
        "SPINT2_%=:\n\t"
        "ds_read_b32 v164, %[vctl]\n\t"
        "s_waitcnt lgkmcnt(0)\n\t"
        "v_cmp_gt_u32_e64 s[86:87], s89, v164\n\t"
        "s_and_b64 s[86:87], s[86:87], 0x400\n\t"
        "s_cbranch_scc0 TAIL2_%=\n\t"
        SC3("s_add_u32 %[nfix], %[nfix], 1\n\t")
        "s_sleep 1\n\t"
        "s_add_u32 s94, s94, 1\n\t"
        "s_and_b32 s92, s94, 0x3ff\n\t"
        "s_cbranch_scc1 SPINT2_%=\n\t"
        "s_memrealtime s[92:93]\n\t"
        "s_waitcnt lgkmcnt(0)\n\t"
        "s_cmp_eq_u32 s94, 0x400\n\t"
        "s_cselect_b32 s90, s92, s90\n\t"
        "s_sub_u32 s92, s92, s90\n\t"
        "s_cmp_lt_u32 s92, " KDL3 "\n\t"
        "s_cbranch_scc1 SPINT2_%=\n\t"
        "v_mov_b32 v166, %[bad]\n\t"
        "ds_write_b32 v166, v166\n\t"
        "s_branch TAIL2_%=\n\t"
        "FIXT3_%=:\n\t"
        "s_cmp_eq_u64 s[84:85], 0\n\t"
        "s_cbranch_scc1 NOFIXT3_%=\n\t"
        "s_add_u32 %[nfix], %[nfix], 1\n\t"
        "v_mov_b32 v166, %[bad]\n\t"
        "ds_read_b32 v164, v166\n\t"
        "s_mov_b64 s[86:87], %[xtail]\n\t"
        "s_mov_b32 s92, 24576\n\t"
        "s_add_u32 s86, s86, s92\n\t"
        "s_addc_u32 s87, s87, 0\n\t"
        "s_memrealtime s[90:91]\n\t"
        "s_waitcnt lgkmcnt(0)\n\t"
        "v_cmp_ne_u32_e64 s[92:93], 0, v164\n\t"
        "s_cmp_lg_u64 s[92:93], 0\n\t"
        "s_cbranch_scc1 NOFIXT3_%=\n\t"
        "FIXLT3_%=:\n\t"
        "global_load_dwordx4 v[88:91], %[l16], s[86:87] offset:0 sc0 sc1\n\t"
        "global_load_dwordx4 v[92:95], %[l16], s[86:87] offset:1024 sc0 sc1\n\t"
        "s_waitcnt vmcnt(0)\n\t"
        "v_bitop3_b32 v166, v88, v92, %[tag] bitop3:0x7e\n\t"
        "v_and_b32 v166, 1, v166\n\t"
        "v_cmp_ne_u32_e64 s[92:93], 0, v166\n\t"
        "s_cmp_eq_u64 s[92:93], 0\n\t"
        "s_cbranch_scc1 NOFIXT3_%=\n\t"
        "s_sleep 1\n\t"
        "s_memrealtime s[92:93]\n\t"
        "s_waitcnt lgkmcnt(0)\n\t"
        "s_sub_u32 s92, s92, s90\n\t"
        "s_cmp_lt_u32 s92, " KDL "\n\t"
        "s_cbranch_scc1 FIXLT3_%=\n\t"
        "v_mov_b32 v166, %[bad]\n\t"
        "ds_write_b32 v166, v166\n\t"
        "NOFIXT3_%=:\n\t"
        "s_mov_b32 s94, 0\n\t"
        "SPINT3_%=:\n\t"
        "ds_read_b32 v164, %[vctl]\n\t"
        "s_waitcnt lgkmcnt(0)\n\t"
        "v_cmp_gt_u32_e64 s[86:87], s89, v164\n\t"
        "s_and_b64 s[86:87], s[86:87], 0x400\n\t"
        "s_cbranch_scc0 TAIL3_%=\n\t"
        SC3("s_add_u32 %[nfix], %[nfix], 1\n\t")
        "s_sleep 1\n\t"
        "s_add_u32 s94, s94, 1\n\t"
        "s_and_b32 s92, s94, 0x3ff\n\t"
        "s_cbranch_scc1 SPINT3_%=\n\t"
        "s_memrealtime s[92:93]\n\t"
        "s_waitcnt lgkmcnt(0)\n\t"
        "s_cmp_eq_u32 s94, 0x400\n\t"
        "s_cselect_b32 s90, s92, s90\n\t"
        "s_sub_u32 s92, s92, s90\n\t"
        "s_cmp_lt_u32 s92, " KDL3 "\n\t"
        "s_cbranch_scc1 SPINT3_%=\n\t"
        "v_mov_b32 v166, %[bad]\n\t"
        "ds_write_b32 v166, v166\n\t"
        "s_branch TAIL3_%=\n\t"
        "FIXT4_%=:\n\t"
        "s_cmp_eq_u64 s[84:85], 0\n\t"
        "s_cbranch_scc1 NOFIXT4_%=\n\t"
        "s_add_u32 %[nfix], %[nfix], 1\n\t"
        "v_mov_b32 v166, %[bad]\n\t"
        "ds_read_b32 v164, v166\n\t"
        "s_mov_b64 s[86:87], %[xtail]\n\t"
        "s_mov_b32 s92, 32768\n\t"
        "s_add_u32 s86, s86, s92\n\t"
        "s_addc_u32 s87, s87, 0\n\t"
        "s_memrealtime s[90:91]\n\t"
        "s_waitcnt lgkmcnt(0)\n\t"
        "v_cmp_ne_u32_e64 s[92:93], 0, v164\n\t"
        "s_cmp_lg_u64 s[92:93], 0\n\t"
        "s_cbranch_scc1 NOFIXT4_%=\n\t"
        "FIXLT4_%=:\n\t"
        "global_load_dwordx4 v[96:99], %[l16], s[86:87] offset:0 sc0 sc1\n\t"
        "global_load_dwordx4 v[100:103], %[l16], s[86:87] offset:1024 sc0 sc1\n\t"
        "s_waitcnt vmcnt(0)\n\t"
        "v_bitop3_b32 v166, v96, v100, %[tag] bitop3:0x7e\n\t"
        "v_and_b32 v166, 1, v166\n\t"
        "v_cmp_ne_u32_e64 s[92:93], 0, v166\n\t"
        "s_cmp_eq_u64 s[92:93], 0\n\t"
        "s_cbranch_scc1 NOFIXT4_%=\n\t"
        "s_sleep 1\n\t"
        "s_memrealtime s[92:93]\n\t"
        "s_waitcnt lgkmcnt(0)\n\t"
        "s_sub_u32 s92, s92, s90\n\t"
        "s_cmp_lt_u32 s92, " KDL "\n\t"
        "s_cbranch_scc1 FIXLT4_%=\n\t"
        "v_mov_b32 v166, %[bad]\n\t"
        "ds_write_b32 v166, v166\n\t"
        "NOFIXT4_%=:\n\t"
        "s_mov_b32 s94, 0\n\t"
        "SPINT4_%=:\n\t"
        "ds_read_b32 v164, %[vctl]\n\t"
        "s_waitcnt lgkmcnt(0)\n\t"
        "v_cmp_gt_u32_e64 s[86:87], s89, v164\n\t"
        "s_and_b64 s[86:87], s[86:87], 0x400\n\t"
        "s_cbranch_scc0 TAIL4_%=\n\t"
        SC3("s_add_u32 %[nfix], %[nfix], 1\n\t")
        "s_sleep 1\n\t"
        "s_add_u32 s94, s94, 1\n\t"
        "s_and_b32 s92, s94, 0x3ff\n\t"
        "s_cbranch_scc1 SPINT4_%=\n\t"
        "s_memrealtime s[92:93]\n\t"
        "s_waitcnt lgkmcnt(0)\n\t"
        "s_cmp_eq_u32 s94, 0x400\n\t"
        "s_cselect_b32 s90, s92, s90\n\t"
        "s_sub_u32 s92, s92, s90\n\t"
        "s_cmp_lt_u32 s92, " KDL3 "\n\t"
        "s_cbranch_scc1 SPINT4_%=\n\t"
        "v_mov_b32 v166, %[bad]\n\t"
        "ds_write_b32 v166, v166\n\t"
        "s_branch TAIL4_%=\n\t"
        "FIXT5_%=:\n\t"
        "s_cmp_eq_u64 s[84:85], 0\n\t"
        "s_cbranch_scc1 NOFIXT5_%=\n\t"
        "s_add_u32 %[nfix], %[nfix], 1\n\t"
        "v_mov_b32 v166, %[bad]\n\t"
        "ds_read_b32 v164, v166\n\t"
        "s_mov_b64 s[86:87], %[xtail]\n\t"
        "s_mov_b32 s92, 40960\n\t"
        "s_add_u32 s86, s86, s92\n\t"
        "s_addc_u32 s87, s87, 0\n\t"
        "s_memrealtime s[90:91]\n\t"
        "s_waitcnt lgkmcnt(0)\n\t"
        "v_cmp_ne_u32_e64 s[92:93], 0, v164\n\t"
        "s_cmp_lg_u64 s[92:93], 0\n\t"
        "s_cbranch_scc1 NOFIXT5_%=\n\t"
        "FIXLT5_%=:\n\t"
        "global_load_dwordx4 v[104:107], %[l16], s[86:87] offset:0 sc0 sc1\n\t"
        "global_load_dwordx4 v[108:111], %[l16], s[86:87] offset:1024 sc0 sc1\n\t"
        "s_waitcnt vmcnt(0)\n\t"
        "v_bitop3_b32 v166, v104, v108, %[tag] bitop3:0x7e\n\t"
        "v_and_b32 v166, 1, v166\n\t"
        "v_cmp_ne_u32_e64 s[92:93], 0, v166\n\t"
        "s_cmp_eq_u64 s[92:93], 0\n\t"
        "s_cbranch_scc1 NOFIXT5_%=\n\t"
        "s_sleep 1\n\t"
        "s_memrealtime s[92:93]\n\t"
        "s_waitcnt lgkmcnt(0)\n\t"
        "s_sub_u32 s92, s92, s90\n\t"
        "s_cmp_lt_u32 s92, " KDL "\n\t"
        "s_cbranch_scc1 FIXLT5_%=\n\t"
        "v_mov_b32 v166, %[bad]\n\t"
        "ds_write_b32 v166, v166\n\t"
        "NOFIXT5_%=:\n\t"
        "s_mov_b32 s94, 0\n\t"
        "SPINT5_%=:\n\t"
        "ds_read_b32 v164, %[vctl]\n\t"
        "s_waitcnt lgkmcnt(0)\n\t"
        "v_cmp_gt_u32_e64 s[86:87], s89, v164\n\t"
        "s_and_b64 s[86:87], s[86:87], 0x400\n\t"
        "s_cbranch_scc0 TAIL5_%=\n\t"
        SC3("s_add_u32 %[nfix], %[nfix], 1\n\t")
        "s_sleep 1\n\t"
        "s_add_u32 s94, s94, 1\n\t"
        "s_and_b32 s92, s94, 0x3ff\n\t"
        "s_cbranch_scc1 SPINT5_%=\n\t"
        "s_memrealtime s[92:93]\n\t"
        "s_waitcnt lgkmcnt(0)\n\t"
        "s_cmp_eq_u32 s94, 0x400\n\t"
        "s_cselect_b32 s90, s92, s90\n\t"
        "s_sub_u32 s92, s92, s90\n\t"
        "s_cmp_lt_u32 s92, " KDL3 "\n\t"
        "s_cbranch_scc1 SPINT5_%=\n\t"
        "v_mov_b32 v166, %[bad]\n\t"
        "ds_write_b32 v166, v166\n\t"
        "s_branch TAIL5_%=\n\t"
        "FIXT6_%=:\n\t"
        "s_cmp_eq_u64 s[84:85], 0\n\t"
        "s_cbranch_scc1 NOFIXT6_%=\n\t"
        "s_add_u32 %[nfix], %[nfix], 1\n\t"
        "v_mov_b32 v166, %[bad]\n\t"
        "ds_read_b32 v164, v166\n\t"
        "s_mov_b64 s[86:87], %[xtail]\n\t"
        "s_mov_b32 s92, 49152\n\t"
        "s_add_u32 s86, s86, s92\n\t"
        "s_addc_u32 s87, s87, 0\n\t"
        "s_memrealtime s[90:91]\n\t"
        "s_waitcnt lgkmcnt(0)\n\t"
        "v_cmp_ne_u32_e64 s[92:93], 0, v164\n\t"
        "s_cmp_lg_u64 s[92:93], 0\n\t"
        "s_cbranch_scc1 NOFIXT6_%=\n\t"
        "FIXLT6_%=:\n\t"
        "global_load_dwordx4 v[112:115], %[l16], s[86:87] offset:0 sc0 sc1\n\t"
        "global_load_dwordx4 v[116:119], %[l16], s[86:87] offset:1024 sc0 sc1\n\t"
        "s_waitcnt vmcnt(0)\n\t"
        "v_bitop3_b32 v166, v112, v116, %[tag] bitop3:0x7e\n\t"
        "v_and_b32 v166, 1, v166\n\t"
        "v_cmp_ne_u32_e64 s[92:93], 0, v166\n\t"
        "s_cmp_eq_u64 s[92:93], 0\n\t"
        "s_cbranch_scc1 NOFIXT6_%=\n\t"
        "s_sleep 1\n\t"
        "s_memrealtime s[92:93]\n\t"
        "s_waitcnt lgkmcnt(0)\n\t"
        "s_sub_u32 s92, s92, s90\n\t"
        "s_cmp_lt_u32 s92, " KDL "\n\t"
        "s_cbranch_scc1 FIXLT6_%=\n\t"
        "v_mov_b32 v166, %[bad]\n\t"
        "ds_write_b32 v166, v166\n\t"
        "NOFIXT6_%=:\n\t"
        "s_mov_b32 s94, 0\n\t"
        "SPINT6_%=:\n\t"
        "ds_read_b32 v164, %[vctl]\n\t"
        "s_waitcnt lgkmcnt(0)\n\t"
        "v_cmp_gt_u32_e64 s[86:87], s89, v164\n\t"
        "s_and_b64 s[86:87], s[86:87], 0x400\n\t"
        "s_cbranch_scc0 TAIL6_%=\n\t"
        SC3("s_add_u32 %[nfix], %[nfix], 1\n\t")
        "s_sleep 1\n\t"
        "s_add_u32 s94, s94, 1\n\t"
        "s_and_b32 s92, s94, 0x3ff\n\t"
        "s_cbranch_scc1 SPINT6_%=\n\t"
        "s_memrealtime s[92:93]\n\t"
        "s_waitcnt lgkmcnt(0)\n\t"
        "s_cmp_eq_u32 s94, 0x400\n\t"
        "s_cselect_b32 s90, s92, s90\n\t"
        "s_sub_u32 s92, s92, s90\n\t"
        "s_cmp_lt_u32 s92, " KDL3 "\n\t"
        "s_cbranch_scc1 SPINT6_%=\n\t"
        "v_mov_b32 v166, %[bad]\n\t"
        "ds_write_b32 v166, v166\n\t"
        "s_branch TAIL6_%=\n\t"
        "FIXT7_%=:\n\t"
        "s_cmp_eq_u64 s[84:85], 0\n\t"
        "s_cbranch_scc1 NOFIXT7_%=\n\t"
        "s_add_u32 %[nfix], %[nfix], 1\n\t"
        "v_mov_b32 v166, %[bad]\n\t"
        "ds_read_b32 v164, v166\n\t"
        "s_mov_b64 s[86:87], %[xtail]\n\t"
        "s_mov_b32 s92, 57344\n\t"
        "s_add_u32 s86, s86, s92\n\t"
        "s_addc_u32 s87, s87, 0\n\t"
        "s_memrealtime s[90:91]\n\t"
        "s_waitcnt lgkmcnt(0)\n\t"
        "v_cmp_ne_u32_e64 s[92:93], 0, v164\n\t"
        "s_cmp_lg_u64 s[92:93], 0\n\t"
        "s_cbranch_scc1 NOFIXT7_%=\n\t"
        "FIXLT7_%=:\n\t"
        "global_load_dwordx4 v[120:123], %[l16], s[86:87] offset:0 sc0 sc1\n\t"
        "global_load_dwordx4 v[124:127], %[l16], s[86:87] offset:1024 sc0 sc1\n\t"
        "s_waitcnt vmcnt(0)\n\t"
        "v_bitop3_b32 v166, v120, v124, %[tag] bitop3:0x7e\n\t"
        "v_and_b32 v166, 1, v166\n\t"
        "v_cmp_ne_u32_e64 s[92:93], 0, v166\n\t"
        "s_cmp_eq_u64 s[92:93], 0\n\t"
        "s_cbranch_scc1 NOFIXT7_%=\n\t"
        "s_sleep 1\n\t"
        "s_memrealtime s[92:93]\n\t"
        "s_waitcnt lgkmcnt(0)\n\t"
        "s_sub_u32 s92, s92, s90\n\t"
        "s_cmp_lt_u32 s92, " KDL "\n\t"
        "s_cbranch_scc1 FIXLT7_%=\n\t"
        "v_mov_b32 v166, %[bad]\n\t"
        "ds_write_b32 v166, v166\n\t"
        "NOFIXT7_%=:\n\t"
        "s_mov_b32 s94, 0\n\t"
        "SPINT7_%=:\n\t"
        "ds_read_b32 v164, %[vctl]\n\t"
        "s_waitcnt lgkmcnt(0)\n\t"
        "v_cmp_gt_u32_e64 s[86:87], s89, v164\n\t"
        "s_and_b64 s[86:87], s[86:87], 0x400\n\t"
        "s_cbranch_scc0 RESPEC_%=\n\t"
        SC3("s_add_u32 %[nfix], %[nfix], 1\n\t")
        "s_sleep 1\n\t"
        "s_add_u32 s94, s94, 1\n\t"
        "s_and_b32 s92, s94, 0x3ff\n\t"
        "s_cbranch_scc1 SPINT7_%=\n\t"
        "s_memrealtime s[92:93]\n\t"
        "s_waitcnt lgkmcnt(0)\n\t"
        "s_cmp_eq_u32 s94, 0x400\n\t"
        "s_cselect_b32 s90, s92, s90\n\t"
        "s_sub_u32 s92, s92, s90\n\t"
        "s_cmp_lt_u32 s92, " KDL3 "\n\t"
        "s_cbranch_scc1 SPINT7_%=\n\t"
        "v_mov_b32 v166, %[bad]\n\t"
        "ds_write_b32 v166, v166\n\t"
        "s_branch RESPEC_%=\n\t"
        "RESPEC_%=:\n\t"
        "v_mov_b32 v164, %[bad]\n\t"
        MF("v_mfma_f32_16x16x32_f16 v[116:119], v[128:131], v[120:123], v[40:43]\n\t")
        MF("v_mfma_f32_16x16x32_f16 v[116:119], v[132:135], v[124:127], v[116:119]\n\t")
        "s_branch TAILS_%=\n\t"
        "ENTRY2_%=:\n\t"
        "s_mov_b64 s[96:97], %[xtail]\n\t"
        "s_movk_i32 s95, 0x2000\n\t"
        "s_lshl_b32 s89, %[stage], 2\n\t"
        "v_mov_b32 v165, %[bad]\n\t"
        "s_setprio " KPRM "\n\t"
        "s_mov_b32 s94, 0\n\t"
        "SPINTK_%=:\n\t"
        "ds_read_b32 v164, v165 offset:" KTOK "\n\t"
        "s_waitcnt lgkmcnt(0)\n\t"
        "v_cmp_gt_u32_e64 s[86:87], s89, v164\n\t"
        "s_cmp_eq_u64 s[86:87], 0\n\t"
        "s_cbranch_scc1 GOTK_%=\n\t"
        SC3("s_add_u32 %[nfix], %[nfix], 1\n\t")
        "s_sleep 1\n\t"
        "s_add_u32 s94, s94, 1\n\t"
        "s_and_b32 s92, s94, 0x3ff\n\t"
        "s_cbranch_scc1 SPINTK_%=\n\t"
        "s_memrealtime s[92:93]\n\t"
        "s_waitcnt lgkmcnt(0)\n\t"
        "s_cmp_eq_u32 s94, 0x400\n\t"
        "s_cselect_b32 s90, s92, s90\n\t"
        "s_sub_u32 s92, s92, s90\n\t"
        "s_cmp_lt_u32 s92, " KDL3 "\n\t"
        "s_cbranch_scc1 SPINTK_%=\n\t"
        "v_mov_b32 v166, %[bad]\n\t"
        "ds_write_b32 v166, v166\n\t"
        "s_branch GOTK_%=\n\t"
        "GOTK_%=:\n\t"
        "global_load_dwordx4 v[112:115], %[l16], s[96:97] offset:0 " POL_FAR "\n\t"
        "global_load_dwordx4 v[116:119], %[l16], s[96:97] offset:1024 " POL_FAR "\n\t"
        "s_add_u32 s96, s96, s95\n\t"
        "s_addc_u32 s97, s97, 0\n\t"
        "s_add_u32 s92, 0, 0\n\t"
        "v_add_u32 v165, s92, %[l16]\n\t"
        "ds_read_b128 v[128:131], v165 offset:0\n\t"
        "ds_read_b128 v[132:135], v165 offset:1024\n\t"
        "ds_read_b128 v[136:139], v165 offset:2048\n\t"
        "ds_read_b128 v[140:143], v165 offset:3072\n\t"
        "ds_read_b128 v[144:147], v165 offset:4096\n\t"
        "ds_read_b128 v[148:151], v165 offset:5120\n\t"
        "ds_read_b128 v[152:155], v165 offset:6144\n\t"
        "ds_read_b128 v[156:159], v165 offset:7168\n\t"
        "ds_read_b128 v[160:163], v165 offset:8192\n\t"
        "s_waitcnt lgkmcnt(0)\n\t"
        "global_load_dwordx4 v[120:123], %[l16], s[96:97] offset:0 " POL_FAR "\n\t"
        "global_load_dwordx4 v[124:127], %[l16], s[96:97] offset:1024 " POL_FAR "\n\t"
        "s_add_u32 s96, s96, s95\n\t"
        "s_addc_u32 s97, s97, 0\n\t"
        "s_waitcnt vmcnt(2)\n\t"
        "s_setprio " KPRC "\n\t"
        "v_bitop3_b32 v166, v112, v116, %[tag] bitop3:0x7e\n\t"
        "v_and_b32 v166, 1, v166\n\t"
        TG("v_cmp_ne_u32_e64 s[84:85], 0, v166\n\t")
        NTG("s_mov_b64 s[84:85], 0\n\t")
        "s_waitcnt lgkmcnt(0)\n\t"
        "s_cmp_eq_u64 s[84:85], 0\n\t"
        "s_cbranch_scc0 FIXT0_%=\n\t"
        "GO1_%=:\n\t"
        "s_cmp_eq_u32 %[stage], 1\n\t"
        "s_cbranch_scc1 TAIL0_%=\n\t"
        TRM("s_memrealtime s[90:91]\n\t")
        TRM("s_mov_b32 s93, 16\n\t")
        TRM("s_mov_b32 m0, s93\n\t")
        TRM("s_waitcnt lgkmcnt(0)\n\t")
        TRM("v_writelane_b32 %[tr], s90, m0\n\t")
        MF("v_mfma_f32_16x16x32_f16 v[8:11], v[112:115], v[128:131], v[44:47]\n\t")
        "global_load_dwordx4 v[80:83], %[l16], s[96:97] offset:0 " POL_FAR "\n\t"
        "ds_read_b128 v[128:131], v165 offset:9216\n\t"
        MF("v_mfma_f32_16x16x32_f16 v[12:15], v[112:115], v[132:135], v[48:51]\n\t")
        "global_load_dwordx4 v[84:87], %[l16], s[96:97] offset:1024 " POL_FAR "\n\t"
        "ds_read_b128 v[132:135], v165 offset:10240\n\t"
        MF("v_mfma_f32_16x16x32_f16 v[16:19], v[112:115], v[136:139], v[52:55]\n\t")
        "s_add_u32 s96, s96, s95\n\t"
        "s_addc_u32 s97, s97, 0\n\t"
        "ds_read_b128 v[136:139], v165 offset:11264\n\t"
        MF("v_mfma_f32_16x16x32_f16 v[20:23], v[112:115], v[140:143], v[56:59]\n\t")
        "ds_read_b128 v[140:143], v165 offset:12288\n\t"
        MF("v_mfma_f32_16x16x32_f16 v[24:27], v[112:115], v[144:147], v[60:63]\n\t")
        "ds_read_b128 v[144:147], v165 offset:13312\n\t"
        MF("v_mfma_f32_16x16x32_f16 v[28:31], v[112:115], v[148:151], v[64:67]\n\t")
        "ds_read_b128 v[148:151], v165 offset:14336\n\t"
        MF("v_mfma_f32_16x16x32_f16 v[32:35], v[112:115], v[152:155], v[68:71]\n\t")
        "ds_read_b128 v[152:155], v165 offset:15360\n\t"
        MF("v_mfma_f32_16x16x32_f16 v[36:39], v[112:115], v[156:159], v[72:75]\n\t")
        "ds_read_b128 v[156:159], v165 offset:16384\n\t"
        MF("v_mfma_f32_16x16x32_f16 v[40:43], v[160:163], v[112:115], v[76:79]\n\t")
        "ds_read_b128 v[160:163], v165 offset:17408\n\t"
        "s_branch TAILH_%=\n\t"
        "DONE_%=:\n\t"
        : [nfix] "+s"(nfix), [stage] "+s"(k.stage), [xtail] "+s"(k.xtail), [outp] "+s"(k.outp) TRO
        : [vctl] "v"(k.vctl), [l16] "v"(k.l16), [tag] "s"(k.tag), [bad] "s"(k.bad), [xc] "s"(k.xc),
          [xloc] "s"(k.xloc), [xfar] "s"(k.xfar), [xt0] "s"(k.xt0), [out2] "s"(k.out2), [xtl2] "s"(k.xtl2)
        : "memory", "scc", "v8", "v9", "v10", "v11", "v12", "v13", "v14", "v15", "v16", "v17", "v18", "v19", "v20", "v21", "v22", "v23", "v24", "v25", "v26", "v27", "v28", "v29", "v30", "v31", "v32", "v33", "v34", "v35", "v36", "v37", "v38", "v39", "v40", "v41", "v42", "v43", "v44", "v45", "v46", "v47", "v48", "v49", "v50", "v51", "v52", "v53", "v54", "v55", "v56", "v57", "v58", "v59", "v60", "v61", "v62", "v63", "v64", "v65", "v66", "v67", "v68", "v69", "v70", "v71", "v72", "v73", "v74", "v75", "v76", "v77", "v78", "v79", "v80", "v81", "v82", "v83", "v84", "v85", "v86", "v87", "v88", "v89", "v90", "v91", "v92", "v93", "v94", "v95", "v96", "v97", "v98", "v99", "v100", "v101", "v102", "v103", "v104", "v105", "v106", "v107", "v108", "v109", "v110", "v111", "v112", "v113", "v114", "v115", "v116", "v117", "v118", "v119", "v120", "v121", "v122", "v123", "v124", "v125", "v126", "v127", "v128", "v129", "v130", "v131", "v132", "v133", "v134", "v135", "v136", "v137", "v138", "v139", "v140", "v141", "v142", "v143", "v144", "v145", "v146", "v147", "v148", "v149", "v150", "v151", "v152", "v153", "v154", "v155", "v156", "v157", "v158", "v159", "v160", "v161", "v162", "v163", "v164", "v165", "v166", "v167", "s84", "s85", "s86", "s87", "s88", "s89", "s90", "s91", "s92", "s93", "s94", "s95", "s96", "s97", "s98", "s99");
}

__device__ __forceinline__ int xcol(int col0) { return 1920 + (col0 >> 4) - col0; }
__device__ __forceinline__ void conv_w64(const Params& p, int tj, int col0, int h, int cg, int lane, const f32x4* smp, int& se) {
    const int g = lane >> 4, q = lane & 15;
    const float* wsrc = p.w + (size_t)(h * 32 + 4 * g) * NL + col0 + cg * 64 + 4 * q;
    f32x4 v[8];
#pragma unroll
    for (int i = 0; i < 8; ++i)
        v[i] = __builtin_nontemporal_load((const f32x4*)(wsrc + (size_t)((i & 3) + 16 * (i >> 2)) * NL));
    se = scale_exp(*smp);
    const float sc = exp2i(se);
#pragma unroll
    for (int i = 0; i < 8; ++i) v[i] *= sc;
    char* dst = p.myws + OFF_W16 + (size_t)(tj * NH + h) * WB + cg * 4096 + lane * 16;
#pragma unroll
    for (int f = 0; f < 4; ++f) {
        u32x4 o;
        o.x = pk16(v[0][f], v[1][f]); o.y = pk16(v[2][f], v[3][f]);
        o.z = pk16(v[4][f], v[5][f]); o.w = pk16(v[6][f], v[7][f]);
        st16_sys_p(dst + f * 1024, o);
    }
}
__device__ __forceinline__ void conv_w16(const Params& p, int tj, int col0, int h, int lane, const f32x4* smp, int& se) {
    const int g = lane >> 4, q = lane & 15;
    const float* wsrc = p.w + (size_t)(h * 32 + 4 * g) * NL + col0 + xcol(col0) + q;
    float e[8];
#pragma unroll
    for (int i = 0; i < 8; ++i) e[i] = __builtin_nontemporal_load(wsrc + (size_t)((i & 3) + 16 * (i >> 2)) * NL);
    se = scale_exp(*smp);
    const float sc = exp2i(se);
#pragma unroll
    for (int i = 0; i < 8; ++i) e[i] *= sc;
    u32x4 o;
    o.x = pk16(e[0], e[1]); o.y = pk16(e[2], e[3]); o.z = pk16(e[4], e[5]); o.w = pk16(e[6], e[7]);
    st16_sys_p(p.myws + OFF_W16 + (size_t)(tj * NH + h) * WB + 8192 + lane * 16, o);
}
__device__ __forceinline__ void conv_wblock(const Params& p, int tj, int col0, bool wide, int h, int lane, float sc) {
    const int g = lane >> 4, q = lane & 15;
    const float* wrow = p.w + (size_t)(h * 32 + 4 * g) * NL + col0;
    f32x4 v[2][8];
    float e[8];
#pragma unroll
    for (int cg = 0; cg < 2; ++cg)
#pragma unroll
        for (int i = 0; i < 8; ++i)
            v[cg][i] = __builtin_nontemporal_load((const f32x4*)(wrow + 4 * q + (size_t)((i & 3) + 16 * (i >> 2)) * NL + cg * 64));
    if (wide) {
#pragma unroll
        for (int i = 0; i < 8; ++i)
            e[i] = __builtin_nontemporal_load(wrow + (size_t)((i & 3) + 16 * (i >> 2)) * NL + xcol(col0) + q);
    }
    char* dst = p.myws + OFF_W16 + (size_t)(tj * NH + h) * WB + lane * 16;
#pragma unroll
    for (int cg = 0; cg < 2; ++cg)
#pragma unroll
        for (int f = 0; f < 4; ++f) {
            u32x4 o;
            o.x = pk16s(v[cg][0][f], v[cg][1][f], sc); o.y = pk16s(v[cg][2][f], v[cg][3][f], sc);
            o.z = pk16s(v[cg][4][f], v[cg][5][f], sc); o.w = pk16s(v[cg][6][f], v[cg][7][f], sc);
            st16_sys_p(dst + (cg * 4 + f) * 1024, o);
        }
    if (wide) {
        u32x4 o;
        o.x = pk16s(e[0], e[1], sc); o.y = pk16s(e[2], e[3], sc);
        o.z = pk16s(e[4], e[5], sc); o.w = pk16s(e[6], e[7], sc);
        st16_sys_p(dst + 8192, o);
    }
}

template <int NB>
__device__ __forceinline__ void consume(const Params& p, const unsigned ldsb, const int wave, const int lane,
                                        const int ti, const int col0, const unsigned call, const char* far, const unsigned xoff) {
    const int gr = wave >> 2;
    KArgs k;
    k.vctl = ldsb + CTRL + (lane == 0 ? C_FREE + wave * 4 : (lane == 1 ? C_RDYA : (lane < 10 ? C_FREE - 8 + lane * 4 : C_FULL)));
    k.l16 = lane * 16;
    k.tag = ((call + 1u) >> 1) & 1u;
    k.bad = ldsb + CTRL + C_BAD;
    k.xloc = p.myws + rfl(xoff + wave * 2048);
    k.xfar = far + (PF * 16384 + wave * 2048);
    k.xtail = far + (NM * 16384 + gr * (NT * 4096) + (wave & 3) * 2048);
    k.xt0 = gr ? far + ((NM - 1) * 16384 + wave * 2048) : k.xtail;
    k.stage = gr;
    k.outp = p.out + (size_t)((ti * 256 + wave * 32) * NL + col0);
    k.out2 = k.outp + 16 * NL;
    k.xtl2 = k.xtail + NT * 8192;
    k.xc = (unsigned)xcol(col0) * 4u;
    unsigned nfix = 0;
    __builtin_amdgcn_s_setprio(PRIO_C);
    unsigned tr = 0;
    if constexpr (NB == 9) kall9(k, nfix, tr);
    else kall8(k, nfix, tr);
    asm volatile("s_endpgm" ::: "memory");
    __builtin_unreachable();
}

template <int O> __device__ __forceinline__ void xld(f32x4& d0, f32x4& d1, unsigned v, const float* x) {
    asm volatile("global_load_dwordx4 %0, %2, %3 offset:%4 nt\n\tglobal_load_dwordx4 %1, %2, %3 offset:%5 nt"
                 : "=&v"(d0), "=&v"(d1) : "v"(v), "s"(x), "i"(O), "i"(O + 64) : "memory");
}
template <int W> __device__ __forceinline__ void far_fetch(unsigned v, const char* far, unsigned sink) {
    asm volatile("s_mov_b32 m0, %2\n\ts_waitcnt vmcnt(%3)\n\tglobal_load_lds_dwordx4 %0, %1 " POL_FAR :: "v"(v), "s"(far), "s"(sink), "i"(W) : "memory");
}
typedef unsigned u32x16 __attribute__((ext_vector_type(16)));
typedef unsigned u32x8 __attribute__((ext_vector_type(8)));
template <class T> __device__ __forceinline__ T* mkptr(unsigned lo, unsigned hi) { return (T*)(((unsigned long long)hi << 32) | lo); }

struct SendArgs {
    u32x16 a; u32x8 b; unsigned rank;
    __device__ __forceinline__ void ask() {
        static_assert(offsetof(Params, myws) == 24 && offsetof(Params, dst) == 32 && offsetof(Params, rank) == 96, "argument layout");
        asm volatile("s_load_dwordx16 %0, %3, 0x0\n\ts_load_dwordx8 %1, %3, 0x40\n\ts_load_dword %2, %3, 0x60"
                     : "=&s"(a), "=&s"(b), "=&s"(rank) : "s"(__builtin_amdgcn_kernarg_segment_ptr()) : "memory");
    }
    __device__ __forceinline__ void wait() { asm volatile("s_waitcnt lgkmcnt(0)" : "+s"(a), "+s"(b), "+s"(rank) :: "memory"); }
    __device__ __forceinline__ const float* x() const { return mkptr<const float>(a[0], a[1]); }
    __device__ __forceinline__ char* myws() const { return mkptr<char>(a[6], a[7]); }
    __device__ __forceinline__ char* dst(int j) const { return j < 4 ? mkptr<char>(a[8 + 2 * j], a[9 + 2 * j]) : mkptr<char>(b[2 * j - 8], b[2 * j - 7]); }
};

__global__ void __launch_bounds__(NWAVE * 64, 3) ag_gemm_silu(Params p) {
    extern __shared__ __attribute__((aligned(16))) char lds_raw[];
    LAS char* lds = (LAS char*)lds_raw;
    constexpr unsigned ldsb = 0;
    asm volatile("s_load_dword s100, %0, 0x0\n\ts_load_dword s101, %0, 0x40" :: "s"(__builtin_amdgcn_kernarg_segment_ptr()) : "memory", "s100", "s101");
    SB0();
    const int tid = threadIdx.x;
    const int wave = rfl(tid >> 6), lane = tid & 63;
    const int b = blockIdx.x;
    const int g = lane >> 4, q = lane & 15;
    const unsigned cnto = (unsigned)OFF_CNT + (unsigned)(b * 16 + wave) * 4u;

    const unsigned role = b < NCWG ? (unsigned)wave : (b < NCWG + SW_N && (unsigned)(wave - SW_W) < 2u ? (unsigned)(wave - SW_W + 1) : 8u);
    if (__builtin_expect(role < 4u, 1)) do {
        {
            unsigned long long pc; unsigned v;
            asm volatile("s_cmp_eq_u32 %3, 3\n\ts_cselect_b64 exec, -1, 0\n\ts_getpc_b64 %1\n\tglobal_load_dword %0, %2, %1\n\ts_mov_b64 exec, -1"
                         : "=v"(v), "=&s"(pc) : "v"(lane * CODE_PF), "s"(role) : "memory", "scc");
        }
        SendArgs ka;
        ka.ask();
        fp16_saturate();
        constexpr int P = COMM_P, NF = COMM_NF;
        constexpr int ND = NR - 1;
        constexpr int H0 = SW_N / 32 * SW_I;
        static_assert(DRY_N <= NF * P && ND + (P - 1) * (3 + ND) <= 63 && NF >= 2 && H0 + NF * P <= PF && NCW == 32 && CWPG == 2 && SW_N % 32 == 0 && SW_I <= P, "comm window");
        const bool dry = role == 2u;
        const unsigned cw = b < NCWG ? (unsigned)(wave * COMM_WGS + b) : (unsigned)(b - NCWG);
        const unsigned h0 = b < NCWG ? (unsigned)H0 : (cw >> 5) * SW_I;
        const unsigned last = rfl(b < NCWG ? (dry ? (unsigned)DRY_N : ~0u) : (unsigned)SW_I);
        unsigned vx = (((cw & 31u) * 16u + q) * KD + h0 * 32u + 4u * g) * 4u;
        unsigned vs = x_soff(lane);
        unsigned vc = cnto;
        unsigned vd = OFF_RX + TILE_GAP + ((cw >> 4) & 1u) * RT_B + h0 * 16384u + (cw & 15u) * 1024u + lane * 16u;
        unsigned eall;
        asm volatile("s_ashr_i32 %0, %1, 31" : "=s"(eall) : "s"(role - 2u) : "scc");
        f32x4 xa[P][2];
        unsigned av[P];
        unsigned cur[P];
#pragma unroll
        for (int s = 0; s < P; ++s) { cur[s] = 0u; av[s] = 0u; }
        asm volatile("s_mov_b32 exec_lo, %4\n\ts_mov_b32 exec_hi, %4" : "+v"(vx), "+v"(vs), "+v"(vc), "+v"(vd) : "s"(eall) : "memory");
        ka.wait();
        const float* xp = ka.x();
        char* myws = ka.myws();
#define XLD(d0_, d1_, v_, o_) xld<(o_)>(d0_, d1_, v_, xp)
#define VOFF(i_, tb_) ((((i_) >> 4) & 1u) * (tb_) + ((i_) >> 5) * 16384u + ((i_) & 15u) * 1024u + lane * 16u)
        unsigned cv;
        f32x4 sv;
        asm volatile("global_load_dword %0, %1, %2 sc1" : "=v"(cv) : "v"(vc), "s"(myws) : "memory");
        XLD(xa[0][0], xa[0][1], vx, 0);
        asm volatile("global_load_dwordx4 %0, %1, %2" : "=v"(sv) : "v"(vs), "s"(xp) : "memory");
        static_for<P>([&](auto sc) { constexpr int s = decltype(sc)::value; if (s || !LD_ORD) XLD(xa[s][0], xa[s][1], vx, s * 128); });
        asm volatile("s_cmp_lg_u32 %8, 3\n\ts_cbranch_scc1 GO_%=\n\t"
                     "s_load_dword s100, %0, %7\n\ts_load_dword s100, %1, %7\n\ts_load_dword s100, %2, %7\n\ts_load_dword s100, %3, %7\n\t"
                     "s_load_dword s100, %4, %7\n\ts_load_dword s100, %5, %7\n\ts_load_dword s100, %6, %7\n\t"
                     "s_waitcnt vmcnt(0) lgkmcnt(0)\n\t"
                     "s_endpgm\n\t"
                     "GO_%=:"
                     :: "s"(ka.dst(0)), "s"(ka.dst(1)), "s"(ka.dst(2)), "s"(ka.dst(3)), "s"(ka.dst(4)), "s"(ka.dst(5)), "s"(ka.dst(6)),
                        "s"(OFF_RX + (ka.rank << 21) + 128u * b), "s"(role) : "memory", "scc", "s100");
        if (!DRY_N) {
            unsigned l = role;
            asm volatile("s_mov_b64 exec, -1" : "+s"(l) :: "memory");
            if (l == 2u) { STAMP(0); if (b < NCWG) return; break; }
        }
        vd += ka.rank << 21;
        asm volatile("s_waitcnt vmcnt(%3)" : "+v"(cv), "+v"(sv), "+v"(vd) : "i"(LD_ORD ? 2 * P - 2 : 2 * P) : "memory");
        const float xsc = exp2i(scale_exp(sv));
        const unsigned call = rfl(cv) + 1u;
        const unsigned tag = ((call + 1u) >> 1) & 1u;
        vd += (call & 1u) * 2u * RT_B;
        const unsigned vzero = 0u;
        int left = EARLY_N && b >= 8 ? EARLY_N : 0x10000;
        unsigned ib[P];
#pragma unroll
        for (int s = 0; s < P; ++s) ib[s] = 0u;
        auto body = [&](auto sc, auto rc) -> bool {
            constexpr int s = decltype(sc)::value, r = decltype(rc)::value;
            constexpr int OPR = ND + (r == NF - 2 || (COMM_DIR && r > NF - 2) ? 3 : 2), OPP = ND + (r - 1 == NF - 2 || (COMM_DIR && r - 1 > NF - 2) ? 3 : 2);
            constexpr int W = r == 0 ? 2 * (P - 1 - s) + s * OPR : ND + (P - 1 - s) * OPP + s * OPR;
            if constexpr (r < NF && r * P + s > 0 && (r * P + s == DRY_N || (SW_N && r * P + s == SW_I))) {
                unsigned l = last;
                asm volatile("s_mov_b64 exec, -1" : "+s"(l) :: "memory");
                if (__builtin_expect(l == (unsigned)(r * P + s), 0)) return false;
                asm volatile("s_mov_b32 exec_lo, %0\n\ts_mov_b32 exec_hi, %0" :: "s"(eall + 0u) : "memory");
            }
            if constexpr (r < NF || COMM_DIR)
                asm volatile("s_waitcnt vmcnt(%3)" : "+v"(xa[s][0]), "+v"(xa[s][1]), "+v"(av[s]) : "i"(W) : "memory");
            else
                asm volatile("s_waitcnt vmcnt(%2)" : "+v"(xa[s][0]), "+v"(av[s]) : "i"(W) : "memory");
            const unsigned it = cur[s];
            if constexpr (r >= NF) if (it >= (unsigned)NITEM) return false;
            u32x4 o;
            unsigned voff;
            if constexpr (r < NF) {
                o = cvt_tag(xa[s][0] * xsc, xa[s][1] * xsc, tag);
                voff = vd + (unsigned)((r * P + s) * 16384);
            } else if constexpr (COMM_DIR) {
                o = cvt_tag(xa[s][0] * xsc, xa[s][1] * xsc, tag);
                voff = VOFF(it, RT_B) + (OFF_RX + TILE_GAP + (ka.rank << 21) + (call & 1u) * 2u * RT_B);
            } else {
                asm volatile("v_mov_b32 %0, %4\n\tv_mov_b32 %1, %5\n\tv_mov_b32 %2, %6\n\tv_mov_b32 %3, %7"
                             : "=&v"(o.x), "=&v"(o.y), "=&v"(o.z), "=&v"(o.w)
                             : "v"(xa[s][0].x), "v"(xa[s][0].y), "v"(xa[s][0].z), "v"(xa[s][0].w));
                if (__ballot((o.x & 1u) != tag) != 0ull) {
                    f32x4 y0, y1;
                    XLD(y0, y1, (((it & 31u) * 16u + q) * KD + (it >> 5) * 32u + 4u * g) * 4u, 0);
                    asm volatile("s_waitcnt vmcnt(0)" : "+v"(y0), "+v"(y1) :: "memory");
                    o = cvt_tag(y0 * xsc, y1 * xsc, tag);
                }
                voff = VOFF(it, RT_B) + (OFF_RX + TILE_GAP + (ka.rank << 21) + (call & 1u) * 2u * RT_B);
            }
            asm volatile("" : "+v"(o));
            if constexpr (r >= NF - 1) cur[s] = ib[s] + rfl(av[s]);
            if constexpr (r >= NF - 2) {
                const unsigned vone = left > 0 ? 1u : 0u;
                ib[s] = left > 0 ? (unsigned)((H0 + NF * P) * 32) : (unsigned)NITEM;
                --left;
                asm volatile("s_and_b64 exec, exec, 1\n\tglobal_atomic_add %0, %1, %2, %3 sc0 sc1\n\ts_mov_b32 exec_lo, %4\n\ts_mov_b32 exec_hi, %4"
                             : "=&v"(av[s]) : "v"(vzero), "v"(vone), "s"(myws + OFF_ICTR + (call & 1u) * 256u), "s"(eall + 0u) : "memory", "scc");
            }
            if constexpr (r < NF - 1) {
                XLD(xa[s][0], xa[s][1], vx, ((r + 1) * P + s) * 128);
            } else {
                const unsigned in = cur[s] < (unsigned)NITEM ? cur[s] : (unsigned)NITEM - 1u;
                if constexpr (COMM_DIR) XLD(xa[s][0], xa[s][1], (((in & 31u) * 16u + q) * KD + (in >> 5) * 32u + 4u * g) * 4u, 0);
                else
                asm volatile("global_load_dwordx4 %0, %1, %2 sc0 sc1" : "+v"(xa[s][0]) : "v"(VOFF(in, TILE_B)), "s"(myws + (OFF_X16 + (size_t)(call & 1u) * X16_B)) : "memory");
            }
#define ST1(n_) "global_store_dwordx4 %0, %1, %" #n_ " " POL_COMM "\n\t"
            asm volatile(ST1(2) ST1(3) ST1(4) ST1(5) ST1(6) ST1(7) ST1(8) "s_nop 1"
                         :: "v"(voff), "v"(o), "s"(ka.dst(0)), "s"(ka.dst(1)), "s"(ka.dst(2)), "s"(ka.dst(3)), "s"(ka.dst(4)), "s"(ka.dst(5)), "s"(ka.dst(6)) : "memory");
            return true;
        };
        bool go = true;
        static_for<NF>([&](auto rc) { static_for<P>([&](auto sc) { if (go) go = body(sc, rc); }); });
        if (DRY_N == NF * P) {
            unsigned l = last;
            asm volatile("s_mov_b64 exec, -1" : "+s"(l) :: "memory");
            if (l == (unsigned)DRY_N) go = false;
        }
        while (go) static_for<P>([&](auto sc) { if (go) go = body(sc, IC<NF>{}); });
#pragma unroll
        for (int s = 0; s < P; ++s)
            asm volatile("s_waitcnt vmcnt(0)" : "+v"(xa[s][0]), "+v"(xa[s][1]), "+v"(av[s]) :: "memory");
        if (b >= NCWG) break;
        if (lane == 0 && !dry) {
            __hip_atomic_store((unsigned*)(myws + cnto), call, __ATOMIC_RELAXED, __HIP_MEMORY_SCOPE_AGENT);
            if (cw == 0) st4_sys_p(myws + OFF_ICTR + ((call + 1u) & 1u) * 256u, 0u);
        }
        STAMP(1);
        return;
    } while (0);
    asm volatile("" :: "s"(p.x), "s"(p.w), "s"(p.out), "s"(p.myws), "s"(p.rank), "s"(p.map), "s"(p.dst[0]), "s"(p.dst[1]), "s"(p.dst[2]),
                       "s"(p.dst[3]), "s"(p.dst[4]), "s"(p.dst[5]), "s"(p.dst[6]), "s"(p.dst[7]));
    char* myws = p.myws;
    unsigned* cntp = (unsigned*)(myws + cnto);
    STAMP(0);
    const unsigned tstart = (unsigned)rt();
    if (b < NCWG) {
        if (wave >= 4 && wave < 4 + PF_WV) {
            const int sr = my_src(p.map);
            const unsigned call = rfl(__hip_atomic_load(cntp, __ATOMIC_RELAXED, __HIP_MEMORY_SCOPE_AGENT)) + 1u;
            if (lane == 0) __hip_atomic_store(cntp, call, __ATOMIC_RELAXED, __HIP_MEMORY_SCOPE_AGENT);
            if (sr == p.rank) return;
            const char* far = rank_ws(p, sr) + (OFF_X16 + (size_t)(call & 1u) * X16_B);
            const char* look = myws + (lane != 1 || !PF_PROBE ? OFF_XEXP + 4096u + (call & 1u) * 1024u + sr * 128 + (lane & 2) * 2
                                                 : OFF_RX + TILE_GAP + ((unsigned)sr << 21) + (call & 1u) * 2u * RT_B + (PF - PF_K) * 16384 - 16);
            const unsigned sink = ldsb + wave * 1024;
            const unsigned long long t0 = rt();
            __builtin_amdgcn_s_sleep(100);
            if (lane < 3) asm volatile("s_mov_b32 m0, %1\n\ts_nop 0\n\tglobal_load_lds_dword %0, off" :: "v"(far + ((size_t)lane * TILE_B - 128)), "s"(sink) : "memory");
            while (rt() - t0 < (unsigned)(PF_PROBE ? (PF - PF_K - 3) * W_TICK : T_POLL)) __builtin_amdgcn_s_sleep(16);
            for (;;) {
                unsigned v = 0;
                if (lane < 3) asm volatile("global_load_dword %0, %1, off sc0 sc1" : "=v"(v) : "v"(look) : "memory");
                asm volatile("s_waitcnt vmcnt(0)" : "+v"(v) :: "memory");
                if (umin(__builtin_amdgcn_readlane(v, 0) - call * 256u, __builtin_amdgcn_readlane(v, 2) - call * 256u) < 256u &&
                    (!PF_PROBE || ((__builtin_amdgcn_readlane(v, 1) ^ (call + 1u) >> 1) & 1u) == 0u)) break;
                if (rt() - t0 > DEADLINE) return;
                __builtin_amdgcn_s_sleep(PF_SLEEP);
            }
            STAMP(2);
            unsigned i;
            i = (PF + (EARLY_N ? EARLY_PF : 0)) * 32 + (b >> 3) * PF_WV + wave - 4;
#define PF_NEXT i += 2 * PF_WV
#define PF_WW decltype(wc)::value
            auto fetch = [&](unsigned end, auto wc) {
                for (; i < end; PF_NEXT) {
                    far_fetch<PF_WW>(PF_IL ? ((i >> 3) & 1u) * TILE_B + (i >> 4) * 8192u + (i & 7u) * 1024u + lane * 16u
                                                         : ((i >> 4) & 1u) * TILE_B + (i >> 5) * 16384u + (i & 15u) * 1024u + lane * 16u, far, sink);
                }
            };
            static_assert(PF_S2 <= NA - 2 && PF_S3 >= NA, "phases");
            fetch(PF_S2 * 32, IC<PF_W>{});
            fetch((NA - 2) * 32, IC<PF_W2>{});
            unsigned v = 0;
            if (lane == 0) asm volatile("global_load_dword %0, %1, off sc0 sc1" : "=v"(v) : "v"(look) : "memory");
            fetch(NA * 32, IC<PF_W2>{});
            asm volatile("s_waitcnt vmcnt(%1)" : "+v"(v) : "i"(PF_W2 < 64 / (2 * PF_WV) ? PF_W2 : 64 / (2 * PF_WV)) : "memory");
            while (rfl(v) - call * 256u >= 256u) {
                if (rt() - t0 > DEADLINE) { VMW(0); return; }
                __builtin_amdgcn_s_sleep(4);
                if (lane == 0) asm volatile("global_load_dword %0, %1, off sc0 sc1" : "=v"(v) : "v"(look) : "memory");
                asm volatile("s_waitcnt vmcnt(0)" : "+v"(v) :: "memory");
            }
            fetch(PF_S3 * 32, IC<PF_W2>{});
            fetch((PF + PF_N) * 32, IC<PF_W3>{});
            VMW(0);
            STAMP(3);
            return;
        }
        return;
    }

    const unsigned cv0 = __hip_atomic_load(cntp, __ATOMIC_RELAXED, __HIP_MEMORY_SCOPE_AGENT);
    const int tj = map_tj(b);
    const int ti = my_src(p.map) * 2 + ((b >> 3) & 1);
    const bool wide = !(tj & 1);
    const int col0 = tj * 128;
    if (tid < 128) *(LAS unsigned*)(lds + CTRL + tid * 4) = (tid >= C_FREE / 4 && tid < 64) ? (unsigned)LSTEP : (tid == C_ALL / 4 ? NS : 0u);
    asm volatile("s_waitcnt lgkmcnt(0)" ::: "memory");
    __builtin_amdgcn_s_barrier();
    asm volatile("" ::: "memory");
    fp16_saturate();
    char* wflag = myws + OFF_WFLAG + (size_t)(tj * 16 + ti) * 128;
    if (wave < 3) {
        {
            while ((unsigned)rt() - tstart < (unsigned)(W0_DLY + (ti < W0_CAP ? ti : W0_CAP) * W0_STEP)) __builtin_amdgcn_s_sleep(8);
        }
        int wexp = 0;
        const f32x4* smp = w_sample(p.w, lane);
        if (wave < 2) conv_w64(p, tj, col0, ti, wave, lane, smp, wexp);
        else if (wide) conv_w16(p, tj, col0, ti, lane, smp, wexp);
        if (wave == 0) lds_wr32(ldsb + CTRL + C_XEXP + 4, (unsigned)wexp);
        unsigned n;
        asm volatile("s_waitcnt vmcnt(0)\n\ts_mov_b64 exec, 1\n\tds_add_rtn_u32 %0, %1, %2\n\ts_mov_b64 exec, -1\n\ts_waitcnt lgkmcnt(0)"
                     : "=&v"(n) : "v"(ldsb + CTRL + C_WCNT), "v"(1u) : "memory");
        if (rfl(n) == 2u && lane == 0) st4_sys_p(wflag + 4, (cv0 + 1u) * 256u + 1u);
    }
#pragma clang loop unroll(disable)
    for (unsigned xs_ = 0; xs_ < 5u; ++xs_) {
        if ((((unsigned)XC_MAP >> (4u * xs_)) & 15u) != (unsigned)wave) continue;
        const unsigned u = xs_ * (256 - NCWG) + (b - NCWG);
        const float* xs = p.x + (size_t)((u & 31u) * 16u + q) * KD + ((u >> 5) & 31u) * 128u + 4 * g;
        f32x4 v[8];
        {
            const unsigned dly = (u >> 5) >= (unsigned)X_DK ? X_DLY + ((u >> 5) - X_DK) * X_DSTEP + ((u >> 5) >= (unsigned)X_DK2 ? X_DLY2 : 0) : X0_DLY;
            while ((unsigned)rt() - tstart < dly) __builtin_amdgcn_s_sleep(8);
        }
#pragma unroll
        for (int i = 0; i < 8; ++i) v[i] = XC_NT ? __builtin_nontemporal_load((const f32x4*)(xs + i * 16)) : *(const f32x4*)(xs + i * 16);
        const int xe = scale_exp(*x_sample(p.x, lane));
        const unsigned call = rfl(cv0) + 1u;
        if (u < 1024u) {
            const float xsc = exp2i(xe);
            const unsigned tag = ((call + 1u) >> 1) & 1u;
            char* dst = myws + OFF_X16 + (size_t)(call & 1u) * X16_B + (size_t)((u >> 4) & 1u) * TILE_B + lane * 16;
#pragma unroll
            for (int i = 0; i < 4; ++i)
                st16_sys_p(dst + piece_pos((u >> 5) * 4u + i, u & 15u) * 1024u, cvt_tag(v[2 * i] * xsc, v[2 * i + 1] * xsc, tag));
        }
        unsigned n;
        asm volatile("s_waitcnt vmcnt(0)\n\ts_mov_b64 exec, 1\n\tds_add_rtn_u32 %0, %1, %2\n\ts_mov_b64 exec, -1\n\ts_waitcnt lgkmcnt(0)"
                     : "=&v"(n) : "v"(ldsb + CTRL + C_XCNT), "v"(xs_ < (unsigned)X_NA ? 0x101u : 1u) : "memory");
        const unsigned nw = rfl(n);
#pragma clang loop unroll(disable)
        for (unsigned lv = 0; lv < 2u; ++lv) {
            if (lv ? (nw & 255u) != 4u : (xs_ >= (unsigned)X_NA || (nw >> 8) != (unsigned)(X_NA - 1))) continue;
            asm volatile("s_mov_b64 exec, 1\n\tglobal_atomic_add %0, %1, %2, off sc0 sc1\n\ts_mov_b64 exec, -1\n\ts_waitcnt vmcnt(0)"
                         : "=&v"(n) : "v"(myws + OFF_XCNT + (call & 1u) * 128u + lv * 256u), "v"(1u) : "memory");
            if (rfl(n) == (unsigned)(255 - NCWG)) {
                const char* d = p.dst[0];
#pragma unroll
                for (int j = 1; j < NR; ++j) d = ((lane & 7) == j) ? p.dst[j] : d;
                while ((unsigned)rt() - tstart < (unsigned)T_GO) __builtin_amdgcn_s_sleep(2);
                if (lane < 2 * NR) st4_sys_p(d + OFF_XEXP + (lane >> 3) * 4096 + (call & 1u) * 1024u + p.rank * 128 + (4u - lv * 4u), call * 256u + (unsigned)(xe + 64));
                if (lane == 0) st4_sys_p(myws + OFF_XCNT + ((call + 1u) & 1u) * 128u + lv * 256u, 0u);
            }
        }
    }
    const unsigned call = rfl(cv0) + 1u;
    if (lane == 0) __hip_atomic_store(cntp, call, __ATOMIC_RELAXED, __HIP_MEMORY_SCOPE_AGENT);
    const unsigned xown = (unsigned)OFF_X16 + (call & 1u) * (unsigned)X16_B + (unsigned)(ti & 1) * TILE_B;
    const unsigned xoff = rfl((ti >> 1) == p.rank ? xown
                                                  : OFF_RX + TILE_GAP + ((unsigned)(ti >> 1) << 21) + ((call & 1u) * 2u + (ti & 1)) * RT_B);
    const unsigned wbase = call * 256u;
    STAMP(1);

    if (wave < 8) {
        const char* far = rank_ws(p, ti >> 1) + xown;
        DBGRET(1);
        VMW(0);
        if (wide) consume<9>(p, ldsb, wave, lane, ti, col0, call, far, xoff);
        else consume<8>(p, ldsb, wave, lane, ti, col0, call, far, xoff);
    } else if (wave == 8) {
        DBGRET(1);
        __builtin_amdgcn_s_setprio(PRIO_L);
        const unsigned ctrl = ldsb + CTRL;
        const char* src = myws + rfl((unsigned)OFF_W16 + (unsigned)tj * (NH * WB));
        const unsigned ctl = ctrl + (lane < 8 ? C_FREE + lane * 4 : C_RDYB);
        if (wide) glds_run<9, 0>(src, WB, ldsb, lane * 16, ctrl, ctl);
        else glds_run<8, 0>(src, WB, ldsb, lane * 16, ctrl, ctl);
        VMW(0);
        STAMP(2);
    } else if (wave == 9) {
        __builtin_amdgcn_s_setprio(PRIO_P);
        unsigned* tmo = (unsigned*)(myws + OFF_TMO);
        const unsigned tag = ((call + 1u) >> 1) & 1u;
        const char* wflagp = myws + OFF_WFLAG + (unsigned)(tj * 16 + (lane & 15)) * 128u;
        const char* xsrc = myws + xoff;
        unsigned ra = 0;
        unsigned rbw = 0, pw = POLL_W0;
        bool wok = false;
        const char* xexpp = myws + OFF_XEXP + (call & 1u) * 1024u + (ti >> 1) * 128;
        bool xok = false, xoka = false;
        unsigned long long tw = 0; unsigned sp = 0;
        auto expired = [&]() -> bool {
            if ((++sp & 63u) != 0u) return false;
            const unsigned tv = rfl(__hip_atomic_load(tmo, __ATOMIC_RELAXED, __HIP_MEMORY_SCOPE_AGENT));
            const unsigned long long t = rt();
            if (sp == 64u) tw = t;
            if (tv != call && t - tw <= DEADLINE) return false;
            if (lane == 0) __hip_atomic_store(tmo, call, __ATOMIC_RELAXED, __HIP_MEMORY_SCOPE_AGENT);
            lds_wr32(ldsb + CTRL + C_BAD, 1u);
            lds_wr32(ldsb + CTRL + C_RDYA, NS);
            lds_wr32(ldsb + CTRL + C_RDYB, NS);
            return true;
        };
        while (ra < NH || !wok) {
            unsigned fv = tag ^ 1u;
            u32x2 wv = {0u, 0u}, xv = {0u, 0u};
            if (!wok && lane < 16)
                asm volatile("global_load_dwordx2 %0, %1, off sc0 sc1" : "=v"(wv) : "v"(wflagp) : "memory");
            if (!xok && lane == 0 && ra + XP_LATE >= (unsigned)(XP_LATE ? PF : 0))
                asm volatile("global_load_dwordx2 %0, %1, off sc0 sc1" : "=v"(xv) : "v"(xexpp) : "memory");
            if (lane < pw && ra + lane < (unsigned)PF)
                asm volatile("global_load_dword %0, %1, %2 sc0 sc1" : "=v"(fv) : "v"((ra + lane) * 16384u + 16368u), "s"(xsrc) : "memory");
            asm volatile("s_waitcnt vmcnt(0)" : "+v"(fv), "+v"(wv), "+v"(xv) :: "memory");
            bool act = false;
            if (!xok) {
                const unsigned xb = rfl(xv.x) - wbase, xe = umin(xb, rfl(xv.y) - wbase);
                xok = xb < 256u;
                if (!xoka && xe < 256u) {
                    xoka = true; lds_wr32(ldsb + CTRL + C_XEXP, xe);
                    STAMP(2);
                }
            }
            if (!wok) {
                unsigned c = wv.x - wbase;
                c = (c <= 8u ? c : (wv.y - wbase == 1u ? 1u : 0u)) * 16u + (lane & 15);
                const unsigned nb = umin(rfl(dppmin<0x140>(dppmin<0x141>(dppmin<0x4E>(dppmin<0xB1>(c))))), (unsigned)NH);
                if (nb > rbw) { rbw = nb; lds_wr32(ldsb + CTRL + C_RDYB, nb); act = true; }
                wok = nb == (unsigned)NH;
            }
            if (ra < NH) {
                const unsigned long long m = ~__ballot((fv & 1u) == tag);
                const unsigned np = (unsigned)__builtin_ctzll(m | 0x10000ull);
                pw = (np == pw) ? umin(2u * pw, 16u) : (unsigned)POLL_W;
                unsigned rn = ra + np;
                if (rn >= (unsigned)PF && xoka) rn = xok ? NS : (unsigned)NA;
                if (rn > ra) {
                    ra = rn; act = true;
                    if (rn > (unsigned)(KD_A - 2)) lds_wr32(ldsb + CTRL + C_RDYA, rn - (KD_A - 2));
                    if (rn == NS) STAMP(3);
                }
            }
            if (act) { sp = 0; continue; }
            if (expired()) break;
            if (ra < NH) __builtin_amdgcn_s_sleep(POLL_SLEEP);
            else __builtin_amdgcn_s_sleep(POLL_SLEEP2);
        }
        STAMP(4);
    } else if (wave == 11) {
        __builtin_amdgcn_s_setprio(PRIO_P);
#pragma clang loop unroll(disable)
        for (int jn = 1; jn < 8; ++jn) {
            const int hw = 16 * jn + ti, due = (hw * W_NUM >> 4) - W_LEAD;
            for (;;) {
                const int pos = (int)rfl(lds_rd32_wait(ldsb + CTRL + C_FREE)) - LSTEP;
                const int tnow = (int)((unsigned)rt() - tstart);
                if ((pos >= due || tnow >= due * W_TICK) && tnow >= W_TMIN + (jn - 1) * W_TSTEP && rfl(lds_rd32_wait(ldsb + CTRL + C_WCNT)) == 3u) break;
                __builtin_amdgcn_s_sleep(POLL_SLEEP2);
            }
            conv_wblock(p, tj, col0, wide, hw, lane, exp2i((int)rfl(lds_rd32_wait(ldsb + CTRL + C_XEXP + 4))));
            VMW(0);
            if (lane == 0) st4_sys_p(wflag, wbase + (unsigned)jn + 1u);
        }
        STAMP(4);
    }
}

}

extern "C" void kernel_launch(int rank, int nranks, ncclComm_t comm,
                              hipStream_t stream,
                              void* const* d_in, const int* in_sizes, int n_in,
                              void* d_out, int out_size,
                              const int* in_dtypes, int out_dtype,
                              void* const* d_out_all,
                              void* const* d_ws_all, size_t ws_bytes) {
    using namespace agk;
    if (nranks != NR || (unsigned)rank >= (unsigned)NR || ws_bytes < WS_NEED || n_in < 2) {
        fprintf(stderr, "ag_gemm_silu: unexpected configuration\n");
        return;
    }
    static bool attr_done = false;
    if (!attr_done) {
        HIP_CHECK(hipFuncSetAttribute((const void*)ag_gemm_silu, hipFuncAttributeMaxDynamicSharedMemorySize, LDS_B));
        attr_done = true;
    }
    Params p;
    p.x = (const float*)d_in[0];
    p.w = (const float*)d_in[1];
    p.out = (float*)d_out;
    p.myws = (char*)d_ws_all[rank];
    for (int i = 0; i < NR; ++i) p.dst[i] = (char*)d_ws_all[(rank + 1 + i) & 7];
    p.rank = rank;
    p.map = xcd_map(rank);
    p.dbg = 0; p.pad = 0;
    ag_gemm_silu<<<dim3(256), dim3(NWAVE * 64), LDS_B, stream>>>(p);
    HIP_CHECK(hipGetLastError());
}
```
